```python
import numpy as np

kernel_name = "gpu_gemm_a2a_m4096_k4096_n4096_bf16_qe4m3"


ABSMAX_RTOL = 0.09

COMPUTE_DTYPE = "bfloat16"
EPILOGUE = "quant_e4m3"
IN_QUANT = None
OUT_QUANT = "e4m3"
IN_DTYPES = {"x": "bfloat16", "w_mat": "bfloat16"}
WORLD_SIZE = 8

def _to_bf16(x):
    u = np.ascontiguousarray(x, dtype=np.float32).view(np.uint32)
    return ((u + 0x7FFF + ((u >> 16) & 1)) & 0xFFFF0000).view(np.float32)

def setup_inputs():

    x = _to_bf16(np.random.randn(4096, 4096).astype(np.float32))
    w_mat = _to_bf16((np.random.randn(4096, 4096) / np.sqrt(4096)).astype(np.float32))
    return {"x": x, "w_mat": w_mat}

def _snap_fp8(x, mant_bits, max_abs, min_normal):
    a = np.abs(x.astype(np.float64))
    _m, _e = np.frexp(a)
    q = np.where(a >= min_normal, np.ldexp(1.0, _e - 1 - mant_bits),
                 min_normal * 2.0 ** -mant_bits)
    snapped = np.minimum(np.round(a / q) * q, max_abs)
    return (np.sign(x) * snapped).astype(np.float32)
def _quant_dequant(y, amax):
    scale = amax / 448.0
    q = _snap_fp8(y / scale, 3, 448.0, 2.0 ** -6)
    return (q * scale).astype(np.float32)
def _epilogue(y):
    return _quant_dequant(y, np.abs(y).max())

def reference(x, w_mat):
    y = (x @ w_mat).astype(np.float32)
    return _epilogue(y)
```

```cpp
#include <hip/hip_runtime.h>
#include <rccl/rccl.h>
#include <cstdio>
#include <cstdint>

#define HIP_CHECK(expr) do { hipError_t _e = (expr); if (_e != hipSuccess) { \
    fprintf(stderr, "HIP error %s at %s:%d\n", hipGetErrorString(_e), __FILE__, __LINE__); \
    return; } } while (0)

namespace {

typedef __attribute__((ext_vector_type(8))) __bf16 bf16x8;
typedef __attribute__((ext_vector_type(4))) short s16x4;
typedef __attribute__((ext_vector_type(8))) short s16x8;
typedef __attribute__((ext_vector_type(4))) float f32x4;
typedef __attribute__((ext_vector_type(4))) unsigned u32x4;
typedef __attribute__((ext_vector_type(2))) unsigned u32x2;
typedef __attribute__((address_space(3))) s16x4* lds_s16x4_ptr;

constexpr int NR = 8;
constexpr int M_LOC = 512;
constexpr int K_TOT = 4096;
constexpr int N_TOT = 4096;
constexpr int N_LOC = N_TOT / NR;
constexpr int BM = 64, BN = 128, BK = 64;
constexpr int NS = 6;
constexpr int A_BYTES = BM * BK * 2;
constexpr int B_BYTES = BK * BN * 2;
constexpr int STAGE_BYTES = A_BYTES + B_BYTES;
constexpr int NKT = K_TOT / BK;
constexpr int NWG = (M_LOC / BM) * (N_TOT / BN);
constexpr int NTHREADS = 256;
constexpr int GLDS_PER_STAGE = STAGE_BYTES / 1024 / 4;
constexpr int EPI_PITCH = BN * 4 + 16;
constexpr int LDS_BYTES = NS * STAGE_BYTES + 64;
static_assert(NWG == 256, "one workgroup per CU");
static_assert(BM * EPI_PITCH <= NS * STAGE_BYTES, "epilogue staging must fit");

constexpr size_t WS_GRAN = 0;
constexpr size_t WS_DONE = WS_GRAN + 2 * NR * NWG * 8;
constexpr size_t WS_CTR = WS_DONE + NWG * 128;
constexpr size_t WS_TMO = WS_CTR + NWG * 128;
constexpr size_t WS_DIAG = WS_TMO + 128;
constexpr size_t WS_END = WS_DIAG + NWG * 16 * 8;

constexpr unsigned long long DEADLINE_TICKS = 50000000ull;

struct Params {
    const void* x;
    const void* w;
    void* out[NR];
    void* ws[NR];
    int rank;
};

__device__ __forceinline__ void glds16(unsigned voff, const void* sbase, unsigned lds_dst) {
    asm volatile("s_mov_b32 m0, %2\n\ts_nop 0\n\tglobal_load_lds_dwordx4 %0, %1"
                 :: "v"(voff), "s"(sbase), "s"(lds_dst) : "memory");
}

#define WAIT_VM(n) asm volatile("s_waitcnt vmcnt(%0)" :: "i"(n) : "memory")
#define WAIT_LGKM0() asm volatile("s_waitcnt lgkmcnt(0)" ::: "memory")
#define BARRIER() do { asm volatile("" ::: "memory"); __builtin_amdgcn_s_barrier(); asm volatile("" ::: "memory"); } while (0)

__device__ __forceinline__ unsigned wave_max_u32(unsigned v) {
    v = max(v, (unsigned)__builtin_amdgcn_update_dpp(0, (int)v, 0x111, 0xf, 0xf, false));
    v = max(v, (unsigned)__builtin_amdgcn_update_dpp(0, (int)v, 0x112, 0xf, 0xf, false));
    v = max(v, (unsigned)__builtin_amdgcn_update_dpp(0, (int)v, 0x114, 0xf, 0xf, false));
    v = max(v, (unsigned)__builtin_amdgcn_update_dpp(0, (int)v, 0x118, 0xf, 0xf, false));
    v = max(v, (unsigned)__builtin_amdgcn_update_dpp(0, (int)v, 0x142, 0xa, 0xf, false));
    v = max(v, (unsigned)__builtin_amdgcn_update_dpp(0, (int)v, 0x143, 0xc, 0xf, false));
    return (unsigned)__builtin_amdgcn_readlane((int)v, 63);
}

__device__ __forceinline__ u32x4 load16_sys(const void* p) {
    u32x4 r;
    asm volatile("global_load_dwordx4 %0, %1, off sc0 sc1" : "=v"(r) : "v"(p) : "memory");
    return r;
}
__device__ __forceinline__ void store16_sys(void* p, u32x4 v) {
    asm volatile("global_store_dwordx4 %0, %1, off sc0 sc1\n\ts_nop 1" :: "v"(p), "v"(v) : "memory");
}

#define STAMP(i) do {} while (0)

__global__ void __launch_bounds__(NTHREADS, 1) gemm_a2a_kernel(const Params p) {
    __shared__ __attribute__((aligned(1024))) char lds[LDS_BYTES];
    const int tid = threadIdx.x;
    const int lane = tid & 63;
    const int wave = __builtin_amdgcn_readfirstlane(tid >> 6);
    const int wg = blockIdx.x;
    const int rank = p.rank;
    char* const my_ws = (char*)p.ws[rank];
    STAMP(0);

    const int dest = wg & 7;
    const int idx = wg >> 3;
    const int tm = idx & 7, tn = idx >> 3;
    const int row0 = tm * BM;
    const int col0 = dest * N_LOC + tn * BN;

    unsigned* ctr = (unsigned*)(my_ws + WS_CTR + (size_t)wg * 128);
    const unsigned call = __hip_atomic_load(ctr, __ATOMIC_RELAXED, __HIP_MEMORY_SCOPE_AGENT) + 1u;

    const unsigned lds_base = (unsigned)(uintptr_t)lds;

    unsigned voffA[2], voffB[4];
#pragma unroll
    for (int i = 0; i < 2; ++i) {
        const int r = wave * 16 + i * 8 + (lane >> 3);
        const int c = (lane & 7) ^ ((r >> 1) & 7);
        voffA[i] = (unsigned)(r * (K_TOT * 2) + c * 16);
    }
#pragma unroll
    for (int j = 0; j < 4; ++j) {
        const int k = wave * 16 + j * 4 + (lane >> 4);
        const int h = (k & 3) | (((k >> 3) & 1) << 2);
        const int c = (lane & 15) ^ (2 * h);
        voffB[j] = (unsigned)(k * (N_TOT * 2) + c * 16);
    }
    const char* gA = (const char*)p.x + (size_t)row0 * (K_TOT * 2);
    const char* gB = (const char*)p.w + (size_t)col0 * 2;

    auto issue_stage = [&](int kt, int slot) {
        const char* a = gA + (size_t)kt * (BK * 2);
        const char* b = gB + (size_t)kt * ((size_t)BK * N_TOT * 2);
        const unsigned sa = lds_base + slot * STAGE_BYTES + wave * 2048;
        const unsigned sb = lds_base + slot * STAGE_BYTES + A_BYTES + wave * 4096;
        glds16(voffA[0], a, sa);
        glds16(voffA[1], a, sa + 1024);
        glds16(voffB[0], b, sb);
        glds16(voffB[1], b, sb + 1024);
        glds16(voffB[2], b, sb + 2048);
        glds16(voffB[3], b, sb + 3072);
    };

    unsigned rdA[2], rdB[2];
#pragma unroll
    for (int kk = 0; kk < 2; ++kk) {
        const int r = lane & 15;
        const int c = 4 * kk + (lane >> 4);
        rdA[kk] = (unsigned)(r * 128 + ((c ^ ((r >> 1) & 7)) * 16));
    }
#pragma unroll
    for (int cn = 0; cn < 2; ++cn) {
        const int g = lane >> 4, q = (lane & 15) >> 2, pp = lane & 3;
        const int k = 8 * g + q;
        const int h = (k & 3) | (((k >> 3) & 1) << 2);
        const int u = (wave * 2 + cn) ^ h;
        rdB[cn] = (unsigned)(A_BYTES + k * 256 + u * 32 + pp * 8);
    }

    f32x4 acc[4][2];
#pragma unroll
    for (int i = 0; i < 4; ++i)
#pragma unroll
        for (int j = 0; j < 2; ++j) acc[i][j] = (f32x4){0.f, 0.f, 0.f, 0.f};

    struct Frags { bf16x8 a[2][4]; bf16x8 b[2][2]; };

    auto read_frags = [&](Frags& f, int slot) {
        const char* base = lds + slot * STAGE_BYTES;
#pragma unroll
        for (int kk = 0; kk < 2; ++kk) {
#pragma unroll
            for (int cn = 0; cn < 2; ++cn) {
                s16x4 lo = __builtin_amdgcn_ds_read_tr16_b64_v4i16(
                    (lds_s16x4_ptr)(base + rdB[cn] + kk * 8192));
                s16x4 hi = __builtin_amdgcn_ds_read_tr16_b64_v4i16(
                    (lds_s16x4_ptr)(base + rdB[cn] + kk * 8192 + 1024));
                s16x8 v = __builtin_shufflevector(lo, hi, 0, 1, 2, 3, 4, 5, 6, 7);
                f.b[kk][cn] = __builtin_bit_cast(bf16x8, v);
            }
#pragma unroll
            for (int mt = 0; mt < 4; ++mt)
                f.a[kk][mt] = *(const bf16x8*)(base + rdA[kk] + mt * 2048);
        }
    };
    auto mma = [&](const Frags& f) {
#pragma unroll
        for (int kk = 0; kk < 2; ++kk)
#pragma unroll
            for (int mt = 0; mt < 4; ++mt)
#pragma unroll
                for (int cn = 0; cn < 2; ++cn)
                    acc[mt][cn] = __builtin_amdgcn_mfma_f32_16x16x32_bf16(f.b[kk][cn], f.a[kk][mt],
                                                                         acc[mt][cn], 0, 0, 0);
    };

#pragma unroll
    for (int s = 0; s < NS - 1; ++s) issue_stage(s, s);

    Frags f0, f1;
    WAIT_VM((NS - 2) * GLDS_PER_STAGE);
    BARRIER();
    read_frags(f0, 0);

    int slot_next = 1;
    int slot_fill = NS - 1;
    auto step = [&](Frags& cur, Frags& nxt, int kt, bool fill) {
        if (fill) { WAIT_VM((NS - 3) * GLDS_PER_STAGE); }
        else { WAIT_VM(0); }
        WAIT_LGKM0();
        BARRIER();
        if (fill) issue_stage(kt + NS - 1, slot_fill);
        read_frags(nxt, slot_next);
        mma(cur);
        slot_next = (slot_next == NS - 1) ? 0 : slot_next + 1;
        slot_fill = (slot_fill == NS - 1) ? 0 : slot_fill + 1;
    };
    constexpr int NFILL = NKT - NS + 1;
    static_assert((NFILL & 1) == 1, "loop structure assumes odd NFILL");
    for (int kt = 0; kt < NFILL - 1; kt += 2) {
        step(f0, f1, kt, true);
        step(f1, f0, kt + 1, true);
    }
    step(f0, f1, NFILL - 1, true);
    static_assert(NKT - 1 - NFILL == 4, "tail written for NS == 6");
    step(f1, f0, NFILL, false);
    step(f0, f1, NFILL + 1, false);
    step(f1, f0, NFILL + 2, false);
    step(f0, f1, NFILL + 3, false);
    mma(f1);
    STAMP(1);

    float am = 0.f;
#pragma unroll
    for (int i = 0; i < 4; ++i)
#pragma unroll
        for (int j = 0; j < 2; ++j)
#pragma unroll
            for (int e = 0; e < 4; ++e) am = fmaxf(am, fabsf(acc[i][j][e]));
    unsigned amb = wave_max_u32(__float_as_uint(am));
    unsigned* red = (unsigned*)(lds + NS * STAGE_BYTES);
    if (lane == 0) red[wave] = amb;
    __syncthreads();
    const unsigned par = call & 1u;
    if (wave == 0) {
        const unsigned m = max(max(red[0], red[1]), max(red[2], red[3]));
        if (lane < NR) {
            unsigned long long* g = (unsigned long long*)((char*)p.ws[lane] + WS_GRAN) +
                                    ((size_t)par * NR + rank) * NWG + wg;
            __hip_atomic_store(g, ((unsigned long long)call << 32) | m, __ATOMIC_RELAXED,
                               __HIP_MEMORY_SCOPE_SYSTEM);
        }
        if (lane == 0) __hip_atomic_store(ctr, call, __ATOMIC_RELAXED, __HIP_MEMORY_SCOPE_AGENT);
    }

#pragma unroll
    for (int mt = 0; mt < 4; ++mt)
#pragma unroll
        for (int cn = 0; cn < 2; ++cn)
            *(f32x4*)(lds + (mt * 16 + (lane & 15)) * EPI_PITCH + (wave * 32 + cn * 16 + (lane >> 4) * 4) * 4) =
                acc[mt][cn];
    __syncthreads();
    f32x4 yv[2][4];
#pragma unroll
    for (int i = 0; i < 2; ++i)
#pragma unroll
        for (int j = 0; j < 4; ++j)
            yv[i][j] = *(const f32x4*)(lds + (wave * 16 + i * 8 + (lane >> 3)) * EPI_PITCH + (lane & 7) * 64 + j * 16);

    unsigned* tmo = (unsigned*)(my_ws + WS_TMO);
    unsigned gmax = 0;
    {
        const char* gbase = my_ws + WS_GRAN + (size_t)par * NR * NWG * 8 + (size_t)tid * 16;
        const unsigned long long t0 = __builtin_amdgcn_s_memrealtime();
        for (;;) {
            u32x4 v0 = load16_sys(gbase);
            u32x4 v1 = load16_sys(gbase + 4096);
            u32x4 v2 = load16_sys(gbase + 8192);
            u32x4 v3 = load16_sys(gbase + 12288);
            asm volatile("s_waitcnt vmcnt(0)" : "+v"(v0), "+v"(v1), "+v"(v2), "+v"(v3) :: "memory");
            const bool ok = v0[1] == call && v0[3] == call && v1[1] == call && v1[3] == call &&
                            v2[1] == call && v2[3] == call && v3[1] == call && v3[3] == call;
            gmax = max(max(max(v0[0], v0[2]), max(v1[0], v1[2])), max(max(v2[0], v2[2]), max(v3[0], v3[2])));
            if (__all(ok)) break;
            if (__hip_atomic_load(tmo, __ATOMIC_RELAXED, __HIP_MEMORY_SCOPE_AGENT) == call ||
                __builtin_amdgcn_s_memrealtime() - t0 > DEADLINE_TICKS) {
                __hip_atomic_store(tmo, call, __ATOMIC_RELAXED, __HIP_MEMORY_SCOPE_AGENT);
                gmax = 0xffffffffu;
                break;
            }
            __builtin_amdgcn_s_sleep(1);
        }
    }
    gmax = wave_max_u32(gmax);
    if (lane == 0) red[4 + wave] = gmax;
    __syncthreads();
    gmax = max(max(red[4], red[5]), max(red[6], red[7]));
    STAMP(2);
    const bool bad = gmax == 0xffffffffu;
    const float amax = __uint_as_float(gmax);
    const float scale = amax / 448.0f;
    const float inv = scale > 0.f ? 1.0f / scale : 0.f;

    char* obase = (char*)p.out[dest] + (size_t)(rank * M_LOC + row0) * N_LOC + tn * BN;
#pragma unroll
    for (int i = 0; i < 2; ++i) {
        u32x4 q;
#pragma unroll
        for (int j = 0; j < 4; ++j) {
            int w = __builtin_amdgcn_cvt_pk_fp8_f32(yv[i][j][0] * inv, yv[i][j][1] * inv, 0, false);
            w = __builtin_amdgcn_cvt_pk_fp8_f32(yv[i][j][2] * inv, yv[i][j][3] * inv, w, true);
            q[j] = bad ? 0x7f7f7f7fu : (unsigned)w;
        }
        store16_sys(obase + (size_t)(wave * 16 + i * 8 + (lane >> 3)) * N_LOC + (lane & 7) * 16, q);
    }
    asm volatile("s_waitcnt vmcnt(0)" ::: "memory");
    __syncthreads();
    STAMP(3);
    if (tid == 0) {
        if (dest != rank) {
            unsigned* flag = (unsigned*)((char*)p.ws[dest] + WS_DONE + (size_t)(rank * 32 + idx) * 128);
            __hip_atomic_store(flag, call, __ATOMIC_RELAXED, __HIP_MEMORY_SCOPE_SYSTEM);
        }
        if ((wg >> 5) != rank) {
            unsigned* flag = (unsigned*)(my_ws + WS_DONE + (size_t)wg * 128);
            const unsigned long long t0 = __builtin_amdgcn_s_memrealtime();
            while (__hip_atomic_load(flag, __ATOMIC_RELAXED, __HIP_MEMORY_SCOPE_SYSTEM) < call) {
                if (__hip_atomic_load(tmo, __ATOMIC_RELAXED, __HIP_MEMORY_SCOPE_AGENT) == call ||
                    __builtin_amdgcn_s_memrealtime() - t0 > DEADLINE_TICKS) {
                    __hip_atomic_store(tmo, call, __ATOMIC_RELAXED, __HIP_MEMORY_SCOPE_AGENT);
                    __hip_atomic_store((unsigned char*)p.out[rank] + (size_t)rank * M_LOC * N_LOC,
                                       (unsigned char)0x7f, __ATOMIC_RELAXED, __HIP_MEMORY_SCOPE_SYSTEM);
                    break;
                }
                __builtin_amdgcn_s_sleep(1);
            }
        }
    }
    STAMP(4);
}

}

extern "C" void kernel_launch(int rank, int nranks, ncclComm_t comm,
                              hipStream_t stream,
                              void* const* d_in, const int* in_sizes, int n_in,
                              void* d_out, int out_size,
                              const int* in_dtypes, int out_dtype,
                              void* const* d_out_all,
                              void* const* d_ws_all, size_t ws_bytes) {
    (void)comm; (void)in_dtypes; (void)out_dtype; (void)d_out;
    if (nranks != NR || n_in < 2 || (unsigned)rank >= (unsigned)NR || ws_bytes < WS_END ||
        in_sizes[0] != M_LOC * K_TOT || in_sizes[1] != K_TOT * N_TOT || out_size != N_TOT * N_LOC) {
        fprintf(stderr, "kernel_launch: unexpected problem shape\n");
        return;
    }
    Params p;
    p.x = d_in[0];
    p.w = d_in[1];
    for (int i = 0; i < NR; ++i) {
        p.out[i] = d_out_all[i];
        p.ws[i] = d_ws_all[i];
    }
    p.rank = rank;
    hipLaunchKernelGGL(gemm_a2a_kernel, dim3(NWG), dim3(NTHREADS), 0, stream, p);
    HIP_CHECK(hipGetLastError());
}
```

```cpp
#include <hip/hip_runtime.h>
#include <rccl/rccl.h>
#include <cstdio>
#include <cstdint>

#define HIP_CHECK(expr) do { hipError_t _e = (expr); if (_e != hipSuccess) { \
    fprintf(stderr, "HIP error %s at %s:%d\n", hipGetErrorString(_e), __FILE__, __LINE__); \
    return; } } while (0)

#define SG_R 8
#define SG_CM 1
#define SG_CN 2
#define SG_GOFF 4
#define SG_ROTX 0

namespace {

typedef __attribute__((ext_vector_type(4))) short s16x4;
typedef __attribute__((ext_vector_type(8))) short s16x8;
typedef __attribute__((ext_vector_type(4))) float f32x4;
typedef __attribute__((ext_vector_type(4))) unsigned u32x4;
typedef __attribute__((address_space(3))) s16x4* lds_s16x4_ptr;

constexpr int NR = 8;
constexpr int M_LOC = 512;
constexpr int K_TOT = 4096;
constexpr int N_TOT = 4096;
constexpr int N_LOC = N_TOT / NR;
constexpr int BM = 64, BN = 128, BK = 64;
constexpr int NS = 6;
constexpr int A_BYTES = BM * BK * 2;
constexpr int B_BYTES = BK * BN * 2;
constexpr int STAGE_BYTES = A_BYTES + B_BYTES;
constexpr int NKT = K_TOT / BK;
constexpr int NWG = (M_LOC / BM) * (N_TOT / BN);
constexpr int NTHREADS = 512;
constexpr int EPI_PITCH = BN * 4 + 16;
constexpr int EPI_OFF = 4 * STAGE_BYTES;
constexpr int LDS_RED = NS * STAGE_BYTES;
constexpr int LDS_PF = LDS_RED + 256;
constexpr int LDS_BYTES = LDS_PF + 256;
static_assert(NWG == 256, "one workgroup per CU");
static_assert(BM * EPI_PITCH <= 2 * STAGE_BYTES, "epilogue staging must fit");
static_assert((NKT - 1) % NS == 3 && (NKT - 2) % NS == 2, "epilogue staging slots");

constexpr size_t WS_GRAN = 0;
constexpr size_t WS_DONE = WS_GRAN + 2 * NR * NWG * 8;
constexpr size_t WS_CTR = WS_DONE + NWG * 128;
constexpr size_t WS_TMO = WS_CTR + NWG * 128;
constexpr size_t WS_DIAG = WS_TMO + 128;
constexpr size_t WS_END = WS_DIAG + NWG * 16 * 8;

constexpr unsigned long long DEADLINE_TICKS = 50000000ull;

struct Params {
    const void* x;
    const void* w;
    void* my_ws;
    int rank;
    int pad;
    void* out[NR];
    void* ws[NR];
};

__device__ __forceinline__ void glds16(unsigned voff, const void* sbase, unsigned lds_dst) {
    asm volatile("s_mov_b32 m0, %2\n\ts_nop 0\n\tglobal_load_lds_dwordx4 %0, %1"
                 :: "v"(voff), "s"(sbase), "s"(lds_dst) : "memory");
}
#define WAIT_VM(n) asm volatile("s_waitcnt vmcnt(%0)" :: "i"(n) : "memory")
#define BARRIER() asm volatile("s_barrier" ::: "memory")
#define SB() __builtin_amdgcn_sched_barrier(0)
#define MFMA(acc, a, b) asm volatile("v_mfma_f32_16x16x32_bf16 %0, %1, %2, %0" : "+v"(acc) : "v"(a), "v"(b))

__device__ __forceinline__ unsigned wave_max_u32(unsigned v) {
    v = max(v, (unsigned)__builtin_amdgcn_update_dpp(0, (int)v, 0x111, 0xf, 0xf, false));
    v = max(v, (unsigned)__builtin_amdgcn_update_dpp(0, (int)v, 0x112, 0xf, 0xf, false));
    v = max(v, (unsigned)__builtin_amdgcn_update_dpp(0, (int)v, 0x114, 0xf, 0xf, false));
    v = max(v, (unsigned)__builtin_amdgcn_update_dpp(0, (int)v, 0x118, 0xf, 0xf, false));
    v = max(v, (unsigned)__builtin_amdgcn_update_dpp(0, (int)v, 0x142, 0xa, 0xf, false));
    v = max(v, (unsigned)__builtin_amdgcn_update_dpp(0, (int)v, 0x143, 0xc, 0xf, false));
    return (unsigned)__builtin_amdgcn_readlane((int)v, 63);
}

__device__ __forceinline__ u32x4 load16_sys(const void* p) {
    u32x4 r;
    asm volatile("global_load_dwordx4 %0, %1, off sc0 sc1" : "=v"(r) : "v"(p) : "memory");
    return r;
}
__device__ __forceinline__ void store16_sys(void* p, u32x4 v) {
    asm volatile("global_store_dwordx4 %0, %1, off sc0 sc1\n\ts_nop 1" :: "v"(p), "v"(v) : "memory");
}

#define STAMP(i) do {} while (0)

struct Frags { s16x8 a[2][4]; s16x8 b[2][2]; };

__global__ void __launch_bounds__(NTHREADS, 2) gemm_a2a_kernel(const Params p) {
    __shared__ __attribute__((aligned(1024))) char lds[LDS_BYTES];
    const int tid = threadIdx.x;
    const int lane = tid & 63;
    const int wave = __builtin_amdgcn_readfirstlane(tid >> 6);
    const int wg = blockIdx.x;
    const int rank = p.rank;
    char* const my_ws = (char*)p.my_ws;
    const char* const gx = (const char*)p.x;
    const char* const gw = (const char*)p.w;
    asm volatile("" :: "s"(gx), "s"(gw), "s"(my_ws), "s"(rank));
    STAMP(0);

    const int dest = wg & 7;
    const int idx = wg >> 3;
    const int tm = idx & 7, tn = idx >> 3;
    const int row0 = tm * BM;
    const int col0 = dest * N_LOC + tn * BN;

    unsigned* ctr = (unsigned*)(my_ws + WS_CTR + (size_t)wg * 128);
    const unsigned lds_base = (unsigned)(uintptr_t)lds;
    unsigned* red = (unsigned*)(lds + LDS_RED);

    f32x4 acc[4][2];

    if (wave < 4) {
        unsigned rdA[2], rdB[2];
#pragma unroll
        for (int kk = 0; kk < 2; ++kk) {
            const int r = lane & 15;
            const int c = 4 * kk + (lane >> 4);
            rdA[kk] = (unsigned)(r * 128 + ((c ^ ((r >> 1) & 7)) * 16));
        }
#pragma unroll
        for (int cn = 0; cn < 2; ++cn) {
            const int g = lane >> 4, q = (lane & 15) >> 2, pp = lane & 3;
            const int k = 8 * g + q;
            const int h = (k & 3) | (((k >> 3) & 1) << 2);
            const int u = (wave * 2 + cn) ^ h;
            rdB[cn] = (unsigned)(A_BYTES + k * 256 + u * 32 + pp * 8);
        }
#pragma unroll
        for (int i = 0; i < 4; ++i)
#pragma unroll
            for (int j = 0; j < 2; ++j) acc[i][j] = (f32x4){0.f, 0.f, 0.f, 0.f};

        auto rd_b = [&](Frags& f, const char* base, int kk, int cn) {
            s16x4 lo = __builtin_amdgcn_ds_read_tr16_b64_v4i16((lds_s16x4_ptr)(base + rdB[cn] + kk * 8192));
            s16x4 hi = __builtin_amdgcn_ds_read_tr16_b64_v4i16((lds_s16x4_ptr)(base + rdB[cn] + kk * 8192 + 1024));
            f.b[kk][cn] = __builtin_shufflevector(lo, hi, 0, 1, 2, 3, 4, 5, 6, 7);
        };
        auto rd_a = [&](Frags& f, const char* base, int kk, int mt) {
            f.a[kk][mt] = *(const s16x8*)(base + rdA[kk] + mt * 2048);
        };
        auto read_all = [&](Frags& f, int slot) {
            const char* base = lds + slot * STAGE_BYTES;
#pragma unroll
            for (int kk = 0; kk < 2; ++kk) {
                rd_b(f, base, kk, 0); rd_a(f, base, kk, 0); rd_b(f, base, kk, 1);
                rd_a(f, base, kk, 1); rd_a(f, base, kk, 2); rd_a(f, base, kk, 3);
            }
        };
#define MM(f, i) MFMA(acc[((i) >> 1) & 3][(i) & 1], f.b[(i) >> 3][(i) & 1], f.a[(i) >> 3][((i) >> 1) & 3])
        auto step = [&](Frags& cur, Frags& nxt, int slot) {
            const char* base = lds + slot * STAGE_BYTES;
            BARRIER();
            SB(); MM(cur, 0);  SB(); rd_b(nxt, base, 0, 0);
            SB(); MM(cur, 1);  SB(); rd_a(nxt, base, 0, 0); rd_a(nxt, base, 0, 1);
            SB(); MM(cur, 2);  SB(); rd_b(nxt, base, 0, 1);
            SB(); MM(cur, 3);  SB(); rd_a(nxt, base, 0, 2); rd_a(nxt, base, 0, 3);
            SB(); MM(cur, 4);  SB(); rd_b(nxt, base, 1, 0);
            SB(); MM(cur, 5);  SB(); rd_a(nxt, base, 1, 0); rd_a(nxt, base, 1, 1);
            SB(); MM(cur, 6);  SB(); rd_b(nxt, base, 1, 1);
            SB(); MM(cur, 7);  SB(); rd_a(nxt, base, 1, 2); rd_a(nxt, base, 1, 3);
            SB(); MM(cur, 8);  MM(cur, 9);  MM(cur, 10); MM(cur, 11);
            MM(cur, 12); MM(cur, 13); MM(cur, 14); MM(cur, 15);
            SB();
        };
        Frags f0, f1;
        BARRIER();
        read_all(f0, 0);
        int slot = 1;
        for (int kt = 0; kt < NKT - 2; kt += 2) {
            step(f0, f1, slot);
            slot = (slot == NS - 1) ? 0 : slot + 1;
            step(f1, f0, slot);
            slot = (slot == NS - 1) ? 0 : slot + 1;
        }
        step(f0, f1, slot);
        MM(f1, 0); MM(f1, 1); MM(f1, 2); MM(f1, 3); MM(f1, 4); MM(f1, 5); MM(f1, 6); MM(f1, 7);
        MM(f1, 8); MM(f1, 9); MM(f1, 10); MM(f1, 11); MM(f1, 12); MM(f1, 13); MM(f1, 14); MM(f1, 15);
        asm volatile("s_nop 15\n\ts_nop 7" : "+v"(acc[0][0]), "+v"(acc[0][1]), "+v"(acc[1][0]), "+v"(acc[1][1]),
                     "+v"(acc[2][0]), "+v"(acc[2][1]), "+v"(acc[3][0]), "+v"(acc[3][1]));
        float am = 0.f;
#pragma unroll
        for (int i = 0; i < 4; ++i)
#pragma unroll
            for (int j = 0; j < 2; ++j)
#pragma unroll
                for (int e = 0; e < 4; ++e) am = fmaxf(am, fabsf(acc[i][j][e]));
        const unsigned amb = wave_max_u32(__float_as_uint(am));
        if (lane == 0) red[wave] = amb;
    } else if (wave < 7) {
        unsigned voff[8];
        const char* gsrc;
        size_t gstep;
        unsigned ldst;
        if (wave == 4) {
#pragma unroll
            for (int i = 0; i < 8; ++i) {
                const int r = i * 8 + (lane >> 3);
                const int c = (lane & 7) ^ ((r >> 1) & 7);
                voff[i] = (unsigned)(r * (K_TOT * 2) + c * 16);
            }
            gsrc = gx + (size_t)row0 * (K_TOT * 2);
            gstep = BK * 2;
            ldst = lds_base;
        } else {
#pragma unroll
            for (int j = 0; j < 8; ++j) {
                const int k = (wave - 5) * 32 + j * 4 + (lane >> 4);
                const int h = (k & 3) | (((k >> 3) & 1) << 2);
                const int c = (lane & 15) ^ (2 * h);
                voff[j] = (unsigned)(k * (N_TOT * 2) + c * 16);
            }
            gsrc = gw + (size_t)col0 * 2;
            gstep = (size_t)BK * N_TOT * 2;
            ldst = lds_base + A_BYTES + (wave - 5) * 8192;
        }
        int sp = (tn >= 2 ? SG_GOFF : 0) + SG_ROTX * dest;
        const int rot = SG_CM * tm + SG_CN * tn;
        auto issue = [&](int slot) {
            const int spm = sp & (NKT - 1);
            const int kt = (spm & ~(SG_R - 1)) | ((spm + rot) & (SG_R - 1));
            const char* g = gsrc + (size_t)kt * gstep;
            const unsigned d = ldst + slot * STAGE_BYTES;
#pragma unroll
            for (int i = 0; i < 8; ++i) glds16(voff[i], g, d + i * 1024);
            ++sp;
        };
#pragma unroll
        for (int s = 0; s < NS - 1; ++s) issue(s);
        WAIT_VM((NS - 2) * 8);
        BARRIER();
        int slot = NS - 1;
        for (int kt = 0; kt < NKT - NS + 1; ++kt) {
            WAIT_VM((NS - 3) * 8);
            BARRIER();
            issue(slot);
            slot = (slot == NS - 1) ? 0 : slot + 1;
        }
        WAIT_VM(3 * 8); BARRIER();
        WAIT_VM(2 * 8); BARRIER();
        WAIT_VM(1 * 8); BARRIER();
        WAIT_VM(0);     BARRIER();
    } else {
        unsigned c;
        asm volatile("global_load_dword %0, %1, off sc1\n\ts_waitcnt vmcnt(0)" : "=&v"(c) : "v"(ctr) : "memory");
        if (lane == 0) red[16] = c + 1u;
        for (int kt = 0; kt < NKT; ++kt) BARRIER();
    }
    STAMP(1);
    asm volatile("s_waitcnt lgkmcnt(0)" ::: "memory");
    BARRIER();
    const unsigned call = __builtin_amdgcn_readfirstlane(red[16]);
    const unsigned par = call & 1u;
    if (wave == 4) {
        const unsigned m = max(max(red[0], red[1]), max(red[2], red[3]));
        if (lane < NR) {
            unsigned long long* g = (unsigned long long*)((char*)p.ws[lane] + WS_GRAN) +
                                    ((size_t)par * NR + rank) * NWG + wg;
            __hip_atomic_store(g, ((unsigned long long)call << 32) | m, __ATOMIC_RELAXED,
                               __HIP_MEMORY_SCOPE_SYSTEM);
        }
    }
    if (wave < 4) {
#pragma unroll
        for (int mt = 0; mt < 4; ++mt)
#pragma unroll
            for (int cn = 0; cn < 2; ++cn)
                *(f32x4*)(lds + EPI_OFF + (mt * 16 + (lane & 15)) * EPI_PITCH +
                          (wave * 32 + cn * 16 + (lane >> 4) * 4) * 4) = acc[mt][cn];
    }
    asm volatile("s_waitcnt lgkmcnt(0)" ::: "memory");
    BARRIER();
    f32x4 yv[4];
#pragma unroll
    for (int j = 0; j < 4; ++j)
        yv[j] = *(const f32x4*)(lds + EPI_OFF + (wave * 8 + (lane >> 3)) * EPI_PITCH + (lane & 7) * 64 + j * 16);

    unsigned* tmo = (unsigned*)(my_ws + WS_TMO);
    unsigned gmax = 0;
    {
        const char* gbase = my_ws + WS_GRAN + (size_t)par * NR * NWG * 8 + (size_t)wave * 2048 + lane * 16;
        const unsigned long long t0 = __builtin_amdgcn_s_memrealtime();
        for (;;) {
            u32x4 v0 = load16_sys(gbase);
            u32x4 v1 = load16_sys(gbase + 1024);
            asm volatile("s_waitcnt vmcnt(0)" : "+v"(v0), "+v"(v1) :: "memory");
            const bool ok = v0[1] == call && v0[3] == call && v1[1] == call && v1[3] == call;
            gmax = max(max(v0[0], v0[2]), max(v1[0], v1[2]));
            if (__all(ok)) break;
            if (__hip_atomic_load(tmo, __ATOMIC_RELAXED, __HIP_MEMORY_SCOPE_AGENT) == call ||
                __builtin_amdgcn_s_memrealtime() - t0 > DEADLINE_TICKS) {
                __hip_atomic_store(tmo, call, __ATOMIC_RELAXED, __HIP_MEMORY_SCOPE_AGENT);
                gmax = 0xffffffffu;
                break;
            }
            __builtin_amdgcn_s_sleep(1);
        }
    }
    gmax = wave_max_u32(gmax);
    if (lane == 0) red[8 + wave] = gmax;
    asm volatile("s_waitcnt lgkmcnt(0)" ::: "memory");
    BARRIER();
    {
        const u32x4 r0 = *(const u32x4*)(red + 8), r1 = *(const u32x4*)(red + 12);
        gmax = max(max(max(r0[0], r0[1]), max(r0[2], r0[3])), max(max(r1[0], r1[1]), max(r1[2], r1[3])));
    }
    STAMP(2);
    const bool bad = gmax == 0xffffffffu;
    const float amax = __uint_as_float(gmax);
    const float scale = amax / 448.0f;
    const float inv = scale > 0.f ? 1.0f / scale : 0.f;

    char* obase = (char*)p.out[dest] + (size_t)(rank * M_LOC + row0) * N_LOC + tn * BN;
    {
        u32x4 q;
#pragma unroll
        for (int j = 0; j < 4; ++j) {
            int w = __builtin_amdgcn_cvt_pk_fp8_f32(yv[j][0] * inv, yv[j][1] * inv, 0, false);
            w = __builtin_amdgcn_cvt_pk_fp8_f32(yv[j][2] * inv, yv[j][3] * inv, w, true);
            q[j] = bad ? 0x7f7f7f7fu : (unsigned)w;
        }
        store16_sys(obase + (size_t)(wave * 8 + (lane >> 3)) * N_LOC + (lane & 7) * 16, q);
    }
    asm volatile("s_waitcnt vmcnt(0)" ::: "memory");
    BARRIER();
    STAMP(3);
    if (tid == 0) {
        if (dest != rank) {
            unsigned* flag = (unsigned*)((char*)p.ws[dest] + WS_DONE + (size_t)(rank * 32 + idx) * 128);
            __hip_atomic_store(flag, call, __ATOMIC_RELAXED, __HIP_MEMORY_SCOPE_SYSTEM);
        }
        __hip_atomic_store(ctr, call, __ATOMIC_RELAXED, __HIP_MEMORY_SCOPE_AGENT);
        if ((wg >> 5) != rank) {
            unsigned* flag = (unsigned*)(my_ws + WS_DONE + (size_t)wg * 128);
            const unsigned long long t0 = __builtin_amdgcn_s_memrealtime();
            while (__hip_atomic_load(flag, __ATOMIC_RELAXED, __HIP_MEMORY_SCOPE_SYSTEM) < call) {
                if (__hip_atomic_load(tmo, __ATOMIC_RELAXED, __HIP_MEMORY_SCOPE_AGENT) == call ||
                    __builtin_amdgcn_s_memrealtime() - t0 > DEADLINE_TICKS) {
                    __hip_atomic_store(tmo, call, __ATOMIC_RELAXED, __HIP_MEMORY_SCOPE_AGENT);
                    __hip_atomic_store((unsigned char*)p.out[rank] + (size_t)rank * M_LOC * N_LOC,
                                       (unsigned char)0x7f, __ATOMIC_RELAXED, __HIP_MEMORY_SCOPE_SYSTEM);
                    break;
                }
                __builtin_amdgcn_s_sleep(1);
            }
        }
    }
    STAMP(4);
}

}

extern "C" void kernel_launch(int rank, int nranks, ncclComm_t comm,
                              hipStream_t stream,
                              void* const* d_in, const int* in_sizes, int n_in,
                              void* d_out, int out_size,
                              const int* in_dtypes, int out_dtype,
                              void* const* d_out_all,
                              void* const* d_ws_all, size_t ws_bytes) {
    (void)comm; (void)in_dtypes; (void)out_dtype; (void)d_out;
    if (nranks != NR || n_in < 2 || (unsigned)rank >= (unsigned)NR || ws_bytes < WS_END ||
        in_sizes[0] != M_LOC * K_TOT || in_sizes[1] != K_TOT * N_TOT || out_size != N_TOT * N_LOC) {
        fprintf(stderr, "kernel_launch: unexpected problem shape\n");
        return;
    }
    Params p;
    p.x = d_in[0];
    p.w = d_in[1];
    for (int i = 0; i < NR; ++i) {
        p.out[i] = d_out_all[i];
        p.ws[i] = d_ws_all[i];
    }
    p.rank = rank;
    p.pad = 0;
    p.my_ws = d_ws_all[rank];
    hipLaunchKernelGGL(gemm_a2a_kernel, dim3(NWG), dim3(NTHREADS), 0, stream, p);
    HIP_CHECK(hipGetLastError());
}
```

```cpp
#include <hip/hip_runtime.h>
#include <rccl/rccl.h>
#include <cstdio>
#include <cstdint>

#define HIP_CHECK(expr) do { hipError_t _e = (expr); if (_e != hipSuccess) { \
    fprintf(stderr, "HIP error %s at %s:%d\n", hipGetErrorString(_e), __FILE__, __LINE__); \
    return; } } while (0)

#define MAP 0
#define SG_R 4
#define SG_CM 1
#define SG_CN 1
#define SG_GOFF 2
#define PWAVE 5
#define G0 0
#define NS 4
#define LAG (NS - 1)
#define HB 6
#define XST_POL ""
#define XLD_POL "sc1"

namespace {

typedef __attribute__((ext_vector_type(4))) short s16x4;
typedef __attribute__((ext_vector_type(8))) short s16x8;
typedef __attribute__((ext_vector_type(4))) float f32x4;
typedef __attribute__((ext_vector_type(4))) unsigned u32x4;
typedef __attribute__((address_space(3))) s16x4* lds_s16x4_ptr;

constexpr int NR = 8;
constexpr int M_LOC = 512;
constexpr int K_TOT = 4096;
constexpr int N_TOT = 4096;
constexpr int N_LOC = N_TOT / NR;
constexpr int BM = 128, BN = 128, BK = 64;
constexpr int KS = 2;
constexpr int A_BYTES = BM * BK * 2;
constexpr int B_BYTES = BK * BN * 2;
constexpr int STAGE_BYTES = A_BYTES + B_BYTES;
constexpr int NKT = K_TOT / KS / BK;
constexpr int NWG = (M_LOC / BM) * (N_TOT / BN) * KS;
constexpr int NTHREADS = 512;
constexpr int EPI_PITCH = BN * 4 + 16;
constexpr int LAST_SLOT = (NKT - 1) % NS;
constexpr int EPI_OFF = (LAST_SLOT >= 2 ? 0 : 2) * STAGE_BYTES;
constexpr int LDS_RED = EPI_OFF + 64 * EPI_PITCH;
constexpr int LDS_XCC = NS * STAGE_BYTES;
constexpr int LDS_CALL = LDS_XCC + 256;
constexpr int LDS_BYTES = LDS_CALL + 64;
static_assert(NWG == 256, "one workgroup per CU");
static_assert(64 * EPI_PITCH + 256 <= 2 * STAGE_BYTES, "epilogue staging must fit");
static_assert(LAG >= 2 && LAG <= NS - 1 && NKT % NS == 0, "ring depth");

constexpr size_t WS_XG = 0;
constexpr size_t WS_LGF = WS_XG + 2 * 4096;
constexpr size_t WS_LGS = WS_LGF + 8192;
constexpr size_t WS_DONE = 32768;
constexpr size_t WS_CTR = WS_DONE + NWG * 128;
constexpr size_t WS_TMO = WS_CTR + NWG * 128;
constexpr size_t WS_DIAG = WS_TMO + 128;
constexpr size_t WS_XB = 256 * 1024;
constexpr size_t WS_END = WS_XB + (size_t)NWG * 32768;
static_assert(WS_DIAG + NWG * 16 * 8 <= WS_XB, "");

constexpr unsigned long long DEADLINE_TICKS = 50000000ull;

struct Params {
    const void* x;
    const void* w;
    void* my_ws;
    int rank;
    int pad;
    void* out[NR];
    void* ws[NR];
};

__device__ __forceinline__ void glds16(unsigned voff, const void* sbase, unsigned lds_dst) {
    asm volatile("s_mov_b32 m0, %2\n\ts_nop 0\n\tglobal_load_lds_dwordx4 %0, %1"
                 :: "v"(voff), "s"(sbase), "s"(lds_dst) : "memory");
}
#define WAIT_VM(n) asm volatile("s_waitcnt vmcnt(%0)" :: "i"(n) : "memory")
#define WAIT_LGKM0() asm volatile("s_waitcnt lgkmcnt(0)" ::: "memory")
#define BARRIER() asm volatile("s_barrier" ::: "memory")
#define SB() __builtin_amdgcn_sched_barrier(0)
#define MFMA(acc, a, b) asm volatile("v_mfma_f32_16x16x32_bf16 %0, %1, %2, %0" : "+v"(acc) : "v"(a), "v"(b))

__device__ __forceinline__ unsigned wave_max_u32(unsigned v) {
    v = max(v, (unsigned)__builtin_amdgcn_update_dpp(0, (int)v, 0x111, 0xf, 0xf, false));
    v = max(v, (unsigned)__builtin_amdgcn_update_dpp(0, (int)v, 0x112, 0xf, 0xf, false));
    v = max(v, (unsigned)__builtin_amdgcn_update_dpp(0, (int)v, 0x114, 0xf, 0xf, false));
    v = max(v, (unsigned)__builtin_amdgcn_update_dpp(0, (int)v, 0x118, 0xf, 0xf, false));
    v = max(v, (unsigned)__builtin_amdgcn_update_dpp(0, (int)v, 0x142, 0xa, 0xf, false));
    v = max(v, (unsigned)__builtin_amdgcn_update_dpp(0, (int)v, 0x143, 0xc, 0xf, false));
    return (unsigned)__builtin_amdgcn_readlane((int)v, 63);
}

__device__ __forceinline__ void store16_sys(void* p, u32x4 v) {
    asm volatile("global_store_dwordx4 %0, %1, off sc0 sc1\n\ts_nop 1" :: "v"(p), "v"(v) : "memory");
}

#define STAMP(i) do {} while (0)

struct Frags { s16x8 a[4]; s16x4 bl[4], bh[4]; };

__global__ void __launch_bounds__(NTHREADS, 2) gemm_a2a_kernel(const Params p) {
    __shared__ __attribute__((aligned(1024))) char lds[LDS_BYTES];
    const int tid = threadIdx.x;
    const int lane = tid & 63;
    const int wave = __builtin_amdgcn_readfirstlane(tid >> 6);
    const int wg = blockIdx.x;
    const int rank = p.rank;
    char* const my_ws = (char*)p.my_ws;
    const char* const gx = (const char*)p.x;
    const char* const gw = (const char*)p.w;
    asm volatile("" :: "s"(gx), "s"(gw), "s"(my_ws), "s"(rank));
    STAMP(0);

    const int xcd = wg & 7, idx = wg >> 3;
    const int tm = idx & 3;
    const int ks = idx >> 4, tnl = (idx >> 2) & 3;
    const int tn = xcd * 4 + tnl;
    const int partner = wg ^ 128;
    const int ghalf = tnl >= 2;
    const int dest = tn >> 2;
    const int didx = (ks * 4 + tm) * 4 + (tn & 3);
    const int row0 = tm * BM;
    const int col0 = tn * BN;

    unsigned* ctr = (unsigned*)(my_ws + WS_CTR + (size_t)wg * 128);
    unsigned* tmo = (unsigned*)(my_ws + WS_TMO);
    const unsigned lds_base = (unsigned)(uintptr_t)lds;
    unsigned* red = (unsigned*)(lds + LDS_RED);

    if (wave < 4) {
        const int wm = wave >> 1, wn = wave & 1;
        unsigned call;
        asm volatile("global_load_dword %0, %1, off sc1\n\ts_waitcnt vmcnt(0)" : "=&v"(call) : "v"(ctr) : "memory");
        call = __builtin_amdgcn_readfirstlane(call) + 1u;
        const unsigned xcc = __builtin_amdgcn_s_getreg((20 << 0) | (0 << 6) | (3 << 11));
        if (tid == 0)
            asm volatile("global_store_dwordx2 %0, %1, off offset:64 sc1"
                         :: "v"(ctr), "v"(((unsigned long long)call << 32) | xcc) : "memory");
        if (tid == 0) *(unsigned*)(lds + LDS_CALL) = call;
        WAIT_LGKM0();
        unsigned rdA[2][2], rdB[4];
#pragma unroll
        for (int kk = 0; kk < 2; ++kk) {
            const int r = lane & 15;
            const int c = 4 * kk + (lane >> 4);
            const unsigned a = (unsigned)(wm * 8192 + r * 128 + ((c ^ ((r >> 1) & 7)) * 16));
            rdA[kk][0] = a + (ks ^ 1) * 4096;
            rdA[kk][1] = a + ks * 4096;
        }
#pragma unroll
        for (int cn = 0; cn < 4; ++cn) {
            const int g = lane >> 4, q = (lane & 15) >> 2, pp = lane & 3;
            const int k = 8 * g + q;
            const int h = (k & 3) | (((k >> 3) & 1) << 2);
            const int u = (wn * 4 + cn) ^ h;
            rdB[cn] = (unsigned)(A_BYTES + k * 256 + u * 32 + pp * 8);
        }
        f32x4 acc[4][4];
#pragma unroll
        for (int i = 0; i < 4; ++i)
#pragma unroll
            for (int j = 0; j < 4; ++j) acc[i][j] = (f32x4){0.f, 0.f, 0.f, 0.f};
        asm volatile("s_nop 1" : "+v"(acc[0][0]), "+v"(acc[0][1]), "+v"(acc[0][2]), "+v"(acc[0][3]),
                     "+v"(acc[1][0]), "+v"(acc[1][1]), "+v"(acc[1][2]), "+v"(acc[1][3]),
                     "+v"(acc[2][0]), "+v"(acc[2][1]), "+v"(acc[2][2]), "+v"(acc[2][3]),
                     "+v"(acc[3][0]), "+v"(acc[3][1]), "+v"(acc[3][2]), "+v"(acc[3][3]));

        auto rd1 = [&](Frags& f, const char* base, int kk, int g) {
            constexpr int kind[12] = {0, 1, 8, 2, 3, 4, 5, 6, 7, 9, 10, 11};
            const int k = kind[g];
            if (k < 8) {
                const s16x4 v = __builtin_amdgcn_ds_read_tr16_b64_v4i16(
                    (lds_s16x4_ptr)(base + rdB[k >> 1] + kk * 8192 + (k & 1) * 1024));
                if (k & 1) f.bh[k >> 1] = v; else f.bl[k >> 1] = v;
            } else {
                f.a[k - 8] = *(const s16x8*)(base + rdA[kk][(k - 8) >> 1] + ((k - 8) & 1) * 2048);
            }
        };
#define MM(f, i) MFMA(acc[(i) >> 2][(i) & 3], __builtin_shufflevector(f.bl[(i) & 3], f.bh[(i) & 3], 0, 1, 2, 3, 4, 5, 6, 7), f.a[(i) >> 2])
        Frags f0, f1;
#pragma unroll
        for (int i = 0; i < 4; ++i) {
            f1.a[i] = (s16x8){0, 0, 0, 0, 0, 0, 0, 0};
            f1.bl[i] = f1.bh[i] = (s16x4){0, 0, 0, 0};
        }
        asm volatile("s_nop 1" : "+v"(f1.a[0]), "+v"(f1.a[1]), "+v"(f1.a[2]), "+v"(f1.a[3]),
                     "+v"(f1.bl[0]), "+v"(f1.bl[1]), "+v"(f1.bl[2]), "+v"(f1.bl[3]),
                     "+v"(f1.bh[0]), "+v"(f1.bh[1]), "+v"(f1.bh[2]), "+v"(f1.bh[3]));
        auto step = [&](const int slot) {
            const char* base = lds + slot * STAGE_BYTES;
            BARRIER();
            SB();
#pragma unroll
            for (int i = 0; i < 16; ++i) {
                MM(f1, i); SB();
                if (i >= G0 && i < G0 + 12) { rd1(f0, base, 0, i - G0); SB(); }
            }
#pragma unroll
            for (int i = 0; i < 16; ++i) {
                MM(f0, i); SB();
                if (i < 12) { rd1(f1, base, 1, i); SB(); }
            }
            if (LAG == NS - 1) WAIT_LGKM0();
        };
        for (int kt = 0; kt < NKT; kt += NS) {
#pragma unroll
            for (int s = 0; s < NS; ++s) step(s);
        }
        const unsigned long long pg = *(const unsigned long long*)(lds + LDS_XCC);
#pragma unroll
        for (int i = 0; i < 16; ++i) MM(f1, i);
        asm volatile("s_nop 15\n\ts_nop 7" : "+v"(acc[0][0]), "+v"(acc[0][1]), "+v"(acc[0][2]), "+v"(acc[0][3]),
                     "+v"(acc[1][0]), "+v"(acc[1][1]), "+v"(acc[1][2]), "+v"(acc[1][3]),
                     "+v"(acc[2][0]), "+v"(acc[2][1]), "+v"(acc[2][2]), "+v"(acc[2][3]),
                     "+v"(acc[3][0]), "+v"(acc[3][1]), "+v"(acc[3][2]), "+v"(acc[3][3]));
        const bool same_xcd = __builtin_amdgcn_readfirstlane(pg == (((unsigned long long)call << 32) | xcc));
        STAMP(1);
        const unsigned par = call & 1u;
        f32x4 keep[8];
        {
            char* xo = my_ws + WS_XB + (size_t)wg * 32768 + wave * 8192 + lane * 16;
#pragma unroll
            for (int i = 0; i < 8; ++i) {
                const f32x4 snd = acc[i >> 2][i & 3];
                keep[i] = acc[2 + (i >> 2)][i & 3];
                u32x4 t;
#pragma unroll
                for (int e = 0; e < 4; ++e) t[e] = (__float_as_uint(snd[e]) & ~1u) | par;
                if (same_xcd)
                    asm volatile("global_store_dwordx4 %0, %1, off offset:%2 " XST_POL "\n\ts_nop 1"
                                 :: "v"(xo + (i >> 2) * 4096), "v"(t), "i"((i & 3) * 1024) : "memory");
                else
                    asm volatile("global_store_dwordx4 %0, %1, off offset:%2 sc1\n\ts_nop 1"
                                 :: "v"(xo + (i >> 2) * 4096), "v"(t), "i"((i & 3) * 1024) : "memory");
            }
        }
        bool bad = false;
        {
            const char* xi = my_ws + WS_XB + (size_t)partner * 32768 + wave * 8192 + lane * 16;
            const char* xi2 = xi + 4096;
            const unsigned pm = 0u - par;
            unsigned it = 0;
            unsigned long long t0 = 0;
            u32x4 v[8];
            for (;;) {
                asm volatile("global_load_dwordx4 %0, %8, off " XLD_POL "\n\t"
                             "global_load_dwordx4 %1, %8, off offset:1024 " XLD_POL "\n\t"
                             "global_load_dwordx4 %2, %8, off offset:2048 " XLD_POL "\n\t"
                             "global_load_dwordx4 %3, %8, off offset:3072 " XLD_POL "\n\t"
                             "global_load_dwordx4 %4, %9, off " XLD_POL "\n\t"
                             "global_load_dwordx4 %5, %9, off offset:1024 " XLD_POL "\n\t"
                             "global_load_dwordx4 %6, %9, off offset:2048 " XLD_POL "\n\t"
                             "global_load_dwordx4 %7, %9, off offset:3072 " XLD_POL "\n\t"
                             "s_waitcnt vmcnt(0)"
                             : "=&v"(v[0]), "=&v"(v[1]), "=&v"(v[2]), "=&v"(v[3]),
                               "=&v"(v[4]), "=&v"(v[5]), "=&v"(v[6]), "=&v"(v[7])
                             : "v"(xi), "v"(xi2) : "memory");
                unsigned mism = 0;
#pragma unroll
                for (int i = 0; i < 8; ++i)
#pragma unroll
                    for (int e = 0; e < 4; ++e) mism |= v[i][e] ^ pm;
                if (__all((mism & 1u) == 0u)) break;
                if ((++it & 63u) == 0u) {
                    const unsigned long long t = __builtin_amdgcn_s_memrealtime();
                    if (t0 == 0) t0 = t;
                    if (t - t0 > DEADLINE_TICKS) { bad = true; break; }
                }
            }
#pragma unroll
            for (int i = 0; i < 8; ++i)
#pragma unroll
                for (int e = 0; e < 4; ++e) keep[i][e] += __uint_as_float(v[i][e] & ~1u);
        }
        float am = 0.f;
#pragma unroll
        for (int i = 0; i < 8; ++i)
#pragma unroll
            for (int e = 0; e < 4; ++e) am = fmaxf(am, fabsf(keep[i][e]));
        unsigned amb = wave_max_u32(__float_as_uint(am));
        if (bad) amb = 0xffffffffu;
        if (lane == 0) {
            const char* lg = my_ws + WS_LGF + xcd * 1024 + (idx * 4 + wave) * 8;
            const unsigned long long g = ((unsigned long long)call << 32) | amb;
            asm volatile("global_store_dwordx2 %0, %1, off\n\t"
                         "global_store_dwordx2 %2, %1, off sc1"
                         :: "v"(lg), "v"(g), "v"(lg + (WS_LGS - WS_LGF)) : "memory");
        }
        STAMP(2);
#pragma unroll
        for (int i = 0; i < 8; ++i)
            *(f32x4*)(lds + EPI_OFF + (wm * 32 + (i >> 2) * 16 + (lane & 15)) * EPI_PITCH +
                      (wn * 64 + (i & 3) * 16 + (lane >> 4) * 4) * 4) = keep[i];
    } else {
        unsigned voff[8];
        const char* gsrc;
        size_t gstep;
        unsigned ldst;
        if (wave < 6) {
#pragma unroll
            for (int i = 0; i < 8; ++i) {
                const int r = (wave - 4) * 64 + i * 8 + (lane >> 3);
                const int c = (lane & 7) ^ ((r >> 1) & 7);
                voff[i] = (unsigned)(r * (K_TOT * 2) + c * 16);
            }
            gsrc = gx + (size_t)row0 * (K_TOT * 2) + (size_t)ks * (K_TOT / KS * 2);
            gstep = BK * 2;
            ldst = lds_base + (wave - 4) * 8192;
        } else {
#pragma unroll
            for (int j = 0; j < 8; ++j) {
                const int k = (wave - 6) * 32 + j * 4 + (lane >> 4);
                const int h = (k & 3) | (((k >> 3) & 1) << 2);
                const int c = (lane & 15) ^ (2 * h);
                voff[j] = (unsigned)(k * (N_TOT * 2) + c * 16);
            }
            gsrc = gw + (size_t)col0 * 2 + (size_t)ks * ((size_t)(K_TOT / KS) * N_TOT * 2);
            gstep = (size_t)BK * N_TOT * 2;
            ldst = lds_base + A_BYTES + (wave - 6) * 8192;
        }
        int sp = ghalf ? SG_GOFF : 0;
        const int rot = SG_CM * tm + SG_CN * tnl;
        auto issue = [&](int slot) {
            const int spm = sp & (NKT - 1);
            const int kt = (spm & ~(SG_R - 1)) | ((spm + rot) & (SG_R - 1));
            const char* g = gsrc + (size_t)kt * gstep;
            const unsigned d = ldst + slot * STAGE_BYTES;
#pragma unroll
            for (int i = 0; i < 8; ++i) glds16(voff[i], g, d + i * 1024);
            ++sp;
        };
#pragma unroll
        for (int s = 0; s < LAG; ++s) issue(s);
        int slot = LAG % NS;
        for (int kt = 0; kt < NKT - LAG; ++kt) {
            WAIT_VM((LAG - 1) * 8);
            BARRIER();
            if (kt == NKT / 2 && wave == 4) {
                const char* pc = my_ws + WS_CTR + (size_t)partner * 128 + 64;
                asm volatile("s_mov_b32 m0, %2\n\ts_nop 0\n\tglobal_load_lds_dword %0, %1 sc1"
                             :: "v"((unsigned)(lane & 1) * 4u), "s"(pc), "s"(lds_base + LDS_XCC) : "memory");
            }
            issue(slot);
            slot = (slot == NS - 1) ? 0 : slot + 1;
        }
        if (LAG >= 4) { WAIT_VM(3 * 8); BARRIER(); }
        if (LAG >= 3) { WAIT_VM(2 * 8); BARRIER(); }
        WAIT_VM(1 * 8); BARRIER();
        WAIT_VM(0);     BARRIER();
        if (wave == 7 && idx == 0) {
            const unsigned call = __builtin_amdgcn_readfirstlane(*(const unsigned*)(lds + LDS_CALL));
            char* dst = (char*)p.ws[lane >> 3] + WS_XG + (call & 1u) * 4096 + (lane & 7) * 512 + rank * 64 + xcd * 8;
            const char* lg = my_ws + WS_LGF + xcd * 1024 + lane * 16;
            unsigned m0 = 0, m1 = 0;
            bool ok0 = false, ok1 = false;
            unsigned it = 0;
            unsigned long long t0 = 0;
            for (;;) {
                const char* src = lg + (((it & 3u) == 3u) ? (WS_LGS - WS_LGF) : 0);
                u32x4 v;
                asm volatile("global_load_dwordx4 %0, %1, off sc1\n\ts_waitcnt vmcnt(0)"
                             : "=&v"(v) : "v"(src) : "memory");
                if (v[1] == call) { m0 = v[0]; ok0 = true; }
                if (v[3] == call) { m1 = v[2]; ok1 = true; }
                if (__all(ok0 && ok1)) break;
                if ((++it & 63u) == 0u) {
                    const unsigned long long t = __builtin_amdgcn_s_memrealtime();
                    if (t0 == 0) t0 = t;
                    if (t - t0 > DEADLINE_TICKS) { m0 = 0xffffffffu; break; }
                }
            }
            const unsigned m = wave_max_u32(max(m0, m1));
            asm volatile("global_store_dwordx2 %0, %1, off sc0 sc1"
                         :: "v"(dst), "v"(((unsigned long long)call << 32) | m) : "memory");
        }
    }
    WAIT_LGKM0();
    BARRIER();
    const unsigned call = __builtin_amdgcn_readfirstlane(*(const unsigned*)(lds + LDS_CALL));
    const unsigned par = call & 1u;
    f32x4 yv[4];
#pragma unroll
    for (int j = 0; j < 4; ++j)
        yv[j] = *(const f32x4*)(lds + EPI_OFF + (wave * 8 + (lane >> 3)) * EPI_PITCH + (lane & 7) * 64 + j * 16);

    if (wave == PWAVE) {
        const char* gsrc = my_ws + WS_XG + par * 4096 + xcd * 512 + lane * 8;
        unsigned it = 0, gm;
        unsigned long long t0 = 0;
        for (;;) {
            unsigned long long v;
            asm volatile("global_load_dwordx2 %0, %1, off sc0 sc1\n\ts_waitcnt vmcnt(0)"
                         : "=&v"(v) : "v"(gsrc) : "memory");
            gm = (unsigned)v;
            if (__all((unsigned)(v >> 32) == call)) break;
            if ((++it & 63u) == 0u) {
                const unsigned long long t = __builtin_amdgcn_s_memrealtime();
                if (t0 == 0) t0 = t;
                if (t - t0 > DEADLINE_TICKS ||
                    __hip_atomic_load(tmo, __ATOMIC_RELAXED, __HIP_MEMORY_SCOPE_AGENT) == call) {
                    __hip_atomic_store(tmo, call, __ATOMIC_RELAXED, __HIP_MEMORY_SCOPE_AGENT);
                    gm = 0xffffffffu;
                    break;
                }
            }
        }
        gm = wave_max_u32(gm);
        if (lane == 0) red[0] = gm;
        WAIT_LGKM0();
    }
    BARRIER();
    const unsigned gmax = red[0];
    STAMP(3);
    const bool bad = gmax == 0xffffffffu;
    const float amax = __uint_as_float(gmax);
    const float scale = amax / 448.0f;
    const float inv = scale > 0.f ? 1.0f / scale : 0.f;

    {
        const int lr = wave * 8 + (lane >> 3);
        const int orow = rank * M_LOC + row0 + (lr >> 5) * 64 + ks * 32 + (lr & 31);
        char* o = (char*)p.out[dest] + (size_t)orow * N_LOC + (tn & 3) * BN + (lane & 7) * 16;
        u32x4 q;
#pragma unroll
        for (int j = 0; j < 4; ++j) {
            int w = __builtin_amdgcn_cvt_pk_fp8_f32(yv[j][0] * inv, yv[j][1] * inv, 0, false);
            w = __builtin_amdgcn_cvt_pk_fp8_f32(yv[j][2] * inv, yv[j][3] * inv, w, true);
            q[j] = bad ? 0x7f7f7f7fu : (unsigned)w;
        }
        store16_sys(o, q);
    }
    if (wave == 1 && idx == 1) {
        asm volatile("buffer_wbl2 sc1\n\ts_waitcnt vmcnt(1)" ::: "memory");
    } else
    asm volatile("s_waitcnt vmcnt(0)" ::: "memory");
    BARRIER();
    STAMP(4);
    if (tid == 0) {
        if (dest != rank) {
            unsigned* flag = (unsigned*)((char*)p.ws[dest] + WS_DONE + (size_t)(rank * 32 + didx) * 128);
            __hip_atomic_store(flag, call, __ATOMIC_RELAXED, __HIP_MEMORY_SCOPE_SYSTEM);
        }
        __hip_atomic_store(ctr, call, __ATOMIC_RELAXED, __HIP_MEMORY_SCOPE_AGENT);
        if ((wg >> 5) != rank) {
            unsigned* flag = (unsigned*)(my_ws + WS_DONE + (size_t)wg * 128);
            unsigned it = 0;
            unsigned long long t0 = 0;
            while (__hip_atomic_load(flag, __ATOMIC_RELAXED, __HIP_MEMORY_SCOPE_SYSTEM) < call) {
                if ((++it & 63u) == 0u) {
                    const unsigned long long t = __builtin_amdgcn_s_memrealtime();
                    if (t0 == 0) t0 = t;
                    if (t - t0 > DEADLINE_TICKS ||
                        __hip_atomic_load(tmo, __ATOMIC_RELAXED, __HIP_MEMORY_SCOPE_AGENT) == call) {
                        __hip_atomic_store(tmo, call, __ATOMIC_RELAXED, __HIP_MEMORY_SCOPE_AGENT);
                        __hip_atomic_store((unsigned char*)p.out[rank] + (size_t)rank * M_LOC * N_LOC,
                                           (unsigned char)0x7f, __ATOMIC_RELAXED, __HIP_MEMORY_SCOPE_SYSTEM);
                        break;
                    }
                }
            }
        }
    }
    STAMP(5);
}

}

extern "C" void kernel_launch(int rank, int nranks, ncclComm_t comm,
                              hipStream_t stream,
                              void* const* d_in, const int* in_sizes, int n_in,
                              void* d_out, int out_size,
                              const int* in_dtypes, int out_dtype,
                              void* const* d_out_all,
                              void* const* d_ws_all, size_t ws_bytes) {
    (void)comm; (void)in_dtypes; (void)out_dtype; (void)d_out;
    if (nranks != NR || n_in < 2 || (unsigned)rank >= (unsigned)NR || ws_bytes < WS_END ||
        in_sizes[0] != M_LOC * K_TOT || in_sizes[1] != K_TOT * N_TOT || out_size != N_TOT * N_LOC) {
        fprintf(stderr, "kernel_launch: unexpected problem shape\n");
        return;
    }
    Params p;
    p.x = d_in[0];
    p.w = d_in[1];
    for (int i = 0; i < NR; ++i) {
        p.out[i] = d_out_all[i];
        p.ws[i] = d_ws_all[i];
    }
    p.rank = rank;
    p.pad = 0;
    p.my_ws = d_ws_all[rank];
    hipLaunchKernelGGL(gemm_a2a_kernel, dim3(NWG), dim3(NTHREADS), 0, stream, p);
    HIP_CHECK(hipGetLastError());
}
```

```cpp
#include <hip/hip_runtime.h>
#include <rccl/rccl.h>
#include <cstdio>
#include <cstdint>

#define HIP_CHECK(expr) do { hipError_t _e = (expr); if (_e != hipSuccess) { \
    fprintf(stderr, "HIP error %s at %s:%d\n", hipGetErrorString(_e), __FILE__, __LINE__); \
    return; } } while (0)

#define SWAPX 0
#define XPIPE 0
#define SWIDX 0
#define EARLY 0
#define EPOL "sc0 sc1"
#define ESLEEP 8
#define FSLEEP 64
#define SG 4312431, 3, 2, 1, 2, 1, 5, 0, 0, 0
namespace {

typedef __attribute__((ext_vector_type(4))) short s16x4;
typedef __attribute__((ext_vector_type(8))) short s16x8;
typedef __attribute__((ext_vector_type(2))) __bf16 bf16x2;
typedef __attribute__((ext_vector_type(2))) float f32x2;
typedef __attribute__((ext_vector_type(4))) float f32x4;
typedef __attribute__((ext_vector_type(2))) unsigned u32x2;
typedef __attribute__((ext_vector_type(4))) unsigned u32x4;
typedef __attribute__((address_space(3))) s16x4* lds_s16x4_ptr;
typedef __attribute__((address_space(3))) s16x8* lds_s16x8_ptr;

constexpr int NR = 8;
constexpr int M_LOC = 512;
constexpr int K_TOT = 4096;
constexpr int N_TOT = 4096;
constexpr int N_LOC = N_TOT / NR;
constexpr int BM = 256, BN = 128, BK = 64;
constexpr int NS = 3;
constexpr int A_BYTES = BM * BK * 2;
constexpr int B_BYTES = BK * BN * 2;
constexpr int STAGE_BYTES = A_BYTES + B_BYTES;
constexpr int NSTEP = 16;
constexpr int NWG = 256;
constexpr int NTHREADS = 512;
constexpr int LDS_CALL = NS * STAGE_BYTES;
constexpr int LDS_RED = LDS_CALL + 16;
constexpr int LDS_BYTES = LDS_RED + 16;

constexpr size_t WS_XG = 0;
constexpr size_t WS_LGF = WS_XG + 2 * 4096;
constexpr size_t WS_LGS = WS_LGF + 8192;
constexpr size_t WS_DONE = 32768;
constexpr size_t WS_CTR = WS_DONE + NWG * 128;
constexpr size_t WS_DIAG = WS_CTR + NWG * 128;
constexpr size_t WS_XB = 256 * 1024;
#define XB_STRIDE 49408
constexpr size_t WS_EB = WS_XB + (size_t)NWG * XB_STRIDE;
constexpr size_t WS_END = WS_EB + 2 * (size_t)NWG * 8192;
static_assert(WS_DIAG + NWG * 16 * 8 <= WS_XB, "");

constexpr unsigned POLL_LIMIT = 1u << 20;

struct Params {
    const void* x;
    const void* w;
    void* my_ws;
    int rank;
    int pad;
    void* out[NR];
    void* ws[NR];
    unsigned long long ord[NWG];
};

__device__ __forceinline__ void glds16(unsigned voff, const void* sbase, unsigned lds_dst) {
    asm volatile("s_mov_b32 m0, %2\n\ts_nop 0\n\tglobal_load_lds_dwordx4 %0, %1"
                 :: "v"(voff), "s"(sbase), "s"(lds_dst) : "memory");
}
#define WAIT_VM(n) asm volatile("s_waitcnt vmcnt(%0)" :: "i"(n) : "memory")
#define WAIT_LGKM0() asm volatile("s_waitcnt lgkmcnt(0)" ::: "memory")
#define BARRIER() asm volatile("s_barrier" ::: "memory")
#define SB() __builtin_amdgcn_sched_barrier(0)
#define MFMA(acc, a, b) asm volatile("v_mfma_f32_16x16x32_bf16 %0, %1, %2, %0" : "+v"(acc) : "v"(a), "v"(b))
#define MFMA0(acc, a, b) asm volatile("v_mfma_f32_16x16x32_bf16 %0, %1, %2, 0" : "+v"(acc) : "v"(a), "v"(b))

__device__ __forceinline__ unsigned wave_max_u32(unsigned v) {
    v = max(v, (unsigned)__builtin_amdgcn_update_dpp(0, (int)v, 0x111, 0xf, 0xf, false));
    v = max(v, (unsigned)__builtin_amdgcn_update_dpp(0, (int)v, 0x112, 0xf, 0xf, false));
    v = max(v, (unsigned)__builtin_amdgcn_update_dpp(0, (int)v, 0x114, 0xf, 0xf, false));
    v = max(v, (unsigned)__builtin_amdgcn_update_dpp(0, (int)v, 0x118, 0xf, 0xf, false));
    v = max(v, (unsigned)__builtin_amdgcn_update_dpp(0, (int)v, 0x142, 0xa, 0xf, false));
    v = max(v, (unsigned)__builtin_amdgcn_update_dpp(0, (int)v, 0x143, 0xc, 0xf, false));
    return (unsigned)__builtin_amdgcn_readlane((int)v, 63);
}

__device__ __forceinline__ unsigned pk_bf16(float a, float b) {
    const bf16x2 v = __builtin_convertvector((f32x2){a, b}, bf16x2);
    return __builtin_bit_cast(unsigned, v);
}
__device__ __forceinline__ unsigned pk_bf16_tag(float a, float b, unsigned tag) {
    const unsigned t = (__float_as_uint(a) + ((1u - tag) << 16)) & 0xfffe0000u;
    return (pk_bf16(__uint_as_float(t), b) & ~1u) | tag;
}
__device__ __forceinline__ u32x4 pack_granule(const f32x4& a, const f32x4& b, unsigned tag) {
    u32x4 g;
    g[0] = pk_bf16_tag(a[0], a[1], tag);
    g[1] = pk_bf16(a[2], a[3]);
    g[2] = pk_bf16(b[0], b[1]);
    g[3] = pk_bf16(b[2], b[3]);
    return g;
}
__device__ __forceinline__ float add_lo(unsigned w, float c) {
    unsigned one_lo = 0x00003f80u;
    asm("" : "+s"(one_lo));
    return __builtin_amdgcn_fdot2_f32_bf16(__builtin_bit_cast(bf16x2, w), __builtin_bit_cast(bf16x2, one_lo), c, false);
}
__device__ __forceinline__ float add_hi(unsigned w, float c) {
    return __builtin_amdgcn_fdot2_f32_bf16(__builtin_bit_cast(bf16x2, w), __builtin_bit_cast(bf16x2, 0x3f800000u), c, false);
}
__device__ __forceinline__ void add_granule(f32x4& a, f32x4& b, const u32x4& g) {
    a[0] = add_lo(g[0], a[0]); a[1] = add_hi(g[0], a[1]); a[2] = add_lo(g[1], a[2]); a[3] = add_hi(g[1], a[3]);
    b[0] = add_lo(g[2], b[0]); b[1] = add_hi(g[2], b[1]); b[2] = add_lo(g[3], b[2]); b[3] = add_hi(g[3], b[3]);
}
__device__ __forceinline__ void tr4(unsigned& r0, unsigned& r1, unsigned& r2, unsigned& r3) {
    u32x2 t;
    t = __builtin_amdgcn_permlane32_swap(r0, r2, false, false); r0 = t[0]; r2 = t[1];
    t = __builtin_amdgcn_permlane32_swap(r1, r3, false, false); r1 = t[0]; r3 = t[1];
    t = __builtin_amdgcn_permlane16_swap(r0, r1, false, false); r0 = t[0]; r1 = t[1];
    t = __builtin_amdgcn_permlane16_swap(r2, r3, false, false); r2 = t[0]; r3 = t[1];
}

#define STAMPT(i, t) do {} while (0)
#define STAMP(i) STAMPT(i, 256)

__device__ __forceinline__ unsigned wait_amax(unsigned lds_base, unsigned call) {
    for (unsigned it = 0;; ++it) {
        u32x2 r;
        asm volatile("ds_read_b64 %0, %1\n\ts_waitcnt lgkmcnt(0)" : "=v"(r) : "v"(lds_base + LDS_RED) : "memory");
        if (__builtin_amdgcn_readfirstlane(r[1]) == call) return __builtin_amdgcn_readfirstlane(r[0]);
        if (it > 512 * POLL_LIMIT) return 0xffffffffu;
    }
}
__device__ __forceinline__ float inv_scale(unsigned gmax) {
    const float scale = __uint_as_float(gmax) / 448.0f;
    return scale > 0.f ? 1.0f / scale : 0.f;
}

template <int V> struct IntC { static constexpr int value = V; };

__global__ void __launch_bounds__(NTHREADS, 2) gemm_a2a_kernel(const Params p) {
    __shared__ __attribute__((aligned(1024))) char lds[LDS_BYTES];
    const int tid = threadIdx.x;
    const int lane = tid & 63;
    const int wave = __builtin_amdgcn_readfirstlane(tid >> 6);
    const int wg = blockIdx.x;
    const int rank = p.rank;
    char* const my_ws = (char*)p.my_ws;
    const char* const gx = (const char*)p.x;
    const char* const gw = (const char*)p.w;
    asm volatile("" :: "s"(gx), "s"(gw), "s"(my_ws), "s"(rank));
    STAMP(0);

    const int xcd = wg & 7, idx = wg >> 3;
    const int tnl = idx & 3, tm = (idx >> 2) & 1, ks = idx >> 3;
    const int row0 = tm * BM;
    const int col0 = xcd * N_LOC + tnl * BN;

    const unsigned long long otab = p.ord[wg];
    const int of = 0;
    auto ktile = [&](int s) {
        return ks * NSTEP + (((int)(otab >> (4 * (s & 15))) + of) & 15);
    };
    unsigned* ctr = (unsigned*)(my_ws + WS_CTR + (size_t)wg * 128);
    const unsigned lds_base = (unsigned)(uintptr_t)lds;

    if (__builtin_expect(wave < 4, 1)) {
        const int hi = wave & 1;
        const int ordB = ktile(lane), ordA = ktile(lane - hi);
        unsigned voffA[8], voffB[4];
#pragma unroll
        for (int i = 0; i < 8; ++i) {
            const int r = wave * 64 + i * 8 + (lane >> 3);
            const int gr = r ^ ((3 ^ ks) << 5);
            const int c = (lane & 7) ^ ((r >> 1) & 7);
            voffA[i] = (unsigned)(gr * (K_TOT * 2) + c * 16);
        }
#pragma unroll
        for (int j = 0; j < 4; ++j) {
            const int k = wave * 16 + j * 4 + (lane >> 4);
            const int h = (k & 3) | (((k >> 3) & 1) << 2);
            const int c = (lane & 15) ^ (2 * h);
            voffB[j] = (unsigned)(k * (N_TOT * 2) + c * 16);
        }
        const char* const ga = gx + (size_t)row0 * (K_TOT * 2);
        const char* const gb = gw + (size_t)col0 * 2;
        const unsigned dA = lds_base + wave * 8192;
        const unsigned dB = lds_base + A_BYTES + wave * 4096;
        auto issue = [&](int u, unsigned so) {
            const char* a = ga + (unsigned)__builtin_amdgcn_readlane(ordA, u) * (BK * 2);
            const char* b = gb + (unsigned)__builtin_amdgcn_readlane(ordB, u) * (BK * N_TOT * 2);
#pragma unroll
            for (int j = 0; j < 4; ++j) {
                glds16(voffB[j], b, dB + so + j * 1024);
                glds16(voffA[2 * j], a, dA + so + 2 * j * 1024);
                glds16(voffA[2 * j + 1], a, dA + so + (2 * j + 1) * 1024);
            }
        };
        if (hi) {
            const char* b = gb + (unsigned)__builtin_amdgcn_readlane(ordB, 0) * (BK * N_TOT * 2);
#pragma unroll
            for (int j = 0; j < 4; ++j) glds16(voffB[j], b, dB + j * 1024);
        } else {
            issue(0, 0);
        }
        issue(1, STAGE_BYTES);
        unsigned so = 2 * STAGE_BYTES;
#define TLS(k)
        for (int k = 0; k < NSTEP - 2; ++k) {
            WAIT_VM(12);
            BARRIER();
            TLS(k);
            issue(k + 2, so);
            so = (so == 2 * STAGE_BYTES) ? 0 : so + STAGE_BYTES;
        }
        WAIT_VM(12); BARRIER();
        TLS(14);
        if (hi) {
            const char* a = ga + (unsigned)__builtin_amdgcn_readlane(ordA, 16) * (BK * 2);
#pragma unroll
            for (int j = 0; j < 8; ++j) glds16(voffA[j], a, dA + STAGE_BYTES + j * 1024);
            WAIT_VM(8);
        } else {
            WAIT_VM(0);
        }
        BARRIER();
        TLS(15);
        WAIT_VM(0); BARRIER();
        TLS(16);
        const unsigned call = __builtin_amdgcn_readfirstlane(*(const unsigned*)(lds + LDS_CALL));
        if (wave == 0 && idx == SWIDX) {
            char* dst = (char*)p.ws[lane >> 3] + WS_XG + (call & 1u) * 4096 + (lane & 7) * 512 + rank * 64 + xcd * 8;
            const char* lg = my_ws + WS_LGF + xcd * 1024 + lane * 16;
            asm volatile("" :: "v"(dst));
            unsigned m0 = 0, m1 = 0;
            bool ok0 = false, ok1 = false;
            unsigned it = 0;
            for (;;) {
                const char* src = lg + (((it & 63u) == 63u) ? (WS_LGS - WS_LGF) : 0);
                u32x4 v;
                asm volatile("global_load_dwordx4 %0, %1, off sc1\n\ts_waitcnt vmcnt(0)"
                             : "=&v"(v) : "v"(src) : "memory");
                if (v[1] == call) { m0 = v[0]; ok0 = true; }
                if (v[3] == call) { m1 = v[2]; ok1 = true; }
                if (__all(ok0 && ok1)) break;
                if (++it > 4 * POLL_LIMIT) { m0 = 0xffffffffu; break; }
            }
            const unsigned m = wave_max_u32(max(m0, m1));
            asm volatile("global_store_dwordx2 %0, %1, off sc0 sc1"
                         :: "v"(dst), "v"(((unsigned long long)call << 32) | m) : "memory");
            STAMPT(6, 0);
        }
        if (wave == 0) {
            const char* gsrc = my_ws + WS_XG + (call & 1u) * 4096 + xcd * 512 + lane * 8;
            for (unsigned it = 0; it < 64 * POLL_LIMIT; ++it) {
                unsigned c;
                asm volatile("ds_read_b32 %0, %1\n\ts_waitcnt lgkmcnt(0)" : "=v"(c) : "v"(lds_base + LDS_RED + 8) : "memory");
                if (__builtin_amdgcn_readfirstlane(c) == 4u) break;
            }
            unsigned gm, it = 0;
            for (;;) {
                unsigned long long v;
                asm volatile("global_load_dwordx2 %0, %1, off sc0 sc1\n\ts_waitcnt vmcnt(0)"
                             : "=&v"(v) : "v"(gsrc) : "memory");
                gm = (unsigned)v;
                if (__all((unsigned)(v >> 32) == call)) break;
                if (++it > 4 * POLL_LIMIT) { gm = 0xffffffffu; break; }
                if ((it & 255u) == 0u) asm volatile("buffer_wbl2 sc1\n\ts_waitcnt vmcnt(0)" ::: "memory");
            }
            const unsigned gmax = wave_max_u32(gm);
            asm volatile("ds_write_b64 %0, %1" :: "v"(lds_base + LDS_RED), "v"(((unsigned long long)call << 32) | gmax) : "memory");
        }
        const int src = wg >> 5;
        if (src == rank) return;
        if ((EARLY >> wave) & 1) {
            const unsigned etag = ((call + 1u) >> 1) & 1u;
            const char* e = my_ws + WS_EB + (size_t)(call & 1u) * (NWG * 8192) + (size_t)(wg * 4 + wave) * 2048 + lane * 16;
            const int si = wg & 31;
            const int orow = src * M_LOC + ((si >> 2) & 1) * BM + (wave >> 1) * 128 + (si >> 3) * 32 + 16 + (lane & 15);
            char* o = (char*)p.out[rank] + (size_t)orow * N_LOC + (si & 3) * BN + (wave & 1) * 64 + (lane >> 4) * 16;
            asm volatile("" :: "v"(o));
            u32x4 g0, g1;
            bool bad = false;
            for (unsigned it = 0;;) {
                asm volatile("global_load_dwordx4 %0, %2, off sc0 sc1\n\t"
                             "global_load_dwordx4 %1, %2, off offset:1024 sc0 sc1\n\ts_waitcnt vmcnt(0)"
                             : "=&v"(g0), "=&v"(g1) : "v"(e) : "memory");
                if (__all((((g0[0] ^ etag) | (g1[0] ^ etag)) & 1u) == 0u)) break;
                if (++it > POLL_LIMIT) { bad = true; break; }
                __builtin_amdgcn_s_sleep(ESLEEP);
            }
            const unsigned gmax = wait_amax(lds_base, call);
            const float inv = inv_scale(gmax);
            bad = bad || gmax == 0xffffffffu;
            u32x4 q;
#pragma unroll
            for (int j = 0; j < 4; ++j) {
                const unsigned w0 = j < 2 ? g0[2 * j] : g1[2 * j - 4], w1 = j < 2 ? g0[2 * j + 1] : g1[2 * j - 3];
                int w = __builtin_amdgcn_cvt_pk_fp8_f32(__uint_as_float(w0 << 16) * inv, __uint_as_float(w0 & 0xffff0000u) * inv, 0, false);
                w = __builtin_amdgcn_cvt_pk_fp8_f32(__uint_as_float(w1 << 16) * inv, __uint_as_float(w1 & 0xffff0000u) * inv, w, true);
                q[j] = bad ? 0x7f7f7f7fu : (unsigned)w;
            }
            asm volatile("global_store_dwordx4 %0, %1, off sc0 sc1\n\ts_nop 1" :: "v"(o), "v"(q) : "memory");
        }
        if (wave == 1) {
            wait_amax(lds_base, call);
            __builtin_amdgcn_s_sleep(FSLEEP);
            const char* f = my_ws + WS_DONE + (size_t)wg * 128 + (lane & 3) * 8;
            for (unsigned it = 0;; ++it) {
                unsigned long long v;
                asm volatile("global_load_dwordx2 %0, %1, off sc0 sc1\n\ts_waitcnt vmcnt(0)" : "=&v"(v) : "v"(f) : "memory");
                if (__all((unsigned)(v >> 32) >= call)) break;
                if (it > POLL_LIMIT) {
                    if (lane == 0)
                        __hip_atomic_store((unsigned char*)p.out[rank] + (size_t)rank * M_LOC * N_LOC,
                                           (unsigned char)0x7f, __ATOMIC_RELAXED, __HIP_MEMORY_SCOPE_SYSTEM);
                    break;
                }
            }
            STAMPT(5, 64);
        }
        return;
    }
    const int cw = wave - 4;
    const int wm = cw >> 1, wn = cw & 1;
    unsigned call;
    asm volatile("global_load_dword %0, %1, off sc1\n\ts_waitcnt vmcnt(0)" : "=&v"(call) : "v"(ctr) : "memory");
    call = __builtin_amdgcn_readfirstlane(call) + 1u;
    if (tid == 256) {
        *(u32x4*)(lds + LDS_CALL) = (u32x4){call, 0u, 0u, 0u};
        *(u32x4*)(lds + LDS_RED) = (u32x4){0u, 0u, 0u, 0u};
    }
    WAIT_LGKM0();
    const unsigned par = call & 1u;
    auto pf = [&](int) {};
    const int pfd = 0;
    unsigned rdA[2], rdB[4];
#pragma unroll
    for (int kk = 0; kk < 2; ++kk) {
        const int r = lane & 15;
        const int c = 4 * kk + (lane >> 4);
        rdA[kk] = lds_base + (unsigned)(wm * 16384 + r * 128 + ((c ^ ((r >> 1) & 7)) * 16));
    }
#pragma unroll
    for (int cn = 0; cn < 4; ++cn) {
        const int g = lane >> 4, q = (lane & 15) >> 2, pp = lane & 3;
        const int k = 8 * g + q;
        const int h = (k & 3) | (((k >> 3) & 1) << 2);
        const int u = (wn * 4 + cn) ^ h;
        rdB[cn] = lds_base + (unsigned)(A_BYTES + k * 256 + u * 32 + pp * 8);
    }
    f32x4 acc[8][4];
#pragma unroll
    for (int i = 0; i < 8; ++i) {
#pragma unroll
        for (int j = 0; j < 4; ++j) acc[i][j] = (f32x4){0.f, 0.f, 0.f, 0.f};
        asm volatile("s_nop 1" : "+v"(acc[i][0]), "+v"(acc[i][1]), "+v"(acc[i][2]), "+v"(acc[i][3]));
    }
    s16x8 aq[2][2];
    s16x4 bl[2][4][2], bh[2][4][2];
    const unsigned xv = cw * 4096 + lane * 16;
    const __amdgpu_buffer_rsrc_t xr = __builtin_amdgcn_make_buffer_rsrc(my_ws + WS_XB, 0, NWG * XB_STRIDE, 0x00020000);
    const unsigned xo = wg * XB_STRIDE;
    unsigned tmask = 0xfffe0000u;
    const unsigned tc1 = (1u - par) << 16;
    asm volatile("" : "+v"(tmask) : "v"(xv));

#define BFR(s, n, kk) __builtin_shufflevector(bl[s][n][kk], bh[s][n][kk], 0, 1, 2, 3, 4, 5, 6, 7)
#define RDA(kk, mt) (*(lds_s16x8_ptr)(uintptr_t)(rdA[kk] + (mt) * 2048))
#define RDB(kk, n, h) __builtin_amdgcn_ds_read_tr16_b64_v4i16((lds_s16x4_ptr)(uintptr_t)(rdB[n] + (kk) * 8192 + (h) * 1024))
    auto rdb = [&](auto sc, int j) {
        constexpr int s = decltype(sc)::value;
        const int n = j >> 2, kk = ((j >> 1) & 1) ^ (n & 1);
        if (j & 1) bh[s][n][kk] = RDB(kk, n, 1); else bl[s][n][kk] = RDB(kk, n, 0);
    };
    auto grp = [&](auto mtc, auto abc, auto bsc, int na, auto&& fill) {
        constexpr int mt = decltype(mtc)::value, ab = decltype(abc)::value, bs = decltype(bsc)::value;
#pragma unroll
        for (int i = 0; i < 8; ++i) {
            const int nt = i >> 1, kk = (i & 1) ^ (nt & 1);
            MFMA(acc[mt][nt], BFR(bs, nt, kk), aq[ab][kk]);
            SB();
            if (i < 2 && na >= 0) aq[1 - ab][i] = RDA(i, na);
            fill(i);
            SB();
        }
    };
#define I(v) IntC<v>{}
    auto none = [](int) {};
    unsigned slot = 0, sd = 0;
    auto next_slot = [&](int i) {
        if (i == 0) { sd = slot == NS - 1 ? (unsigned)(-(NS - 1) * STAGE_BYTES) : (unsigned)STAGE_BYTES; slot = slot == NS - 1 ? 0 : slot + 1; }
        else if (i < 5) rdB[i - 1] += sd;
        else rdA[i - 5] += sd;
    };
    auto interval = [&](auto sc, int k) {
        constexpr int s = decltype(sc)::value;
        BARRIER();
        SB();
        grp(I(3), I(1), I(1 - s), 4, [&](int i) { if (i >= 2 && i < 6) rdb(sc, i - 2); });
        grp(I(4), I(0), I(1 - s), 5, [&](int i) { if (i >= 2 && i < 6) rdb(sc, i + 2); });
        grp(I(5), I(1), I(1 - s), 6, [&](int i) { if (i >= 2 && i < 6) rdb(sc, i + 6); });
        grp(I(6), I(0), I(1 - s), 7, [&](int i) { if (i >= 2 && i < 6) rdb(sc, i + 10); });
        grp(I(7), I(1), I(1 - s), 0, [&](int i) { if (s == 1 && i == 4) pf(k + pfd); });
        grp(I(0), I(0), sc, 1, none);
        grp(I(1), I(1), sc, 2, [&](int i) { if (i >= 2 && i < 7) next_slot(i - 2); });
        grp(I(2), I(0), sc, 3, [&](int i) { if (i == 2 || i == 4) next_slot(4 + (i >> 1)); });
        WAIT_LGKM0();
        SB();
    };
    BARRIER();
    SB();
    aq[0][0] = RDA(0, 0);
    rdb(I(0), 0); rdb(I(0), 1);
    aq[0][1] = RDA(1, 0);
#pragma unroll
    for (int j = 2; j < 16; ++j) rdb(I(0), j);
    SB();
    grp(I(0), I(0), I(0), 1, none);
    grp(I(1), I(1), I(0), 2, [&](int i) { if (i >= 2 && i < 7) next_slot(i - 2); });
    grp(I(2), I(0), I(0), 3, [&](int i) { if (i == 2 || i == 4) next_slot(4 + (i >> 1)); });
    WAIT_LGKM0();
    SB();
    for (int k = 1; k < NSTEP - 1; k += 2) {
        interval(I(1), k);
        interval(I(0), k + 1);
    }
    u32x4 g;
    auto xst = [&](auto mtc, int i) {
        constexpr int mt = decltype(mtc)::value;
        const int nt = i >= 4 ? 2 : 0;
        const int j = i & 3;
        if (j == 1) {
            g[0] = (__float_as_uint(acc[mt][nt][0]) + tc1) & tmask;
            g[1] = pk_bf16(acc[mt][nt][2], acc[mt][nt][3]);
        } else if (j == 2) {
            g[0] = (pk_bf16(__uint_as_float(g[0]), acc[mt][nt][1]) & ~1u) | par;
        } else if (j == 3) {
            g[2] = pk_bf16(acc[mt][nt + 1][0], acc[mt][nt + 1][1]);
            g[3] = pk_bf16(acc[mt][nt + 1][2], acc[mt][nt + 1][3]);
            __builtin_amdgcn_raw_buffer_store_b128(g, xr, xv + ((mt & 1) * 2 + (nt >> 1)) * 1024, xo + (mt >> 1) * 16384, 0);
        }
    };
    u32x4 v[12];
    auto xld = [&](int b, int q) {
        v[4 * b + q] = __builtin_amdgcn_raw_buffer_load_b128(xr, xv + q * 1024, (wg ^ ((3 - b) << 6)) * XB_STRIDE + b * 16384, 16);
    };
    constexpr int S1 = STAGE_BYTES / 2048;
    BARRIER();
    SB();
    grp(I(3), I(1), I(0), 4, [&](int i) { rdb(I(1), i); });
    grp(I(4), I(0), I(0), 0, [&](int i) { rdb(I(1), i + 8); });
    grp(I(0), I(1), I(1), 1, none);
    grp(I(1), I(0), I(1), 2, [&](int i) { xst(I(0), i); });
    grp(I(2), I(1), I(1), 3, [&](int i) { xst(I(1), i); });
    grp(I(3), I(0), I(1), 5, [&](int i) { xst(I(2), i); });
    BARRIER();
    SB();
    grp(I(5), I(1), I(0), S1 + 4, [&](int i) { xst(I(3), i); });
    grp(I(4), I(0), I(1), S1 + 5, none);
    grp(I(5), I(1), I(1), 6, [&](int i) { xst(I(4), i); });
    grp(I(6), I(0), I(0), S1 + 6, [&](int i) { xst(I(5), i); });
    grp(I(6), I(1), I(1), 7, none);
    grp(I(7), I(0), I(0), S1 + 7, [&](int i) { if (i >= 4) xld(0, i - 4); });
    grp(I(7), I(1), I(1), -1, [&](int i) { if (i >= 4) xld(1, i - 4); });
    xld(2, 0); xld(2, 1); xld(2, 2); xld(2, 3);
    asm volatile("s_nop 7" : "+v"(acc[6][0]), "+v"(acc[6][1]), "+v"(acc[6][2]), "+v"(acc[6][3]),
                 "+v"(acc[7][0]), "+v"(acc[7][1]), "+v"(acc[7][2]), "+v"(acc[7][3]));
    SB();
    STAMP(1);
    bool bad = false;
#pragma unroll
    for (int b = 0; b < 3; ++b) {
        for (unsigned it = 0;;) {
            const unsigned mism = (v[4 * b][0] ^ par) | (v[4 * b + 1][0] ^ par) | (v[4 * b + 2][0] ^ par) | (v[4 * b + 3][0] ^ par);
            if (__all((mism & 1u) == 0u)) break;
            if (++it > POLL_LIMIT) { bad = true; break; }
            if ((it & 255u) == 0u) asm volatile("buffer_wbl2 sc1\n\ts_waitcnt vmcnt(0)" ::: "memory");
            asm volatile("" ::: "memory");
            xld(b, 0); xld(b, 1); xld(b, 2); xld(b, 3);
        }
#pragma unroll
        for (int q = 0; q < 4; ++q)
            add_granule(acc[6 + (q >> 1)][2 * (q & 1)], acc[6 + (q >> 1)][2 * (q & 1) + 1], v[4 * b + q]);
    }
    float am = 0.f;
#pragma unroll
    for (int i = 0; i < 8; ++i)
#pragma unroll
        for (int e = 0; e < 4; ++e) am = fmaxf(am, fabsf(acc[6 + (i >> 2)][i & 3][e]));
    unsigned amb = wave_max_u32(__float_as_uint(am));
    if (bad) amb = 0xffffffffu;
    if (lane == 0) {
        const char* lg = my_ws + WS_LGF + xcd * 1024 + (idx * 4 + cw) * 8;
        const unsigned long long g = ((unsigned long long)call << 32) | amb;
        asm volatile("global_store_dwordx2 %0, %1, off\n\t"
                     "global_store_dwordx2 %2, %1, off sc1"
                     :: "v"(lg), "v"(g), "v"(lg + (WS_LGS - WS_LGF)) : "memory");
        asm volatile("ds_add_u32 %0, %1" :: "v"(lds_base + LDS_RED + 8), "v"(1u) : "memory");
    }
    STAMP(2);
    f32x4 yv[2][4];
#pragma unroll
    for (int m = 0; m < 2; ++m)
#pragma unroll
        for (int e = 0; e < 4; ++e) {
            unsigned r0 = __float_as_uint(acc[6 + m][0][e]), r1 = __float_as_uint(acc[6 + m][1][e]);
            unsigned r2 = __float_as_uint(acc[6 + m][2][e]), r3 = __float_as_uint(acc[6 + m][3][e]);
            tr4(r0, r1, r2, r3);
            yv[m][0][e] = __uint_as_float(r0); yv[m][1][e] = __uint_as_float(r1);
            yv[m][2][e] = __uint_as_float(r2); yv[m][3][e] = __uint_as_float(r3);
        }
    const int orow = rank * M_LOC + row0 + wm * 128 + ks * 32 + (lane & 15);
    char* o = (char*)p.out[xcd] + (size_t)orow * N_LOC + tnl * BN + wn * 64 + (lane >> 4) * 16;
    char* o2 = o + 16 * N_LOC;
    asm volatile("" :: "v"(o), "v"(o2));
    const bool early = ((EARLY >> cw) & 1) && xcd != rank;
    if (early) {
        const unsigned etag = ((call + 1u) >> 1) & 1u;
        char* e = (char*)p.ws[xcd] + WS_EB + (size_t)par * (NWG * 8192) + (size_t)((rank * 32 + idx) * 4 + cw) * 2048 + lane * 16;
        const u32x4 g0 = pack_granule(yv[1][0], yv[1][1], etag), g1 = pack_granule(yv[1][2], yv[1][3], etag);
        asm volatile("global_store_dwordx4 %0, %1, off " EPOL "\n\t"
                     "global_store_dwordx4 %0, %2, off offset:1024 " EPOL "\n\ts_nop 1" :: "v"(e), "v"(g0), "v"(g1) : "memory");
    }
    char* dflag = (char*)p.ws[xcd] + WS_DONE + (size_t)(rank * 32 + idx) * 128 + cw * 8;
    asm volatile("" :: "v"(dflag));
    const unsigned gmax = wait_amax(lds_base, call);
    STAMP(3);
    bad = gmax == 0xffffffffu;
    const float inv = inv_scale(gmax);
#pragma unroll
    for (int m = 0; m < 2; ++m) {
        if (m == 1 && early) break;
        u32x4 q;
#pragma unroll
        for (int j = 0; j < 4; ++j) {
            int w = __builtin_amdgcn_cvt_pk_fp8_f32(yv[m][j][0] * inv, yv[m][j][1] * inv, 0, false);
            w = __builtin_amdgcn_cvt_pk_fp8_f32(yv[m][j][2] * inv, yv[m][j][3] * inv, w, true);
            q[j] = bad ? 0x7f7f7f7fu : (unsigned)w;
        }
        asm volatile("global_store_dwordx4 %0, %1, off sc0 sc1\n\ts_nop 1" :: "v"(m ? o2 : o), "v"(q) : "memory");
    }
    if (cw == 2 && idx == 1) {
        asm volatile("buffer_wbl2 sc1\n\ts_waitcnt vmcnt(1)" ::: "memory");
    } else {
        asm volatile("s_waitcnt vmcnt(0)" ::: "memory");
    }
    STAMP(4);
    if (lane == 0 && xcd != rank)
        asm volatile("global_store_dwordx2 %0, %1, off sc0 sc1"
                     :: "v"(dflag), "v"((unsigned long long)call << 32) : "memory");
    if (tid == 256) __hip_atomic_store(ctr, call, __ATOMIC_RELAXED, __HIP_MEMORY_SCOPE_AGENT);
}

}

extern "C" void kernel_launch(int rank, int nranks, ncclComm_t comm,
                              hipStream_t stream,
                              void* const* d_in, const int* in_sizes, int n_in,
                              void* d_out, int out_size,
                              const int* in_dtypes, int out_dtype,
                              void* const* d_out_all,
                              void* const* d_ws_all, size_t ws_bytes) {
    (void)comm; (void)in_dtypes; (void)out_dtype; (void)d_out;
    if (nranks != NR || n_in < 2 || (unsigned)rank >= (unsigned)NR || ws_bytes < WS_END ||
        in_sizes[0] != M_LOC * K_TOT || in_sizes[1] != K_TOT * N_TOT || out_size != N_TOT * N_LOC) {
        fprintf(stderr, "kernel_launch: unexpected problem shape\n");
        return;
    }
    Params p;
    p.x = d_in[0];
    p.w = d_in[1];
    for (int i = 0; i < NR; ++i) {
        p.out[i] = d_out_all[i];
        p.ws[i] = d_ws_all[i];
    }
    p.rank = rank;
    p.pad = 0;
    p.my_ws = d_ws_all[rank];
    const int sg[10] = {SG};
    {
        int seq[10], ns = 0;
        for (int d = sg[0]; d > 0; d /= 10) seq[ns++] = d % 10 - 1;
        for (int w = 0; w < NWG; ++w) {
            const int xcd = w & 7, tnl = (w >> 3) & 3, tm = (w >> 5) & 1, ks = w >> 6;
            int key[16];
            for (int k = 0; k < 16; ++k) {
                const int ph = k + sg[2] * ks + sg[3] * xcd;
                const int mine = seq[(ph + sg[1] * tm) % ns], other = seq[(ph + sg[1] * (1 - tm)) % ns];
                int lead = 0;
                if (mine == tnl && other == tnl) lead = ((ph / ns + tm) & 1) ? sg[4] : sg[6];
                else if (mine == tnl) lead = sg[4];
                else if (other == tnl) lead = 0;
                else lead = (((ph + tnl * sg[7]) >> sg[8]) + tm + sg[9]) & 1 ? sg[5] : 0;
                key[k] = (4 * k - 4 * lead - (lead ? 2 : 0)) * 16 + k;
            }
            unsigned long long t = 0;
            for (int k = 0; k < 16; ++k) {
                int pos = 0;
                for (int q = 0; q < 16; ++q) pos += key[q] < key[k];
                t |= (unsigned long long)k << (4 * pos);
            }
            p.ord[w] = t;
        }
    }
    hipLaunchKernelGGL(gemm_a2a_kernel, dim3(NWG), dim3(NTHREADS), 0, stream, p);
    HIP_CHECK(hipGetLastError());
}
```

```cpp
#include <hip/hip_runtime.h>
#include <rccl/rccl.h>
#include <cstdio>
#include <cstdint>

#define HIP_CHECK(expr) do { hipError_t _e = (expr); if (_e != hipSuccess) { \
    fprintf(stderr, "HIP error %s at %s:%d\n", hipGetErrorString(_e), __FILE__, __LINE__); \
    return; } } while (0)

#define SWAPX 0
#define TLW 2
#define SWIDX 0
#define PREHEAT 16
#define SPEC 1
#define SPERIOD 256
#define WBMODE 3
#define XST_AUX 0
#define WBDELAY 48
#define POLLGATE 1
#define RECORD 1
#define SGRACE 0
#define XSLEEP 0
#define XSLEEP1 0
#define XSLEEP2 0
#define SPRIO 0
#define SWQUIET 1
#define SPINS 0
#define PREF 7
#define FSLEEP 16
#define SG 4312431, 3, 2, 1, 2, 1, 5, 0, 0, 0
namespace {

typedef __attribute__((ext_vector_type(4))) short s16x4;
typedef __attribute__((ext_vector_type(8))) short s16x8;
typedef __attribute__((ext_vector_type(2))) __bf16 bf16x2;
typedef __attribute__((ext_vector_type(2))) float f32x2;
typedef __attribute__((ext_vector_type(4))) float f32x4;
typedef __attribute__((ext_vector_type(2))) unsigned u32x2;
typedef __attribute__((ext_vector_type(4))) unsigned u32x4;
typedef __attribute__((address_space(3))) s16x4* lds_s16x4_ptr;
typedef __attribute__((address_space(3))) s16x8* lds_s16x8_ptr;
typedef __attribute__((address_space(3))) f32x4* lds_f32x4_ptr;
typedef __attribute__((address_space(3))) volatile unsigned* lds_u32_ptr;

constexpr int NR = 8;
constexpr int M_LOC = 512;
constexpr int K_TOT = 4096;
constexpr int N_TOT = 4096;
constexpr int N_LOC = N_TOT / NR;
constexpr int BM = 256, BN = 128, BK = 64;
constexpr int NS = 3;
constexpr int A_BYTES = BM * BK * 2;
constexpr int B_BYTES = BK * BN * 2;
constexpr int STAGE_BYTES = A_BYTES + B_BYTES;
constexpr int NSTEP = 16;
constexpr int NWG = 256;
constexpr int NTHREADS = 512;
constexpr int LDS_CALL = NS * STAGE_BYTES;
constexpr int LDS_RED = LDS_CALL + 16;
constexpr int LDS_HF = LDS_RED + 16;
constexpr int LDS_RDY = LDS_CALL + 4;
constexpr int LDS_L16 = LDS_CALL + 8;
constexpr int LDS_BYTES = LDS_HF + 32;
constexpr int LDS_HO = 2 * STAGE_BYTES;

constexpr size_t WS_XG = 0;
constexpr size_t WS_LGF = WS_XG + 2 * 8192;
constexpr size_t WS_LGS = WS_LGF + 16384;
constexpr size_t WS_DONE = WS_LGS + 16384;
constexpr size_t WS_CTR = WS_DONE + NWG * 128;
constexpr size_t WS_DIAG = WS_CTR + NWG * 128;
constexpr size_t WS_XB = 256 * 1024;
#define XB_STRIDE 49408
constexpr size_t WS_END = WS_XB + (size_t)NWG * XB_STRIDE;
static_assert(WS_DIAG + NWG * 16 * 8 <= WS_XB, "");

constexpr unsigned POLL_LIMIT = 1u << 20;

struct Params {
    const void* x;
    const void* w;
    void* my_ws;
    int rank;
    int pad;
    int pace[8];
    void* out[NR];
    void* ws[NR];
    unsigned long long ord[NWG];
};

__device__ __forceinline__ void glds16(unsigned voff, const void* sbase, unsigned lds_dst) {
    asm volatile("s_mov_b32 m0, %2\n\ts_nop 0\n\tglobal_load_lds_dwordx4 %0, %1"
                 :: "v"(voff), "s"(sbase), "s"(lds_dst) : "memory");
}
#define WAIT_VM(n) asm volatile("s_waitcnt vmcnt(%0)" :: "i"(n) : "memory")
#define WAIT_LGKM0() asm volatile("s_waitcnt lgkmcnt(0)" ::: "memory")
#define BARRIER() asm volatile("s_barrier" ::: "memory")
#define SB() __builtin_amdgcn_sched_barrier(0)
#define MFMA(acc, a, b) asm volatile("v_mfma_f32_16x16x32_bf16 %0, %1, %2, %0" : "+v"(acc) : "v"(a), "v"(b))
#define MFMA0(acc, a, b) asm volatile("v_mfma_f32_16x16x32_bf16 %0, %1, %2, 0" : "+v"(acc) : "v"(a), "v"(b))

__device__ __forceinline__ unsigned wave_max_u32(unsigned v) {
    v = max(v, (unsigned)__builtin_amdgcn_update_dpp(0, (int)v, 0x111, 0xf, 0xf, false));
    v = max(v, (unsigned)__builtin_amdgcn_update_dpp(0, (int)v, 0x112, 0xf, 0xf, false));
    v = max(v, (unsigned)__builtin_amdgcn_update_dpp(0, (int)v, 0x114, 0xf, 0xf, false));
    v = max(v, (unsigned)__builtin_amdgcn_update_dpp(0, (int)v, 0x118, 0xf, 0xf, false));
    v = max(v, (unsigned)__builtin_amdgcn_update_dpp(0, (int)v, 0x142, 0xa, 0xf, false));
    v = max(v, (unsigned)__builtin_amdgcn_update_dpp(0, (int)v, 0x143, 0xc, 0xf, false));
    return (unsigned)__builtin_amdgcn_readlane((int)v, 63);
}

__device__ __forceinline__ unsigned pk_bf16(float a, float b) {
    const bf16x2 v = __builtin_convertvector((f32x2){a, b}, bf16x2);
    return __builtin_bit_cast(unsigned, v);
}
__device__ __forceinline__ unsigned pk_bf16_tag(float a, float b, unsigned tag) {
    const unsigned t = (__float_as_uint(a) + ((1u - tag) << 16)) & 0xfffe0000u;
    return (pk_bf16(__uint_as_float(t), b) & ~1u) | tag;
}
__device__ __forceinline__ u32x4 pack_granule(const f32x4& a, const f32x4& b, unsigned tag) {
    u32x4 g;
    g[0] = pk_bf16_tag(a[0], a[1], tag);
    g[1] = pk_bf16(a[2], a[3]);
    g[2] = pk_bf16(b[0], b[1]);
    g[3] = pk_bf16(b[2], b[3]);
    return g;
}
__device__ __forceinline__ float add_lo(unsigned w, float c) {
    unsigned one_lo = 0x00003f80u;
    asm("" : "+s"(one_lo));
    return __builtin_amdgcn_fdot2_f32_bf16(__builtin_bit_cast(bf16x2, w), __builtin_bit_cast(bf16x2, one_lo), c, false);
}
__device__ __forceinline__ float add_hi(unsigned w, float c) {
    return __builtin_amdgcn_fdot2_f32_bf16(__builtin_bit_cast(bf16x2, w), __builtin_bit_cast(bf16x2, 0x3f800000u), c, false);
}
__device__ __forceinline__ void add_granule(f32x4& a, f32x4& b, const u32x4& g) {
    a[0] = add_lo(g[0], a[0]); a[1] = add_hi(g[0], a[1]); a[2] = add_lo(g[1], a[2]); a[3] = add_hi(g[1], a[3]);
    b[0] = add_lo(g[2], b[0]); b[1] = add_hi(g[2], b[1]); b[2] = add_lo(g[3], b[2]); b[3] = add_hi(g[3], b[3]);
}
__device__ __forceinline__ void tr4(unsigned& r0, unsigned& r1, unsigned& r2, unsigned& r3) {
    u32x2 t;
    t = __builtin_amdgcn_permlane32_swap(r0, r2, false, false); r0 = t[0]; r2 = t[1];
    t = __builtin_amdgcn_permlane32_swap(r1, r3, false, false); r1 = t[0]; r3 = t[1];
    t = __builtin_amdgcn_permlane16_swap(r0, r1, false, false); r0 = t[0]; r1 = t[1];
    t = __builtin_amdgcn_permlane16_swap(r2, r3, false, false); r2 = t[0]; r3 = t[1];
}

#define NCYC 2
#define STAMPT(i, t) do {} while (0)
#define STAMP(i) STAMPT(i, 256)

__device__ __forceinline__ unsigned wait_amax(unsigned lds_base, unsigned call) {
    for (unsigned it = 0;; ++it) {
        u32x2 r;
        asm volatile("ds_read_b64 %0, %1\n\ts_waitcnt lgkmcnt(0)" : "=v"(r) : "v"(lds_base + LDS_RED) : "memory");
        if (__builtin_amdgcn_readfirstlane(r[1]) == call) return __builtin_amdgcn_readfirstlane(r[0]);
        if (it > 64 * POLL_LIMIT) return 0xffffffffu;
        if (SPINS) __builtin_amdgcn_s_sleep(4);
    }
}
__device__ __forceinline__ float inv_scale(unsigned gmax) {
    const float scale = __uint_as_float(gmax) / 448.0f;
    return scale > 0.f ? 1.0f / scale : 0.f;
}

__device__ __forceinline__ char* xg_entry(const Params& p, int lane, int g) {
    char* b = (char*)p.ws[0];
#pragma unroll
    for (int i = 1; i < NR; ++i) {
        char* w = (char*)p.ws[i];
        asm("" : "+s"(w));
        b = (lane >> 3) == i ? w : b;
    }
    return b + WS_XG + (lane & 7) * 1024 + (p.rank * 8 + g) * 16;
}

template <int V> struct IntC { static constexpr int value = V; };

__global__ void __launch_bounds__(NTHREADS, 2) gemm_a2a_kernel(const Params p) {
    __shared__ __attribute__((aligned(1024))) char lds[LDS_BYTES];
    const int tid = threadIdx.x;
    const int lane = tid & 63;
    const int wave = __builtin_amdgcn_readfirstlane(tid >> 6);
    const int wg = blockIdx.x;
    const int rank = p.rank;
    char* const my_ws = (char*)p.my_ws;
    const char* const gx = (const char*)p.x;
    const char* const gw = (const char*)p.w;
    asm volatile("" :: "s"(gx), "s"(gw), "s"(my_ws), "s"(rank));
    STAMP(0);

    const int xcd = wg & 7, idx = wg >> 3;
    const int tnl = idx & 3, tm = (idx >> 2) & 1, ks = idx >> 3;
    const int row0 = tm * BM;
    const int col0 = xcd * N_LOC + tnl * BN;

    const unsigned long long otab = p.ord[wg];
    const int of = 0;
    auto ktile = [&](int s) {
        return ks * NSTEP + (((int)(otab >> (4 * (s & 15))) + of) & 15);
    };
    unsigned* ctr = (unsigned*)(my_ws + WS_CTR + (size_t)wg * 128);
    const unsigned lds_base = (unsigned)(uintptr_t)lds;

    if (__builtin_expect(wave < 4, 1)) {
        const int hi = wave & 1;
        const int ordB = ktile(lane), ordA = ktile(lane - hi);
        unsigned voffA[8], voffB[4];
#pragma unroll
        for (int i = 0; i < 8; ++i) {
            const int r = wave * 64 + i * 8 + (lane >> 3);
            const int gr = (r & 0x8f) | ((((r >> 4) ^ 3 ^ ks) & 3) << 5) | (hi << 4);
            const int c = (lane & 7) ^ ((r >> 1) & 7);
            voffA[i] = (unsigned)(gr * (K_TOT * 2) + c * 16);
        }
#pragma unroll
        for (int j = 0; j < 4; ++j) {
            const int k = wave * 16 + j * 4 + (lane >> 4);
            const int h = (k & 3) | (((k >> 3) & 1) << 2);
            const int c = (lane & 15) ^ (2 * h);
            voffB[j] = (unsigned)(k * (N_TOT * 2) + c * 16);
        }
        const char* const ga = gx + (size_t)row0 * (K_TOT * 2);
        const char* const gb = gw + (size_t)col0 * 2;
        const unsigned dA = lds_base + wave * 8192;
        const unsigned dB = lds_base + A_BYTES + wave * 4096;
        auto issue = [&](int u, unsigned so) {
            const char* a = ga + (unsigned)__builtin_amdgcn_readlane(ordA, u) * (BK * 2);
            const char* b = gb + (unsigned)__builtin_amdgcn_readlane(ordB, u) * (BK * N_TOT * 2);
#pragma unroll
            for (int j = 0; j < 4; ++j) {
                glds16(voffB[j], b, dB + so + j * 1024);
                glds16(voffA[2 * j], a, dA + so + 2 * j * 1024);
                glds16(voffA[2 * j + 1], a, dA + so + (2 * j + 1) * 1024);
            }
        };
        if (hi) {
            const char* b = gb + (unsigned)__builtin_amdgcn_readlane(ordB, 0) * (BK * N_TOT * 2);
#pragma unroll
            for (int j = 0; j < 4; ++j) glds16(voffB[j], b, dB + j * 1024);
        } else {
            issue(0, 0);
        }
        issue(1, STAGE_BYTES);
        unsigned so = 2 * STAGE_BYTES;
#define TLS(k)
        for (int k = 0; k < NSTEP - 2; ++k) {
            WAIT_VM(12);
            BARRIER();
            TLS(k);
            issue(k + 2, so);
            so = (so == 2 * STAGE_BYTES) ? 0 : so + STAGE_BYTES;
        }
        WAIT_VM(12); BARRIER();
        TLS(14);
        if (hi) {
            const char* a = ga + (unsigned)__builtin_amdgcn_readlane(ordA, 16) * (BK * 2);
#pragma unroll
            for (int j = 0; j < 8; ++j) glds16(voffA[j], a, dA + STAGE_BYTES + j * 1024);
        }
        const unsigned call = __builtin_amdgcn_readfirstlane(*(const unsigned*)(lds + LDS_CALL));
        const int wm = wave & 1;
        const unsigned par = call & 1u;
        const unsigned xv = wm * 8192 + lane * 16;
        char* const xge = xg_entry(p, lane, xcd) + par * 8192;
        const int orow = rank * M_LOC + row0 + wm * 128 + ks * 32 + (lane & 15);
        char* const o = (char*)p.out[xcd] + (size_t)orow * N_LOC + tnl * BN + (lane >> 4) * 16;
        char* const o2 = o + 16 * N_LOC;
        char* const dflag = (char*)p.ws[xcd] + WS_DONE + (size_t)(rank * 32 + idx) * 128 + wm * 8;
        const char* const lg = my_ws + WS_LGF + xcd * 2048 + 1024 + (idx * 2 + wm) * 8;
        const char* const gsrc = my_ws + WS_XG + par * 8192 + xcd * 1024 + lane * 16;
        asm volatile("" :: "v"(xge), "v"(o), "v"(o2), "v"(dflag), "v"(lg), "v"(xv), "v"(gsrc));
        if (hi) WAIT_VM(8); else WAIT_VM(0);
        BARRIER();
        TLS(15);
        if (hi) {
            WAIT_VM(0);
            asm volatile("ds_write_b32 %0, %1" :: "v"(lds_base + LDS_L16 + (wave >> 1) * 4), "v"(1u) : "memory");
        }
        if (wave >= 2) {
            const __amdgpu_buffer_rsrc_t xr = __builtin_amdgcn_make_buffer_rsrc(my_ws + WS_XB, 0, NWG * XB_STRIDE, 0x00020000);
            TLS(16);
            u32x4 v[12];
            auto xld = [&](int b) {
#pragma unroll
                for (int q = 0; q < 4; ++q)
                    v[4 * b + q] = __builtin_amdgcn_raw_buffer_load_b128(xr, xv + (q >> 1) * 4096 + (q & 1) * 1024, (wg ^ ((3 - b) << 6)) * XB_STRIDE + b * 16384, 16);
            };
            f32x4 y[2][2][4];
            if (SPRIO) __builtin_amdgcn_s_setprio(SPRIO);
            bool bad = false;
            for (unsigned it = 0;; ++it) {
                u32x2 f;
                asm volatile("ds_read_b64 %0, %1\n\ts_waitcnt lgkmcnt(0)" : "=v"(f) : "v"(lds_base + LDS_HF + wm * 8) : "memory");
                if (__builtin_amdgcn_readfirstlane(f[0] & f[1])) break;
                if (it > 64 * POLL_LIMIT) { bad = true; break; }
                if (SPINS) __builtin_amdgcn_s_sleep(SPINS);
            }
            if (XSLEEP) __builtin_amdgcn_s_sleep(XSLEEP);
            xld(0); xld(1);
            if (XSLEEP1) __builtin_amdgcn_s_sleep(XSLEEP1);
            xld(2);
            for (unsigned it = 0;; ++it) {
                u32x2 f;
                asm volatile("ds_read_b64 %0, %1\n\ts_waitcnt lgkmcnt(0)" : "=v"(f) : "v"(lds_base + LDS_HF + wm * 8) : "memory");
                if (__builtin_amdgcn_readfirstlane(f[0] & f[1] & 2u)) break;
                if (it > 64 * POLL_LIMIT) { bad = true; break; }
            }
#pragma unroll
            for (int n = 0; n < 8; ++n)
                y[0][n >> 2][n & 3] = *(lds_f32x4_ptr)(uintptr_t)(lds_base + LDS_HO + xv + n * 1024);
            unsigned pend = 7;
            for (unsigned it = 0; pend; ++it) {
                unsigned got = 0;
#pragma unroll
                for (int b = 0; b < 3; ++b)
                    if (pend >> b & 1) {
                        const unsigned mism = (v[4 * b][0] ^ par) | (v[4 * b + 1][0] ^ par) | (v[4 * b + 2][0] ^ par) | (v[4 * b + 3][0] ^ par);
                        if (__all((mism & 1u) == 0u)) got |= 1u << b;
                        else {
                            if ((it & 255u) == 255u) asm volatile("buffer_wbl2 sc1\n\ts_waitcnt vmcnt(0)" ::: "memory");
                            asm volatile("" ::: "memory");
                            xld(b);
                        }
                    }
#pragma unroll
                for (int b = 0; b < 3; ++b)
                    if (got >> b & 1) {
#pragma unroll
                        for (int q = 0; q < 4; ++q)
                            add_granule(y[0][q >> 1][2 * (q & 1)], y[0][q >> 1][2 * (q & 1) + 1], v[4 * b + q]);
                    }
                pend &= ~got;
                if (it > POLL_LIMIT) { bad = true; break; }
            }
            float am = 0.f;
#pragma unroll
            for (int i = 0; i < 8; ++i)
#pragma unroll
                for (int e = 0; e < 4; ++e) am = fmaxf(am, fabsf(y[0][i >> 2][i & 3][e]));
            unsigned amb = wave_max_u32(__float_as_uint(am));
            if (bad) amb = 0xffffffffu;
            if (lane == 0) {
                const unsigned long long g = ((unsigned long long)call << 32) | amb;
                asm volatile("global_store_dwordx2 %0, %1, off\n\t"
                             "global_store_dwordx2 %2, %1, off sc1"
                             :: "v"(lg), "v"(g), "v"(lg + (WS_LGS - WS_LGF)) : "memory");
                asm volatile("ds_add_u32 %0, %1" :: "v"(lds_base + LDS_RED + 8), "v"(1u) : "memory");
            }
            STAMPT(7, 128);
            auto send = [&](auto hc, unsigned val) {
                constexpr int h = decltype(hc)::value;
                const float inv = inv_scale(val);
                char* const dst = h ? o2 : o;
                u32x4 q[2];
#pragma unroll
                for (int c = 0; c < 2; ++c) {
                    unsigned r[4];
#pragma unroll
                    for (int j = 0; j < 4; ++j) {
                        int w = __builtin_amdgcn_cvt_pk_fp8_f32(y[h][c][j][0] * inv, y[h][c][j][1] * inv, 0, false);
                        w = __builtin_amdgcn_cvt_pk_fp8_f32(y[h][c][j][2] * inv, y[h][c][j][3] * inv, w, true);
                        r[j] = val == 0xffffffffu ? 0x7f7f7f7fu : (unsigned)w;
                    }
                    tr4(r[0], r[1], r[2], r[3]);
                    q[c] = (u32x4){r[0], r[1], r[2], r[3]};
                }
                asm volatile("global_store_dwordx4 %0, %1, off sc0 sc1\n\t"
                             "global_store_dwordx4 %0, %2, off offset:64 sc0 sc1\n\ts_nop 1" :: "v"(dst), "v"(q[0]), "v"(q[1]) : "memory");
            };
            unsigned s0 = 0xfffffffeu, s1 = 0xfffffffdu, flagged = 0xfffffffeu, told = 0;
            bool have1 = false;
            unsigned ts = (unsigned)__builtin_amdgcn_s_memrealtime() - SGRACE;
            ts += (2u * SPERIOD + (unsigned)((idx * 2 + wm) * SPERIOD >> 6) - ts % SPERIOD) % SPERIOD;
            for (unsigned it = 0;; ++it) {
                u32x2 r, f;
                unsigned long long now;
                asm volatile("ds_read_b64 %0, %3\n\tds_read_b64 %1, %4\n\ts_memrealtime %2\n\ts_waitcnt lgkmcnt(0)"
                             : "=&v"(r), "=&v"(f), "=s"(now) : "v"(lds_base + LDS_RED), "v"(lds_base + LDS_HF + 16 + wm * 8) : "memory");
                unsigned pa = __builtin_amdgcn_readfirstlane(r[0]);
                const unsigned st = __builtin_amdgcn_readfirstlane(r[1]);
                bool fin = st == call;
                if (it > 64 * POLL_LIMIT) { fin = true; pa = 0xffffffffu; }
                if (!have1) {
                    const unsigned f0 = __builtin_amdgcn_readfirstlane(f[0]), f1 = __builtin_amdgcn_readfirstlane(f[1]);
                    if (f0 & f1 & 0x80000000u) {
                        const unsigned fm = max(f0, f1);
                        const unsigned a1 = fm == 0xffffffffu ? fm : fm & 0x7fffffffu;
                        if (pa != 0u) told = max(told, a1);
                        amb = max(amb, a1);
                        have1 = true;
                        if (lane == 0) asm volatile("ds_add_u32 %0, %1" :: "v"(lds_base + LDS_RED + 8), "v"(16u) : "memory");
#pragma unroll
                        for (int n = 0; n < 8; ++n)
                            y[1][n >> 2][n & 3] = *(lds_f32x4_ptr)(uintptr_t)(lds_base + LDS_HO + 16384 + xv + n * 1024);
                    }
                }
                if (!have1 && pa != 0xffffffffu) fin = false;
                if (RECORD && pa != 0u && amb > max(pa, told)) {
                    told = amb;
                    asm volatile("global_store_dwordx2 %0, %1, off offset:8 sc0 sc1" :: "v"(xge), "v"(((unsigned long long)call << 32) | amb) : "memory");
                }
                const unsigned val = fin ? pa : max(pa, amb);
                bool go = fin;
                if (SPEC && (int)((unsigned)now - ts) >= 0) {
                    go = go || (st == ~call && pa != 0u && (RECORD || pa >= amb));
                    if ((int)((unsigned)now - ts) >= SGRACE || (go && (s0 != val || (s1 != val && have1)))) ts += SPERIOD;
                }
                if (go && s0 != val) {
                    send(IntC<0>{}, val);
                    s0 = val;
                    go = fin;
                }
                if (go && s1 != val && (have1 || fin)) {
                    send(IntC<1>{}, val);
                    s1 = val;
                }
                if (s0 == s1 && s0 != flagged) {
                    flagged = s0;
                    asm volatile("s_waitcnt vmcnt(0)" ::: "memory");
                    STAMPT(4, 128);
                    if (lane == 0 && xcd != rank)
                        asm volatile("global_store_dwordx2 %0, %1, off sc0 sc1"
                                     :: "v"(dflag), "v"(((unsigned long long)call << 32) | s0) : "memory");
                }
                if (fin) break;
            }
            return;
        }
        if (wave == 0) {
            u32x4 gv;
            unsigned best = 0;
            auto gstate = [&]() {
                const bool ok = gv[1] == call, started = gv[3] == call;
                const unsigned long long seen = __ballot(started);
                best = max(best, max(ok ? gv[0] : 0u, started ? gv[2] : 0u));
                const unsigned gmax = wave_max_u32(best);
                const unsigned st = __all(ok) ? call : ((seen >> (8 * xcd)) & 0xffu) ? ~call : 0u;
                asm volatile("ds_write_b64 %0, %1" :: "v"(lds_base + LDS_RED), "v"(((unsigned long long)st << 32) | gmax) : "memory");
                return st == call;
            };
            if (idx != SWIDX || !SWQUIET) {
                asm volatile("global_load_dwordx4 %0, %1, off sc0 sc1\n\ts_waitcnt vmcnt(0)" : "=&v"(gv) : "v"(gsrc) : "memory");
                gstate();
            }
            if (idx == SWIDX) {
                const char* lgs = my_ws + WS_LGF + xcd * 2048 + lane * 16;
                const char* lgt = my_ws + WS_LGF + xcd * 2048 + 1024 + lane * 8;
                unsigned m0 = 0, m1 = 0, m2 = 0;
                bool ok0 = false, ok1 = false, ok2 = false;
                unsigned it = 0;
                for (;;) {
                    const unsigned alt = ((it & 63u) == 63u) ? (WS_LGS - WS_LGF) : 0;
                    u32x4 v;
                    u32x2 u;
                    asm volatile("global_load_dwordx4 %0, %2, off sc1\n\tglobal_load_dwordx2 %1, %3, off sc1\n\ts_waitcnt vmcnt(0)"
                                 : "=&v"(v), "=&v"(u) : "v"(lgs + alt), "v"(lgt + alt) : "memory");
                    if (v[1] == call) { m0 = v[0]; ok0 = true; }
                    if (v[3] == call) { m1 = v[2]; ok1 = true; }
                    if (u[1] == call) { m2 = u[0]; ok2 = true; }
                    if (__all(ok0 && ok1 && ok2)) break;
                    if (++it > 4 * POLL_LIMIT) { m0 = 0xffffffffu; break; }
                }
                const unsigned m = wave_max_u32(max(max(m0, m1), m2));
                asm volatile("global_store_dwordx2 %0, %1, off sc0 sc1"
                             :: "v"(xge), "v"(((unsigned long long)call << 32) | m) : "memory");
                STAMPT(6, 0);
            }
            for (unsigned it = 0; POLLGATE && it < 64 * POLL_LIMIT; ++it) {
                unsigned c;
                asm volatile("ds_read_b32 %0, %1\n\ts_waitcnt lgkmcnt(0)" : "=v"(c) : "v"(lds_base + LDS_RED + 8) : "memory");
                c = __builtin_amdgcn_readfirstlane(c);
                if (POLLGATE == 1 ? (c & 15u) == 2u : c == 34u) break;
                if (SPINS) __builtin_amdgcn_s_sleep(2);
            }
            for (unsigned it = 0;;) {
                asm volatile("global_load_dwordx4 %0, %1, off sc0 sc1\n\ts_waitcnt vmcnt(0)"
                             : "=&v"(gv) : "v"(gsrc) : "memory");
                if (++it > 4 * POLL_LIMIT) { gv[0] = 0xffffffffu; gv[1] = call; }
                if (gstate()) break;
                if ((it & 255u) == 0u) asm volatile("buffer_wbl2 sc1\n\ts_waitcnt vmcnt(0)" ::: "memory");
            }
            return;
        }
        if ((wg >> 5) == rank) return;
        const unsigned long long want = ((unsigned long long)call << 32) | wait_amax(lds_base, call);
        __builtin_amdgcn_s_sleep(FSLEEP);
        const char* f = my_ws + WS_DONE + (size_t)wg * 128 + (lane & 1) * 8;
        for (unsigned it = 0;; ++it) {
            unsigned long long v;
            asm volatile("global_load_dwordx2 %0, %1, off sc0 sc1\n\ts_waitcnt vmcnt(0)" : "=&v"(v) : "v"(f) : "memory");
            if (__all(v == want)) break;
            if (it > POLL_LIMIT) {
                if (lane == 0)
                    __hip_atomic_store((unsigned char*)p.out[rank] + (size_t)rank * M_LOC * N_LOC,
                                       (unsigned char)0x7f, __ATOMIC_RELAXED, __HIP_MEMORY_SCOPE_SYSTEM);
                break;
            }
        }
        STAMPT(5, 64);
        return;
    }
    const int cw = wave - 4;
    const int wm = cw >> 1, wn = cw & 1;
    unsigned call;
    asm volatile("global_load_dword %0, %1, off sc1" : "=&v"(call) : "v"(ctr) : "memory");
    char* xge = xg_entry(p, lane, xcd);
    {
        u32x4 ra[4];
#pragma unroll
        for (int i = 0; i < 4; ++i)
#pragma unroll
            for (int e = 0; e < 4; ++e) ra[i][e] = (((unsigned)tid * 2654435761u + (i * 4 + e) * 0x9e3779b9u) * 0x85ebca6bu >> 3 & 0x80ff80ffu) | 0x3f003f00u;
        f32x4 d[4] = {};
        const int rounds = PREHEAT;
#pragma unroll 1
        for (int r = 0; r < rounds; ++r) {
#pragma unroll
            for (int i = 0; i < 8; ++i) MFMA(d[i & 3], ra[i & 3], ra[(i + 1 + (i >> 2)) & 3]);
        }
        asm volatile("" : "+v"(xge) : "v"(d[0]), "v"(d[1]), "v"(d[2]), "v"(d[3]));
    }
    asm volatile("s_waitcnt vmcnt(0)" : "+v"(call) :: "memory");
    call = __builtin_amdgcn_readfirstlane(call) + 1u;
    if (tid == 256) {
        *(u32x4*)(lds + LDS_CALL) = (u32x4){call, 0u, 0u, 0u};
        *(u32x4*)(lds + LDS_RED) = (u32x4){0u, 0u, 0u, 0u};
        *(u32x4*)(lds + LDS_HF) = (u32x4){0u, 0u, 0u, 0u};
        *(u32x4*)(lds + LDS_HF + 16) = (u32x4){0u, 0u, 0u, 0u};
    }
    WAIT_LGKM0();
    const unsigned par = call & 1u;
    xge += par * 8192;
    asm volatile("" : "+v"(xge));
    if (cw == 0 && idx == SWIDX) {
        asm volatile("global_store_dwordx2 %0, %1, off offset:8 sc0 sc1" :: "v"(xge), "v"((unsigned long long)call << 32) : "memory");
    }
    unsigned rdA[2], rdB[4];
#pragma unroll
    for (int kk = 0; kk < 2; ++kk) {
        const int r = lane & 15;
        const int c = 4 * kk + (lane >> 4);
        rdA[kk] = lds_base + (unsigned)(wm * 16384 + r * 128 + ((c ^ ((r >> 1) & 7)) * 16));
    }
#pragma unroll
    for (int cn = 0; cn < 4; ++cn) {
        const int g = lane >> 4, q = (lane & 15) >> 2, pp = lane & 3;
        const int k = 8 * g + q;
        const int h = (k & 3) | (((k >> 3) & 1) << 2);
        const int u = (wn * 4 + cn) ^ h;
        rdB[cn] = lds_base + (unsigned)(A_BYTES + k * 256 + u * 32 + pp * 8);
    }
    f32x4 acc[8][4];
#pragma unroll
    for (int i = 0; i < 8; ++i) {
#pragma unroll
        for (int j = 0; j < 4; ++j) acc[i][j] = (f32x4){0.f, 0.f, 0.f, 0.f};
        asm volatile("s_nop 1" : "+v"(acc[i][0]), "+v"(acc[i][1]), "+v"(acc[i][2]), "+v"(acc[i][3]));
    }
    s16x8 aq[2][2];
    s16x4 bl[2][4][2], bh[2][4][2];
    const unsigned xv = cw * 4096 + lane * 16;
    const __amdgpu_buffer_rsrc_t xr = __builtin_amdgcn_make_buffer_rsrc(my_ws + WS_XB, 0, NWG * XB_STRIDE, 0x00020000);
    const unsigned xo = wg * XB_STRIDE;
    unsigned tmask = 0xfffe0000u;
    const unsigned tc1 = (1u - par) << 16;
    asm volatile("" : "+v"(tmask) : "v"(xv));

#define BFR(s, n, kk) __builtin_shufflevector(bl[s][n][kk], bh[s][n][kk], 0, 1, 2, 3, 4, 5, 6, 7)
#define RDA(kk, mt) (*(lds_s16x8_ptr)(uintptr_t)(rdA[kk] + (mt) * 2048))
#define RDB(kk, n, h) __builtin_amdgcn_ds_read_tr16_b64_v4i16((lds_s16x4_ptr)(uintptr_t)(rdB[n] + (kk) * 8192 + (h) * 1024))
    auto rdb = [&](auto sc, int j) {
        constexpr int s = decltype(sc)::value;
        const int n = j >> 2, kk = ((j >> 1) & 1) ^ (n & 1);
        if (j & 1) bh[s][n][kk] = RDB(kk, n, 1); else bl[s][n][kk] = RDB(kk, n, 0);
    };
    auto grp = [&](auto mtc, auto abc, auto bsc, int na, auto&& fill) {
        constexpr int mt = decltype(mtc)::value, ab = decltype(abc)::value, bs = decltype(bsc)::value;
#pragma unroll
        for (int i = 0; i < 8; ++i) {
            const int nt = i >> 1, kk = (i & 1) ^ (nt & 1);
            MFMA(acc[mt][nt], BFR(bs, nt, kk), aq[ab][kk]);
            SB();
            if (i < 2 && na >= 0) aq[1 - ab][i] = RDA(i, na);
            fill(i);
            SB();
        }
    };
#define I(v) IntC<v>{}
    auto none = [](int) {};
    unsigned slot = 0, sd = 0;
    auto next_slot = [&](int i) {
        if (i == 0) { sd = slot == NS - 1 ? (unsigned)(-(NS - 1) * STAGE_BYTES) : (unsigned)STAGE_BYTES; slot = slot == NS - 1 ? 0 : slot + 1; }
        else if (i < 5) rdB[i - 1] += sd;
        else rdA[i - 5] += sd;
    };
    auto interval = [&](auto sc, int k) {
        constexpr int s = decltype(sc)::value;
        BARRIER();
        SB();
        grp(I(3), I(1), I(1 - s), 4, [&](int i) { if (i >= 2 && i < 6) rdb(sc, i - 2); });
        grp(I(4), I(0), I(1 - s), 5, [&](int i) { if (i >= 2 && i < 6) rdb(sc, i + 2); });
        grp(I(5), I(1), I(1 - s), 6, [&](int i) { if (i >= 2 && i < 6) rdb(sc, i + 6); });
        grp(I(6), I(0), I(1 - s), 7, [&](int i) { if (i >= 2 && i < 6) rdb(sc, i + 10); });
        grp(I(7), I(1), I(1 - s), 0, none);
        grp(I(0), I(0), sc, 1, none);
        grp(I(1), I(1), sc, 2, [&](int i) { if (i >= 2 && i < 7) next_slot(i - 2); });
        grp(I(2), I(0), sc, 3, [&](int i) { if (i == 2 || i == 4) next_slot(4 + (i >> 1)); });
        WAIT_LGKM0();
        SB();
    };
    BARRIER();
    SB();
    aq[0][0] = RDA(0, 0);
    rdb(I(0), 0); rdb(I(0), 1);
    aq[0][1] = RDA(1, 0);
#pragma unroll
    for (int j = 2; j < 16; ++j) rdb(I(0), j);
    SB();
    grp(I(0), I(0), I(0), 1, none);
    grp(I(1), I(1), I(0), 2, [&](int i) { if (i >= 2 && i < 7) next_slot(i - 2); });
    grp(I(2), I(0), I(0), 3, [&](int i) { if (i == 2 || i == 4) next_slot(4 + (i >> 1)); });
    WAIT_LGKM0();
    SB();
    for (int k = 1; k < NSTEP - 1; k += 2) {
        interval(I(1), k);
        interval(I(0), k + 1);
    }
    u32x4 g;
    auto xst = [&](auto mtc, int i) {
        constexpr int mt = decltype(mtc)::value;
        const int nt = i >= 4 ? 2 : 0;
        const int j = i & 3;
        if (j == 1) {
            g[0] = (__float_as_uint(acc[mt][nt][0]) + tc1) & tmask;
            g[1] = pk_bf16(acc[mt][nt][2], acc[mt][nt][3]);
        } else if (j == 2) {
            g[0] = (pk_bf16(__uint_as_float(g[0]), acc[mt][nt][1]) & ~1u) | par;
        } else if (j == 3) {
            g[2] = pk_bf16(acc[mt][nt + 1][0], acc[mt][nt + 1][1]);
            g[3] = pk_bf16(acc[mt][nt + 1][2], acc[mt][nt + 1][3]);
            __builtin_amdgcn_raw_buffer_store_b128(g, xr, xv + ((mt >> 2) * 2 + (nt >> 1)) * 1024, xo + (mt & 3) * 16384, XST_AUX);
        }
    };
    u32x4 v[6];
    auto xld = [&](int b, int q) {
        v[2 * b + q] = __builtin_amdgcn_raw_buffer_load_b128(xr, xv + (2 + q) * 1024, (wg ^ ((3 - b) << 6)) * XB_STRIDE + b * 16384, 16);
    };
    const unsigned ho = lds_base + LDS_HO + xv;
    constexpr int S1 = STAGE_BYTES / 2048;
    BARRIER();
    SB();
    grp(I(3), I(1), I(0), 4, [&](int i) { rdb(I(1), i); });
    grp(I(4), I(0), I(0), 0, [&](int i) { rdb(I(1), i + 8); });
    grp(I(0), I(1), I(1), 1, none);
    grp(I(1), I(0), I(1), 2, [&](int i) { xst(I(0), i); });
    grp(I(2), I(1), I(1), 3, [&](int i) { xst(I(1), i); });
    unsigned l16;
    grp(I(3), I(0), I(1), 5, [&](int i) { xst(I(2), i); if (i == 4) l16 = *(lds_u32_ptr)(uintptr_t)(lds_base + LDS_L16 + wm * 4);
                                          if (i == PREF) *(lds_u32_ptr)(uintptr_t)(lds_base + LDS_HF + cw * 4) = 1u; });
    for (unsigned it = 0; __builtin_amdgcn_readfirstlane(l16) == 0u && it < 64 * POLL_LIMIT; ++it)
        l16 = *(lds_u32_ptr)(uintptr_t)(lds_base + LDS_L16 + wm * 4);
    SB();
    grp(I(5), I(1), I(0), S1 + 4, [&](int i) {
        if (i >= 1 && i < 5) *(lds_f32x4_ptr)(uintptr_t)(ho + (i - 1) * 1024) = acc[3][i - 1];
        if (i == 5) *(lds_u32_ptr)(uintptr_t)(lds_base + LDS_HF + cw * 4) = 3u;
    });
    grp(I(4), I(0), I(1), S1 + 5, none);
    grp(I(5), I(1), I(1), 6, [&](int i) { xst(I(4), i); });
    grp(I(6), I(0), I(0), S1 + 6, [&](int i) { xst(I(5), i); });
    grp(I(6), I(1), I(1), 7, none);
    grp(I(7), I(0), I(0), S1 + 7, [&](int i) { xst(I(6), i); });
    grp(I(7), I(1), I(1), -1, [&](int i) { if (i >= 4) xld((i - 4) >> 1, i & 1); });
    if (XSLEEP2) __builtin_amdgcn_s_sleep(XSLEEP2);
    xld(2, 0); xld(2, 1);
    asm volatile("s_nop 7" : "+v"(acc[7][0]), "+v"(acc[7][1]), "+v"(acc[7][2]), "+v"(acc[7][3]));
    SB();
    STAMP(1);
    bool bad = false;
#pragma unroll
    for (int b = 0; b < 3; ++b) {
        for (unsigned it = 0;;) {
            const unsigned mism = (v[2 * b][0] ^ par) | (v[2 * b + 1][0] ^ par);
            if (__all((mism & 1u) == 0u)) break;
            if (++it > POLL_LIMIT) { bad = true; break; }
            if ((it & 255u) == 0u) asm volatile("buffer_wbl2 sc1\n\ts_waitcnt vmcnt(0)" ::: "memory");
            asm volatile("" ::: "memory");
            xld(b, 0); xld(b, 1);
        }
#pragma unroll
        for (int q = 0; q < 2; ++q) add_granule(acc[7][2 * q], acc[7][2 * q + 1], v[2 * b + q]);
    }
#pragma unroll
    for (int n = 0; n < 4; ++n) *(lds_f32x4_ptr)(uintptr_t)(ho + 16384 + n * 1024) = acc[7][n];
    float am = 0.f;
#pragma unroll
    for (int i = 0; i < 4; ++i)
#pragma unroll
        for (int e = 0; e < 4; ++e) am = fmaxf(am, fabsf(acc[7][i][e]));
    const unsigned pa = __builtin_amdgcn_readfirstlane(*(lds_u32_ptr)(uintptr_t)(lds_base + LDS_RED));
    unsigned amb = wave_max_u32(__float_as_uint(am));
    if (bad) amb = 0xffffffffu;
    {
        const unsigned long long g = ((unsigned long long)call << 32) | amb;
        if (RECORD && pa != 0u && amb > pa)
            asm volatile("global_store_dwordx2 %0, %1, off offset:8 sc0 sc1" :: "v"(xge), "v"(g) : "memory");
        if (lane == 0) {
            const char* lg = my_ws + WS_LGF + xcd * 2048 + (idx * 4 + cw) * 8;
            asm volatile("global_store_dwordx2 %0, %1, off\n\t"
                         "global_store_dwordx2 %2, %1, off sc1"
                         :: "v"(lg), "v"(g), "v"(lg + (WS_LGS - WS_LGF)) : "memory");
        }
    }
    asm volatile("" ::: "memory");
    *(lds_u32_ptr)(uintptr_t)(lds_base + LDS_HF + 16 + cw * 4) = amb | 0x80000000u;
    STAMP(2);
    if (tid == 256) __hip_atomic_store(ctr, call, __ATOMIC_RELAXED, __HIP_MEMORY_SCOPE_AGENT);
    if (WBMODE != 2 && cw == 2 && idx == 1) {
        wait_amax(lds_base, call);
        __builtin_amdgcn_s_sleep(WBDELAY);
        asm volatile("buffer_wbl2 sc1" ::: "memory");
    }
}

}

#define PACE 0, 0, -1, 0, 0, 0, 0, 0
static const int g_pace[8] = {PACE};
extern "C" void kernel_launch(int rank, int nranks, ncclComm_t comm,
                              hipStream_t stream,
                              void* const* d_in, const int* in_sizes, int n_in,
                              void* d_out, int out_size,
                              const int* in_dtypes, int out_dtype,
                              void* const* d_out_all,
                              void* const* d_ws_all, size_t ws_bytes) {
    (void)comm; (void)in_dtypes; (void)out_dtype; (void)d_out;
    if (nranks != NR || n_in < 2 || (unsigned)rank >= (unsigned)NR || ws_bytes < WS_END ||
        in_sizes[0] != M_LOC * K_TOT || in_sizes[1] != K_TOT * N_TOT || out_size != N_TOT * N_LOC) {
        fprintf(stderr, "kernel_launch: unexpected problem shape\n");
        return;
    }
    Params p;
    p.x = d_in[0];
    p.w = d_in[1];
    for (int i = 0; i < NR; ++i) {
        p.out[i] = d_out_all[i];
        p.ws[i] = d_ws_all[i];
    }
    p.rank = rank;
    p.pad = 0;
    p.my_ws = d_ws_all[rank];
    for (int i = 0; i < 8; ++i) p.pace[i] = g_pace[i];
    const int sg[10] = {SG};
    {
        int seq[10], ns = 0;
        for (int d = sg[0]; d > 0; d /= 10) seq[ns++] = d % 10 - 1;
        for (int w = 0; w < NWG; ++w) {
            const int xcd = w & 7, tnl = (w >> 3) & 3, tm = (w >> 5) & 1, ks = w >> 6;
            int key[16];
            for (int k = 0; k < 16; ++k) {
                const int ph = k + sg[2] * ks + sg[3] * xcd;
                const int mine = seq[(ph + sg[1] * tm) % ns], other = seq[(ph + sg[1] * (1 - tm)) % ns];
                int lead = 0;
                if (mine == tnl && other == tnl) lead = ((ph / ns + tm) & 1) ? sg[4] : sg[6];
                else if (mine == tnl) lead = sg[4];
                else if (other == tnl) lead = 0;
                else lead = (((ph + tnl * sg[7]) >> sg[8]) + tm + sg[9]) & 1 ? sg[5] : 0;
                key[k] = (4 * k - 4 * lead - (lead ? 2 : 0)) * 16 + k;
            }
            unsigned long long t = 0;
            for (int k = 0; k < 16; ++k) {
                int pos = 0;
                for (int q = 0; q < 16; ++q) pos += key[q] < key[k];
                t |= (unsigned long long)k << (4 * pos);
            }
            p.ord[w] = t;
        }
    }
    hipLaunchKernelGGL(gemm_a2a_kernel, dim3(NWG), dim3(NTHREADS), 0, stream, p);
    HIP_CHECK(hipGetLastError());
}
```

```cpp
#include <hip/hip_runtime.h>
#include <rccl/rccl.h>
#include <cstdio>
#include <cstdint>

#define HIP_CHECK(expr) do { hipError_t _e = (expr); if (_e != hipSuccess) { \
    fprintf(stderr, "HIP error %s at %s:%d\n", hipGetErrorString(_e), __FILE__, __LINE__); \
    return; } } while (0)

#define SWAPX 0
#define TLW 2
#define SWIDX 0
#define PREHEAT 12
#define SPEC 1
#define SPERIOD 256
#define WBMODE 3
#define XST_AUX 0
#define WBDELAY 48
#define POLLGATE 1
#define RECORD 1
#define SGRACE 0
#define XSLEEP 0
#define XSLEEP1 0
#define XSLEEP2 0
#define SPRIO 0
#define SWQUIET 1
#define SPINS 0
#define LPRIO 0
#define LATEHEAT 1
#define PREF 7
#define FSLEEP 16
#define SG 4312431, 3, 2, 1, 2, 1, 5, 0, 0, 0
namespace {

typedef __attribute__((ext_vector_type(4))) short s16x4;
typedef __attribute__((ext_vector_type(8))) short s16x8;
typedef __attribute__((ext_vector_type(2))) __bf16 bf16x2;
typedef __attribute__((ext_vector_type(2))) float f32x2;
typedef __attribute__((ext_vector_type(4))) float f32x4;
typedef __attribute__((ext_vector_type(2))) unsigned u32x2;
typedef __attribute__((ext_vector_type(4))) unsigned u32x4;
typedef __attribute__((address_space(3))) s16x4* lds_s16x4_ptr;
typedef __attribute__((address_space(3))) s16x8* lds_s16x8_ptr;
typedef __attribute__((address_space(3))) f32x4* lds_f32x4_ptr;
typedef __attribute__((address_space(3))) volatile unsigned* lds_u32_ptr;

constexpr int NR = 8;
constexpr int M_LOC = 512;
constexpr int K_TOT = 4096;
constexpr int N_TOT = 4096;
constexpr int N_LOC = N_TOT / NR;
constexpr int BM = 256, BN = 128, BK = 64;
constexpr int NS = 3;
constexpr int A_BYTES = BM * BK * 2;
constexpr int B_BYTES = BK * BN * 2;
constexpr int STAGE_BYTES = A_BYTES + B_BYTES;
constexpr int NSTEP = 16;
constexpr int NWG = 256;
constexpr int NTHREADS = 512;
constexpr int LDS_CALL = NS * STAGE_BYTES;
constexpr int LDS_RED = LDS_CALL + 16;
constexpr int LDS_HF = LDS_RED + 16;
constexpr int LDS_RDY = LDS_CALL + 4;
constexpr int LDS_L16 = LDS_CALL + 8;
constexpr int LDS_BYTES = LDS_HF + 32;
constexpr int LDS_HO = 2 * STAGE_BYTES;

constexpr size_t WS_XG = 0;
constexpr size_t WS_LGF = WS_XG + 2 * 8192;
constexpr size_t WS_LGS = WS_LGF + 16384;
constexpr size_t WS_DONE = WS_LGS + 16384;
constexpr size_t WS_CTR = WS_DONE + NWG * 128;
constexpr size_t WS_DIAG = WS_CTR + NWG * 128;
constexpr size_t WS_XB = 256 * 1024;
#define XB_STRIDE 49408
constexpr size_t WS_END = WS_XB + (size_t)NWG * XB_STRIDE;
static_assert(WS_DIAG + NWG * 16 * 8 <= WS_XB, "");

constexpr unsigned POLL_LIMIT = 1u << 20;

struct Params {
    const void* x;
    const void* w;
    void* my_ws;
    int rank;
    int pad;
    int pace[8];
    void* out[NR];
    void* ws[NR];
    unsigned long long ord[NWG];
};

__device__ __forceinline__ void glds16(unsigned voff, const void* sbase, unsigned lds_dst) {
    asm volatile("s_mov_b32 m0, %2\n\ts_nop 0\n\tglobal_load_lds_dwordx4 %0, %1"
                 :: "v"(voff), "s"(sbase), "s"(lds_dst) : "memory");
}
#define WAIT_VM(n) asm volatile("s_waitcnt vmcnt(%0)" :: "i"(n) : "memory")
#define WAIT_LGKM0() asm volatile("s_waitcnt lgkmcnt(0)" ::: "memory")
#define BARRIER() asm volatile("s_barrier" ::: "memory")
#define SB() __builtin_amdgcn_sched_barrier(0)
#define MFMA(acc, a, b) asm volatile("v_mfma_f32_16x16x32_bf16 %0, %1, %2, %0" : "+v"(acc) : "v"(a), "v"(b))
#define MFMA0(acc, a, b) asm volatile("v_mfma_f32_16x16x32_bf16 %0, %1, %2, 0" : "+v"(acc) : "v"(a), "v"(b))

__device__ __forceinline__ unsigned wave_max_u32(unsigned v) {
    v = max(v, (unsigned)__builtin_amdgcn_update_dpp(0, (int)v, 0x111, 0xf, 0xf, false));
    v = max(v, (unsigned)__builtin_amdgcn_update_dpp(0, (int)v, 0x112, 0xf, 0xf, false));
    v = max(v, (unsigned)__builtin_amdgcn_update_dpp(0, (int)v, 0x114, 0xf, 0xf, false));
    v = max(v, (unsigned)__builtin_amdgcn_update_dpp(0, (int)v, 0x118, 0xf, 0xf, false));
    v = max(v, (unsigned)__builtin_amdgcn_update_dpp(0, (int)v, 0x142, 0xa, 0xf, false));
    v = max(v, (unsigned)__builtin_amdgcn_update_dpp(0, (int)v, 0x143, 0xc, 0xf, false));
    return (unsigned)__builtin_amdgcn_readlane((int)v, 63);
}

__device__ __forceinline__ unsigned pk_bf16(float a, float b) {
    const bf16x2 v = __builtin_convertvector((f32x2){a, b}, bf16x2);
    return __builtin_bit_cast(unsigned, v);
}
__device__ __forceinline__ unsigned pk_bf16_tag(float a, float b, unsigned tag) {
    const unsigned t = (__float_as_uint(a) + ((1u - tag) << 16)) & 0xfffe0000u;
    return (pk_bf16(__uint_as_float(t), b) & ~1u) | tag;
}
__device__ __forceinline__ u32x4 pack_granule(const f32x4& a, const f32x4& b, unsigned tag) {
    u32x4 g;
    g[0] = pk_bf16_tag(a[0], a[1], tag);
    g[1] = pk_bf16(a[2], a[3]);
    g[2] = pk_bf16(b[0], b[1]);
    g[3] = pk_bf16(b[2], b[3]);
    return g;
}
__device__ __forceinline__ float add_lo(unsigned w, float c) {
    unsigned one_lo = 0x00003f80u;
    asm("" : "+s"(one_lo));
    return __builtin_amdgcn_fdot2_f32_bf16(__builtin_bit_cast(bf16x2, w), __builtin_bit_cast(bf16x2, one_lo), c, false);
}
__device__ __forceinline__ float add_hi(unsigned w, float c) {
    return __builtin_amdgcn_fdot2_f32_bf16(__builtin_bit_cast(bf16x2, w), __builtin_bit_cast(bf16x2, 0x3f800000u), c, false);
}
__device__ __forceinline__ void add_granule(f32x4& a, f32x4& b, const u32x4& g) {
    a[0] = add_lo(g[0], a[0]); a[1] = add_hi(g[0], a[1]); a[2] = add_lo(g[1], a[2]); a[3] = add_hi(g[1], a[3]);
    b[0] = add_lo(g[2], b[0]); b[1] = add_hi(g[2], b[1]); b[2] = add_lo(g[3], b[2]); b[3] = add_hi(g[3], b[3]);
}
__device__ __forceinline__ void tr4(unsigned& r0, unsigned& r1, unsigned& r2, unsigned& r3) {
    u32x2 t;
    t = __builtin_amdgcn_permlane32_swap(r0, r2, false, false); r0 = t[0]; r2 = t[1];
    t = __builtin_amdgcn_permlane32_swap(r1, r3, false, false); r1 = t[0]; r3 = t[1];
    t = __builtin_amdgcn_permlane16_swap(r0, r1, false, false); r0 = t[0]; r1 = t[1];
    t = __builtin_amdgcn_permlane16_swap(r2, r3, false, false); r2 = t[0]; r3 = t[1];
}

#define NCYC 2
#define STAMPT(i, t) do {} while (0)
#define STAMP(i) STAMPT(i, 256)

__device__ __forceinline__ unsigned wait_amax(unsigned lds_base, unsigned call) {
    for (unsigned it = 0;; ++it) {
        u32x2 r;
        asm volatile("ds_read_b64 %0, %1\n\ts_waitcnt lgkmcnt(0)" : "=v"(r) : "v"(lds_base + LDS_RED) : "memory");
        if (__builtin_amdgcn_readfirstlane(r[1]) == call) return __builtin_amdgcn_readfirstlane(r[0]);
        if (it > 64 * POLL_LIMIT) return 0xffffffffu;
        if (SPINS) __builtin_amdgcn_s_sleep(4);
    }
}
__device__ __forceinline__ float inv_scale(unsigned gmax) {
    const float scale = __uint_as_float(gmax) / 448.0f;
    return scale > 0.f ? 1.0f / scale : 0.f;
}

__device__ __forceinline__ char* xg_entry(const Params& p, int lane, int g) {
    char* b = (char*)p.ws[0];
#pragma unroll
    for (int i = 1; i < NR; ++i) {
        char* w = (char*)p.ws[i];
        asm("" : "+s"(w));
        b = (lane >> 3) == i ? w : b;
    }
    return b + WS_XG + (lane & 7) * 1024 + (p.rank * 8 + g) * 16;
}

template <int V> struct IntC { static constexpr int value = V; };

__global__ void __launch_bounds__(NTHREADS, 2) gemm_a2a_kernel(const Params p) {
    __shared__ __attribute__((aligned(1024))) char lds[LDS_BYTES];
    const int tid = threadIdx.x;
    const int lane = tid & 63;
    const int wave = __builtin_amdgcn_readfirstlane(tid >> 6);
    const int wg = blockIdx.x;
    const int rank = p.rank;
    char* const my_ws = (char*)p.my_ws;
    const char* const gx = (const char*)p.x;
    const char* const gw = (const char*)p.w;
    const unsigned long long otab = p.ord[wg];

    const int xcd = wg & 7, idx = wg >> 3;
    const int tnl = idx & 3, tm = (idx >> 2) & 1, ks = idx >> 3;
    const int row0 = tm * BM;
    const int col0 = xcd * N_LOC + tnl * BN;

    const int of = 0;
    const int hi = wave & 1;
    unsigned voffA[8], voffB[4];
    {
        unsigned ve = (unsigned)(((wave & 2) * 64 + hi * 16 + (lane >> 3)) * (K_TOT * 2) + (((lane & 7) ^ (lane >> 4)) * 16));
        asm("" : "+v"(ve));
        unsigned vo = (ve ^ 64u) + 8 * (K_TOT * 2);
        asm("" : "+v"(vo));
#pragma unroll
        for (int i = 0; i < 8; ++i) voffA[i] = (i & 1 ? vo : ve) | (unsigned)((((i >> 1) ^ 3 ^ ks) & 3) * 32 * (K_TOT * 2));
        unsigned vb = (unsigned)(((wave & 3) * 16 + (lane >> 4)) * (N_TOT * 2) + (((lane & 15) ^ (2 * (lane >> 4))) * 16));
        asm("" : "+v"(vb));
        voffB[0] = vb; voffB[1] = vb | 4 * (N_TOT * 2); voffB[2] = vb ^ (8 * (N_TOT * 2) + 128); voffB[3] = vb ^ (12 * (N_TOT * 2) + 128);
    }
    asm volatile("" : "+v"(voffA[0]), "+v"(voffA[1]), "+v"(voffA[2]), "+v"(voffA[3]), "+v"(voffA[4]), "+v"(voffA[5]), "+v"(voffA[6]), "+v"(voffA[7]),
                      "+v"(voffB[0]), "+v"(voffB[1]), "+v"(voffB[2]), "+v"(voffB[3]));
    unsigned offA = row0 * (K_TOT * 2) + ks * NSTEP * (BK * 2), offB = col0 * 2 + ks * NSTEP * (BK * N_TOT * 2);
    unsigned dA = (unsigned)(uintptr_t)lds + wave * 8192, dB = (unsigned)(uintptr_t)lds + A_BYTES + wave * 4096;
    asm volatile("" : "+s"(offA), "+s"(offB), "+s"(dA), "+s"(dB));
    asm volatile("" :: "s"(gx), "s"(gw), "s"(my_ws), "s"(rank), "s"(otab));
    auto ktile = [&](int s) {
        return ((int)(otab >> (4 * (s & 15))) + of) & 15;
    };
    unsigned* ctr = (unsigned*)(my_ws + WS_CTR + (size_t)wg * 128);
    const unsigned lds_base = (unsigned)(uintptr_t)lds;

    if (__builtin_expect(wave < 4, 1)) {
        if (LPRIO) __builtin_amdgcn_s_setprio(LPRIO);
        const char* const ga = gx + offA;
        const char* const gb = gw + offB;
        auto issue_at = [&](unsigned ka, unsigned kb, unsigned so) {
            const char* a = ga + ka * (BK * 2);
            const char* b = gb + kb * (BK * N_TOT * 2);
#pragma unroll
            for (int j = 0; j < 4; ++j) {
                glds16(voffB[j], b, dB + so + j * 1024);
                glds16(voffA[2 * j], a, dA + so + 2 * j * 1024);
                glds16(voffA[2 * j + 1], a, dA + so + (2 * j + 1) * 1024);
            }
        };
        {
            const char* b = gb + (unsigned)ktile(0) * (BK * N_TOT * 2);
#pragma unroll
            for (int j = 0; j < 4; ++j) glds16(voffB[j], b, dB + j * 1024);
        }
        if (!hi) {
            const char* a = ga + (unsigned)ktile(0) * (BK * 2);
#pragma unroll
            for (int j = 0; j < 8; ++j) glds16(voffA[j], a, dA + j * 1024);
        }
        issue_at(ktile(1 - hi), ktile(1), STAGE_BYTES);
        int ordB = ktile(lane), ordA = ktile(lane - hi);
        asm volatile("" : "+v"(ordA), "+v"(ordB));
        auto issue = [&](int u, unsigned so) { issue_at(__builtin_amdgcn_readlane(ordA, u), __builtin_amdgcn_readlane(ordB, u), so); };
        unsigned so = 2 * STAGE_BYTES;
#define TLS(k)
        for (int k = 0; k < NSTEP - 2; ++k) {
            WAIT_VM(12);
            BARRIER();
            TLS(k);
            issue(k + 2, so);
            so = (so == 2 * STAGE_BYTES) ? 0 : so + STAGE_BYTES;
        }
        WAIT_VM(12); BARRIER();
        TLS(14);
        if (hi) {
            const char* a = ga + (unsigned)__builtin_amdgcn_readlane(ordA, 16) * (BK * 2);
#pragma unroll
            for (int j = 0; j < 8; ++j) glds16(voffA[j], a, dA + STAGE_BYTES + j * 1024);
        }
        const unsigned call = __builtin_amdgcn_readfirstlane(*(const unsigned*)(lds + LDS_CALL));
        const int wm = wave & 1;
        const unsigned par = call & 1u;
        const unsigned xv = wm * 8192 + lane * 16;
        char* const xge = xg_entry(p, lane, xcd) + par * 8192;
        const int orow = rank * M_LOC + row0 + wm * 128 + ks * 32 + (lane & 15);
        char* const o = (char*)p.out[xcd] + (size_t)orow * N_LOC + tnl * BN + (lane >> 4) * 16;
        char* const o2 = o + 16 * N_LOC;
        char* const dflag = (char*)p.ws[xcd] + WS_DONE + (size_t)(rank * 32 + idx) * 128 + wm * 8;
        const char* const lg = my_ws + WS_LGF + xcd * 2048 + 1024 + (idx * 2 + wm) * 8;
        const char* const gsrc = my_ws + WS_XG + par * 8192 + xcd * 1024 + lane * 16;
        asm volatile("" :: "v"(xge), "v"(o), "v"(o2), "v"(dflag), "v"(lg), "v"(xv), "v"(gsrc));
        if (hi) WAIT_VM(8); else WAIT_VM(0);
        BARRIER();
        TLS(15);
        if (hi) {
            WAIT_VM(0);
            asm volatile("ds_write_b32 %0, %1" :: "v"(lds_base + LDS_L16 + (wave >> 1) * 4), "v"(1u) : "memory");
        }
        if (wave >= 2) {
            const __amdgpu_buffer_rsrc_t xr = __builtin_amdgcn_make_buffer_rsrc(my_ws + WS_XB, 0, NWG * XB_STRIDE, 0x00020000);
            TLS(16);
            u32x4 v[12];
            auto xld = [&](int b) {
#pragma unroll
                for (int q = 0; q < 4; ++q)
                    v[4 * b + q] = __builtin_amdgcn_raw_buffer_load_b128(xr, xv + (q >> 1) * 4096 + (q & 1) * 1024, (wg ^ ((3 - b) << 6)) * XB_STRIDE + b * 16384, 16);
            };
            f32x4 y[2][2][4];
            if (SPRIO) __builtin_amdgcn_s_setprio(SPRIO);
            bool bad = false;
            for (unsigned it = 0;; ++it) {
                u32x2 f;
                asm volatile("ds_read_b64 %0, %1\n\ts_waitcnt lgkmcnt(0)" : "=v"(f) : "v"(lds_base + LDS_HF + wm * 8) : "memory");
                if (__builtin_amdgcn_readfirstlane(f[0] & f[1])) break;
                if (it > 64 * POLL_LIMIT) { bad = true; break; }
                if (SPINS) __builtin_amdgcn_s_sleep(SPINS);
            }
            if (XSLEEP) __builtin_amdgcn_s_sleep(XSLEEP);
            xld(0); xld(1);
            if (XSLEEP1) __builtin_amdgcn_s_sleep(XSLEEP1);
            xld(2);
            for (unsigned it = 0;; ++it) {
                u32x2 f;
                asm volatile("ds_read_b64 %0, %1\n\ts_waitcnt lgkmcnt(0)" : "=v"(f) : "v"(lds_base + LDS_HF + wm * 8) : "memory");
                if (__builtin_amdgcn_readfirstlane(f[0] & f[1] & 2u)) break;
                if (it > 64 * POLL_LIMIT) { bad = true; break; }
            }
#pragma unroll
            for (int n = 0; n < 8; ++n)
                y[0][n >> 2][n & 3] = *(lds_f32x4_ptr)(uintptr_t)(lds_base + LDS_HO + xv + n * 1024);
            unsigned pend = 7;
            for (unsigned it = 0; pend; ++it) {
                unsigned got = 0;
#pragma unroll
                for (int b = 0; b < 3; ++b)
                    if (pend >> b & 1) {
                        const unsigned mism = (v[4 * b][0] ^ par) | (v[4 * b + 1][0] ^ par) | (v[4 * b + 2][0] ^ par) | (v[4 * b + 3][0] ^ par);
                        if (__all((mism & 1u) == 0u)) got |= 1u << b;
                        else {
                            if ((it & 255u) == 255u) asm volatile("buffer_wbl2 sc1\n\ts_waitcnt vmcnt(0)" ::: "memory");
                            asm volatile("" ::: "memory");
                            xld(b);
                        }
                    }
#pragma unroll
                for (int b = 0; b < 3; ++b)
                    if (got >> b & 1) {
#pragma unroll
                        for (int q = 0; q < 4; ++q)
                            add_granule(y[0][q >> 1][2 * (q & 1)], y[0][q >> 1][2 * (q & 1) + 1], v[4 * b + q]);
                    }
                pend &= ~got;
                if (it > POLL_LIMIT) { bad = true; break; }
            }
            float am = 0.f;
#pragma unroll
            for (int i = 0; i < 8; ++i)
#pragma unroll
                for (int e = 0; e < 4; ++e) am = fmaxf(am, fabsf(y[0][i >> 2][i & 3][e]));
            unsigned amb = wave_max_u32(__float_as_uint(am));
            if (bad) amb = 0xffffffffu;
            if (lane == 0) {
                const unsigned long long g = ((unsigned long long)call << 32) | amb;
                asm volatile("global_store_dwordx2 %0, %1, off\n\t"
                             "global_store_dwordx2 %2, %1, off sc1"
                             :: "v"(lg), "v"(g), "v"(lg + (WS_LGS - WS_LGF)) : "memory");
                asm volatile("ds_add_u32 %0, %1" :: "v"(lds_base + LDS_RED + 8), "v"(1u) : "memory");
            }
            STAMPT(7, 128);
            auto send = [&](auto hc, unsigned val) {
                constexpr int h = decltype(hc)::value;
                const float inv = inv_scale(val);
                char* const dst = h ? o2 : o;
                u32x4 q[2];
#pragma unroll
                for (int c = 0; c < 2; ++c) {
                    unsigned r[4];
#pragma unroll
                    for (int j = 0; j < 4; ++j) {
                        int w = __builtin_amdgcn_cvt_pk_fp8_f32(y[h][c][j][0] * inv, y[h][c][j][1] * inv, 0, false);
                        w = __builtin_amdgcn_cvt_pk_fp8_f32(y[h][c][j][2] * inv, y[h][c][j][3] * inv, w, true);
                        r[j] = val == 0xffffffffu ? 0x7f7f7f7fu : (unsigned)w;
                    }
                    tr4(r[0], r[1], r[2], r[3]);
                    q[c] = (u32x4){r[0], r[1], r[2], r[3]};
                }
                asm volatile("global_store_dwordx4 %0, %1, off sc0 sc1\n\t"
                             "global_store_dwordx4 %0, %2, off offset:64 sc0 sc1\n\ts_nop 1" :: "v"(dst), "v"(q[0]), "v"(q[1]) : "memory");
            };
            unsigned s0 = 0xfffffffeu, s1 = 0xfffffffdu, flagged = 0xfffffffeu, told = 0;
            bool have1 = false;
            unsigned ts = (unsigned)__builtin_amdgcn_s_memrealtime() - SGRACE;
            ts += (2u * SPERIOD + (unsigned)((idx * 2 + wm) * SPERIOD >> 6) - ts % SPERIOD) % SPERIOD;
            for (unsigned it = 0;; ++it) {
                u32x2 r, f;
                unsigned long long now;
                asm volatile("ds_read_b64 %0, %3\n\tds_read_b64 %1, %4\n\ts_memrealtime %2\n\ts_waitcnt lgkmcnt(0)"
                             : "=&v"(r), "=&v"(f), "=s"(now) : "v"(lds_base + LDS_RED), "v"(lds_base + LDS_HF + 16 + wm * 8) : "memory");
                unsigned pa = __builtin_amdgcn_readfirstlane(r[0]);
                const unsigned st = __builtin_amdgcn_readfirstlane(r[1]);
                bool fin = st == call;
                if (it > 64 * POLL_LIMIT) { fin = true; pa = 0xffffffffu; }
                if (!have1) {
                    const unsigned f0 = __builtin_amdgcn_readfirstlane(f[0]), f1 = __builtin_amdgcn_readfirstlane(f[1]);
                    if (f0 & f1 & 0x80000000u) {
                        const unsigned fm = max(f0, f1);
                        const unsigned a1 = fm == 0xffffffffu ? fm : fm & 0x7fffffffu;
                        if (pa != 0u) told = max(told, a1);
                        amb = max(amb, a1);
                        have1 = true;
                        if (lane == 0) asm volatile("ds_add_u32 %0, %1" :: "v"(lds_base + LDS_RED + 8), "v"(16u) : "memory");
#pragma unroll
                        for (int n = 0; n < 8; ++n)
                            y[1][n >> 2][n & 3] = *(lds_f32x4_ptr)(uintptr_t)(lds_base + LDS_HO + 16384 + xv + n * 1024);
                    }
                }
                if (!have1 && pa != 0xffffffffu) fin = false;
                if (RECORD && pa != 0u && amb > max(pa, told)) {
                    told = amb;
                    asm volatile("global_store_dwordx2 %0, %1, off offset:8 sc0 sc1" :: "v"(xge), "v"(((unsigned long long)call << 32) | amb) : "memory");
                }
                const unsigned val = fin ? pa : max(pa, amb);
                bool go = fin;
                if (SPEC && (int)((unsigned)now - ts) >= 0) {
                    go = go || (st == ~call && pa != 0u && (RECORD || pa >= amb));
                    if ((int)((unsigned)now - ts) >= SGRACE || (go && (s0 != val || (s1 != val && have1)))) ts += SPERIOD;
                }
                if (go && s0 != val) {
                    send(IntC<0>{}, val);
                    s0 = val;
                    go = fin;
                }
                if (go && s1 != val && (have1 || fin)) {
                    send(IntC<1>{}, val);
                    s1 = val;
                }
                if (s0 == s1 && s0 != flagged) {
                    flagged = s0;
                    asm volatile("s_waitcnt vmcnt(0)" ::: "memory");
                    STAMPT(4, 128);
                    if (lane == 0 && xcd != rank)
                        asm volatile("global_store_dwordx2 %0, %1, off sc0 sc1"
                                     :: "v"(dflag), "v"(((unsigned long long)call << 32) | s0) : "memory");
                }
                if (fin) break;
            }
            return;
        }
        if (wave == 0) {
            u32x4 gv;
            unsigned best = 0;
            auto gstate = [&]() {
                const bool ok = gv[1] == call, started = gv[3] == call;
                const unsigned long long seen = __ballot(started);
                best = max(best, max(ok ? gv[0] : 0u, started ? gv[2] : 0u));
                const unsigned gmax = wave_max_u32(best);
                const unsigned st = __all(ok) ? call : ((seen >> (8 * xcd)) & 0xffu) ? ~call : 0u;
                asm volatile("ds_write_b64 %0, %1" :: "v"(lds_base + LDS_RED), "v"(((unsigned long long)st << 32) | gmax) : "memory");
                return st == call;
            };
            if (idx != SWIDX || !SWQUIET) {
                asm volatile("global_load_dwordx4 %0, %1, off sc0 sc1\n\ts_waitcnt vmcnt(0)" : "=&v"(gv) : "v"(gsrc) : "memory");
                gstate();
            }
            if (idx == SWIDX) {
                const char* lgs = my_ws + WS_LGF + xcd * 2048 + lane * 16;
                const char* lgt = my_ws + WS_LGF + xcd * 2048 + 1024 + lane * 8;
                unsigned m0 = 0, m1 = 0, m2 = 0;
                bool ok0 = false, ok1 = false, ok2 = false;
                unsigned it = 0;
                for (;;) {
                    const unsigned alt = ((it & 63u) == 63u) ? (WS_LGS - WS_LGF) : 0;
                    u32x4 v;
                    u32x2 u;
                    asm volatile("global_load_dwordx4 %0, %2, off sc1\n\tglobal_load_dwordx2 %1, %3, off sc1\n\ts_waitcnt vmcnt(0)"
                                 : "=&v"(v), "=&v"(u) : "v"(lgs + alt), "v"(lgt + alt) : "memory");
                    if (v[1] == call) { m0 = v[0]; ok0 = true; }
                    if (v[3] == call) { m1 = v[2]; ok1 = true; }
                    if (u[1] == call) { m2 = u[0]; ok2 = true; }
                    if (__all(ok0 && ok1 && ok2)) break;
                    if (++it > 4 * POLL_LIMIT) { m0 = 0xffffffffu; break; }
                }
                const unsigned m = wave_max_u32(max(max(m0, m1), m2));
                asm volatile("global_store_dwordx2 %0, %1, off sc0 sc1"
                             :: "v"(xge), "v"(((unsigned long long)call << 32) | m) : "memory");
                STAMPT(6, 0);
            }
            for (unsigned it = 0; POLLGATE && it < 64 * POLL_LIMIT; ++it) {
                unsigned c;
                asm volatile("ds_read_b32 %0, %1\n\ts_waitcnt lgkmcnt(0)" : "=v"(c) : "v"(lds_base + LDS_RED + 8) : "memory");
                c = __builtin_amdgcn_readfirstlane(c);
                if (POLLGATE == 1 ? (c & 15u) == 2u : c == 34u) break;
                if (SPINS) __builtin_amdgcn_s_sleep(2);
            }
            for (unsigned it = 0;;) {
                asm volatile("global_load_dwordx4 %0, %1, off sc0 sc1\n\ts_waitcnt vmcnt(0)"
                             : "=&v"(gv) : "v"(gsrc) : "memory");
                if (++it > 4 * POLL_LIMIT) { gv[0] = 0xffffffffu; gv[1] = call; }
                if (gstate()) break;
                if ((it & 255u) == 0u) asm volatile("buffer_wbl2 sc1\n\ts_waitcnt vmcnt(0)" ::: "memory");
            }
            return;
        }
        if ((wg >> 5) == rank) return;
        const unsigned long long want = ((unsigned long long)call << 32) | wait_amax(lds_base, call);
        __builtin_amdgcn_s_sleep(FSLEEP);
        const char* f = my_ws + WS_DONE + (size_t)wg * 128 + (lane & 1) * 8;
        for (unsigned it = 0;; ++it) {
            unsigned long long v;
            asm volatile("global_load_dwordx2 %0, %1, off sc0 sc1\n\ts_waitcnt vmcnt(0)" : "=&v"(v) : "v"(f) : "memory");
            if (__all(v == want)) break;
            if (it > POLL_LIMIT) {
                if (lane == 0)
                    __hip_atomic_store((unsigned char*)p.out[rank] + (size_t)rank * M_LOC * N_LOC,
                                       (unsigned char)0x7f, __ATOMIC_RELAXED, __HIP_MEMORY_SCOPE_SYSTEM);
                break;
            }
        }
        STAMPT(5, 64);
        return;
    }
    const int cw = wave - 4;
    const int wm = cw >> 1, wn = cw & 1;
    unsigned call = *(const __attribute__((address_space(4))) unsigned*)(uintptr_t)ctr;
    char* xge = xg_entry(p, lane, xcd);
    unsigned rdA[2], rdB[4];
#pragma unroll
    for (int kk = 0; kk < 2; ++kk) {
        const int r = lane & 15;
        const int c = 4 * kk + (lane >> 4);
        rdA[kk] = lds_base + (unsigned)(wm * 16384 + r * 128 + ((c ^ ((r >> 1) & 7)) * 16));
    }
#pragma unroll
    for (int cn = 0; cn < 4; ++cn) {
        const int g = lane >> 4, q = (lane & 15) >> 2, pp = lane & 3;
        const int k = 8 * g + q;
        const int h = (k & 3) | (((k >> 3) & 1) << 2);
        const int u = (wn * 4 + cn) ^ h;
        rdB[cn] = lds_base + (unsigned)(A_BYTES + k * 256 + u * 32 + pp * 8);
    }
    f32x4 acc[8][4];
#pragma unroll
    for (int i = 0; i < 8; ++i) {
#pragma unroll
        for (int j = 0; j < 4; ++j) acc[i][j] = (f32x4){0.f, 0.f, 0.f, 0.f};
        asm volatile("s_nop 1" : "+v"(acc[i][0]), "+v"(acc[i][1]), "+v"(acc[i][2]), "+v"(acc[i][3]));
    }
    s16x8 aq[2][2];
    s16x4 bl[2][4][2], bh[2][4][2];
    const unsigned xv = cw * 4096 + lane * 16;
    const __amdgpu_buffer_rsrc_t xr = __builtin_amdgcn_make_buffer_rsrc(my_ws + WS_XB, 0, NWG * XB_STRIDE, 0x00020000);
    const unsigned xo = wg * XB_STRIDE;
    unsigned tmask = 0xfffe0000u;
    asm volatile("" : "+s"(call));
    ++call;
    if (tid < 256 + 4) *(u32x4*)(lds + LDS_CALL + lane * 16) = (u32x4){lane ? 0u : call, 0u, 0u, 0u};
    const unsigned par = call & 1u;
    const unsigned tc1 = (1u - par) << 16;
    xge += par * 8192;
    asm volatile("" : "+v"(tmask), "+v"(xge), "+v"(rdA[0]), "+v"(rdA[1]), "+v"(rdB[0]), "+v"(rdB[1]), "+v"(rdB[2]), "+v"(rdB[3]) : "v"(xv));
    {
        u32x4 ra[4];
#pragma unroll
        for (int i = 0; i < 4; ++i)
#pragma unroll
            for (int e = 0; e < 4; ++e) ra[i][e] = (((unsigned)tid * 2654435761u + (i * 4 + e) * 0x9e3779b9u) * 0x85ebca6bu >> 3 & 0x80ff80ffu) | 0x3f003f00u;
        f32x4 d[4] = {};
        const int rounds = PREHEAT;
#pragma unroll 1
        for (int r = 0; r < rounds; ++r) {
#pragma unroll
            for (int i = 0; i < 8; ++i) MFMA(d[i & 3], ra[i & 3], ra[(i + 1 + (i >> 2)) & 3]);
        }
        asm volatile("" :: "v"(d[0]), "v"(d[1]), "v"(d[2]), "v"(d[3]));
    }
    WAIT_LGKM0();

#define BFR(s, n, kk) __builtin_shufflevector(bl[s][n][kk], bh[s][n][kk], 0, 1, 2, 3, 4, 5, 6, 7)
#define RDA(kk, mt) (*(lds_s16x8_ptr)(uintptr_t)(rdA[kk] + (mt) * 2048))
#define RDB(kk, n, h) __builtin_amdgcn_ds_read_tr16_b64_v4i16((lds_s16x4_ptr)(uintptr_t)(rdB[n] + (kk) * 8192 + (h) * 1024))
    auto rdb = [&](auto sc, int j) {
        constexpr int s = decltype(sc)::value;
        const int n = j >> 2, kk = ((j >> 1) & 1) ^ (n & 1);
        if (j & 1) bh[s][n][kk] = RDB(kk, n, 1); else bl[s][n][kk] = RDB(kk, n, 0);
    };
    auto grp = [&](auto mtc, auto abc, auto bsc, int na, auto&& fill) {
        constexpr int mt = decltype(mtc)::value, ab = decltype(abc)::value, bs = decltype(bsc)::value;
#pragma unroll
        for (int i = 0; i < 8; ++i) {
            const int nt = i >> 1, kk = (i & 1) ^ (nt & 1);
            MFMA(acc[mt][nt], BFR(bs, nt, kk), aq[ab][kk]);
            SB();
            if (i < 2 && na >= 0) aq[1 - ab][i] = RDA(i, na);
            fill(i);
            SB();
        }
    };
#define I(v) IntC<v>{}
    auto none = [](int) {};
    unsigned slot = 0, sd = 0;
    auto next_slot = [&](int i) {
        if (i == 0) { sd = slot == NS - 1 ? (unsigned)(-(NS - 1) * STAGE_BYTES) : (unsigned)STAGE_BYTES; slot = slot == NS - 1 ? 0 : slot + 1; }
        else if (i < 5) rdB[i - 1] += sd;
        else rdA[i - 5] += sd;
    };
    auto interval = [&](auto sc, int k) {
        constexpr int s = decltype(sc)::value;
        BARRIER();
        SB();
        grp(I(3), I(1), I(1 - s), 4, [&](int i) { if (i >= 2 && i < 6) rdb(sc, i - 2); });
        grp(I(4), I(0), I(1 - s), 5, [&](int i) { if (i >= 2 && i < 6) rdb(sc, i + 2); });
        grp(I(5), I(1), I(1 - s), 6, [&](int i) { if (i >= 2 && i < 6) rdb(sc, i + 6); });
        grp(I(6), I(0), I(1 - s), 7, [&](int i) { if (i >= 2 && i < 6) rdb(sc, i + 10); });
        grp(I(7), I(1), I(1 - s), 0, none);
        grp(I(0), I(0), sc, 1, none);
        grp(I(1), I(1), sc, 2, [&](int i) { if (i >= 2 && i < 7) next_slot(i - 2); });
        grp(I(2), I(0), sc, 3, [&](int i) { if (i == 2 || i == 4) next_slot(4 + (i >> 1)); });
        WAIT_LGKM0();
        SB();
    };
    BARRIER();
    SB();
    aq[0][0] = RDA(0, 0);
    rdb(I(0), 0); rdb(I(0), 1);
    SB();
    aq[0][1] = RDA(1, 0);
    SB();
#pragma unroll
    for (int j = 2; j < 16; ++j) { rdb(I(0), j); if (j & 1) SB(); }
    grp(I(0), I(0), I(0), 1, none);
    grp(I(1), I(1), I(0), 2, [&](int i) { if (i >= 2 && i < 7) next_slot(i - 2); });
    grp(I(2), I(0), I(0), 3, [&](int i) { if (i == 2 || i == 4) next_slot(4 + (i >> 1)); });
    if (cw == 0 && idx == SWIDX) {
        asm volatile("global_store_dwordx2 %0, %1, off offset:8 sc0 sc1" :: "v"(xge), "v"((unsigned long long)call << 32) : "memory");
    }
    WAIT_LGKM0();
    SB();
    for (int k = 1; k < NSTEP - 1; k += 2) {
        interval(I(1), k);
        interval(I(0), k + 1);
    }
    u32x4 g;
    auto xst = [&](auto mtc, int i) {
        constexpr int mt = decltype(mtc)::value;
        const int nt = i >= 4 ? 2 : 0;
        const int j = i & 3;
        if (j == 1) {
            g[0] = (__float_as_uint(acc[mt][nt][0]) + tc1) & tmask;
            g[1] = pk_bf16(acc[mt][nt][2], acc[mt][nt][3]);
        } else if (j == 2) {
            g[0] = (pk_bf16(__uint_as_float(g[0]), acc[mt][nt][1]) & ~1u) | par;
        } else if (j == 3) {
            g[2] = pk_bf16(acc[mt][nt + 1][0], acc[mt][nt + 1][1]);
            g[3] = pk_bf16(acc[mt][nt + 1][2], acc[mt][nt + 1][3]);
            __builtin_amdgcn_raw_buffer_store_b128(g, xr, xv + ((mt >> 2) * 2 + (nt >> 1)) * 1024, xo + (mt & 3) * 16384, XST_AUX);
        }
    };
    u32x4 v[6];
    auto xld = [&](int b, int q) {
        v[2 * b + q] = __builtin_amdgcn_raw_buffer_load_b128(xr, xv + (2 + q) * 1024, (wg ^ ((3 - b) << 6)) * XB_STRIDE + b * 16384, 16);
    };
    const unsigned ho = lds_base + LDS_HO + xv;
    constexpr int S1 = STAGE_BYTES / 2048;
    BARRIER();
    SB();
    grp(I(3), I(1), I(0), 4, [&](int i) { rdb(I(1), i); });
    grp(I(4), I(0), I(0), 0, [&](int i) { rdb(I(1), i + 8); });
    grp(I(0), I(1), I(1), 1, none);
    grp(I(1), I(0), I(1), 2, [&](int i) { xst(I(0), i); });
    grp(I(2), I(1), I(1), 3, [&](int i) { xst(I(1), i); });
    unsigned l16;
    grp(I(3), I(0), I(1), 5, [&](int i) { xst(I(2), i); if (i == 4) l16 = *(lds_u32_ptr)(uintptr_t)(lds_base + LDS_L16 + wm * 4);
                                          if (i == PREF) *(lds_u32_ptr)(uintptr_t)(lds_base + LDS_HF + cw * 4) = 1u; });
    for (unsigned it = 0; __builtin_amdgcn_readfirstlane(l16) == 0u && it < 64 * POLL_LIMIT; ++it)
        l16 = *(lds_u32_ptr)(uintptr_t)(lds_base + LDS_L16 + wm * 4);
    SB();
    grp(I(5), I(1), I(0), S1 + 4, [&](int i) {
        if (i >= 1 && i < 5) *(lds_f32x4_ptr)(uintptr_t)(ho + (i - 1) * 1024) = acc[3][i - 1];
        if (i == 5) *(lds_u32_ptr)(uintptr_t)(lds_base + LDS_HF + cw * 4) = 3u;
    });
    grp(I(4), I(0), I(1), S1 + 5, none);
    grp(I(5), I(1), I(1), 6, [&](int i) { xst(I(4), i); });
    grp(I(6), I(0), I(0), S1 + 6, [&](int i) { xst(I(5), i); });
    grp(I(6), I(1), I(1), 7, none);
    grp(I(7), I(0), I(0), S1 + 7, [&](int i) { xst(I(6), i); });
    grp(I(7), I(1), I(1), -1, [&](int i) { if (i >= 4) xld((i - 4) >> 1, i & 1); });
    if (XSLEEP2) __builtin_amdgcn_s_sleep(XSLEEP2);
    xld(2, 0); xld(2, 1);
    asm volatile("s_nop 7" : "+v"(acc[7][0]), "+v"(acc[7][1]), "+v"(acc[7][2]), "+v"(acc[7][3]));
    SB();
    STAMP(1);
    bool bad = false;
#pragma unroll
    for (int b = 0; b < 3; ++b) {
        for (unsigned it = 0;;) {
            const unsigned mism = (v[2 * b][0] ^ par) | (v[2 * b + 1][0] ^ par);
            if (__all((mism & 1u) == 0u)) break;
            if (++it > POLL_LIMIT) { bad = true; break; }
            if ((it & 255u) == 0u) asm volatile("buffer_wbl2 sc1\n\ts_waitcnt vmcnt(0)" ::: "memory");
            asm volatile("" ::: "memory");
            xld(b, 0); xld(b, 1);
        }
#pragma unroll
        for (int q = 0; q < 2; ++q) add_granule(acc[7][2 * q], acc[7][2 * q + 1], v[2 * b + q]);
    }
#pragma unroll
    for (int n = 0; n < 4; ++n) *(lds_f32x4_ptr)(uintptr_t)(ho + 16384 + n * 1024) = acc[7][n];
    float am = 0.f;
#pragma unroll
    for (int i = 0; i < 4; ++i)
#pragma unroll
        for (int e = 0; e < 4; ++e) am = fmaxf(am, fabsf(acc[7][i][e]));
    const unsigned pa = __builtin_amdgcn_readfirstlane(*(lds_u32_ptr)(uintptr_t)(lds_base + LDS_RED));
    unsigned amb = wave_max_u32(__float_as_uint(am));
    if (bad) amb = 0xffffffffu;
    {
        const unsigned long long g = ((unsigned long long)call << 32) | amb;
        if (RECORD && pa != 0u && amb > pa)
            asm volatile("global_store_dwordx2 %0, %1, off offset:8 sc0 sc1" :: "v"(xge), "v"(g) : "memory");
        if (lane == 0) {
            const char* lg = my_ws + WS_LGF + xcd * 2048 + (idx * 4 + cw) * 8;
            asm volatile("global_store_dwordx2 %0, %1, off\n\t"
                         "global_store_dwordx2 %2, %1, off sc1"
                         :: "v"(lg), "v"(g), "v"(lg + (WS_LGS - WS_LGF)) : "memory");
        }
    }
    asm volatile("" ::: "memory");
    *(lds_u32_ptr)(uintptr_t)(lds_base + LDS_HF + 16 + cw * 4) = amb | 0x80000000u;
    STAMP(2);
    if (tid == 256) __hip_atomic_store(ctr, call, __ATOMIC_RELAXED, __HIP_MEMORY_SCOPE_AGENT);
    if (WBMODE != 2 && cw == 2 && idx == 1) {
        wait_amax(lds_base, call);
        __builtin_amdgcn_s_sleep(WBDELAY);
        asm volatile("buffer_wbl2 sc1" ::: "memory");
    }
}

}

#define PACE 0, 0, -1, 0, 0, 0, 0, 0
static const int g_pace[8] = {PACE};
extern "C" void kernel_launch(int rank, int nranks, ncclComm_t comm,
                              hipStream_t stream,
                              void* const* d_in, const int* in_sizes, int n_in,
                              void* d_out, int out_size,
                              const int* in_dtypes, int out_dtype,
                              void* const* d_out_all,
                              void* const* d_ws_all, size_t ws_bytes) {
    (void)comm; (void)in_dtypes; (void)out_dtype; (void)d_out;
    if (nranks != NR || n_in < 2 || (unsigned)rank >= (unsigned)NR || ws_bytes < WS_END ||
        in_sizes[0] != M_LOC * K_TOT || in_sizes[1] != K_TOT * N_TOT || out_size != N_TOT * N_LOC) {
        fprintf(stderr, "kernel_launch: unexpected problem shape\n");
        return;
    }
    Params p;
    p.x = d_in[0];
    p.w = d_in[1];
    for (int i = 0; i < NR; ++i) {
        p.out[i] = d_out_all[i];
        p.ws[i] = d_ws_all[i];
    }
    p.rank = rank;
    p.pad = 0;
    p.my_ws = d_ws_all[rank];
    for (int i = 0; i < 8; ++i) p.pace[i] = g_pace[i];
    const int sg[10] = {SG};
    {
        int seq[10], ns = 0;
        for (int d = sg[0]; d > 0; d /= 10) seq[ns++] = d % 10 - 1;
        for (int w = 0; w < NWG; ++w) {
            const int xcd = w & 7, tnl = (w >> 3) & 3, tm = (w >> 5) & 1, ks = w >> 6;
            int key[16];
            for (int k = 0; k < 16; ++k) {
                const int ph = k + sg[2] * ks + sg[3] * xcd;
                const int mine = seq[(ph + sg[1] * tm) % ns], other = seq[(ph + sg[1] * (1 - tm)) % ns];
                int lead = 0;
                if (mine == tnl && other == tnl) lead = ((ph / ns + tm) & 1) ? sg[4] : sg[6];
                else if (mine == tnl) lead = sg[4];
                else if (other == tnl) lead = 0;
                else lead = (((ph + tnl * sg[7]) >> sg[8]) + tm + sg[9]) & 1 ? sg[5] : 0;
                key[k] = (4 * k - 4 * lead - (lead ? 2 : 0)) * 16 + k;
            }
            unsigned long long t = 0;
            for (int k = 0; k < 16; ++k) {
                int pos = 0;
                for (int q = 0; q < 16; ++q) pos += key[q] < key[k];
                t |= (unsigned long long)k << (4 * pos);
            }
            p.ord[w] = t;
        }
    }
    hipLaunchKernelGGL(gemm_a2a_kernel, dim3(NWG), dim3(NTHREADS), 0, stream, p);
    HIP_CHECK(hipGetLastError());
}
```

```cpp
#include <hip/hip_runtime.h>
#include <rccl/rccl.h>
#include <cstdio>
#include <cstdint>

#define HIP_CHECK(expr) do { hipError_t _e = (expr); if (_e != hipSuccess) { \
    fprintf(stderr, "HIP error %s at %s:%d\n", hipGetErrorString(_e), __FILE__, __LINE__); \
    return; } } while (0)

#define SWAPX 0
#define TLW 2
#define SWIDX 0
#define PREHEAT 12
#define SPEC 1
#define SPERIOD 256
#define WBMODE 3
#define XST_AUX 0
#define WBDELAY 48
#define POLLGATE 1
#define RECORD 1
#define SGRACE 0
#define XSLEEP 0
#define XSLEEP1 0
#define XSLEEP2 0
#define SPRIO 0
#define SWQUIET 1
#define SPINS 0
#define LPRIO 0
#define LATEHEAT 1
#define PREF 7
#define FSLEEP 16
#define PHDEL 0
#define PHGAP 0
#define PHDG 0
#define LATEZ 8
#define TOUCH 0
#define TSLEEP 2
#define TMASK 2
#define SG 4312431, 3, 2, 1, 2, 1, 5, 0, 0, 0
namespace {

typedef __attribute__((ext_vector_type(4))) short s16x4;
typedef __attribute__((ext_vector_type(8))) short s16x8;
typedef __attribute__((ext_vector_type(2))) __bf16 bf16x2;
typedef __attribute__((ext_vector_type(2))) float f32x2;
typedef __attribute__((ext_vector_type(4))) float f32x4;
typedef __attribute__((ext_vector_type(2))) unsigned u32x2;
typedef __attribute__((ext_vector_type(4))) unsigned u32x4;
typedef __attribute__((address_space(3))) s16x4* lds_s16x4_ptr;
typedef __attribute__((address_space(3))) s16x8* lds_s16x8_ptr;
typedef __attribute__((address_space(3))) f32x4* lds_f32x4_ptr;
typedef __attribute__((address_space(3))) volatile unsigned* lds_u32_ptr;

constexpr int NR = 8;
constexpr int M_LOC = 512;
constexpr int K_TOT = 4096;
constexpr int N_TOT = 4096;
constexpr int N_LOC = N_TOT / NR;
constexpr int BM = 256, BN = 128, BK = 64;
constexpr int NS = 3;
constexpr int A_BYTES = BM * BK * 2;
constexpr int B_BYTES = BK * BN * 2;
constexpr int STAGE_BYTES = A_BYTES + B_BYTES;
constexpr int NSTEP = 16;
constexpr int NWG = 256;
constexpr int NTHREADS = 512;
constexpr int LDS_CALL = NS * STAGE_BYTES;
constexpr int LDS_RED = LDS_CALL + 16;
constexpr int LDS_HF = LDS_RED + 16;
constexpr int LDS_RDY = LDS_CALL + 4;
constexpr int LDS_L16 = LDS_CALL + 8;
constexpr int LDS_DUMMY = LDS_HF + 32;
constexpr int LDS_BYTES = LDS_DUMMY + 256;
constexpr int LDS_HO = 2 * STAGE_BYTES;

constexpr size_t WS_XG = 0;
constexpr size_t WS_LGF = WS_XG + 2 * 8192;
constexpr size_t WS_LGS = WS_LGF + 16384;
constexpr size_t WS_DONE = WS_LGS + 16384;
constexpr size_t WS_CTR = WS_DONE + NWG * 128;
constexpr size_t WS_DIAG = WS_CTR + NWG * 128;
constexpr size_t WS_XB = 256 * 1024;
#define XB_STRIDE 24832
constexpr size_t WS_END = WS_XB + (size_t)NWG * XB_STRIDE;
static_assert(WS_DIAG + NWG * 16 * 8 <= WS_XB, "");

constexpr unsigned POLL_LIMIT = 1u << 20;

struct Params {
    const void* x;
    const void* w;
    void* my_ws;
    int rank;
    int pad;
    int pace[8];
    void* out[NR];
    void* ws[NR];
    unsigned long long ord[NWG];
};

__device__ __forceinline__ void glds16(unsigned voff, const void* sbase, unsigned lds_dst) {
    asm volatile("s_mov_b32 m0, %2\n\ts_nop 0\n\tglobal_load_lds_dwordx4 %0, %1"
                 :: "v"(voff), "s"(sbase), "s"(lds_dst) : "memory");
}
#define WAIT_VM(n) asm volatile("s_waitcnt vmcnt(%0)" :: "i"(n) : "memory")
#define WAIT_LGKM0() asm volatile("s_waitcnt lgkmcnt(0)" ::: "memory")
#define BARRIER() asm volatile("s_barrier" ::: "memory")
#define SB() __builtin_amdgcn_sched_barrier(0)
#define MFMA(acc, a, b) asm volatile("v_mfma_f32_16x16x32_bf16 %0, %1, %2, %0" : "+v"(acc) : "v"(a), "v"(b))
#define MFMA0(acc, a, b) asm volatile("v_mfma_f32_16x16x32_bf16 %0, %1, %2, 0" : "+v"(acc) : "v"(a), "v"(b))

__device__ __forceinline__ unsigned wave_max_l63(unsigned v) {
    v = max(v, (unsigned)__builtin_amdgcn_update_dpp(0, (int)v, 0x111, 0xf, 0xf, false));
    v = max(v, (unsigned)__builtin_amdgcn_update_dpp(0, (int)v, 0x112, 0xf, 0xf, false));
    v = max(v, (unsigned)__builtin_amdgcn_update_dpp(0, (int)v, 0x114, 0xf, 0xf, false));
    v = max(v, (unsigned)__builtin_amdgcn_update_dpp(0, (int)v, 0x118, 0xf, 0xf, false));
    v = max(v, (unsigned)__builtin_amdgcn_update_dpp(0, (int)v, 0x142, 0xa, 0xf, false));
    v = max(v, (unsigned)__builtin_amdgcn_update_dpp(0, (int)v, 0x143, 0xc, 0xf, false));
    return v;
}
__device__ __forceinline__ unsigned wave_max_u32(unsigned v) { return (unsigned)__builtin_amdgcn_readlane((int)wave_max_l63(v), 63); }
__device__ __forceinline__ void publish_l63(const char* ws, const char* ws2, unsigned off, unsigned v, unsigned call) {
    asm volatile("s_lshl_b64 exec, 1, 63\n\tglobal_store_dwordx2 %0, %1, %2\n\tglobal_store_dwordx2 %0, %1, %3 sc1\n\ts_mov_b64 exec, -1"
                 :: "v"(off), "v"((u32x2){v, call}), "s"(ws), "s"(ws2) : "memory", "scc");
}
__device__ __forceinline__ void add2(f32x4 (&y)[4], const u32x4& s0, const u32x4& s1, const u32x4& g0, const u32x4& g1) {
    asm volatile("s_nop 1\n\tv_mfma_f32_16x16x32_bf16 %0, %4, %6, %0\n\tv_mfma_f32_16x16x32_bf16 %1, %5, %6, %1\n\t"
                 "v_mfma_f32_16x16x32_bf16 %2, %4, %7, %2\n\tv_mfma_f32_16x16x32_bf16 %3, %5, %7, %3\n\ts_nop 7"
                 : "+v"(y[0]), "+v"(y[1]), "+v"(y[2]), "+v"(y[3]) : "v"(s0), "v"(s1), "v"(g0), "v"(g1));
}
__device__ __forceinline__ void make_sel(int lane, u32x4& s0, u32x4& s1) {
    const unsigned one = (lane >> 4) == ((lane & 15) >> 2) ? (lane & 1 ? 0x3f800000u : 0x3f80u) : 0u;
    const unsigned x0 = lane & 2 ? 0u : one, x1 = lane & 2 ? one : 0u;
    s0 = (u32x4){x0, x1, 0u, 0u};
    s1 = (u32x4){0u, 0u, x0, x1};
    asm volatile("s_nop 1" : "+v"(s0), "+v"(s1));
}

__device__ __forceinline__ unsigned pk_bf16(float a, float b) {
    const bf16x2 v = __builtin_convertvector((f32x2){a, b}, bf16x2);
    return __builtin_bit_cast(unsigned, v);
}
__device__ __forceinline__ unsigned pk_bf16_tag(float a, float b, unsigned tag) {
    const unsigned t = (__float_as_uint(a) + ((1u - tag) << 16)) & 0xfffe0000u;
    return (pk_bf16(__uint_as_float(t), b) & ~1u) | tag;
}
__device__ __forceinline__ u32x4 pack_granule(const f32x4& a, const f32x4& b, unsigned tag) {
    u32x4 g;
    g[0] = pk_bf16_tag(a[0], a[1], tag);
    g[1] = pk_bf16(a[2], a[3]);
    g[2] = pk_bf16(b[0], b[1]);
    g[3] = pk_bf16(b[2], b[3]);
    return g;
}
__device__ __forceinline__ float add_lo(unsigned w, float c) {
    unsigned one_lo = 0x00003f80u;
    asm("" : "+s"(one_lo));
    return __builtin_amdgcn_fdot2_f32_bf16(__builtin_bit_cast(bf16x2, w), __builtin_bit_cast(bf16x2, one_lo), c, false);
}
__device__ __forceinline__ float add_hi(unsigned w, float c) {
    return __builtin_amdgcn_fdot2_f32_bf16(__builtin_bit_cast(bf16x2, w), __builtin_bit_cast(bf16x2, 0x3f800000u), c, false);
}
__device__ __forceinline__ void add_granule(f32x4& a, f32x4& b, const u32x4& g) {
    a[0] = add_lo(g[0], a[0]); a[1] = add_hi(g[0], a[1]); a[2] = add_lo(g[1], a[2]); a[3] = add_hi(g[1], a[3]);
    b[0] = add_lo(g[2], b[0]); b[1] = add_hi(g[2], b[1]); b[2] = add_lo(g[3], b[2]); b[3] = add_hi(g[3], b[3]);
}
__device__ __forceinline__ void tr4(unsigned& r0, unsigned& r1, unsigned& r2, unsigned& r3) {
    u32x2 t;
    t = __builtin_amdgcn_permlane32_swap(r0, r2, false, false); r0 = t[0]; r2 = t[1];
    t = __builtin_amdgcn_permlane32_swap(r1, r3, false, false); r1 = t[0]; r3 = t[1];
    t = __builtin_amdgcn_permlane16_swap(r0, r1, false, false); r0 = t[0]; r1 = t[1];
    t = __builtin_amdgcn_permlane16_swap(r2, r3, false, false); r2 = t[0]; r3 = t[1];
}

#define NCYC 2
#define STAMPT(i, t) do {} while (0)
#define STAMP(i) STAMPT(i, 256)

__device__ __forceinline__ unsigned wait_amax(unsigned lds_base, unsigned call) {
    for (unsigned it = 0;; ++it) {
        u32x2 r;
        asm volatile("ds_read_b64 %0, %1\n\ts_waitcnt lgkmcnt(0)" : "=v"(r) : "v"(lds_base + LDS_RED) : "memory");
        if (__builtin_amdgcn_readfirstlane(r[1]) == call) return __builtin_amdgcn_readfirstlane(r[0]);
        if (it > 64 * POLL_LIMIT) return 0xffffffffu;
        if (SPINS) __builtin_amdgcn_s_sleep(4);
    }
}
__device__ __forceinline__ float inv_scale(unsigned gmax) {
    const float scale = __uint_as_float(gmax) / 448.0f;
    return scale > 0.f ? 1.0f / scale : 0.f;
}

__device__ __forceinline__ char* xg_entry(const Params& p, int lane, int g) {
    char* b = (char*)p.ws[0];
#pragma unroll
    for (int i = 1; i < NR; ++i) {
        char* w = (char*)p.ws[i];
        asm("" : "+s"(w));
        b = (lane >> 3) == i ? w : b;
    }
    return b + WS_XG + (lane & 7) * 1024 + (p.rank * 8 + g) * 16;
}

template <int V> struct IntC { static constexpr int value = V; };

__global__ void __launch_bounds__(NTHREADS, 2) gemm_a2a_kernel(const Params p) {
    __shared__ __attribute__((aligned(1024))) char lds[LDS_BYTES];
    const int tid = threadIdx.x;
    const int lane = tid & 63;
    const int wave = __builtin_amdgcn_readfirstlane(tid >> 6);
    const int wg = blockIdx.x;
    const int rank = p.rank;
    char* const my_ws = (char*)p.my_ws;
    const char* const gx = (const char*)p.x;
    const char* const gw = (const char*)p.w;
    const unsigned long long otab = p.ord[wg];

    const int xcd = wg & 7, idx = wg >> 3;
    const int tnl = idx & 3, tm = (idx >> 2) & 1, ks = idx >> 3;
    const int row0 = tm * BM;
    const int col0 = xcd * N_LOC + tnl * BN;

    const int of = 0;
    const int hi = wave & 1;
    unsigned voffA[8], voffB[4];
    {
        unsigned ve = (unsigned)(((wave & 2) * 64 + hi * 16 + (lane >> 3)) * (K_TOT * 2) + (((lane & 7) ^ (lane >> 4)) * 16));
        asm("" : "+v"(ve));
        unsigned vo = (ve ^ 64u) + 8 * (K_TOT * 2);
        asm("" : "+v"(vo));
#pragma unroll
        for (int i = 0; i < 8; ++i) voffA[i] = (i & 1 ? vo : ve) | (unsigned)((((i >> 1) ^ 3 ^ ks) & 3) * 32 * (K_TOT * 2));
        unsigned vb = (unsigned)(((wave & 3) * 16 + (lane >> 4)) * (N_TOT * 2) + (((lane & 15) ^ (2 * (lane >> 4))) * 16));
        asm("" : "+v"(vb));
        voffB[0] = vb; voffB[1] = vb | 4 * (N_TOT * 2); voffB[2] = vb ^ (8 * (N_TOT * 2) + 128); voffB[3] = vb ^ (12 * (N_TOT * 2) + 128);
    }
    asm volatile("" : "+v"(voffA[0]), "+v"(voffA[1]), "+v"(voffA[2]), "+v"(voffA[3]), "+v"(voffA[4]), "+v"(voffA[5]), "+v"(voffA[6]), "+v"(voffA[7]),
                      "+v"(voffB[0]), "+v"(voffB[1]), "+v"(voffB[2]), "+v"(voffB[3]));
    unsigned offA = row0 * (K_TOT * 2) + ks * NSTEP * (BK * 2), offB = col0 * 2 + ks * NSTEP * (BK * N_TOT * 2);
    unsigned dA = (unsigned)(uintptr_t)lds + wave * 8192, dB = (unsigned)(uintptr_t)lds + A_BYTES + wave * 4096;
    asm volatile("" : "+s"(offA), "+s"(offB), "+s"(dA), "+s"(dB));
    asm volatile("" :: "s"(gx), "s"(gw), "s"(my_ws), "s"(rank), "s"(otab));
    auto ktile = [&](int s) {
        return ((int)(otab >> (4 * (s & 15))) + of) & 15;
    };
    unsigned* ctr = (unsigned*)(my_ws + WS_CTR + (size_t)wg * 128);
    const unsigned lds_base = (unsigned)(uintptr_t)lds;

    if (__builtin_expect(wave < 4, 1)) {
        if (LPRIO) __builtin_amdgcn_s_setprio(LPRIO);
        const char* const ga = gx + offA;
        const char* const gb = gw + offB;
        auto issue_at = [&](unsigned ka, unsigned kb, unsigned so) {
            const char* a = ga + ka * (BK * 2);
            const char* b = gb + kb * (BK * N_TOT * 2);
#pragma unroll
            for (int j = 0; j < 4; ++j) {
                glds16(voffB[j], b, dB + so + j * 1024);
                glds16(voffA[2 * j], a, dA + so + 2 * j * 1024);
                glds16(voffA[2 * j + 1], a, dA + so + (2 * j + 1) * 1024);
            }
        };
        {
            const char* b = gb + (unsigned)ktile(0) * (BK * N_TOT * 2);
#pragma unroll
            for (int j = 0; j < 4; ++j) glds16(voffB[j], b, dB + j * 1024);
        }
        if (!hi) {
            const char* a = ga + (unsigned)ktile(0) * (BK * 2);
#pragma unroll
            for (int j = 0; j < 8; ++j) glds16(voffA[j], a, dA + j * 1024);
        }
        issue_at(ktile(1 - hi), ktile(1), STAGE_BYTES);
        int ordB = ktile(lane), ordA = ktile(lane - hi);
        asm volatile("" : "+v"(ordA), "+v"(ordB));
        auto issue = [&](int u, unsigned so) { issue_at(__builtin_amdgcn_readlane(ordA, u), __builtin_amdgcn_readlane(ordB, u), so); };
        unsigned so = 2 * STAGE_BYTES;
#define TLS(k)
        for (int k = 0; k < NSTEP - 2; ++k) {
            WAIT_VM(12);
            BARRIER();
            TLS(k);
            issue(k + 2, so);
            so = (so == 2 * STAGE_BYTES) ? 0 : so + STAGE_BYTES;
        }
        const bool tch = TOUCH && wave == 0 && (TMASK >> tnl & 1);
        auto touch = [&]() {
            const char* const xb = my_ws + WS_XB + (size_t)wg * XB_STRIDE;
            const unsigned tv = lane * 128;
            asm volatile("s_mov_b32 m0, %4\n\ts_nop 0\n\tglobal_load_lds_dword %0, %1\n\tglobal_load_lds_dword %0, %2\n\tglobal_load_lds_dword %0, %3"
                         :: "v"(tv), "s"(xb), "s"(xb + 8192), "s"(xb + 16384), "s"(lds_base + LDS_DUMMY) : "memory");
        };
        if (TOUCH == 13 && tch) { touch(); WAIT_VM(15); } else WAIT_VM(12);
        BARRIER();
        TLS(14);
        if (TOUCH == 14 && tch) { __builtin_amdgcn_s_sleep(TSLEEP); touch(); }
        if (hi) {
            const char* a = ga + (unsigned)__builtin_amdgcn_readlane(ordA, 16) * (BK * 2);
#pragma unroll
            for (int j = 0; j < 8; ++j) glds16(voffA[j], a, dA + STAGE_BYTES + j * 1024);
        }
        const unsigned call = __builtin_amdgcn_readfirstlane(*(const unsigned*)(lds + LDS_CALL));
        const int wm = wave & 1;
        const unsigned par = call & 1u;
        const unsigned xv = wm * 8192 + lane * 16;
        char* const xge = xg_entry(p, lane, xcd) + par * 8192;
        const int orow = rank * M_LOC + row0 + wm * 128 + ks * 32 + (lane & 15);
        char* const o = (char*)p.out[xcd] + (size_t)orow * N_LOC + tnl * BN + (lane >> 4) * 16;
        char* const o2 = o + 16 * N_LOC;
        char* const dflag = (char*)p.ws[xcd] + WS_DONE + (size_t)(rank * 32 + idx) * 128 + wm * 8;
        const unsigned lg = WS_LGF + xcd * 2048 + 1024 + (idx * 2 + wm) * 8;
        const char* const ws2 = my_ws + (WS_LGS - WS_LGF);
        const char* const gsrc = my_ws + WS_XG + par * 8192 + xcd * 1024 + lane * 16;
        asm volatile("" :: "v"(xge), "v"(o), "v"(o2), "v"(dflag), "v"(lg), "s"(ws2), "v"(xv), "v"(gsrc));
        if (hi) WAIT_VM(8); else if (tch) WAIT_VM(3); else WAIT_VM(0);
        BARRIER();
        TLS(15);
        if (hi) {
            WAIT_VM(0);
            asm volatile("ds_write_b32 %0, %1" :: "v"(lds_base + LDS_L16 + (wave >> 1) * 4), "v"(1u) : "memory");
        }
        if (wave >= 2) {
            const __amdgpu_buffer_rsrc_t xr = __builtin_amdgcn_make_buffer_rsrc(my_ws + WS_XB, 0, NWG * XB_STRIDE, 0x00020000);
            TLS(16);
            u32x4 v[12];
            const unsigned xw = wm * 4096 + lane * 16;
            auto xld = [&](int b) {
#pragma unroll
                for (int q = 0; q < 4; ++q)
                    v[4 * b + q] = __builtin_amdgcn_raw_buffer_load_b128(xr, xw + q * 1024, (wg ^ ((3 - b) << 6)) * XB_STRIDE + b * 8192, 16);
            };
            f32x4 y[2][2][4];
            u32x4 sel0, sel1;
            make_sel(lane, sel0, sel1);
            if (SPRIO) __builtin_amdgcn_s_setprio(SPRIO);
            bool bad = false;
            for (unsigned it = 0;; ++it) {
                u32x2 f;
                asm volatile("ds_read_b64 %0, %1\n\ts_waitcnt lgkmcnt(0)" : "=v"(f) : "v"(lds_base + LDS_HF + wm * 8) : "memory");
                if (__builtin_amdgcn_readfirstlane(f[0] & f[1])) break;
                if (it > 64 * POLL_LIMIT) { bad = true; break; }
                if (SPINS) __builtin_amdgcn_s_sleep(SPINS);
            }
            if (XSLEEP) __builtin_amdgcn_s_sleep(XSLEEP);
            xld(0); xld(1);
            if (XSLEEP1) __builtin_amdgcn_s_sleep(XSLEEP1);
            xld(2);
            for (unsigned it = 0;; ++it) {
                u32x2 f;
                asm volatile("ds_read_b64 %0, %1\n\ts_waitcnt lgkmcnt(0)" : "=v"(f) : "v"(lds_base + LDS_HF + wm * 8) : "memory");
                if (__builtin_amdgcn_readfirstlane(f[0] & f[1] & 2u)) break;
                if (it > 64 * POLL_LIMIT) { bad = true; break; }
            }
#pragma unroll
            for (int n = 0; n < 8; ++n)
                y[0][n >> 2][n & 3] = *(lds_f32x4_ptr)(uintptr_t)(lds_base + LDS_HO + xv + n * 1024);
            unsigned pend = 7;
            const unsigned frdy = lds_base + LDS_RDY + wm;
            auto pass = [&]() {
#pragma unroll
                for (int b = 0; b < 3; ++b)
                    if (pend >> b & 1) {
                        const unsigned tags = v[4 * b][0] & v[4 * b + 1][0] & v[4 * b + 2][0] & v[4 * b + 3][0];
                        if (__all(tags & 1u)) {
                            pend &= ~(1u << b);
                            asm volatile("ds_write_b8 %0, %1" :: "v"(frdy), "v"(pend ^ 7u) : "memory");
                        }
                    }
#pragma unroll
                for (int b = 0; b < 3; ++b)
                    if (pend >> b & 1) xld(b);
            };
            pass();
            for (unsigned it = 0; pend; ++it) {
                if ((it & 255u) == 255u) asm volatile("buffer_wbl2 sc1\n\ts_waitcnt vmcnt(0)" ::: "memory");
                pass();
                if (it > POLL_LIMIT) { bad = true; pend = 0; }
            }
#pragma unroll
            for (int q = 0; q < 12; ++q) {
                MFMA(y[0][(q >> 1) & 1][2 * (q & 1)], sel0, v[q]);
                MFMA(y[0][(q >> 1) & 1][2 * (q & 1) + 1], sel1, v[q]);
            }
            asm volatile("s_nop 7" : "+v"(y[0][0][0]), "+v"(y[0][0][1]), "+v"(y[0][0][2]), "+v"(y[0][0][3]),
                                     "+v"(y[0][1][0]), "+v"(y[0][1][1]), "+v"(y[0][1][2]), "+v"(y[0][1][3]));
            float am = 0.f;
#pragma unroll
            for (int i = 0; i < 8; ++i)
#pragma unroll
                for (int e = 0; e < 4; ++e) am = fmaxf(am, fabsf(y[0][i >> 2][i & 3][e]));
            const unsigned av = wave_max_l63(__float_as_uint(am) | (bad ? 0xffffffffu : 0u));
            publish_l63(my_ws, ws2, lg, av, call);
            unsigned amb = (unsigned)__builtin_amdgcn_readlane((int)av, 63);
            if (lane == 0) asm volatile("ds_add_u32 %0, %1" :: "v"(lds_base + LDS_RED + 8), "v"(1u) : "memory");
            STAMPT(7, 128);
            auto send = [&](auto hc, unsigned val) {
                constexpr int h = decltype(hc)::value;
                const float inv = inv_scale(val);
                char* const dst = h ? o2 : o;
                u32x4 q[2];
#pragma unroll
                for (int c = 0; c < 2; ++c) {
                    unsigned r[4];
#pragma unroll
                    for (int j = 0; j < 4; ++j) {
                        int w = __builtin_amdgcn_cvt_pk_fp8_f32(y[h][c][j][0] * inv, y[h][c][j][1] * inv, 0, false);
                        w = __builtin_amdgcn_cvt_pk_fp8_f32(y[h][c][j][2] * inv, y[h][c][j][3] * inv, w, true);
                        r[j] = val == 0xffffffffu ? 0x7f7f7f7fu : (unsigned)w;
                    }
                    tr4(r[0], r[1], r[2], r[3]);
                    q[c] = (u32x4){r[0], r[1], r[2], r[3]};
                }
                asm volatile("global_store_dwordx4 %0, %1, off sc0 sc1\n\t"
                             "global_store_dwordx4 %0, %2, off offset:64 sc0 sc1\n\ts_nop 1" :: "v"(dst), "v"(q[0]), "v"(q[1]) : "memory");
            };
            unsigned s0 = 0xfffffffeu, s1 = 0xfffffffdu, flagged = 0xfffffffeu, told = 0;
            bool have1 = false;
            unsigned ts = (unsigned)__builtin_amdgcn_s_memrealtime() - SGRACE;
            ts += (2u * SPERIOD + (unsigned)((idx * 2 + wm) * SPERIOD >> 6) - ts % SPERIOD) % SPERIOD;
            for (unsigned it = 0;; ++it) {
                u32x2 r, f;
                unsigned long long now;
                asm volatile("ds_read_b64 %0, %3\n\tds_read_b64 %1, %4\n\ts_memrealtime %2\n\ts_waitcnt lgkmcnt(0)"
                             : "=&v"(r), "=&v"(f), "=s"(now) : "v"(lds_base + LDS_RED), "v"(lds_base + LDS_HF + 16 + wm * 8) : "memory");
                unsigned pa = __builtin_amdgcn_readfirstlane(r[0]);
                const unsigned st = __builtin_amdgcn_readfirstlane(r[1]);
                bool fin = st == call;
                if (it > 64 * POLL_LIMIT) { fin = true; pa = 0xffffffffu; }
                if (!have1) {
                    const unsigned f0 = __builtin_amdgcn_readfirstlane(f[0]), f1 = __builtin_amdgcn_readfirstlane(f[1]);
                    if (f0 & f1 & 0x80000000u) {
                        const unsigned fm = max(f0, f1);
                        const unsigned a1 = fm == 0xffffffffu ? fm : fm & 0x7fffffffu;
                        if (pa != 0u) told = max(told, a1);
                        amb = max(amb, a1);
                        have1 = true;
                        if (lane == 0) asm volatile("ds_add_u32 %0, %1" :: "v"(lds_base + LDS_RED + 8), "v"(16u) : "memory");
#pragma unroll
                        for (int n = 0; n < 8; ++n)
                            y[1][n >> 2][n & 3] = *(lds_f32x4_ptr)(uintptr_t)(lds_base + LDS_HO + 16384 + xv + n * 1024);
                    }
                }
                if (!have1 && pa != 0xffffffffu) fin = false;
                if (RECORD && pa != 0u && amb > max(pa, told)) {
                    told = amb;
                    asm volatile("global_store_dwordx2 %0, %1, off offset:8 sc0 sc1" :: "v"(xge), "v"(((unsigned long long)call << 32) | amb) : "memory");
                }
                const unsigned val = fin ? pa : max(pa, amb);
                bool go = fin;
                if (SPEC && (int)((unsigned)now - ts) >= 0) {
                    go = go || (st == ~call && pa != 0u && (RECORD || pa >= amb));
                    if ((int)((unsigned)now - ts) >= SGRACE || (go && (s0 != val || (s1 != val && have1)))) ts += SPERIOD;
                }
                if (go && s0 != val) {
                    send(IntC<0>{}, val);
                    s0 = val;
                    go = fin;
                }
                if (go && s1 != val && (have1 || fin)) {
                    send(IntC<1>{}, val);
                    s1 = val;
                }
                if (s0 == s1 && s0 != flagged) {
                    flagged = s0;
                    asm volatile("s_waitcnt vmcnt(0)" ::: "memory");
                    STAMPT(4, 128);
                    if (lane == 0 && xcd != rank)
                        asm volatile("global_store_dwordx2 %0, %1, off sc0 sc1"
                                     :: "v"(dflag), "v"(((unsigned long long)call << 32) | s0) : "memory");
                }
                if (fin) break;
            }
            return;
        }
        if (wave == 0) {
            u32x4 gv;
            unsigned best = 0;
            auto gstate = [&]() {
                const bool ok = gv[1] == call, started = gv[3] == call;
                const unsigned long long seen = __ballot(started);
                best = max(best, max(ok ? gv[0] : 0u, started ? gv[2] : 0u));
                const unsigned gmax = wave_max_u32(best);
                const unsigned st = __all(ok) ? call : ((seen >> (8 * xcd)) & 0xffu) ? ~call : 0u;
                asm volatile("ds_write_b64 %0, %1" :: "v"(lds_base + LDS_RED), "v"(((unsigned long long)st << 32) | gmax) : "memory");
                return st == call;
            };
            if (idx != SWIDX || !SWQUIET) {
                asm volatile("global_load_dwordx4 %0, %1, off sc0 sc1\n\ts_waitcnt vmcnt(0)" : "=&v"(gv) : "v"(gsrc) : "memory");
                gstate();
            }
            if (idx == SWIDX) {
                const char* lgs = my_ws + WS_LGF + xcd * 2048 + lane * 16;
                const char* lgt = my_ws + WS_LGF + xcd * 2048 + 1024 + lane * 8;
                unsigned m0 = 0, m1 = 0, m2 = 0;
                bool ok0 = false, ok1 = false, ok2 = false;
                unsigned it = 0;
                for (;;) {
                    const unsigned alt = ((it & 63u) == 63u) ? (WS_LGS - WS_LGF) : 0;
                    u32x4 v;
                    u32x2 u;
                    asm volatile("global_load_dwordx4 %0, %2, off sc1\n\tglobal_load_dwordx2 %1, %3, off sc1\n\ts_waitcnt vmcnt(0)"
                                 : "=&v"(v), "=&v"(u) : "v"(lgs + alt), "v"(lgt + alt) : "memory");
                    if (v[1] == call) { m0 = v[0]; ok0 = true; }
                    if (v[3] == call) { m1 = v[2]; ok1 = true; }
                    if (u[1] == call) { m2 = u[0]; ok2 = true; }
                    if (__all(ok0 && ok1 && ok2)) break;
                    if (++it > 4 * POLL_LIMIT) { m0 = 0xffffffffu; break; }
                }
                const unsigned m = wave_max_u32(max(max(m0, m1), m2));
                asm volatile("global_store_dwordx2 %0, %1, off sc0 sc1"
                             :: "v"(xge), "v"(((unsigned long long)call << 32) | m) : "memory");
                STAMPT(6, 0);
            }
            for (unsigned it = 0; POLLGATE && it < 64 * POLL_LIMIT; ++it) {
                unsigned c;
                asm volatile("ds_read_b32 %0, %1\n\ts_waitcnt lgkmcnt(0)" : "=v"(c) : "v"(lds_base + LDS_RED + 8) : "memory");
                c = __builtin_amdgcn_readfirstlane(c);
                if (POLLGATE == 1 ? (c & 15u) == 2u : c == 34u) break;
                if (SPINS) __builtin_amdgcn_s_sleep(2);
            }
            for (unsigned it = 0;;) {
                asm volatile("global_load_dwordx4 %0, %1, off sc0 sc1\n\ts_waitcnt vmcnt(0)"
                             : "=&v"(gv) : "v"(gsrc) : "memory");
                if (++it > 4 * POLL_LIMIT) { gv[0] = 0xffffffffu; gv[1] = call; }
                if (gstate()) break;
                if ((it & 255u) == 0u) asm volatile("buffer_wbl2 sc1\n\ts_waitcnt vmcnt(0)" ::: "memory");
            }
            return;
        }
        if ((wg >> 5) == rank) return;
        const unsigned long long want = ((unsigned long long)call << 32) | wait_amax(lds_base, call);
        __builtin_amdgcn_s_sleep(FSLEEP);
        const char* f = my_ws + WS_DONE + (size_t)wg * 128 + (lane & 1) * 8;
        for (unsigned it = 0;; ++it) {
            unsigned long long v;
            asm volatile("global_load_dwordx2 %0, %1, off sc0 sc1\n\ts_waitcnt vmcnt(0)" : "=&v"(v) : "v"(f) : "memory");
            if (__all(v == want)) break;
            if (it > POLL_LIMIT) {
                if (lane == 0)
                    __hip_atomic_store((unsigned char*)p.out[rank] + (size_t)rank * M_LOC * N_LOC,
                                       (unsigned char)0x7f, __ATOMIC_RELAXED, __HIP_MEMORY_SCOPE_SYSTEM);
                break;
            }
        }
        STAMPT(5, 64);
        return;
    }
    const int cw = wave - 4;
    const int wm = cw >> 1, wn = cw & 1;
    unsigned call = *(const __attribute__((address_space(4))) unsigned*)(uintptr_t)ctr;
    char* wsp[NR];
#pragma unroll
    for (int i = 0; i < NR; ++i) wsp[i] = (char*)p.ws[i];
    unsigned rdA[2], rdB[4];
#pragma unroll
    for (int kk = 0; kk < 2; ++kk) {
        const int r = lane & 15;
        const int c = 4 * kk + (lane >> 4);
        rdA[kk] = lds_base + (unsigned)(wm * 16384 + r * 128 + ((c ^ ((r >> 1) & 7)) * 16));
    }
#pragma unroll
    for (int cn = 0; cn < 4; ++cn) {
        const int g = lane >> 4, q = (lane & 15) >> 2, pp = lane & 3;
        const int k = 8 * g + q;
        const int h = (k & 3) | (((k >> 3) & 1) << 2);
        const int u = (wn * 4 + cn) ^ h;
        rdB[cn] = lds_base + (unsigned)(A_BYTES + k * 256 + u * 32 + pp * 8);
    }
    f32x4 acc[8][4];
#pragma unroll
    for (int i = 0; i < LATEZ; ++i) {
#pragma unroll
        for (int j = 0; j < 4; ++j) acc[i][j] = (f32x4){0.f, 0.f, 0.f, 0.f};
        asm volatile("s_nop 1" : "+v"(acc[i][0]), "+v"(acc[i][1]), "+v"(acc[i][2]), "+v"(acc[i][3]));
    }
    s16x8 aq[2][2];
    s16x4 bl[2][4][2], bh[2][4][2];
    asm volatile("" : "+v"(rdA[0]), "+v"(rdA[1]), "+v"(rdB[0]), "+v"(rdB[1]), "+v"(rdB[2]), "+v"(rdB[3]));
    {
        u32x4 ra[4];
#pragma unroll
        for (int i = 0; i < 4; ++i)
#pragma unroll
            for (int e = 0; e < 4; ++e) ra[i][e] = (((unsigned)tid * 2654435761u + (i * 4 + e) * 0x9e3779b9u) * 0x85ebca6bu >> 3 & 0x80ff80ffu) | 0x3f003f00u;
        f32x4 d[4] = {};
        const int rounds = PREHEAT, pdel = PHDEL, pgap = PHGAP, pdg = PHDG;
#pragma unroll 1
        for (int r = 0; r < pdel; ++r) __builtin_amdgcn_s_sleep(1);
#pragma unroll 1
        for (int r = 0, gp = pgap * 8; r < rounds; ++r, gp += pdg) {
#pragma unroll
            for (int i = 0; i < 8; ++i) MFMA(d[i & 3], ra[i & 3], ra[(i + 1 + (i >> 2)) & 3]);
#pragma unroll 1
            for (int q = 8; q <= gp; q += 8) asm volatile("s_nop 7");
        }
        asm volatile("" :: "v"(d[0]), "v"(d[1]), "v"(d[2]), "v"(d[3]));
    }
    asm volatile("" : "+s"(call), "+s"(wsp[0]), "+s"(wsp[1]), "+s"(wsp[2]), "+s"(wsp[3]), "+s"(wsp[4]), "+s"(wsp[5]), "+s"(wsp[6]), "+s"(wsp[7]));

#define BFR(s, n, kk) __builtin_shufflevector(bl[s][n][kk], bh[s][n][kk], 0, 1, 2, 3, 4, 5, 6, 7)
#define RDA(kk, mt) (*(lds_s16x8_ptr)(uintptr_t)(rdA[kk] + (mt) * 2048))
#define RDB(kk, n, h) __builtin_amdgcn_ds_read_tr16_b64_v4i16((lds_s16x4_ptr)(uintptr_t)(rdB[n] + (kk) * 8192 + (h) * 1024))
    auto rdb = [&](auto sc, int j) {
        constexpr int s = decltype(sc)::value;
        const int n = j >> 2, kk = ((j >> 1) & 1) ^ (n & 1);
        if (j & 1) bh[s][n][kk] = RDB(kk, n, 1); else bl[s][n][kk] = RDB(kk, n, 0);
    };
    auto grp = [&](auto mtc, auto abc, auto bsc, int na, auto&& fill) {
        constexpr int mt = decltype(mtc)::value, ab = decltype(abc)::value, bs = decltype(bsc)::value;
#pragma unroll
        for (int i = 0; i < 8; ++i) {
            const int nt = i >> 1, kk = (i & 1) ^ (nt & 1);
            MFMA(acc[mt][nt], BFR(bs, nt, kk), aq[ab][kk]);
            SB();
            if (i < 2 && na >= 0) aq[1 - ab][i] = RDA(i, na);
            fill(i);
            SB();
        }
    };
#define I(v) IntC<v>{}
    auto none = [](int) {};
    unsigned slot = 0, sd = 0;
    auto next_slot = [&](int i) {
        if (i == 0) { sd = slot == NS - 1 ? (unsigned)(-(NS - 1) * STAGE_BYTES) : (unsigned)STAGE_BYTES; slot = slot == NS - 1 ? 0 : slot + 1; }
        else if (i < 5) rdB[i - 1] += sd;
        else rdA[i - 5] += sd;
    };
    auto interval = [&](auto sc, int k) {
        constexpr int s = decltype(sc)::value;
        BARRIER();
        SB();
        grp(I(3), I(1), I(1 - s), 4, [&](int i) { if (i >= 2 && i < 6) rdb(sc, i - 2); });
        grp(I(4), I(0), I(1 - s), 5, [&](int i) { if (i >= 2 && i < 6) rdb(sc, i + 2); });
        grp(I(5), I(1), I(1 - s), 6, [&](int i) { if (i >= 2 && i < 6) rdb(sc, i + 6); });
        grp(I(6), I(0), I(1 - s), 7, [&](int i) { if (i >= 2 && i < 6) rdb(sc, i + 10); });
        grp(I(7), I(1), I(1 - s), 0, none);
        grp(I(0), I(0), sc, 1, none);
        grp(I(1), I(1), sc, 2, [&](int i) { if (i >= 2 && i < 7) next_slot(i - 2); });
        grp(I(2), I(0), sc, 3, [&](int i) { if (i == 2 || i == 4) next_slot(4 + (i >> 1)); });
        WAIT_LGKM0();
        SB();
    };
    BARRIER();
    SB();
    aq[0][0] = RDA(0, 0);
    rdb(I(0), 0); rdb(I(0), 1);
    SB();
    aq[0][1] = RDA(1, 0);
    SB();
#pragma unroll
    for (int j = 2; j < 16; ++j) { rdb(I(0), j); if (j & 1) SB(); }
    grp(I(0), I(0), I(0), 1, none);
    grp(I(1), I(1), I(0), 2, [&](int i) { if (i >= 2 && i < 7) next_slot(i - 2); });
    grp(I(2), I(0), I(0), 3, [&](int i) { if (i == 2 || i == 4) next_slot(4 + (i >> 1)); });
#pragma unroll
    for (int i = LATEZ; i < 8; ++i) {
#pragma unroll
        for (int j = 0; j < 4; ++j) acc[i][j] = (f32x4){0.f, 0.f, 0.f, 0.f};
        asm volatile("s_nop 1" : "+v"(acc[i][0]), "+v"(acc[i][1]), "+v"(acc[i][2]), "+v"(acc[i][3]));
    }
    const unsigned xv = cw * 4096 + lane * 16;
    const __amdgpu_buffer_rsrc_t xr = __builtin_amdgcn_make_buffer_rsrc(my_ws + WS_XB, 0, NWG * XB_STRIDE, 0x00020000);
    const unsigned xo = wg * XB_STRIDE;
    const unsigned xw = cw * 2048 + lane * 16;
    unsigned lg = WS_LGF + xcd * 2048 + (idx * 4 + cw) * 8;
    const char* ws2 = my_ws + (WS_LGS - WS_LGF);
    unsigned xp[3];
#pragma unroll
    for (int b = 0; b < 3; ++b) xp[b] = (wg ^ ((3 - b) << 6)) * XB_STRIDE + b * 8192;
    unsigned fl16 = lds_base + LDS_L16 + wm * 4, fhf = lds_base + LDS_HF + cw * 4;
    asm volatile("" : "+v"(lg), "+s"(ws2), "+s"(xp[0]), "+s"(xp[1]), "+s"(xp[2]), "+v"(fl16), "+v"(fhf));
    u32x4 sel0, sel1;
    make_sel(lane, sel0, sel1);
    unsigned tmask = 0xfffe0000u;
    ++call;
    if (tid < 256 + 4) *(u32x4*)(lds + LDS_CALL + lane * 16) = (u32x4){lane ? 0u : call, 0u, 0u, 0u};
    const unsigned par = call & 1u;
    char* xge = wsp[0];
#pragma unroll
    for (int i = 1; i < NR; ++i) xge = (lane >> 3) == i ? wsp[i] : xge;
    xge += WS_XG + (lane & 7) * 1024 + (rank * 8 + xcd) * 16 + par * 8192;
    asm volatile("" : "+v"(tmask), "+v"(xge) : "v"(xv), "v"(xw));
    if (cw == 0 && idx == SWIDX) {
        asm volatile("global_store_dwordx2 %0, %1, off offset:8 sc0 sc1" :: "v"(xge), "v"((unsigned long long)call << 32) : "memory");
    }
    WAIT_LGKM0();
    SB();
    for (int k = 1; k < NSTEP - 1; k += 2) {
        interval(I(1), k);
        interval(I(0), k + 1);
    }
    u32x4 gp[3][2];
    auto xst = [&](auto mtc, int i) {
        constexpr int mt = decltype(mtc)::value;
        const int nt = i >= 4 ? 2 : 0;
        const int j = i & 3;
        u32x4& g = gp[mt & 3][nt >> 1];
        if (j == 1) {
            g[0] = (__float_as_uint(acc[mt][nt][0]) + (mt < 4 ? 0u : 0x10000u)) & tmask;
            g[1] = pk_bf16(acc[mt][nt][2], acc[mt][nt][3]);
        } else if (j == 2) {
            g[0] = pk_bf16(__uint_as_float(g[0]), acc[mt][nt][1]);
            g[0] = mt < 4 ? g[0] | 1u : g[0] & ~1u;
        } else if (j == 3) {
            g[2] = pk_bf16(acc[mt][nt + 1][0], acc[mt][nt + 1][1]);
            g[3] = pk_bf16(acc[mt][nt + 1][2], acc[mt][nt + 1][3]);
            if (mt < 4) __builtin_amdgcn_raw_buffer_store_b128(g, xr, xw + (nt >> 1) * 1024, xo + mt * 8192, XST_AUX);
            else asm volatile("" : "+v"(g));
        }
    };
    u32x4 v[6];
    auto xld = [&](int b, int q) {
        v[2 * b + q] = __builtin_amdgcn_raw_buffer_load_b128(xr, xw + q * 1024, xo + b * 8192, 16);
    };
    unsigned ho = lds_base + LDS_HO + xv;
    asm("" : "+v"(ho));
    constexpr int S1 = STAGE_BYTES / 2048;
    BARRIER();
    SB();
    grp(I(3), I(1), I(0), 4, [&](int i) { rdb(I(1), i); });
    grp(I(4), I(0), I(0), 0, [&](int i) { rdb(I(1), i + 8); });
    grp(I(0), I(1), I(1), 1, none);
    grp(I(1), I(0), I(1), 2, [&](int i) { xst(I(0), i); });
    grp(I(2), I(1), I(1), 3, [&](int i) { xst(I(1), i); });
    unsigned l16;
    grp(I(3), I(0), I(1), 5, [&](int i) { xst(I(2), i); if (i == PREF) *(lds_u32_ptr)(uintptr_t)fhf = 1u; });
    grp(I(5), I(1), I(0), 6, [&](int i) {
        if (i >= 1 && i < 5) *(lds_f32x4_ptr)(uintptr_t)(ho + (i - 1) * 1024) = acc[3][i - 1];
        if (i == 5) *(lds_u32_ptr)(uintptr_t)fhf = 3u;
    });
    grp(I(6), I(0), I(0), 7, [&](int i) { if (i == 0) l16 = *(lds_u32_ptr)(uintptr_t)fl16;
                                          if (i == 6) l16 = __builtin_amdgcn_readfirstlane(l16); });
    if (__builtin_expect(l16 == 0u, 0))
        for (unsigned it = 0; it < 64 * POLL_LIMIT; ++it)
            if (__builtin_amdgcn_readfirstlane(*(lds_u32_ptr)(uintptr_t)fl16)) break;
    SB();
    grp(I(7), I(1), I(0), S1 + 4, none);
    grp(I(4), I(0), I(1), S1 + 5, none);
    grp(I(5), I(1), I(1), S1 + 6, [&](int i) { xst(I(4), i); });
    grp(I(6), I(0), I(1), S1 + 7, [&](int i) { xst(I(5), i); });
    grp(I(7), I(1), I(1), -1, [&](int i) { xst(I(6), i); });
    asm volatile("s_nop 5" : "+v"(acc[7][0]), "+v"(acc[7][1]), "+v"(acc[7][2]), "+v"(acc[7][3]));
    SB();
    STAMP(1);
    bool bad = false;
    for (unsigned it = 0, todo = 7; todo; ++it) {
        unsigned n = (__builtin_amdgcn_readfirstlane(*(lds_u32_ptr)(uintptr_t)(lds_base + LDS_RDY)) >> (wm * 8)) & todo;
        if (it > 64 * POLL_LIMIT) { n = todo; bad = true; }
#pragma unroll
        for (int b = 0; b < 3; ++b)
            if (n >> b & 1) {
#pragma unroll
                for (int q = 0; q < 2; ++q)
                    __builtin_amdgcn_raw_buffer_store_b128(gp[b][q], xr, xw + q * 1024, xp[b], XST_AUX);
            }
        todo &= ~n;
    }
    if (XSLEEP2) __builtin_amdgcn_s_sleep(XSLEEP2);
    unsigned pend = 7;
    auto pass = [&]() {
#pragma unroll
        for (int b = 0; b < 3; ++b)
            if (pend >> b & 1) { xld(b, 0); xld(b, 1); }
#pragma unroll
        for (int b = 0; b < 3; ++b)
            if (pend >> b & 1) {
                if (__all(((v[2 * b][0] | v[2 * b + 1][0]) & 1u) == 0u)) {
                    pend &= ~(1u << b);
                    add2(acc[7], sel0, sel1, v[2 * b], v[2 * b + 1]);
                }
            }
    };
    pass();
    for (unsigned it = 0; pend; ++it) {
        if ((it & 255u) == 255u) asm volatile("buffer_wbl2 sc1\n\ts_waitcnt vmcnt(0)" ::: "memory");
        asm volatile("" ::: "memory");
        pass();
        if (it > POLL_LIMIT) { bad = true; pend = 0; }
    }
    unsigned pav = *(lds_u32_ptr)(uintptr_t)(lds_base + LDS_RED);
    float am = 0.f;
#pragma unroll
    for (int i = 0; i < 4; ++i)
#pragma unroll
        for (int e = 0; e < 4; ++e) am = fmaxf(am, fabsf(acc[7][i][e]));
    const unsigned av = wave_max_l63(__float_as_uint(am) | (bad ? 0xffffffffu : 0u));
    publish_l63(my_ws, ws2, lg, av, call);
    const unsigned amb = (unsigned)__builtin_amdgcn_readlane((int)av, 63);
    {
        asm volatile("" : "+v"(pav));
        const unsigned pa = __builtin_amdgcn_readfirstlane(pav);
        if (RECORD && pa != 0u && amb > pa)
            asm volatile("global_store_dwordx2 %0, %1, off offset:8 sc0 sc1" :: "v"(xge), "v"(((unsigned long long)call << 32) | amb) : "memory");
    }
#pragma unroll
    for (int n = 0; n < 4; ++n) *(lds_f32x4_ptr)(uintptr_t)(ho + 16384 + n * 1024) = acc[7][n];
    asm volatile("" ::: "memory");
    *(lds_u32_ptr)(uintptr_t)(fhf + 16) = amb | 0x80000000u;
    STAMP(2);
    if (tid == 256) __hip_atomic_store(ctr, call, __ATOMIC_RELAXED, __HIP_MEMORY_SCOPE_AGENT);
    if (WBMODE != 2 && cw == 2 && idx == 1) {
        wait_amax(lds_base, call);
        __builtin_amdgcn_s_sleep(WBDELAY);
        asm volatile("buffer_wbl2 sc1" ::: "memory");
    }
}

}

#define PACE 0, 0, -1, 0, 0, 0, 0, 0
static const int g_pace[8] = {PACE};
extern "C" void kernel_launch(int rank, int nranks, ncclComm_t comm,
                              hipStream_t stream,
                              void* const* d_in, const int* in_sizes, int n_in,
                              void* d_out, int out_size,
                              const int* in_dtypes, int out_dtype,
                              void* const* d_out_all,
                              void* const* d_ws_all, size_t ws_bytes) {
    (void)comm; (void)in_dtypes; (void)out_dtype; (void)d_out;
    if (nranks != NR || n_in < 2 || (unsigned)rank >= (unsigned)NR || ws_bytes < WS_END ||
        in_sizes[0] != M_LOC * K_TOT || in_sizes[1] != K_TOT * N_TOT || out_size != N_TOT * N_LOC) {
        fprintf(stderr, "kernel_launch: unexpected problem shape\n");
        return;
    }
    Params p;
    p.x = d_in[0];
    p.w = d_in[1];
    for (int i = 0; i < NR; ++i) {
        p.out[i] = d_out_all[i];
        p.ws[i] = d_ws_all[i];
    }
    p.rank = rank;
    p.pad = 0;
    p.my_ws = d_ws_all[rank];
    for (int i = 0; i < 8; ++i) p.pace[i] = g_pace[i];
    const int sg[10] = {SG};
    {
        int seq[10], ns = 0;
        for (int d = sg[0]; d > 0; d /= 10) seq[ns++] = d % 10 - 1;
        for (int w = 0; w < NWG; ++w) {
            const int xcd = w & 7, tnl = (w >> 3) & 3, tm = (w >> 5) & 1, ks = w >> 6;
            int key[16];
            for (int k = 0; k < 16; ++k) {
                const int ph = k + sg[2] * ks + sg[3] * xcd;
                const int mine = seq[(ph + sg[1] * tm) % ns], other = seq[(ph + sg[1] * (1 - tm)) % ns];
                int lead = 0;
                if (mine == tnl && other == tnl) lead = ((ph / ns + tm) & 1) ? sg[4] : sg[6];
                else if (mine == tnl) lead = sg[4];
                else if (other == tnl) lead = 0;
                else lead = (((ph + tnl * sg[7]) >> sg[8]) + tm + sg[9]) & 1 ? sg[5] : 0;
                key[k] = (4 * k - 4 * lead - (lead ? 2 : 0)) * 16 + k;
            }
            unsigned long long t = 0;
            for (int k = 0; k < 16; ++k) {
                int pos = 0;
                for (int q = 0; q < 16; ++q) pos += key[q] < key[k];
                t |= (unsigned long long)k << (4 * pos);
            }
            p.ord[w] = t;
        }
    }
    hipLaunchKernelGGL(gemm_a2a_kernel, dim3(NWG), dim3(NTHREADS), 0, stream, p);
    HIP_CHECK(hipGetLastError());
}
```

```cpp
#include <hip/hip_runtime.h>
#include <rccl/rccl.h>
#include <cstdio>
#include <cstdint>

#define HIP_CHECK(expr) do { hipError_t _e = (expr); if (_e != hipSuccess) { \
    fprintf(stderr, "HIP error %s at %s:%d\n", hipGetErrorString(_e), __FILE__, __LINE__); \
    return; } } while (0)

#define SWAPX 0
#define TLW 2
#define SWIDX 0
#define PREHEAT 12
#define SPEC 1
#define SPERIOD 256
#define WBMODE 3
#define XST_AUX 0
#define WBDELAY 48
#define POLLGATE 3
#define RECORD 1
#define SGRACE 0
#define XSLEEP 0
#define XSLEEP1 0
#define XSLEEP2 0
#define SPRIO 0
#define SWQUIET 1
#define SPINS 0
#define LPRIO 0
#define LATEHEAT 1
#define PREF 7
#define FSLEEP 16
#define PHDEL 0
#define PHGAP 0
#define PHDG 0
#define LATEZ 8
#define TOUCH 0
#define TSLEEP 2
#define TMASK 2
#define TROT 0
#define ZTH 4.9f
#define SG 4312431, 3, 2, 1, 2, 1, 5, 0, 0, 0
namespace {

typedef __attribute__((ext_vector_type(4))) short s16x4;
typedef __attribute__((ext_vector_type(8))) short s16x8;
typedef __attribute__((ext_vector_type(2))) __bf16 bf16x2;
typedef __attribute__((ext_vector_type(2))) float f32x2;
typedef __attribute__((ext_vector_type(4))) float f32x4;
typedef __attribute__((ext_vector_type(2))) unsigned u32x2;
typedef __attribute__((ext_vector_type(4))) unsigned u32x4;
typedef __attribute__((address_space(3))) s16x4* lds_s16x4_ptr;
typedef __attribute__((address_space(3))) s16x8* lds_s16x8_ptr;
typedef __attribute__((address_space(3))) f32x4* lds_f32x4_ptr;
typedef __attribute__((address_space(3))) volatile unsigned* lds_u32_ptr;

constexpr int NR = 8;
constexpr int M_LOC = 512;
constexpr int K_TOT = 4096;
constexpr int N_TOT = 4096;
constexpr int N_LOC = N_TOT / NR;
constexpr int BM = 256, BN = 128, BK = 64;
constexpr int NS = 3;
constexpr int A_BYTES = BM * BK * 2;
constexpr int B_BYTES = BK * BN * 2;
constexpr int STAGE_BYTES = A_BYTES + B_BYTES;
constexpr int NSTEP = 16;
constexpr int NWG = 256;
constexpr int NTHREADS = 512;
constexpr int LDS_CALL = NS * STAGE_BYTES;
constexpr int LDS_RED = LDS_CALL + 16;
constexpr int LDS_HF = LDS_RED + 16;
constexpr int LDS_RDY = LDS_CALL + 4;
constexpr int LDS_L16 = LDS_CALL + 8;
constexpr int LDS_DUMMY = LDS_HF + 32;
constexpr int LDS_BYTES = LDS_DUMMY + 256;
constexpr int LDS_HO = 2 * STAGE_BYTES;

constexpr size_t WS_XG = 0;
constexpr size_t WS_LGF = WS_XG + 2 * 8192;
constexpr size_t WS_LGS = WS_LGF + 16384;
constexpr size_t WS_DONE = WS_LGS + 16384;
constexpr size_t WS_CTR = WS_DONE + NWG * 128;
constexpr size_t WS_DIAG = WS_CTR + NWG * 128;
constexpr size_t WS_XB = 256 * 1024;
#define XB_STRIDE 24832
constexpr size_t WS_END = WS_XB + (size_t)NWG * XB_STRIDE;
static_assert(WS_DIAG + NWG * 16 * 8 <= WS_XB, "");

constexpr unsigned POLL_LIMIT = 1u << 20;

struct Params {
    const void* x;
    const void* w;
    void* my_ws;
    int rank;
    int pad;
    int pace[8];
    void* out[NR];
    void* ws[NR];
    unsigned long long ord[NWG];
};

__device__ __forceinline__ void glds16(unsigned voff, const void* sbase, unsigned lds_dst) {
    asm volatile("s_mov_b32 m0, %2\n\ts_nop 0\n\tglobal_load_lds_dwordx4 %0, %1"
                 :: "v"(voff), "s"(sbase), "s"(lds_dst) : "memory");
}
#define WAIT_VM(n) asm volatile("s_waitcnt vmcnt(%0)" :: "i"(n) : "memory")
#define WAIT_LGKM0() asm volatile("s_waitcnt lgkmcnt(0)" ::: "memory")
#define BARRIER() asm volatile("s_barrier" ::: "memory")
#define SB() __builtin_amdgcn_sched_barrier(0)
#define MFMA(acc, a, b) asm volatile("v_mfma_f32_16x16x32_bf16 %0, %1, %2, %0" : "+v"(acc) : "v"(a), "v"(b))
#define MFMA0(acc, a, b) asm volatile("v_mfma_f32_16x16x32_bf16 %0, %1, %2, 0" : "+v"(acc) : "v"(a), "v"(b))

__device__ __forceinline__ unsigned wave_max_l63(unsigned v) {
    v = max(v, (unsigned)__builtin_amdgcn_update_dpp(0, (int)v, 0x111, 0xf, 0xf, false));
    v = max(v, (unsigned)__builtin_amdgcn_update_dpp(0, (int)v, 0x112, 0xf, 0xf, false));
    v = max(v, (unsigned)__builtin_amdgcn_update_dpp(0, (int)v, 0x114, 0xf, 0xf, false));
    v = max(v, (unsigned)__builtin_amdgcn_update_dpp(0, (int)v, 0x118, 0xf, 0xf, false));
    v = max(v, (unsigned)__builtin_amdgcn_update_dpp(0, (int)v, 0x142, 0xa, 0xf, false));
    v = max(v, (unsigned)__builtin_amdgcn_update_dpp(0, (int)v, 0x143, 0xc, 0xf, false));
    return v;
}
__device__ __forceinline__ float wave_sum_l63(float v) {
    v += __int_as_float(__builtin_amdgcn_update_dpp(0, __float_as_int(v), 0x111, 0xf, 0xf, false));
    v += __int_as_float(__builtin_amdgcn_update_dpp(0, __float_as_int(v), 0x112, 0xf, 0xf, false));
    v += __int_as_float(__builtin_amdgcn_update_dpp(0, __float_as_int(v), 0x114, 0xf, 0xf, false));
    v += __int_as_float(__builtin_amdgcn_update_dpp(0, __float_as_int(v), 0x118, 0xf, 0xf, false));
    v += __int_as_float(__builtin_amdgcn_update_dpp(0, __float_as_int(v), 0x142, 0xa, 0xf, false));
    v += __int_as_float(__builtin_amdgcn_update_dpp(0, __float_as_int(v), 0x143, 0xc, 0xf, false));
    return v;
}
__device__ __forceinline__ unsigned wave_max_u32(unsigned v) { return (unsigned)__builtin_amdgcn_readlane((int)wave_max_l63(v), 63); }
__device__ __forceinline__ void publish_l63(const char* ws, const char* ws2, unsigned off, unsigned v, unsigned call) {
    asm volatile("s_lshl_b64 exec, 1, 63\n\tglobal_store_dwordx2 %0, %1, %2\n\tglobal_store_dwordx2 %0, %1, %3 sc1\n\ts_mov_b64 exec, -1"
                 :: "v"(off), "v"((u32x2){v, call}), "s"(ws), "s"(ws2) : "memory", "scc");
}
__device__ __forceinline__ void add2(f32x4 (&y)[4], const u32x4& s0, const u32x4& s1, const u32x4& g0, const u32x4& g1) {
    asm volatile("s_nop 1\n\tv_mfma_f32_16x16x32_bf16 %0, %4, %6, %0\n\tv_mfma_f32_16x16x32_bf16 %1, %5, %6, %1\n\t"
                 "v_mfma_f32_16x16x32_bf16 %2, %4, %7, %2\n\tv_mfma_f32_16x16x32_bf16 %3, %5, %7, %3\n\ts_nop 7"
                 : "+v"(y[0]), "+v"(y[1]), "+v"(y[2]), "+v"(y[3]) : "v"(s0), "v"(s1), "v"(g0), "v"(g1));
}
__device__ __forceinline__ void make_sel(int lane, u32x4& s0, u32x4& s1) {
    const unsigned one = (lane >> 4) == ((lane & 15) >> 2) ? (lane & 1 ? 0x3f800000u : 0x3f80u) : 0u;
    const unsigned x0 = lane & 2 ? 0u : one, x1 = lane & 2 ? one : 0u;
    s0 = (u32x4){x0, x1, 0u, 0u};
    s1 = (u32x4){0u, 0u, x0, x1};
    asm volatile("s_nop 1" : "+v"(s0), "+v"(s1));
}

__device__ __forceinline__ unsigned pk_bf16(float a, float b) {
    const bf16x2 v = __builtin_convertvector((f32x2){a, b}, bf16x2);
    return __builtin_bit_cast(unsigned, v);
}
__device__ __forceinline__ unsigned pk_bf16_tag(float a, float b, unsigned tag) {
    const unsigned t = (__float_as_uint(a) + ((1u - tag) << 16)) & 0xfffe0000u;
    return (pk_bf16(__uint_as_float(t), b) & ~1u) | tag;
}
__device__ __forceinline__ u32x4 pack_granule(const f32x4& a, const f32x4& b, unsigned tag) {
    u32x4 g;
    g[0] = pk_bf16_tag(a[0], a[1], tag);
    g[1] = pk_bf16(a[2], a[3]);
    g[2] = pk_bf16(b[0], b[1]);
    g[3] = pk_bf16(b[2], b[3]);
    return g;
}
__device__ __forceinline__ float add_lo(unsigned w, float c) {
    unsigned one_lo = 0x00003f80u;
    asm("" : "+s"(one_lo));
    return __builtin_amdgcn_fdot2_f32_bf16(__builtin_bit_cast(bf16x2, w), __builtin_bit_cast(bf16x2, one_lo), c, false);
}
__device__ __forceinline__ float add_hi(unsigned w, float c) {
    return __builtin_amdgcn_fdot2_f32_bf16(__builtin_bit_cast(bf16x2, w), __builtin_bit_cast(bf16x2, 0x3f800000u), c, false);
}
__device__ __forceinline__ void add_granule(f32x4& a, f32x4& b, const u32x4& g) {
    a[0] = add_lo(g[0], a[0]); a[1] = add_hi(g[0], a[1]); a[2] = add_lo(g[1], a[2]); a[3] = add_hi(g[1], a[3]);
    b[0] = add_lo(g[2], b[0]); b[1] = add_hi(g[2], b[1]); b[2] = add_lo(g[3], b[2]); b[3] = add_hi(g[3], b[3]);
}
__device__ __forceinline__ void tr4(unsigned& r0, unsigned& r1, unsigned& r2, unsigned& r3) {
    u32x2 t;
    t = __builtin_amdgcn_permlane32_swap(r0, r2, false, false); r0 = t[0]; r2 = t[1];
    t = __builtin_amdgcn_permlane32_swap(r1, r3, false, false); r1 = t[0]; r3 = t[1];
    t = __builtin_amdgcn_permlane16_swap(r0, r1, false, false); r0 = t[0]; r1 = t[1];
    t = __builtin_amdgcn_permlane16_swap(r2, r3, false, false); r2 = t[0]; r3 = t[1];
}

#define NCYC 2
#define STAMPT(i, t) do {} while (0)
#define STAMP(i) STAMPT(i, 256)

__device__ __forceinline__ unsigned wait_amax(unsigned lds_base, unsigned call) {
    for (unsigned it = 0;; ++it) {
        u32x2 r;
        asm volatile("ds_read_b64 %0, %1\n\ts_waitcnt lgkmcnt(0)" : "=v"(r) : "v"(lds_base + LDS_RED) : "memory");
        if (__builtin_amdgcn_readfirstlane(r[1]) == call) return __builtin_amdgcn_readfirstlane(r[0]);
        if (it > 64 * POLL_LIMIT) return 0xffffffffu;
        if (SPINS) __builtin_amdgcn_s_sleep(4);
    }
}
__device__ __forceinline__ float inv_scale(unsigned gmax) {
    const float scale = __uint_as_float(gmax) / 448.0f;
    return scale > 0.f ? 1.0f / scale : 0.f;
}

__device__ __forceinline__ char* xg_entry(const Params& p, int lane, int g) {
    char* b = (char*)p.ws[0];
#pragma unroll
    for (int i = 1; i < NR; ++i) {
        char* w = (char*)p.ws[i];
        asm("" : "+s"(w));
        b = (lane >> 3) == i ? w : b;
    }
    return b + WS_XG + (lane & 7) * 1024 + (p.rank * 8 + g) * 16;
}

template <int V> struct IntC { static constexpr int value = V; };

__global__ void __launch_bounds__(NTHREADS, 2) gemm_a2a_kernel(const Params p) {
    __shared__ __attribute__((aligned(1024))) char lds[LDS_BYTES];
    const int tid = threadIdx.x;
    const int lane = tid & 63;
    const int wave = __builtin_amdgcn_readfirstlane(tid >> 6);
    const int wg = blockIdx.x;
    const int rank = p.rank;
    char* const my_ws = (char*)p.my_ws;
    const char* const gx = (const char*)p.x;
    const char* const gw = (const char*)p.w;
    const unsigned long long otab = p.ord[wg];

    const int xcd = wg & 7, idx = wg >> 3;
    const int tnl = (idx + TROT) & 3, tm = (idx >> 2) & 1, ks = idx >> 3;
    const int row0 = tm * BM;
    const int col0 = xcd * N_LOC + tnl * BN;

    const int of = 0;
    const int hi = wave & 1;
    unsigned voffA[8], voffB[4];
    {
        unsigned ve = (unsigned)(((wave & 2) * 64 + hi * 16 + (lane >> 3)) * (K_TOT * 2) + (((lane & 7) ^ (lane >> 4)) * 16));
        asm("" : "+v"(ve));
        unsigned vo = (ve ^ 64u) + 8 * (K_TOT * 2);
        asm("" : "+v"(vo));
#pragma unroll
        for (int i = 0; i < 8; ++i) voffA[i] = (i & 1 ? vo : ve) | (unsigned)((((i >> 1) ^ 3 ^ ks) & 3) * 32 * (K_TOT * 2));
        unsigned vb = (unsigned)(((wave & 3) * 16 + (lane >> 4)) * (N_TOT * 2) + (((lane & 15) ^ (2 * (lane >> 4))) * 16));
        asm("" : "+v"(vb));
        voffB[0] = vb; voffB[1] = vb | 4 * (N_TOT * 2); voffB[2] = vb ^ (8 * (N_TOT * 2) + 128); voffB[3] = vb ^ (12 * (N_TOT * 2) + 128);
    }
    asm volatile("" : "+v"(voffA[0]), "+v"(voffA[1]), "+v"(voffA[2]), "+v"(voffA[3]), "+v"(voffA[4]), "+v"(voffA[5]), "+v"(voffA[6]), "+v"(voffA[7]),
                      "+v"(voffB[0]), "+v"(voffB[1]), "+v"(voffB[2]), "+v"(voffB[3]));
    unsigned offA = row0 * (K_TOT * 2) + ks * NSTEP * (BK * 2), offB = col0 * 2 + ks * NSTEP * (BK * N_TOT * 2);
    unsigned dA = (unsigned)(uintptr_t)lds + wave * 8192, dB = (unsigned)(uintptr_t)lds + A_BYTES + wave * 4096;
    asm volatile("" : "+s"(offA), "+s"(offB), "+s"(dA), "+s"(dB));
    asm volatile("" :: "s"(gx), "s"(gw), "s"(my_ws), "s"(rank), "s"(otab));
    auto ktile = [&](int s) {
        return ((int)(otab >> (4 * (s & 15))) + of) & 15;
    };
    unsigned* ctr = (unsigned*)(my_ws + WS_CTR + (size_t)wg * 128);
    const unsigned lds_base = (unsigned)(uintptr_t)lds;

    if (__builtin_expect(wave < 4, 1)) {
        if (LPRIO) __builtin_amdgcn_s_setprio(LPRIO);
        const char* const ga = gx + offA;
        const char* const gb = gw + offB;
        auto issue_at = [&](unsigned ka, unsigned kb, unsigned so) {
            const char* a = ga + ka * (BK * 2);
            const char* b = gb + kb * (BK * N_TOT * 2);
#pragma unroll
            for (int j = 0; j < 4; ++j) {
                glds16(voffB[j], b, dB + so + j * 1024);
                glds16(voffA[2 * j], a, dA + so + 2 * j * 1024);
                glds16(voffA[2 * j + 1], a, dA + so + (2 * j + 1) * 1024);
            }
        };
        {
            const char* b = gb + (unsigned)ktile(0) * (BK * N_TOT * 2);
#pragma unroll
            for (int j = 0; j < 4; ++j) glds16(voffB[j], b, dB + j * 1024);
        }
        if (!hi) {
            const char* a = ga + (unsigned)ktile(0) * (BK * 2);
#pragma unroll
            for (int j = 0; j < 8; ++j) glds16(voffA[j], a, dA + j * 1024);
        }
        issue_at(ktile(1 - hi), ktile(1), STAGE_BYTES);
        int ordB = ktile(lane), ordA = ktile(lane - hi);
        asm volatile("" : "+v"(ordA), "+v"(ordB));
        auto issue = [&](int u, unsigned so) { issue_at(__builtin_amdgcn_readlane(ordA, u), __builtin_amdgcn_readlane(ordB, u), so); };
        unsigned so = 2 * STAGE_BYTES;
#define TLS(k)
        for (int k = 0; k < NSTEP - 2; ++k) {
            WAIT_VM(12);
            BARRIER();
            TLS(k);
            issue(k + 2, so);
            so = (so == 2 * STAGE_BYTES) ? 0 : so + STAGE_BYTES;
        }
        const bool tch = TOUCH && wave == 0 && (TMASK >> tnl & 1);
        auto touch = [&]() {
            const char* const xb = my_ws + WS_XB + (size_t)wg * XB_STRIDE;
            const unsigned tv = lane * 128;
            asm volatile("s_mov_b32 m0, %4\n\ts_nop 0\n\tglobal_load_lds_dword %0, %1\n\tglobal_load_lds_dword %0, %2\n\tglobal_load_lds_dword %0, %3"
                         :: "v"(tv), "s"(xb), "s"(xb + 8192), "s"(xb + 16384), "s"(lds_base + LDS_DUMMY) : "memory");
        };
        if (TOUCH == 13 && tch) { touch(); WAIT_VM(15); } else WAIT_VM(12);
        BARRIER();
        TLS(14);
        if (TOUCH == 14 && tch) { __builtin_amdgcn_s_sleep(TSLEEP); touch(); }
        if (hi) {
            const char* a = ga + (unsigned)__builtin_amdgcn_readlane(ordA, 16) * (BK * 2);
#pragma unroll
            for (int j = 0; j < 8; ++j) glds16(voffA[j], a, dA + STAGE_BYTES + j * 1024);
        }
        const unsigned call = __builtin_amdgcn_readfirstlane(*(const unsigned*)(lds + LDS_CALL));
        const int wm = wave & 1;
        const unsigned par = call & 1u;
        const unsigned xv = wm * 8192 + lane * 16;
        char* const xge = xg_entry(p, lane, xcd) + par * 8192;
        const int orow = rank * M_LOC + row0 + wm * 128 + ks * 32 + (lane & 15);
        char* const o = (char*)p.out[xcd] + (size_t)orow * N_LOC + tnl * BN + (lane >> 4) * 16;
        char* const o2 = o + 16 * N_LOC;
        char* const dflag = (char*)p.ws[xcd] + WS_DONE + (size_t)(rank * 32 + idx) * 128 + wm * 8;
        const unsigned lg = WS_LGF + xcd * 2048 + 1024 + (idx * 2 + wm) * 8;
        const char* const ws2 = my_ws + (WS_LGS - WS_LGF);
        const char* const gsrc = my_ws + WS_XG + par * 8192 + xcd * 1024 + lane * 16;
        asm volatile("" :: "v"(xge), "v"(o), "v"(o2), "v"(dflag), "v"(lg), "s"(ws2), "v"(xv), "v"(gsrc));
        if (hi) WAIT_VM(8); else if (tch) WAIT_VM(3); else WAIT_VM(0);
        BARRIER();
        TLS(15);
        if (hi) {
            WAIT_VM(0);
            asm volatile("ds_write_b32 %0, %1" :: "v"(lds_base + LDS_L16 + (wave >> 1) * 4), "v"(1u) : "memory");
        }
        if (wave >= 2) {
            const __amdgpu_buffer_rsrc_t xr = __builtin_amdgcn_make_buffer_rsrc(my_ws + WS_XB, 0, NWG * XB_STRIDE, 0x00020000);
            TLS(16);
            u32x4 v[12];
            const unsigned xw = wm * 4096 + lane * 16;
            auto xld = [&](int b) {
#pragma unroll
                for (int q = 0; q < 4; ++q)
                    v[4 * b + q] = __builtin_amdgcn_raw_buffer_load_b128(xr, xw + q * 1024, (wg ^ ((3 - b) << 6)) * XB_STRIDE + b * 8192, 16);
            };
            f32x4 y[2][2][4];
            u32x4 sel0, sel1;
            make_sel(lane, sel0, sel1);
            if (SPRIO) __builtin_amdgcn_s_setprio(SPRIO);
            bool bad = false;
            for (unsigned it = 0;; ++it) {
                u32x2 f;
                asm volatile("ds_read_b64 %0, %1\n\ts_waitcnt lgkmcnt(0)" : "=v"(f) : "v"(lds_base + LDS_HF + wm * 8) : "memory");
                if (__builtin_amdgcn_readfirstlane(f[0] & f[1])) break;
                if (it > 64 * POLL_LIMIT) { bad = true; break; }
                if (SPINS) __builtin_amdgcn_s_sleep(SPINS);
            }
            if (XSLEEP) __builtin_amdgcn_s_sleep(XSLEEP);
            xld(0); xld(1);
            if (XSLEEP1) __builtin_amdgcn_s_sleep(XSLEEP1);
            xld(2);
            for (unsigned it = 0;; ++it) {
                u32x2 f;
                asm volatile("ds_read_b64 %0, %1\n\ts_waitcnt lgkmcnt(0)" : "=v"(f) : "v"(lds_base + LDS_HF + wm * 8) : "memory");
                if (__builtin_amdgcn_readfirstlane(f[0] & f[1] & 2u)) break;
                if (it > 64 * POLL_LIMIT) { bad = true; break; }
            }
#pragma unroll
            for (int n = 0; n < 8; ++n)
                y[0][n >> 2][n & 3] = *(lds_f32x4_ptr)(uintptr_t)(lds_base + LDS_HO + xv + n * 1024);
            unsigned pend = 7;
            const unsigned frdy = lds_base + LDS_RDY + wm;
            auto pass = [&]() {
#pragma unroll
                for (int b = 0; b < 3; ++b)
                    if (pend >> b & 1) {
                        const unsigned tags = v[4 * b][0] & v[4 * b + 1][0] & v[4 * b + 2][0] & v[4 * b + 3][0];
                        if (__all(tags & 1u)) {
                            pend &= ~(1u << b);
                            asm volatile("ds_write_b8 %0, %1" :: "v"(frdy), "v"(pend ^ 7u) : "memory");
                        }
                    }
#pragma unroll
                for (int b = 0; b < 3; ++b)
                    if (pend >> b & 1) xld(b);
            };
            pass();
            for (unsigned it = 0; pend; ++it) {
                if ((it & 255u) == 255u) asm volatile("buffer_wbl2 sc1\n\ts_waitcnt vmcnt(0)" ::: "memory");
                pass();
                if (it > POLL_LIMIT) { bad = true; pend = 0; }
            }
            unsigned pav = *(lds_u32_ptr)(uintptr_t)(lds_base + LDS_RED);
#pragma unroll
            for (int q = 0; q < 12; ++q) {
                MFMA(y[0][(q >> 1) & 1][2 * (q & 1)], sel0, v[q]);
                MFMA(y[0][(q >> 1) & 1][2 * (q & 1) + 1], sel1, v[q]);
            }
            asm volatile("s_nop 7" : "+v"(y[0][0][0]), "+v"(y[0][0][1]), "+v"(y[0][0][2]), "+v"(y[0][0][3]),
                                     "+v"(y[0][1][0]), "+v"(y[0][1][1]), "+v"(y[0][1][2]), "+v"(y[0][1][3]));
            float am = 0.f;
#pragma unroll
            for (int i = 0; i < 8; ++i)
#pragma unroll
                for (int e = 0; e < 4; ++e) am = fmaxf(am, fabsf(y[0][i >> 2][i & 3][e]));
            const unsigned av = wave_max_l63(__float_as_uint(am) | (bad ? 0xffffffffu : 0u));
            const float tv = wave_sum_l63(am) * (ZTH / (2.327f * 64.f));
            publish_l63(my_ws, ws2, lg, av, call);
            unsigned amb = (unsigned)__builtin_amdgcn_readlane((int)av, 63);
            unsigned told = 0;
            {
                asm volatile("" : "+v"(pav));
                const unsigned pa = __builtin_amdgcn_readfirstlane(pav);
                if (amb > max(pa, (unsigned)__builtin_amdgcn_readlane(__float_as_int(tv), 63))) {
                    told = amb;
                    asm volatile("global_store_dwordx2 %0, %1, off offset:8 sc0 sc1" :: "v"(xge), "v"(((unsigned long long)call << 32) | amb) : "memory");
                }
            }
            if (lane == 0) asm volatile("ds_add_u32 %0, %1" :: "v"(lds_base + LDS_RED + 8), "v"(1u) : "memory");
            STAMPT(7, 128);
            auto send = [&](auto hc, unsigned val) {
                constexpr int h = decltype(hc)::value;
                const float inv = inv_scale(val);
                char* const dst = h ? o2 : o;
                u32x4 q[2];
#pragma unroll
                for (int c = 0; c < 2; ++c) {
                    unsigned r[4];
#pragma unroll
                    for (int j = 0; j < 4; ++j) {
                        int w = __builtin_amdgcn_cvt_pk_fp8_f32(y[h][c][j][0] * inv, y[h][c][j][1] * inv, 0, false);
                        w = __builtin_amdgcn_cvt_pk_fp8_f32(y[h][c][j][2] * inv, y[h][c][j][3] * inv, w, true);
                        r[j] = val == 0xffffffffu ? 0x7f7f7f7fu : (unsigned)w;
                    }
                    tr4(r[0], r[1], r[2], r[3]);
                    q[c] = (u32x4){r[0], r[1], r[2], r[3]};
                }
                asm volatile("global_store_dwordx4 %0, %1, off sc0 sc1\n\t"
                             "global_store_dwordx4 %0, %2, off offset:64 sc0 sc1\n\ts_nop 1" :: "v"(dst), "v"(q[0]), "v"(q[1]) : "memory");
            };
            unsigned s0 = 0xfffffffeu, s1 = 0xfffffffdu, flagged = 0xfffffffeu;
            bool have1 = false;
            unsigned ts = (unsigned)__builtin_amdgcn_s_memrealtime() - SGRACE;
            ts += (2u * SPERIOD + (unsigned)((idx * 2 + wm) * SPERIOD >> 6) - ts % SPERIOD) % SPERIOD;
            for (unsigned it = 0;; ++it) {
                u32x2 r, f;
                unsigned long long now;
                asm volatile("ds_read_b64 %0, %3\n\tds_read_b64 %1, %4\n\ts_memrealtime %2\n\ts_waitcnt lgkmcnt(0)"
                             : "=&v"(r), "=&v"(f), "=s"(now) : "v"(lds_base + LDS_RED), "v"(lds_base + LDS_HF + 16 + wm * 8) : "memory");
                unsigned pa = __builtin_amdgcn_readfirstlane(r[0]);
                const unsigned st = __builtin_amdgcn_readfirstlane(r[1]);
                bool fin = st == call;
                if (it > 64 * POLL_LIMIT) { fin = true; pa = 0xffffffffu; }
                if (!have1) {
                    const unsigned f0 = __builtin_amdgcn_readfirstlane(f[0]), f1 = __builtin_amdgcn_readfirstlane(f[1]);
                    if (f0 & f1 & 0x80000000u) {
                        const unsigned fm = max(f0, f1);
                        const unsigned a1 = fm == 0xffffffffu ? fm : fm & 0x7fffffffu;
                        if (pa != 0u) told = max(told, a1);
                        amb = max(amb, a1);
                        have1 = true;
                        if (lane == 0) asm volatile("ds_add_u32 %0, %1" :: "v"(lds_base + LDS_RED + 8), "v"(16u) : "memory");
#pragma unroll
                        for (int n = 0; n < 8; ++n)
                            y[1][n >> 2][n & 3] = *(lds_f32x4_ptr)(uintptr_t)(lds_base + LDS_HO + 16384 + xv + n * 1024);
                    }
                }
                if (!have1 && pa != 0xffffffffu) fin = false;
                if (RECORD && pa != 0u && amb > max(pa, told)) {
                    told = amb;
                    asm volatile("global_store_dwordx2 %0, %1, off offset:8 sc0 sc1" :: "v"(xge), "v"(((unsigned long long)call << 32) | amb) : "memory");
                }
                const unsigned val = fin ? pa : max(pa, amb);
                bool go = fin;
                if (SPEC && (int)((unsigned)now - ts) >= 0) {
                    go = go || (st == ~call && pa != 0u && (RECORD || pa >= amb));
                    if ((int)((unsigned)now - ts) >= SGRACE || (go && (s0 != val || (s1 != val && have1)))) ts += SPERIOD;
                }
                if (go && s0 != val) {
                    send(IntC<0>{}, val);
                    s0 = val;
                    go = fin;
                }
                if (go && s1 != val && (have1 || fin)) {
                    send(IntC<1>{}, val);
                    s1 = val;
                }
                if (s0 == s1 && s0 != flagged) {
                    flagged = s0;
                    asm volatile("s_waitcnt vmcnt(0)" ::: "memory");
                    STAMPT(4, 128);
                    if (lane == 0 && xcd != rank)
                        asm volatile("global_store_dwordx2 %0, %1, off sc0 sc1"
                                     :: "v"(dflag), "v"(((unsigned long long)call << 32) | s0) : "memory");
                }
                if (fin) break;
            }
            return;
        }
        if (wave == 0) {
            u32x4 gv;
            unsigned best = 0;
            auto gstate = [&]() {
                const bool ok = gv[1] == call, started = gv[3] == call;
                const unsigned long long seen = __ballot(started);
                best = max(best, max(ok ? gv[0] : 0u, started ? gv[2] : 0u));
                const unsigned gmax = wave_max_u32(best);
                const unsigned st = __all(ok) ? call : ((seen >> (8 * xcd)) & 0xffu) ? ~call : 0u;
                asm volatile("ds_write_b64 %0, %1" :: "v"(lds_base + LDS_RED), "v"(((unsigned long long)st << 32) | gmax) : "memory");
                return st == call;
            };
            if (idx != SWIDX || !SWQUIET) {
                asm volatile("global_load_dwordx4 %0, %1, off sc0 sc1\n\ts_waitcnt vmcnt(0)" : "=&v"(gv) : "v"(gsrc) : "memory");
                gstate();
            }
            if (idx == SWIDX) {
                const char* lgs = my_ws + WS_LGF + xcd * 2048 + lane * 16;
                const char* lgt = my_ws + WS_LGF + xcd * 2048 + 1024 + lane * 8;
                unsigned m0 = 0, m1 = 0, m2 = 0;
                bool ok0 = false, ok1 = false, ok2 = false;
                unsigned it = 0;
                for (;;) {
                    const unsigned alt = ((it & 63u) == 63u) ? (WS_LGS - WS_LGF) : 0;
                    u32x4 v;
                    u32x2 u;
                    asm volatile("global_load_dwordx4 %0, %2, off sc1\n\tglobal_load_dwordx2 %1, %3, off sc1\n\ts_waitcnt vmcnt(0)"
                                 : "=&v"(v), "=&v"(u) : "v"(lgs + alt), "v"(lgt + alt) : "memory");
                    if (v[1] == call) { m0 = v[0]; ok0 = true; }
                    if (v[3] == call) { m1 = v[2]; ok1 = true; }
                    if (u[1] == call) { m2 = u[0]; ok2 = true; }
                    if (__all(ok0 && ok1 && ok2)) break;
                    if (++it > 4 * POLL_LIMIT) { m0 = 0xffffffffu; break; }
                }
                const unsigned m = wave_max_u32(max(max(m0, m1), m2));
                asm volatile("global_store_dwordx2 %0, %1, off sc0 sc1"
                             :: "v"(xge), "v"(((unsigned long long)call << 32) | m) : "memory");
                STAMPT(6, 0);
            }
            for (unsigned it = 0; POLLGATE && it < 64 * POLL_LIMIT; ++it) {
                unsigned c;
                asm volatile("ds_read_b32 %0, %1\n\ts_waitcnt lgkmcnt(0)" : "=v"(c) : "v"(lds_base + (POLLGATE >= 3 ? LDS_RDY : LDS_RED + 8)) : "memory");
                c = __builtin_amdgcn_readfirstlane(c);
                if (POLLGATE == 3 ? c == 0x0707u : POLLGATE == 4 ? __builtin_popcount(c) >= 4 : POLLGATE == 5 ? c != 0u : POLLGATE == 1 ? (c & 15u) == 2u : c == 34u) break;
                if (SPINS) __builtin_amdgcn_s_sleep(2);
            }
            for (unsigned it = 0;;) {
                asm volatile("global_load_dwordx4 %0, %1, off sc0 sc1\n\ts_waitcnt vmcnt(0)"
                             : "=&v"(gv) : "v"(gsrc) : "memory");
                if (++it > 4 * POLL_LIMIT) { gv[0] = 0xffffffffu; gv[1] = call; }
                if (gstate()) break;
                if ((it & 255u) == 0u) asm volatile("buffer_wbl2 sc1\n\ts_waitcnt vmcnt(0)" ::: "memory");
            }
            return;
        }
        if ((wg >> 5) == rank) return;
        const unsigned long long want = ((unsigned long long)call << 32) | wait_amax(lds_base, call);
        __builtin_amdgcn_s_sleep(FSLEEP);
        const char* f = my_ws + WS_DONE + (size_t)wg * 128 + (lane & 1) * 8;
        for (unsigned it = 0;; ++it) {
            unsigned long long v;
            asm volatile("global_load_dwordx2 %0, %1, off sc0 sc1\n\ts_waitcnt vmcnt(0)" : "=&v"(v) : "v"(f) : "memory");
            if (__all(v == want)) break;
            if (it > POLL_LIMIT) {
                if (lane == 0)
                    __hip_atomic_store((unsigned char*)p.out[rank] + (size_t)rank * M_LOC * N_LOC,
                                       (unsigned char)0x7f, __ATOMIC_RELAXED, __HIP_MEMORY_SCOPE_SYSTEM);
                break;
            }
        }
        STAMPT(5, 64);
        return;
    }
    const int cw = wave - 4;
    const int wm = cw >> 1, wn = cw & 1;
    unsigned call = *(const __attribute__((address_space(4))) unsigned*)(uintptr_t)ctr;
    char* wsp[NR];
#pragma unroll
    for (int i = 0; i < NR; ++i) wsp[i] = (char*)p.ws[i];
    unsigned rdA[2], rdB[4];
#pragma unroll
    for (int kk = 0; kk < 2; ++kk) {
        const int r = lane & 15;
        const int c = 4 * kk + (lane >> 4);
        rdA[kk] = lds_base + (unsigned)(wm * 16384 + r * 128 + ((c ^ ((r >> 1) & 7)) * 16));
    }
#pragma unroll
    for (int cn = 0; cn < 4; ++cn) {
        const int g = lane >> 4, q = (lane & 15) >> 2, pp = lane & 3;
        const int k = 8 * g + q;
        const int h = (k & 3) | (((k >> 3) & 1) << 2);
        const int u = (wn * 4 + cn) ^ h;
        rdB[cn] = lds_base + (unsigned)(A_BYTES + k * 256 + u * 32 + pp * 8);
    }
    f32x4 acc[8][4];
#pragma unroll
    for (int i = 0; i < LATEZ; ++i) {
#pragma unroll
        for (int j = 0; j < 4; ++j) acc[i][j] = (f32x4){0.f, 0.f, 0.f, 0.f};
        asm volatile("s_nop 1" : "+v"(acc[i][0]), "+v"(acc[i][1]), "+v"(acc[i][2]), "+v"(acc[i][3]));
    }
    s16x8 aq[2][2];
    s16x4 bl[2][4][2], bh[2][4][2];
    asm volatile("" : "+v"(rdA[0]), "+v"(rdA[1]), "+v"(rdB[0]), "+v"(rdB[1]), "+v"(rdB[2]), "+v"(rdB[3]));
    {
        u32x4 ra[4];
#pragma unroll
        for (int i = 0; i < 4; ++i)
#pragma unroll
            for (int e = 0; e < 4; ++e) ra[i][e] = (((unsigned)tid * 2654435761u + (i * 4 + e) * 0x9e3779b9u) * 0x85ebca6bu >> 3 & 0x80ff80ffu) | 0x3f003f00u;
        f32x4 d[4] = {};
        const int rounds = PREHEAT, pdel = PHDEL, pgap = PHGAP, pdg = PHDG;
#pragma unroll 1
        for (int r = 0; r < pdel; ++r) __builtin_amdgcn_s_sleep(1);
#pragma unroll 1
        for (int r = 0, gp = pgap * 8; r < rounds; ++r, gp += pdg) {
#pragma unroll
            for (int i = 0; i < 8; ++i) MFMA(d[i & 3], ra[i & 3], ra[(i + 1 + (i >> 2)) & 3]);
#pragma unroll 1
            for (int q = 8; q <= gp; q += 8) asm volatile("s_nop 7");
        }
        asm volatile("" :: "v"(d[0]), "v"(d[1]), "v"(d[2]), "v"(d[3]));
    }
    asm volatile("" : "+s"(call), "+s"(wsp[0]), "+s"(wsp[1]), "+s"(wsp[2]), "+s"(wsp[3]), "+s"(wsp[4]), "+s"(wsp[5]), "+s"(wsp[6]), "+s"(wsp[7]));

#define BFR(s, n, kk) __builtin_shufflevector(bl[s][n][kk], bh[s][n][kk], 0, 1, 2, 3, 4, 5, 6, 7)
#define RDA(kk, mt) (*(lds_s16x8_ptr)(uintptr_t)(rdA[kk] + (mt) * 2048))
#define RDB(kk, n, h) __builtin_amdgcn_ds_read_tr16_b64_v4i16((lds_s16x4_ptr)(uintptr_t)(rdB[n] + (kk) * 8192 + (h) * 1024))
    auto rdb = [&](auto sc, int j) {
        constexpr int s = decltype(sc)::value;
        const int n = j >> 2, kk = ((j >> 1) & 1) ^ (n & 1);
        if (j & 1) bh[s][n][kk] = RDB(kk, n, 1); else bl[s][n][kk] = RDB(kk, n, 0);
    };
    auto grp = [&](auto mtc, auto abc, auto bsc, int na, auto&& fill) {
        constexpr int mt = decltype(mtc)::value, ab = decltype(abc)::value, bs = decltype(bsc)::value;
#pragma unroll
        for (int i = 0; i < 8; ++i) {
            const int nt = i >> 1, kk = (i & 1) ^ (nt & 1);
            MFMA(acc[mt][nt], BFR(bs, nt, kk), aq[ab][kk]);
            SB();
            if (i < 2 && na >= 0) aq[1 - ab][i] = RDA(i, na);
            fill(i);
            SB();
        }
    };
#define I(v) IntC<v>{}
    auto none = [](int) {};
    unsigned slot = 0, sd = 0;
    auto next_slot = [&](int i) {
        if (i == 0) { sd = slot == NS - 1 ? (unsigned)(-(NS - 1) * STAGE_BYTES) : (unsigned)STAGE_BYTES; slot = slot == NS - 1 ? 0 : slot + 1; }
        else if (i < 5) rdB[i - 1] += sd;
        else rdA[i - 5] += sd;
    };
    auto interval = [&](auto sc, int k) {
        constexpr int s = decltype(sc)::value;
        BARRIER();
        SB();
        grp(I(3), I(1), I(1 - s), 4, [&](int i) { if (i >= 2 && i < 6) rdb(sc, i - 2); });
        grp(I(4), I(0), I(1 - s), 5, [&](int i) { if (i >= 2 && i < 6) rdb(sc, i + 2); });
        grp(I(5), I(1), I(1 - s), 6, [&](int i) { if (i >= 2 && i < 6) rdb(sc, i + 6); });
        grp(I(6), I(0), I(1 - s), 7, [&](int i) { if (i >= 2 && i < 6) rdb(sc, i + 10); });
        grp(I(7), I(1), I(1 - s), 0, none);
        grp(I(0), I(0), sc, 1, none);
        grp(I(1), I(1), sc, 2, [&](int i) { if (i >= 2 && i < 7) next_slot(i - 2); });
        grp(I(2), I(0), sc, 3, [&](int i) { if (i == 2 || i == 4) next_slot(4 + (i >> 1)); });
        WAIT_LGKM0();
        SB();
    };
    BARRIER();
    SB();
    aq[0][0] = RDA(0, 0);
    rdb(I(0), 0); rdb(I(0), 1);
    SB();
    aq[0][1] = RDA(1, 0);
    SB();
#pragma unroll
    for (int j = 2; j < 16; ++j) { rdb(I(0), j); if (j & 1) SB(); }
    grp(I(0), I(0), I(0), 1, none);
    grp(I(1), I(1), I(0), 2, [&](int i) { if (i >= 2 && i < 7) next_slot(i - 2); });
    grp(I(2), I(0), I(0), 3, [&](int i) { if (i == 2 || i == 4) next_slot(4 + (i >> 1)); });
#pragma unroll
    for (int i = LATEZ; i < 8; ++i) {
#pragma unroll
        for (int j = 0; j < 4; ++j) acc[i][j] = (f32x4){0.f, 0.f, 0.f, 0.f};
        asm volatile("s_nop 1" : "+v"(acc[i][0]), "+v"(acc[i][1]), "+v"(acc[i][2]), "+v"(acc[i][3]));
    }
    const unsigned xv = cw * 4096 + lane * 16;
    const __amdgpu_buffer_rsrc_t xr = __builtin_amdgcn_make_buffer_rsrc(my_ws + WS_XB, 0, NWG * XB_STRIDE, 0x00020000);
    const unsigned xo = wg * XB_STRIDE;
    const unsigned xw = cw * 2048 + lane * 16;
    unsigned lg = WS_LGF + xcd * 2048 + (idx * 4 + cw) * 8;
    const char* ws2 = my_ws + (WS_LGS - WS_LGF);
    unsigned xp[3];
#pragma unroll
    for (int b = 0; b < 3; ++b) xp[b] = (wg ^ ((3 - b) << 6)) * XB_STRIDE + b * 8192;
    unsigned fl16 = lds_base + LDS_L16 + wm * 4, fhf = lds_base + LDS_HF + cw * 4;
    asm volatile("" : "+v"(lg), "+s"(ws2), "+s"(xp[0]), "+s"(xp[1]), "+s"(xp[2]), "+v"(fl16), "+v"(fhf));
    u32x4 sel0, sel1;
    make_sel(lane, sel0, sel1);
    unsigned tmask = 0xfffe0000u;
    ++call;
    if (tid < 256 + 4) *(u32x4*)(lds + LDS_CALL + lane * 16) = (u32x4){lane ? 0u : call, 0u, 0u, 0u};
    const unsigned par = call & 1u;
    char* xge = wsp[0];
#pragma unroll
    for (int i = 1; i < NR; ++i) xge = (lane >> 3) == i ? wsp[i] : xge;
    xge += WS_XG + (lane & 7) * 1024 + (rank * 8 + xcd) * 16 + par * 8192;
    asm volatile("" : "+v"(tmask), "+v"(xge) : "v"(xv), "v"(xw));
    if (cw == 0 && idx == SWIDX) {
        asm volatile("global_store_dwordx2 %0, %1, off offset:8 sc0 sc1" :: "v"(xge), "v"((unsigned long long)call << 32) : "memory");
    }
    WAIT_LGKM0();
    SB();
    for (int k = 1; k < NSTEP - 1; k += 2) {
        interval(I(1), k);
        interval(I(0), k + 1);
    }
    u32x4 gp[3][2];
    auto xst = [&](auto mtc, int i) {
        constexpr int mt = decltype(mtc)::value;
        const int nt = i >= 4 ? 2 : 0;
        const int j = i & 3;
        u32x4& g = gp[mt & 3][nt >> 1];
        if (j == 1) {
            g[0] = (__float_as_uint(acc[mt][nt][0]) + (mt < 4 ? 0u : 0x10000u)) & tmask;
            g[1] = pk_bf16(acc[mt][nt][2], acc[mt][nt][3]);
        } else if (j == 2) {
            g[0] = pk_bf16(__uint_as_float(g[0]), acc[mt][nt][1]);
            g[0] = mt < 4 ? g[0] | 1u : g[0] & ~1u;
        } else if (j == 3) {
            g[2] = pk_bf16(acc[mt][nt + 1][0], acc[mt][nt + 1][1]);
            g[3] = pk_bf16(acc[mt][nt + 1][2], acc[mt][nt + 1][3]);
            if (mt < 4) __builtin_amdgcn_raw_buffer_store_b128(g, xr, xw + (nt >> 1) * 1024, xo + mt * 8192, XST_AUX);
            else asm volatile("" : "+v"(g));
        }
    };
    u32x4 v[6];
    auto xld = [&](int b, int q) {
        v[2 * b + q] = __builtin_amdgcn_raw_buffer_load_b128(xr, xw + q * 1024, xo + b * 8192, 16);
    };
    unsigned ho = lds_base + LDS_HO + xv;
    asm("" : "+v"(ho));
    constexpr int S1 = STAGE_BYTES / 2048;
    BARRIER();
    SB();
    grp(I(3), I(1), I(0), 4, [&](int i) { rdb(I(1), i); });
    grp(I(4), I(0), I(0), 0, [&](int i) { rdb(I(1), i + 8); });
    grp(I(0), I(1), I(1), 1, none);
    grp(I(1), I(0), I(1), 2, [&](int i) { xst(I(0), i); });
    grp(I(2), I(1), I(1), 3, [&](int i) { xst(I(1), i); });
    unsigned l16;
    grp(I(3), I(0), I(1), 5, [&](int i) { xst(I(2), i); if (i == PREF) *(lds_u32_ptr)(uintptr_t)fhf = 1u; });
    grp(I(5), I(1), I(0), 6, [&](int i) {
        if (i >= 1 && i < 5) *(lds_f32x4_ptr)(uintptr_t)(ho + (i - 1) * 1024) = acc[3][i - 1];
        if (i == 5) *(lds_u32_ptr)(uintptr_t)fhf = 3u;
    });
    grp(I(6), I(0), I(0), 7, [&](int i) { if (i == 0) l16 = *(lds_u32_ptr)(uintptr_t)fl16;
                                          if (i == 6) l16 = __builtin_amdgcn_readfirstlane(l16); });
    if (__builtin_expect(l16 == 0u, 0))
        for (unsigned it = 0; it < 64 * POLL_LIMIT; ++it)
            if (__builtin_amdgcn_readfirstlane(*(lds_u32_ptr)(uintptr_t)fl16)) break;
    SB();
    grp(I(7), I(1), I(0), S1 + 4, none);
    grp(I(4), I(0), I(1), S1 + 5, none);
    grp(I(5), I(1), I(1), S1 + 6, [&](int i) { xst(I(4), i); });
    grp(I(6), I(0), I(1), S1 + 7, [&](int i) { xst(I(5), i); });
    grp(I(7), I(1), I(1), -1, [&](int i) { xst(I(6), i); });
    asm volatile("s_nop 5" : "+v"(acc[7][0]), "+v"(acc[7][1]), "+v"(acc[7][2]), "+v"(acc[7][3]));
    SB();
    STAMP(1);
    bool bad = false;
    for (unsigned it = 0, todo = 7; todo; ++it) {
        unsigned n = (__builtin_amdgcn_readfirstlane(*(lds_u32_ptr)(uintptr_t)(lds_base + LDS_RDY)) >> (wm * 8)) & todo;
        if (it > 64 * POLL_LIMIT) { n = todo; bad = true; }
#pragma unroll
        for (int b = 0; b < 3; ++b)
            if (n >> b & 1) {
#pragma unroll
                for (int q = 0; q < 2; ++q)
                    __builtin_amdgcn_raw_buffer_store_b128(gp[b][q], xr, xw + q * 1024, xp[b], XST_AUX);
            }
        todo &= ~n;
    }
    if (XSLEEP2) __builtin_amdgcn_s_sleep(XSLEEP2);
    unsigned pend = 7;
    auto pass = [&]() {
#pragma unroll
        for (int b = 0; b < 3; ++b)
            if (pend >> b & 1) { xld(b, 0); xld(b, 1); }
#pragma unroll
        for (int b = 0; b < 3; ++b)
            if (pend >> b & 1) {
                if (__all(((v[2 * b][0] | v[2 * b + 1][0]) & 1u) == 0u)) {
                    pend &= ~(1u << b);
                    add2(acc[7], sel0, sel1, v[2 * b], v[2 * b + 1]);
                }
            }
    };
    pass();
    for (unsigned it = 0; pend; ++it) {
        if ((it & 255u) == 255u) asm volatile("buffer_wbl2 sc1\n\ts_waitcnt vmcnt(0)" ::: "memory");
        asm volatile("" ::: "memory");
        pass();
        if (it > POLL_LIMIT) { bad = true; pend = 0; }
    }
    unsigned pav = *(lds_u32_ptr)(uintptr_t)(lds_base + LDS_RED);
    float am = 0.f;
#pragma unroll
    for (int i = 0; i < 4; ++i)
#pragma unroll
        for (int e = 0; e < 4; ++e) am = fmaxf(am, fabsf(acc[7][i][e]));
    const unsigned av = wave_max_l63(__float_as_uint(am) | (bad ? 0xffffffffu : 0u));
    const float tv = wave_sum_l63(am) * (ZTH / (2.05f * 64.f));
    publish_l63(my_ws, ws2, lg, av, call);
    const unsigned amb = (unsigned)__builtin_amdgcn_readlane((int)av, 63);
    {
        asm volatile("" : "+v"(pav));
        const unsigned pa = __builtin_amdgcn_readfirstlane(pav);
        if (RECORD && amb > (pa != 0u ? pa : (unsigned)__builtin_amdgcn_readlane(__float_as_int(tv), 63)))
            asm volatile("global_store_dwordx2 %0, %1, off offset:8 sc0 sc1" :: "v"(xge), "v"(((unsigned long long)call << 32) | amb) : "memory");
    }
#pragma unroll
    for (int n = 0; n < 4; ++n) *(lds_f32x4_ptr)(uintptr_t)(ho + 16384 + n * 1024) = acc[7][n];
    asm volatile("" ::: "memory");
    *(lds_u32_ptr)(uintptr_t)(fhf + 16) = amb | 0x80000000u;
    STAMP(2);
    if (tid == 256) __hip_atomic_store(ctr, call, __ATOMIC_RELAXED, __HIP_MEMORY_SCOPE_AGENT);
    if (WBMODE != 2 && cw == 2 && idx == 1) {
        wait_amax(lds_base, call);
        __builtin_amdgcn_s_sleep(WBDELAY);
        asm volatile("buffer_wbl2 sc1" ::: "memory");
    }
}

}

#define PACE 0, 0, -1, 0, 0, 0, 0, 0
static const int g_pace[8] = {PACE};
extern "C" void kernel_launch(int rank, int nranks, ncclComm_t comm,
                              hipStream_t stream,
                              void* const* d_in, const int* in_sizes, int n_in,
                              void* d_out, int out_size,
                              const int* in_dtypes, int out_dtype,
                              void* const* d_out_all,
                              void* const* d_ws_all, size_t ws_bytes) {
    (void)comm; (void)in_dtypes; (void)out_dtype; (void)d_out;
    if (nranks != NR || n_in < 2 || (unsigned)rank >= (unsigned)NR || ws_bytes < WS_END ||
        in_sizes[0] != M_LOC * K_TOT || in_sizes[1] != K_TOT * N_TOT || out_size != N_TOT * N_LOC) {
        fprintf(stderr, "kernel_launch: unexpected problem shape\n");
        return;
    }
    Params p;
    p.x = d_in[0];
    p.w = d_in[1];
    for (int i = 0; i < NR; ++i) {
        p.out[i] = d_out_all[i];
        p.ws[i] = d_ws_all[i];
    }
    p.rank = rank;
    p.pad = 0;
    p.my_ws = d_ws_all[rank];
    for (int i = 0; i < 8; ++i) p.pace[i] = g_pace[i];
    const int sg[10] = {SG};
    {
        int seq[10], ns = 0;
        for (int d = sg[0]; d > 0; d /= 10) seq[ns++] = d % 10 - 1;
        for (int w = 0; w < NWG; ++w) {
            const int xcd = w & 7, tnl = ((w >> 3) + TROT) & 3, tm = (w >> 5) & 1, ks = w >> 6;
            int key[16];
            for (int k = 0; k < 16; ++k) {
                const int ph = k + sg[2] * ks + sg[3] * xcd;
                const int mine = seq[(ph + sg[1] * tm) % ns], other = seq[(ph + sg[1] * (1 - tm)) % ns];
                int lead = 0;
                if (mine == tnl && other == tnl) lead = ((ph / ns + tm) & 1) ? sg[4] : sg[6];
                else if (mine == tnl) lead = sg[4];
                else if (other == tnl) lead = 0;
                else lead = (((ph + tnl * sg[7]) >> sg[8]) + tm + sg[9]) & 1 ? sg[5] : 0;
                key[k] = (4 * k - 4 * lead - (lead ? 2 : 0)) * 16 + k;
            }
            unsigned long long t = 0;
            for (int k = 0; k < 16; ++k) {
                int pos = 0;
                for (int q = 0; q < 16; ++q) pos += key[q] < key[k];
                t |= (unsigned long long)k << (4 * pos);
            }
            p.ord[w] = t;
        }
    }
    hipLaunchKernelGGL(gemm_a2a_kernel, dim3(NWG), dim3(NTHREADS), 0, stream, p);
    HIP_CHECK(hipGetLastError());
}
```

```cpp
#include <hip/hip_runtime.h>
#include <rccl/rccl.h>
#include <cstdio>
#include <cstdint>

#define HIP_CHECK(expr) do { hipError_t _e = (expr); if (_e != hipSuccess) { \
    fprintf(stderr, "HIP error %s at %s:%d\n", hipGetErrorString(_e), __FILE__, __LINE__); \
    return; } } while (0)

#define SWAPX 0
#define TLW 2
#define SWIDX 0
#define PH1 19
#define SPEC 1
#define SPERIOD 256
#define WBMODE 3
#define XST_AUX 0
#define WBDELAY 48
#define POLLGATE 3
#define RECORD 1
#define SGRACE 0
#define XSLEEP 0
#define XSLEEP1 0
#define XSLEEP2 0
#define SPRIO 0
#define SWQUIET 1
#define SPINS 0
#define LPRIO 0
#define LATEHEAT 1
#define PREF 7
#define FSLEEP 16
#define PHDEL 2
#define LATEZ 8
#define TOUCH 0
#define TSLEEP 2
#define TMASK 2
#define TROT 0
#define ZTH 4.9f
#define SG 13, 0, 2, 1, 2, 1, 3, 0, 1, 0
#define ROT2 3
#define ROT3 12
namespace {

typedef __attribute__((ext_vector_type(4))) short s16x4;
typedef __attribute__((ext_vector_type(8))) short s16x8;
typedef __attribute__((ext_vector_type(2))) __bf16 bf16x2;
typedef __attribute__((ext_vector_type(2))) float f32x2;
typedef __attribute__((ext_vector_type(4))) float f32x4;
typedef __attribute__((ext_vector_type(2))) unsigned u32x2;
typedef __attribute__((ext_vector_type(4))) unsigned u32x4;
typedef __attribute__((address_space(3))) s16x4* lds_s16x4_ptr;
typedef __attribute__((address_space(3))) s16x8* lds_s16x8_ptr;
typedef __attribute__((address_space(3))) f32x4* lds_f32x4_ptr;
typedef __attribute__((address_space(3))) volatile unsigned* lds_u32_ptr;

constexpr int NR = 8;
constexpr int M_LOC = 512;
constexpr int K_TOT = 4096;
constexpr int N_TOT = 4096;
constexpr int N_LOC = N_TOT / NR;
constexpr int BM = 256, BN = 128, BK = 64;
constexpr int NS = 3;
constexpr int A_BYTES = BM * BK * 2;
constexpr int B_BYTES = BK * BN * 2;
constexpr int STAGE_BYTES = A_BYTES + B_BYTES;
constexpr int NSTEP = 16;
constexpr int NWG = 256;
constexpr int NTHREADS = 512;
constexpr int LDS_CALL = NS * STAGE_BYTES;
constexpr int LDS_RED = LDS_CALL + 16;
constexpr int LDS_HF = LDS_RED + 16;
constexpr int LDS_RDY = LDS_CALL + 4;
constexpr int LDS_L16 = LDS_CALL + 8;
constexpr int LDS_DUMMY = LDS_HF + 32;
constexpr int LDS_BYTES = LDS_DUMMY + 256;
constexpr int LDS_HO = 2 * STAGE_BYTES;

constexpr size_t WS_XG = 0;
constexpr size_t WS_LGF = WS_XG + 2 * 8192;
constexpr size_t WS_LGS = WS_LGF + 16384;
constexpr size_t WS_DONE = WS_LGS + 16384;
constexpr size_t WS_CTR = WS_DONE + NWG * 128;
constexpr size_t WS_DIAG = WS_CTR + NWG * 128;
constexpr size_t WS_XB = 256 * 1024;
#define XB_STRIDE 24832
constexpr size_t WS_END = WS_XB + (size_t)NWG * XB_STRIDE;
static_assert(WS_DIAG + NWG * 16 * 8 <= WS_XB, "");

constexpr unsigned POLL_LIMIT = 1u << 20;

struct Params {
    const void* x;
    const void* w;
    void* my_ws;
    int rank;
    int pad;
    int pace[8];
    void* out[NR];
    void* ws[NR];
    unsigned long long ord[NWG];
};

__device__ __forceinline__ void glds16(unsigned voff, const void* sbase, unsigned lds_dst) {
    asm volatile("s_mov_b32 m0, %2\n\ts_nop 0\n\tglobal_load_lds_dwordx4 %0, %1"
                 :: "v"(voff), "s"(sbase), "s"(lds_dst) : "memory");
}
#define WAIT_VM(n) asm volatile("s_waitcnt vmcnt(%0)" :: "i"(n) : "memory")
#define WAIT_LGKM0() asm volatile("s_waitcnt lgkmcnt(0)" ::: "memory")
#define BARRIER() asm volatile("s_barrier" ::: "memory")
#define SB() __builtin_amdgcn_sched_barrier(0)
#define MFMA(acc, a, b) asm volatile("v_mfma_f32_16x16x32_bf16 %0, %1, %2, %0" : "+v"(acc) : "v"(a), "v"(b))
#define MFMA0(acc, a, b) asm volatile("v_mfma_f32_16x16x32_bf16 %0, %1, %2, 0" : "+v"(acc) : "v"(a), "v"(b))

__device__ __forceinline__ unsigned wave_max_l63(unsigned v) {
    v = max(v, (unsigned)__builtin_amdgcn_update_dpp(0, (int)v, 0x111, 0xf, 0xf, false));
    v = max(v, (unsigned)__builtin_amdgcn_update_dpp(0, (int)v, 0x112, 0xf, 0xf, false));
    v = max(v, (unsigned)__builtin_amdgcn_update_dpp(0, (int)v, 0x114, 0xf, 0xf, false));
    v = max(v, (unsigned)__builtin_amdgcn_update_dpp(0, (int)v, 0x118, 0xf, 0xf, false));
    v = max(v, (unsigned)__builtin_amdgcn_update_dpp(0, (int)v, 0x142, 0xa, 0xf, false));
    v = max(v, (unsigned)__builtin_amdgcn_update_dpp(0, (int)v, 0x143, 0xc, 0xf, false));
    return v;
}
__device__ __forceinline__ float wave_sum_l63(float v) {
    v += __int_as_float(__builtin_amdgcn_update_dpp(0, __float_as_int(v), 0x111, 0xf, 0xf, false));
    v += __int_as_float(__builtin_amdgcn_update_dpp(0, __float_as_int(v), 0x112, 0xf, 0xf, false));
    v += __int_as_float(__builtin_amdgcn_update_dpp(0, __float_as_int(v), 0x114, 0xf, 0xf, false));
    v += __int_as_float(__builtin_amdgcn_update_dpp(0, __float_as_int(v), 0x118, 0xf, 0xf, false));
    v += __int_as_float(__builtin_amdgcn_update_dpp(0, __float_as_int(v), 0x142, 0xa, 0xf, false));
    v += __int_as_float(__builtin_amdgcn_update_dpp(0, __float_as_int(v), 0x143, 0xc, 0xf, false));
    return v;
}
__device__ __forceinline__ unsigned wave_max_u32(unsigned v) { return (unsigned)__builtin_amdgcn_readlane((int)wave_max_l63(v), 63); }
__device__ __forceinline__ void publish_l63(const char* ws, const char* ws2, unsigned off, unsigned v, unsigned call) {
    asm volatile("s_lshl_b64 exec, 1, 63\n\tglobal_store_dwordx2 %0, %1, %2\n\tglobal_store_dwordx2 %0, %1, %3 sc1\n\ts_mov_b64 exec, -1"
                 :: "v"(off), "v"((u32x2){v, call}), "s"(ws), "s"(ws2) : "memory", "scc");
}
__device__ __forceinline__ void add2(f32x4 (&y)[4], const u32x4& s0, const u32x4& s1, const u32x4& g0, const u32x4& g1) {
    asm volatile("s_nop 1\n\tv_mfma_f32_16x16x32_bf16 %0, %4, %6, %0\n\tv_mfma_f32_16x16x32_bf16 %1, %5, %6, %1\n\t"
                 "v_mfma_f32_16x16x32_bf16 %2, %4, %7, %2\n\tv_mfma_f32_16x16x32_bf16 %3, %5, %7, %3\n\ts_nop 7"
                 : "+v"(y[0]), "+v"(y[1]), "+v"(y[2]), "+v"(y[3]) : "v"(s0), "v"(s1), "v"(g0), "v"(g1));
}
__device__ __forceinline__ void make_sel(int lane, u32x4& s0, u32x4& s1) {
    const unsigned one = (lane >> 4) == ((lane & 15) >> 2) ? (lane & 1 ? 0x3f800000u : 0x3f80u) : 0u;
    const unsigned x0 = lane & 2 ? 0u : one, x1 = lane & 2 ? one : 0u;
    s0 = (u32x4){x0, x1, 0u, 0u};
    s1 = (u32x4){0u, 0u, x0, x1};
    asm volatile("s_nop 1" : "+v"(s0), "+v"(s1));
}

__device__ __forceinline__ unsigned pk_bf16(float a, float b) {
    const bf16x2 v = __builtin_convertvector((f32x2){a, b}, bf16x2);
    return __builtin_bit_cast(unsigned, v);
}
__device__ __forceinline__ unsigned pk_bf16_tag(float a, float b, unsigned tag) {
    const unsigned t = (__float_as_uint(a) + ((1u - tag) << 16)) & 0xfffe0000u;
    return (pk_bf16(__uint_as_float(t), b) & ~1u) | tag;
}
__device__ __forceinline__ u32x4 pack_granule(const f32x4& a, const f32x4& b, unsigned tag) {
    u32x4 g;
    g[0] = pk_bf16_tag(a[0], a[1], tag);
    g[1] = pk_bf16(a[2], a[3]);
    g[2] = pk_bf16(b[0], b[1]);
    g[3] = pk_bf16(b[2], b[3]);
    return g;
}
__device__ __forceinline__ float add_lo(unsigned w, float c) {
    unsigned one_lo = 0x00003f80u;
    asm("" : "+s"(one_lo));
    return __builtin_amdgcn_fdot2_f32_bf16(__builtin_bit_cast(bf16x2, w), __builtin_bit_cast(bf16x2, one_lo), c, false);
}
__device__ __forceinline__ float add_hi(unsigned w, float c) {
    return __builtin_amdgcn_fdot2_f32_bf16(__builtin_bit_cast(bf16x2, w), __builtin_bit_cast(bf16x2, 0x3f800000u), c, false);
}
__device__ __forceinline__ void add_granule(f32x4& a, f32x4& b, const u32x4& g) {
    a[0] = add_lo(g[0], a[0]); a[1] = add_hi(g[0], a[1]); a[2] = add_lo(g[1], a[2]); a[3] = add_hi(g[1], a[3]);
    b[0] = add_lo(g[2], b[0]); b[1] = add_hi(g[2], b[1]); b[2] = add_lo(g[3], b[2]); b[3] = add_hi(g[3], b[3]);
}
__device__ __forceinline__ void tr4(unsigned& r0, unsigned& r1, unsigned& r2, unsigned& r3) {
    u32x2 t;
    t = __builtin_amdgcn_permlane32_swap(r0, r2, false, false); r0 = t[0]; r2 = t[1];
    t = __builtin_amdgcn_permlane32_swap(r1, r3, false, false); r1 = t[0]; r3 = t[1];
    t = __builtin_amdgcn_permlane16_swap(r0, r1, false, false); r0 = t[0]; r1 = t[1];
    t = __builtin_amdgcn_permlane16_swap(r2, r3, false, false); r2 = t[0]; r3 = t[1];
}

#define NCYC 2
#define STAMPT(i, t) do {} while (0)
#define STAMP(i) STAMPT(i, 256)

__device__ __forceinline__ unsigned wait_amax(unsigned lds_base, unsigned call) {
    for (unsigned it = 0;; ++it) {
        u32x2 r;
        asm volatile("ds_read_b64 %0, %1\n\ts_waitcnt lgkmcnt(0)" : "=v"(r) : "v"(lds_base + LDS_RED) : "memory");
        if (__builtin_amdgcn_readfirstlane(r[1]) == call) return __builtin_amdgcn_readfirstlane(r[0]);
        if (it > 64 * POLL_LIMIT) return 0xffffffffu;
        if (SPINS) __builtin_amdgcn_s_sleep(4);
    }
}
__device__ __forceinline__ float inv_scale(unsigned gmax) {
    const float scale = __uint_as_float(gmax) / 448.0f;
    return scale > 0.f ? 1.0f / scale : 0.f;
}

__device__ __forceinline__ char* xg_entry(const Params& p, int lane, int g) {
    char* b = (char*)p.ws[0];
#pragma unroll
    for (int i = 1; i < NR; ++i) {
        char* w = (char*)p.ws[i];
        asm("" : "+s"(w));
        b = (lane >> 3) == i ? w : b;
    }
    return b + WS_XG + (lane & 7) * 1024 + (p.rank * 8 + g) * 16;
}

template <int V> struct IntC { static constexpr int value = V; };

__global__ void __launch_bounds__(NTHREADS, 2) gemm_a2a_kernel(const Params p) {
    __shared__ __attribute__((aligned(1024))) char lds[LDS_BYTES];
    const int tid = threadIdx.x;
    const int lane = tid & 63;
    const int wave = __builtin_amdgcn_readfirstlane(tid >> 6);
    const int wg = blockIdx.x;
    const int rank = p.rank;
    char* const my_ws = (char*)p.my_ws;
    const char* const gx = (const char*)p.x;
    const char* const gw = (const char*)p.w;
    const unsigned long long otab = p.ord[wg];

    const int xcd = wg & 7, idx = wg >> 3;
    const int tnl = (idx + TROT) & 3, tm = (idx >> 2) & 1, ks = idx >> 3;
    const int row0 = tm * BM;
    const int col0 = xcd * N_LOC + tnl * BN;

    const int of = 0;
    const int hi = wave & 1;
    unsigned voffA[8], voffB[4];
    {
        unsigned ve = (unsigned)(((wave & 2) * 64 + hi * 16 + (lane >> 3)) * (K_TOT * 2) + (((lane & 7) ^ (lane >> 4)) * 16));
        asm("" : "+v"(ve));
        unsigned vo = (ve ^ 64u) + 8 * (K_TOT * 2);
        asm("" : "+v"(vo));
#pragma unroll
        for (int i = 0; i < 8; ++i) voffA[i] = (i & 1 ? vo : ve) | (unsigned)((((i >> 1) ^ 3 ^ ks) & 3) * 32 * (K_TOT * 2));
        unsigned vb = (unsigned)(((wave & 3) * 16 + (lane >> 4)) * (N_TOT * 2) + (((lane & 15) ^ (2 * (lane >> 4))) * 16));
        asm("" : "+v"(vb));
        voffB[0] = vb; voffB[1] = vb | 4 * (N_TOT * 2); voffB[2] = vb ^ (8 * (N_TOT * 2) + 128); voffB[3] = vb ^ (12 * (N_TOT * 2) + 128);
    }
    asm volatile("" : "+v"(voffA[0]), "+v"(voffA[1]), "+v"(voffA[2]), "+v"(voffA[3]), "+v"(voffA[4]), "+v"(voffA[5]), "+v"(voffA[6]), "+v"(voffA[7]),
                      "+v"(voffB[0]), "+v"(voffB[1]), "+v"(voffB[2]), "+v"(voffB[3]));
    unsigned offA = row0 * (K_TOT * 2) + ks * NSTEP * (BK * 2), offB = col0 * 2 + ks * NSTEP * (BK * N_TOT * 2);
    unsigned dA = (unsigned)(uintptr_t)lds + wave * 8192, dB = (unsigned)(uintptr_t)lds + A_BYTES + wave * 4096;
    asm volatile("" : "+s"(offA), "+s"(offB), "+s"(dA), "+s"(dB));
    if (wave >= 4) {
        u32x4 ra[2];
#pragma unroll
        for (int i = 0; i < 8; ++i) ra[i >> 2][i & 3] = ((__umul24(tid, 0x9e3779u + i * 0x2c1b3u) + 0x5bd1e9u * i) & 0x80ff80ffu) | 0x3f003f00u;
        f32x4 d[4] = {};
#pragma unroll 1
        for (int r = 0; r < PH1; ++r) {
#pragma unroll
            for (int i = 0; i < 8; ++i) MFMA(d[i & 3], ra[i & 1], ra[(i >> 1) & 1]);
        }
        asm volatile("s_nop 7\n\ts_nop 7" :: "v"(d[0]), "v"(d[1]), "v"(d[2]), "v"(d[3]));
    }
    asm volatile("" :: "s"(gx), "s"(gw), "s"(my_ws), "s"(rank), "s"(otab));
    auto ktile = [&](int s) {
        return ((int)(otab >> (4 * (s & 15))) + of) & 15;
    };
    unsigned* ctr = (unsigned*)(my_ws + WS_CTR + (size_t)wg * 128);
    const unsigned lds_base = (unsigned)(uintptr_t)lds;

    if (__builtin_expect(wave < 4, 1)) {
        if (LPRIO) __builtin_amdgcn_s_setprio(LPRIO);
        const char* const ga = gx + offA;
        const char* const gb = gw + offB;
        auto issue_at = [&](unsigned ka, unsigned kb, unsigned so) {
            const char* a = ga + ka * (BK * 2);
            const char* b = gb + kb * (BK * N_TOT * 2);
#pragma unroll
            for (int j = 0; j < 4; ++j) {
                glds16(voffB[j], b, dB + so + j * 1024);
                glds16(voffA[2 * j], a, dA + so + 2 * j * 1024);
                glds16(voffA[2 * j + 1], a, dA + so + (2 * j + 1) * 1024);
            }
        };
        {
            const char* b = gb + (unsigned)ktile(0) * (BK * N_TOT * 2);
#pragma unroll
            for (int j = 0; j < 4; ++j) glds16(voffB[j], b, dB + j * 1024);
        }
        if (!hi) {
            const char* a = ga + (unsigned)ktile(0) * (BK * 2);
#pragma unroll
            for (int j = 0; j < 8; ++j) glds16(voffA[j], a, dA + j * 1024);
        }
        issue_at(ktile(1 - hi), ktile(1), STAGE_BYTES);
        int ordB = ktile(lane), ordA = ktile(lane - hi);
        asm volatile("" : "+v"(ordA), "+v"(ordB));
        auto issue = [&](int u, unsigned so) { issue_at(__builtin_amdgcn_readlane(ordA, u), __builtin_amdgcn_readlane(ordB, u), so); };
        unsigned so = 2 * STAGE_BYTES;
#define TLS(k)
        for (int k = 0; k < NSTEP - 2; ++k) {
            WAIT_VM(12);
            BARRIER();
            TLS(k);
            issue(k + 2, so);
            so = (so == 2 * STAGE_BYTES) ? 0 : so + STAGE_BYTES;
        }
        const bool tch = TOUCH && wave == 0 && (TMASK >> tnl & 1);
        auto touch = [&]() {
            const char* const xb = my_ws + WS_XB + (size_t)wg * XB_STRIDE;
            const unsigned tv = lane * 128;
            asm volatile("s_mov_b32 m0, %4\n\ts_nop 0\n\tglobal_load_lds_dword %0, %1\n\tglobal_load_lds_dword %0, %2\n\tglobal_load_lds_dword %0, %3"
                         :: "v"(tv), "s"(xb), "s"(xb + 8192), "s"(xb + 16384), "s"(lds_base + LDS_DUMMY) : "memory");
        };
        if (TOUCH == 13 && tch) { touch(); WAIT_VM(15); } else WAIT_VM(12);
        BARRIER();
        TLS(14);
        if (TOUCH == 14 && tch) { __builtin_amdgcn_s_sleep(TSLEEP); touch(); }
        if (hi) {
            const char* a = ga + (unsigned)__builtin_amdgcn_readlane(ordA, 16) * (BK * 2);
#pragma unroll
            for (int j = 0; j < 8; ++j) glds16(voffA[j], a, dA + STAGE_BYTES + j * 1024);
        }
        const unsigned call = __builtin_amdgcn_readfirstlane(*(const unsigned*)(lds + LDS_CALL));
        const int wm = wave & 1;
        const unsigned par = call & 1u;
        const unsigned xv = wm * 8192 + lane * 16;
        char* const xge = xg_entry(p, lane, xcd) + par * 8192;
        const int orow = rank * M_LOC + row0 + wm * 128 + ks * 32 + (lane & 15);
        char* const o = (char*)p.out[xcd] + (size_t)orow * N_LOC + tnl * BN + (lane >> 4) * 16;
        char* const o2 = o + 16 * N_LOC;
        char* const dflag = (char*)p.ws[xcd] + WS_DONE + (size_t)(rank * 32 + idx) * 128 + wm * 8;
        const unsigned lg = WS_LGF + xcd * 2048 + 1024 + (idx * 2 + wm) * 8;
        const char* const ws2 = my_ws + (WS_LGS - WS_LGF);
        const char* const gsrc = my_ws + WS_XG + par * 8192 + xcd * 1024 + lane * 16;
        asm volatile("" :: "v"(xge), "v"(o), "v"(o2), "v"(dflag), "v"(lg), "s"(ws2), "v"(xv), "v"(gsrc));
        if (hi) WAIT_VM(8); else if (tch) WAIT_VM(3); else WAIT_VM(0);
        BARRIER();
        TLS(15);
        if (hi) {
            WAIT_VM(0);
            asm volatile("ds_write_b32 %0, %1" :: "v"(lds_base + LDS_L16 + (wave >> 1) * 4), "v"(1u) : "memory");
        }
        if (wave >= 2) {
            const __amdgpu_buffer_rsrc_t xr = __builtin_amdgcn_make_buffer_rsrc(my_ws + WS_XB, 0, NWG * XB_STRIDE, 0x00020000);
            TLS(16);
            u32x4 v[12];
            const unsigned xw = wm * 4096 + lane * 16;
            auto xld = [&](int b) {
#pragma unroll
                for (int q = 0; q < 4; ++q)
                    v[4 * b + q] = __builtin_amdgcn_raw_buffer_load_b128(xr, xw + q * 1024, (wg ^ ((3 - b) << 6)) * XB_STRIDE + b * 8192, 16);
            };
            f32x4 y[2][2][4];
            u32x4 sel0, sel1;
            make_sel(lane, sel0, sel1);
            if (SPRIO) __builtin_amdgcn_s_setprio(SPRIO);
            bool bad = false;
            for (unsigned it = 0;; ++it) {
                u32x2 f;
                asm volatile("ds_read_b64 %0, %1\n\ts_waitcnt lgkmcnt(0)" : "=v"(f) : "v"(lds_base + LDS_HF + wm * 8) : "memory");
                if (__builtin_amdgcn_readfirstlane(f[0] & f[1])) break;
                if (it > 64 * POLL_LIMIT) { bad = true; break; }
                if (SPINS) __builtin_amdgcn_s_sleep(SPINS);
            }
            if (XSLEEP) __builtin_amdgcn_s_sleep(XSLEEP);
            xld(0); xld(1);
            if (XSLEEP1) __builtin_amdgcn_s_sleep(XSLEEP1);
            xld(2);
            for (unsigned it = 0;; ++it) {
                u32x2 f;
                asm volatile("ds_read_b64 %0, %1\n\ts_waitcnt lgkmcnt(0)" : "=v"(f) : "v"(lds_base + LDS_HF + wm * 8) : "memory");
                if (__builtin_amdgcn_readfirstlane(f[0] & f[1] & 2u)) break;
                if (it > 64 * POLL_LIMIT) { bad = true; break; }
            }
#pragma unroll
            for (int n = 0; n < 8; ++n)
                y[0][n >> 2][n & 3] = *(lds_f32x4_ptr)(uintptr_t)(lds_base + LDS_HO + xv + n * 1024);
            unsigned pend = 7;
            const unsigned frdy = lds_base + LDS_RDY + wm;
            auto pass = [&]() {
#pragma unroll
                for (int b = 0; b < 3; ++b)
                    if (pend >> b & 1) {
                        const unsigned tags = v[4 * b][0] & v[4 * b + 1][0] & v[4 * b + 2][0] & v[4 * b + 3][0];
                        if (__all(tags & 1u)) {
                            pend &= ~(1u << b);
                            asm volatile("ds_write_b8 %0, %1" :: "v"(frdy), "v"(pend ^ 7u) : "memory");
                        }
                    }
#pragma unroll
                for (int b = 0; b < 3; ++b)
                    if (pend >> b & 1) xld(b);
            };
            pass();
            for (unsigned it = 0; pend; ++it) {
                if ((it & 255u) == 255u) asm volatile("buffer_wbl2 sc1\n\ts_waitcnt vmcnt(0)" ::: "memory");
                pass();
                if (it > POLL_LIMIT) { bad = true; pend = 0; }
            }
            unsigned pav = *(lds_u32_ptr)(uintptr_t)(lds_base + LDS_RED);
#pragma unroll
            for (int q = 0; q < 12; ++q) {
                MFMA(y[0][(q >> 1) & 1][2 * (q & 1)], sel0, v[q]);
                MFMA(y[0][(q >> 1) & 1][2 * (q & 1) + 1], sel1, v[q]);
            }
            asm volatile("s_nop 7" : "+v"(y[0][0][0]), "+v"(y[0][0][1]), "+v"(y[0][0][2]), "+v"(y[0][0][3]),
                                     "+v"(y[0][1][0]), "+v"(y[0][1][1]), "+v"(y[0][1][2]), "+v"(y[0][1][3]));
            float am = 0.f;
#pragma unroll
            for (int i = 0; i < 8; ++i)
#pragma unroll
                for (int e = 0; e < 4; ++e) am = fmaxf(am, fabsf(y[0][i >> 2][i & 3][e]));
            const unsigned av = wave_max_l63(__float_as_uint(am) | (bad ? 0xffffffffu : 0u));
            const float tv = wave_sum_l63(am) * (ZTH / (2.327f * 64.f));
            publish_l63(my_ws, ws2, lg, av, call);
            unsigned amb = (unsigned)__builtin_amdgcn_readlane((int)av, 63);
            unsigned told = 0;
            {
                asm volatile("" : "+v"(pav));
                const unsigned pa = __builtin_amdgcn_readfirstlane(pav);
                if (amb > max(pa, (unsigned)__builtin_amdgcn_readlane(__float_as_int(tv), 63))) {
                    told = amb;
                    asm volatile("global_store_dwordx2 %0, %1, off offset:8 sc0 sc1" :: "v"(xge), "v"(((unsigned long long)call << 32) | amb) : "memory");
                }
            }
            if (lane == 0) asm volatile("ds_add_u32 %0, %1" :: "v"(lds_base + LDS_RED + 8), "v"(1u) : "memory");
            STAMPT(7, 128);
            auto send = [&](auto hc, unsigned val) {
                constexpr int h = decltype(hc)::value;
                const float inv = inv_scale(val);
                char* const dst = h ? o2 : o;
                u32x4 q[2];
#pragma unroll
                for (int c = 0; c < 2; ++c) {
                    unsigned r[4];
#pragma unroll
                    for (int j = 0; j < 4; ++j) {
                        int w = __builtin_amdgcn_cvt_pk_fp8_f32(y[h][c][j][0] * inv, y[h][c][j][1] * inv, 0, false);
                        w = __builtin_amdgcn_cvt_pk_fp8_f32(y[h][c][j][2] * inv, y[h][c][j][3] * inv, w, true);
                        r[j] = val == 0xffffffffu ? 0x7f7f7f7fu : (unsigned)w;
                    }
                    tr4(r[0], r[1], r[2], r[3]);
                    q[c] = (u32x4){r[0], r[1], r[2], r[3]};
                }
                asm volatile("global_store_dwordx4 %0, %1, off sc0 sc1\n\t"
                             "global_store_dwordx4 %0, %2, off offset:64 sc0 sc1\n\ts_nop 1" :: "v"(dst), "v"(q[0]), "v"(q[1]) : "memory");
            };
            unsigned s0 = 0xfffffffeu, s1 = 0xfffffffdu, flagged = 0xfffffffeu;
            bool have1 = false;
            unsigned ts = (unsigned)__builtin_amdgcn_s_memrealtime() - SGRACE;
            ts += (2u * SPERIOD + (unsigned)((idx * 2 + wm) * SPERIOD >> 6) - ts % SPERIOD) % SPERIOD;
            for (unsigned it = 0;; ++it) {
                u32x2 r, f;
                unsigned long long now;
                asm volatile("ds_read_b64 %0, %3\n\tds_read_b64 %1, %4\n\ts_memrealtime %2\n\ts_waitcnt lgkmcnt(0)"
                             : "=&v"(r), "=&v"(f), "=s"(now) : "v"(lds_base + LDS_RED), "v"(lds_base + LDS_HF + 16 + wm * 8) : "memory");
                unsigned pa = __builtin_amdgcn_readfirstlane(r[0]);
                const unsigned st = __builtin_amdgcn_readfirstlane(r[1]);
                bool fin = st == call;
                if (it > 64 * POLL_LIMIT) { fin = true; pa = 0xffffffffu; }
                if (!have1) {
                    const unsigned f0 = __builtin_amdgcn_readfirstlane(f[0]), f1 = __builtin_amdgcn_readfirstlane(f[1]);
                    if (f0 & f1 & 0x80000000u) {
                        const unsigned fm = max(f0, f1);
                        const unsigned a1 = fm == 0xffffffffu ? fm : fm & 0x7fffffffu;
                        if (pa != 0u) told = max(told, a1);
                        amb = max(amb, a1);
                        have1 = true;
                        if (lane == 0) asm volatile("ds_add_u32 %0, %1" :: "v"(lds_base + LDS_RED + 8), "v"(16u) : "memory");
#pragma unroll
                        for (int n = 0; n < 8; ++n)
                            y[1][n >> 2][n & 3] = *(lds_f32x4_ptr)(uintptr_t)(lds_base + LDS_HO + 16384 + xv + n * 1024);
                    }
                }
                if (!have1 && pa != 0xffffffffu) fin = false;
                if (RECORD && pa != 0u && amb > max(pa, told)) {
                    told = amb;
                    asm volatile("global_store_dwordx2 %0, %1, off offset:8 sc0 sc1" :: "v"(xge), "v"(((unsigned long long)call << 32) | amb) : "memory");
                }
                const unsigned val = fin ? pa : max(pa, amb);
                bool go = fin;
                if (SPEC && (int)((unsigned)now - ts) >= 0) {
                    go = go || (st == ~call && pa != 0u && (RECORD || pa >= amb));
                    if ((int)((unsigned)now - ts) >= SGRACE || (go && (s0 != val || (s1 != val && have1)))) ts += SPERIOD;
                }
                if (go && s0 != val) {
                    send(IntC<0>{}, val);
                    s0 = val;
                    go = fin;
                }
                if (go && s1 != val && (have1 || fin)) {
                    send(IntC<1>{}, val);
                    s1 = val;
                }
                if (s0 == s1 && s0 != flagged) {
                    flagged = s0;
                    asm volatile("s_waitcnt vmcnt(0)" ::: "memory");
                    STAMPT(4, 128);
                    if (lane == 0 && xcd != rank)
                        asm volatile("global_store_dwordx2 %0, %1, off sc0 sc1"
                                     :: "v"(dflag), "v"(((unsigned long long)call << 32) | s0) : "memory");
                }
                if (fin) break;
            }
            return;
        }
        if (wave == 0) {
            u32x4 gv;
            unsigned best = 0;
            auto gstate = [&]() {
                const bool ok = gv[1] == call, started = gv[3] == call;
                const unsigned long long seen = __ballot(started);
                best = max(best, max(ok ? gv[0] : 0u, started ? gv[2] : 0u));
                const unsigned gmax = wave_max_u32(best);
                const unsigned st = __all(ok) ? call : ((seen >> (8 * xcd)) & 0xffu) ? ~call : 0u;
                asm volatile("ds_write_b64 %0, %1" :: "v"(lds_base + LDS_RED), "v"(((unsigned long long)st << 32) | gmax) : "memory");
                return st == call;
            };
            if (idx != SWIDX || !SWQUIET) {
                asm volatile("global_load_dwordx4 %0, %1, off sc0 sc1\n\ts_waitcnt vmcnt(0)" : "=&v"(gv) : "v"(gsrc) : "memory");
                gstate();
            }
            if (idx == SWIDX) {
                const char* lgs = my_ws + WS_LGF + xcd * 2048 + lane * 16;
                const char* lgt = my_ws + WS_LGF + xcd * 2048 + 1024 + lane * 8;
                unsigned m0 = 0, m1 = 0, m2 = 0;
                bool ok0 = false, ok1 = false, ok2 = false;
                unsigned it = 0;
                for (;;) {
                    const unsigned alt = ((it & 63u) == 63u) ? (WS_LGS - WS_LGF) : 0;
                    u32x4 v;
                    u32x2 u;
                    asm volatile("global_load_dwordx4 %0, %2, off sc1\n\tglobal_load_dwordx2 %1, %3, off sc1\n\ts_waitcnt vmcnt(0)"
                                 : "=&v"(v), "=&v"(u) : "v"(lgs + alt), "v"(lgt + alt) : "memory");
                    if (v[1] == call) { m0 = v[0]; ok0 = true; }
                    if (v[3] == call) { m1 = v[2]; ok1 = true; }
                    if (u[1] == call) { m2 = u[0]; ok2 = true; }
                    if (__all(ok0 && ok1 && ok2)) break;
                    if (++it > 4 * POLL_LIMIT) { m0 = 0xffffffffu; break; }
                }
                const unsigned m = wave_max_u32(max(max(m0, m1), m2));
                asm volatile("global_store_dwordx2 %0, %1, off sc0 sc1"
                             :: "v"(xge), "v"(((unsigned long long)call << 32) | m) : "memory");
                STAMPT(6, 0);
            }
            for (unsigned it = 0; POLLGATE && it < 64 * POLL_LIMIT; ++it) {
                unsigned c;
                asm volatile("ds_read_b32 %0, %1\n\ts_waitcnt lgkmcnt(0)" : "=v"(c) : "v"(lds_base + (POLLGATE >= 3 ? LDS_RDY : LDS_RED + 8)) : "memory");
                c = __builtin_amdgcn_readfirstlane(c);
                if (POLLGATE == 3 ? c == 0x0707u : POLLGATE == 4 ? __builtin_popcount(c) >= 4 : POLLGATE == 5 ? c != 0u : POLLGATE == 1 ? (c & 15u) == 2u : c == 34u) break;
                if (SPINS) __builtin_amdgcn_s_sleep(2);
            }
            for (unsigned it = 0;;) {
                asm volatile("global_load_dwordx4 %0, %1, off sc0 sc1\n\ts_waitcnt vmcnt(0)"
                             : "=&v"(gv) : "v"(gsrc) : "memory");
                if (++it > 4 * POLL_LIMIT) { gv[0] = 0xffffffffu; gv[1] = call; }
                if (gstate()) break;
                if ((it & 255u) == 0u) asm volatile("buffer_wbl2 sc1\n\ts_waitcnt vmcnt(0)" ::: "memory");
            }
            return;
        }
        if ((wg >> 5) == rank) return;
        const unsigned long long want = ((unsigned long long)call << 32) | wait_amax(lds_base, call);
        __builtin_amdgcn_s_sleep(FSLEEP);
        const char* f = my_ws + WS_DONE + (size_t)wg * 128 + (lane & 1) * 8;
        for (unsigned it = 0;; ++it) {
            unsigned long long v;
            asm volatile("global_load_dwordx2 %0, %1, off sc0 sc1\n\ts_waitcnt vmcnt(0)" : "=&v"(v) : "v"(f) : "memory");
            if (__all(v == want)) break;
            if (it > POLL_LIMIT) {
                if (lane == 0)
                    __hip_atomic_store((unsigned char*)p.out[rank] + (size_t)rank * M_LOC * N_LOC,
                                       (unsigned char)0x7f, __ATOMIC_RELAXED, __HIP_MEMORY_SCOPE_SYSTEM);
                break;
            }
        }
        STAMPT(5, 64);
        return;
    }
    const int cw = wave - 4;
    const int wm = cw >> 1, wn = cw & 1;
    unsigned call = *(const __attribute__((address_space(4))) unsigned*)(uintptr_t)ctr;
    char* wsp[NR];
#pragma unroll
    for (int i = 0; i < NR; ++i) wsp[i] = (char*)p.ws[i];
    unsigned rdA[2], rdB[4];
#pragma unroll
    for (int kk = 0; kk < 2; ++kk) {
        const int r = lane & 15;
        const int c = 4 * kk + (lane >> 4);
        rdA[kk] = lds_base + (unsigned)(wm * 16384 + r * 128 + ((c ^ ((r >> 1) & 7)) * 16));
    }
#pragma unroll
    for (int cn = 0; cn < 4; ++cn) {
        const int g = lane >> 4, q = (lane & 15) >> 2, pp = lane & 3;
        const int k = 8 * g + q;
        const int h = (k & 3) | (((k >> 3) & 1) << 2);
        const int u = (wn * 4 + cn) ^ h;
        rdB[cn] = lds_base + (unsigned)(A_BYTES + k * 256 + u * 32 + pp * 8);
    }
    f32x4 acc[8][4];
#pragma unroll
    for (int i = 0; i < LATEZ; ++i) {
#pragma unroll
        for (int j = 0; j < 4; ++j) acc[i][j] = (f32x4){0.f, 0.f, 0.f, 0.f};
        asm volatile("s_nop 1" : "+v"(acc[i][0]), "+v"(acc[i][1]), "+v"(acc[i][2]), "+v"(acc[i][3]));
    }
    s16x8 aq[2][2];
    s16x4 bl[2][4][2], bh[2][4][2];
    asm volatile("" : "+v"(rdA[0]), "+v"(rdA[1]), "+v"(rdB[0]), "+v"(rdB[1]), "+v"(rdB[2]), "+v"(rdB[3]));
    __builtin_amdgcn_s_sleep(PHDEL);
    asm volatile("" : "+s"(call), "+s"(wsp[0]), "+s"(wsp[1]), "+s"(wsp[2]), "+s"(wsp[3]), "+s"(wsp[4]), "+s"(wsp[5]), "+s"(wsp[6]), "+s"(wsp[7]));

#define BFR(s, n, kk) __builtin_shufflevector(bl[s][n][kk], bh[s][n][kk], 0, 1, 2, 3, 4, 5, 6, 7)
#define RDA(kk, mt) (*(lds_s16x8_ptr)(uintptr_t)(rdA[kk] + (mt) * 2048))
#define RDB(kk, n, h) __builtin_amdgcn_ds_read_tr16_b64_v4i16((lds_s16x4_ptr)(uintptr_t)(rdB[n] + (kk) * 8192 + (h) * 1024))
    auto rdb = [&](auto sc, int j) {
        constexpr int s = decltype(sc)::value;
        const int n = j >> 2, kk = ((j >> 1) & 1) ^ (n & 1);
        if (j & 1) bh[s][n][kk] = RDB(kk, n, 1); else bl[s][n][kk] = RDB(kk, n, 0);
    };
    auto grp = [&](auto mtc, auto abc, auto bsc, int na, auto&& fill) {
        constexpr int mt = decltype(mtc)::value, ab = decltype(abc)::value, bs = decltype(bsc)::value;
#pragma unroll
        for (int i = 0; i < 8; ++i) {
            const int nt = i >> 1, kk = (i & 1) ^ (nt & 1);
            MFMA(acc[mt][nt], BFR(bs, nt, kk), aq[ab][kk]);
            SB();
            if (i < 2 && na >= 0) aq[1 - ab][i] = RDA(i, na);
            fill(i);
            SB();
        }
    };
#define I(v) IntC<v>{}
    auto none = [](int) {};
    unsigned slot = 0, sd = 0;
    auto next_slot = [&](int i) {
        if (i == 0) { sd = slot == NS - 1 ? (unsigned)(-(NS - 1) * STAGE_BYTES) : (unsigned)STAGE_BYTES; slot = slot == NS - 1 ? 0 : slot + 1; }
        else if (i < 5) rdB[i - 1] += sd;
        else rdA[i - 5] += sd;
    };
    auto interval = [&](auto sc, int k) {
        constexpr int s = decltype(sc)::value;
        BARRIER();
        SB();
        grp(I(3), I(1), I(1 - s), 4, [&](int i) { if (i >= 2 && i < 6) rdb(sc, i - 2); });
        grp(I(4), I(0), I(1 - s), 5, [&](int i) { if (i >= 2 && i < 6) rdb(sc, i + 2); });
        grp(I(5), I(1), I(1 - s), 6, [&](int i) { if (i >= 2 && i < 6) rdb(sc, i + 6); });
        grp(I(6), I(0), I(1 - s), 7, [&](int i) { if (i >= 2 && i < 6) rdb(sc, i + 10); });
        grp(I(7), I(1), I(1 - s), 0, none);
        grp(I(0), I(0), sc, 1, none);
        grp(I(1), I(1), sc, 2, [&](int i) { if (i >= 2 && i < 7) next_slot(i - 2); });
        grp(I(2), I(0), sc, 3, [&](int i) { if (i == 2 || i == 4) next_slot(4 + (i >> 1)); });
        WAIT_LGKM0();
        SB();
    };
    BARRIER();
    SB();
    aq[0][0] = RDA(0, 0);
    rdb(I(0), 0); rdb(I(0), 1);
    SB();
    aq[0][1] = RDA(1, 0);
    SB();
#pragma unroll
    for (int j = 2; j < 16; ++j) { rdb(I(0), j); if (j & 1) SB(); }
    grp(I(0), I(0), I(0), 1, none);
    grp(I(1), I(1), I(0), 2, [&](int i) { if (i >= 2 && i < 7) next_slot(i - 2); });
    grp(I(2), I(0), I(0), 3, [&](int i) { if (i == 2 || i == 4) next_slot(4 + (i >> 1)); });
#pragma unroll
    for (int i = LATEZ; i < 8; ++i) {
#pragma unroll
        for (int j = 0; j < 4; ++j) acc[i][j] = (f32x4){0.f, 0.f, 0.f, 0.f};
        asm volatile("s_nop 1" : "+v"(acc[i][0]), "+v"(acc[i][1]), "+v"(acc[i][2]), "+v"(acc[i][3]));
    }
    const unsigned xv = cw * 4096 + lane * 16;
    const __amdgpu_buffer_rsrc_t xr = __builtin_amdgcn_make_buffer_rsrc(my_ws + WS_XB, 0, NWG * XB_STRIDE, 0x00020000);
    const unsigned xo = wg * XB_STRIDE;
    const unsigned xw = cw * 2048 + lane * 16;
    unsigned lg = WS_LGF + xcd * 2048 + (idx * 4 + cw) * 8;
    const char* ws2 = my_ws + (WS_LGS - WS_LGF);
    unsigned xp[3];
#pragma unroll
    for (int b = 0; b < 3; ++b) xp[b] = (wg ^ ((3 - b) << 6)) * XB_STRIDE + b * 8192;
    unsigned fl16 = lds_base + LDS_L16 + wm * 4, fhf = lds_base + LDS_HF + cw * 4;
    asm volatile("" : "+v"(lg), "+s"(ws2), "+s"(xp[0]), "+s"(xp[1]), "+s"(xp[2]), "+v"(fl16), "+v"(fhf));
    u32x4 sel0, sel1;
    make_sel(lane, sel0, sel1);
    unsigned tmask = 0xfffe0000u;
    ++call;
    if (tid < 256 + 4) *(u32x4*)(lds + LDS_CALL + lane * 16) = (u32x4){lane ? 0u : call, 0u, 0u, 0u};
    const unsigned par = call & 1u;
    char* xge = wsp[0];
#pragma unroll
    for (int i = 1; i < NR; ++i) xge = (lane >> 3) == i ? wsp[i] : xge;
    xge += WS_XG + (lane & 7) * 1024 + (rank * 8 + xcd) * 16 + par * 8192;
    asm volatile("" : "+v"(tmask), "+v"(xge) : "v"(xv), "v"(xw));
    if (cw == 0 && idx == SWIDX) {
        asm volatile("global_store_dwordx2 %0, %1, off offset:8 sc0 sc1" :: "v"(xge), "v"((unsigned long long)call << 32) : "memory");
    }
    WAIT_LGKM0();
    SB();
    for (int k = 1; k < NSTEP - 1; k += 2) {
        interval(I(1), k);
        interval(I(0), k + 1);
    }
    u32x4 gp[3][2];
    auto xst = [&](auto mtc, int i) {
        constexpr int mt = decltype(mtc)::value;
        const int nt = i >= 4 ? 2 : 0;
        const int j = i & 3;
        u32x4& g = gp[mt & 3][nt >> 1];
        if (j == 1) {
            g[0] = (__float_as_uint(acc[mt][nt][0]) + (mt < 4 ? 0u : 0x10000u)) & tmask;
            g[1] = pk_bf16(acc[mt][nt][2], acc[mt][nt][3]);
        } else if (j == 2) {
            g[0] = pk_bf16(__uint_as_float(g[0]), acc[mt][nt][1]);
            g[0] = mt < 4 ? g[0] | 1u : g[0] & ~1u;
        } else if (j == 3) {
            g[2] = pk_bf16(acc[mt][nt + 1][0], acc[mt][nt + 1][1]);
            g[3] = pk_bf16(acc[mt][nt + 1][2], acc[mt][nt + 1][3]);
            if (mt < 4) __builtin_amdgcn_raw_buffer_store_b128(g, xr, xw + (nt >> 1) * 1024, xo + mt * 8192, XST_AUX);
            else asm volatile("" : "+v"(g));
        }
    };
    u32x4 v[6];
    auto xld = [&](int b, int q) {
        v[2 * b + q] = __builtin_amdgcn_raw_buffer_load_b128(xr, xw + q * 1024, xo + b * 8192, 16);
    };
    unsigned ho = lds_base + LDS_HO + xv;
    asm("" : "+v"(ho));
    constexpr int S1 = STAGE_BYTES / 2048;
    BARRIER();
    SB();
    grp(I(3), I(1), I(0), 4, [&](int i) { rdb(I(1), i); });
    grp(I(4), I(0), I(0), 0, [&](int i) { rdb(I(1), i + 8); });
    grp(I(0), I(1), I(1), 1, none);
    grp(I(1), I(0), I(1), 2, [&](int i) { xst(I(0), i); });
    grp(I(2), I(1), I(1), 3, [&](int i) { xst(I(1), i); });
    unsigned l16;
    grp(I(3), I(0), I(1), 5, [&](int i) { xst(I(2), i); if (i == PREF) *(lds_u32_ptr)(uintptr_t)fhf = 1u; });
    grp(I(5), I(1), I(0), 6, [&](int i) {
        if (i >= 1 && i < 5) *(lds_f32x4_ptr)(uintptr_t)(ho + (i - 1) * 1024) = acc[3][i - 1];
        if (i == 5) *(lds_u32_ptr)(uintptr_t)fhf = 3u;
    });
    grp(I(6), I(0), I(0), 7, [&](int i) { if (i == 0) l16 = *(lds_u32_ptr)(uintptr_t)fl16;
                                          if (i == 6) l16 = __builtin_amdgcn_readfirstlane(l16); });
    if (__builtin_expect(l16 == 0u, 0))
        for (unsigned it = 0; it < 64 * POLL_LIMIT; ++it)
            if (__builtin_amdgcn_readfirstlane(*(lds_u32_ptr)(uintptr_t)fl16)) break;
    SB();
    grp(I(7), I(1), I(0), S1 + 4, none);
    grp(I(4), I(0), I(1), S1 + 5, none);
    grp(I(5), I(1), I(1), S1 + 6, [&](int i) { xst(I(4), i); });
    grp(I(6), I(0), I(1), S1 + 7, [&](int i) { xst(I(5), i); });
    grp(I(7), I(1), I(1), -1, [&](int i) { xst(I(6), i); });
    asm volatile("s_nop 5" : "+v"(acc[7][0]), "+v"(acc[7][1]), "+v"(acc[7][2]), "+v"(acc[7][3]));
    SB();
    STAMP(1);
    bool bad = false;
    for (unsigned it = 0, todo = 7; todo; ++it) {
        unsigned n = (__builtin_amdgcn_readfirstlane(*(lds_u32_ptr)(uintptr_t)(lds_base + LDS_RDY)) >> (wm * 8)) & todo;
        if (it > 64 * POLL_LIMIT) { n = todo; bad = true; }
#pragma unroll
        for (int b = 0; b < 3; ++b)
            if (n >> b & 1) {
#pragma unroll
                for (int q = 0; q < 2; ++q)
                    __builtin_amdgcn_raw_buffer_store_b128(gp[b][q], xr, xw + q * 1024, xp[b], XST_AUX);
            }
        todo &= ~n;
    }
    if (XSLEEP2) __builtin_amdgcn_s_sleep(XSLEEP2);
    unsigned pend = 7;
    auto pass = [&]() {
#pragma unroll
        for (int b = 0; b < 3; ++b)
            if (pend >> b & 1) { xld(b, 0); xld(b, 1); }
#pragma unroll
        for (int b = 0; b < 3; ++b)
            if (pend >> b & 1) {
                if (__all(((v[2 * b][0] | v[2 * b + 1][0]) & 1u) == 0u)) {
                    pend &= ~(1u << b);
                    add2(acc[7], sel0, sel1, v[2 * b], v[2 * b + 1]);
                }
            }
    };
    pass();
    for (unsigned it = 0; pend; ++it) {
        if ((it & 255u) == 255u) asm volatile("buffer_wbl2 sc1\n\ts_waitcnt vmcnt(0)" ::: "memory");
        asm volatile("" ::: "memory");
        pass();
        if (it > POLL_LIMIT) { bad = true; pend = 0; }
    }
    unsigned pav = *(lds_u32_ptr)(uintptr_t)(lds_base + LDS_RED);
    float am = 0.f;
#pragma unroll
    for (int i = 0; i < 4; ++i)
#pragma unroll
        for (int e = 0; e < 4; ++e) am = fmaxf(am, fabsf(acc[7][i][e]));
    const unsigned av = wave_max_l63(__float_as_uint(am) | (bad ? 0xffffffffu : 0u));
    const float tv = wave_sum_l63(am) * (ZTH / (2.05f * 64.f));
    publish_l63(my_ws, ws2, lg, av, call);
    const unsigned amb = (unsigned)__builtin_amdgcn_readlane((int)av, 63);
    {
        asm volatile("" : "+v"(pav));
        const unsigned pa = __builtin_amdgcn_readfirstlane(pav);
        if (RECORD && amb > (pa != 0u ? pa : (unsigned)__builtin_amdgcn_readlane(__float_as_int(tv), 63)))
            asm volatile("global_store_dwordx2 %0, %1, off offset:8 sc0 sc1" :: "v"(xge), "v"(((unsigned long long)call << 32) | amb) : "memory");
    }
#pragma unroll
    for (int n = 0; n < 4; ++n) *(lds_f32x4_ptr)(uintptr_t)(ho + 16384 + n * 1024) = acc[7][n];
    asm volatile("" ::: "memory");
    *(lds_u32_ptr)(uintptr_t)(fhf + 16) = amb | 0x80000000u;
    STAMP(2);
    if (tid == 256) __hip_atomic_store(ctr, call, __ATOMIC_RELAXED, __HIP_MEMORY_SCOPE_AGENT);
    if (WBMODE != 2 && cw == 2 && idx == 1) {
        wait_amax(lds_base, call);
        __builtin_amdgcn_s_sleep(WBDELAY);
        asm volatile("buffer_wbl2 sc1" ::: "memory");
    }
}

}

#define PACE 0, 0, -1, 0, 0, 0, 0, 0
static const int g_pace[8] = {PACE};
extern "C" void kernel_launch(int rank, int nranks, ncclComm_t comm,
                              hipStream_t stream,
                              void* const* d_in, const int* in_sizes, int n_in,
                              void* d_out, int out_size,
                              const int* in_dtypes, int out_dtype,
                              void* const* d_out_all,
                              void* const* d_ws_all, size_t ws_bytes) {
    (void)comm; (void)in_dtypes; (void)out_dtype; (void)d_out;
    if (nranks != NR || n_in < 2 || (unsigned)rank >= (unsigned)NR || ws_bytes < WS_END ||
        in_sizes[0] != M_LOC * K_TOT || in_sizes[1] != K_TOT * N_TOT || out_size != N_TOT * N_LOC) {
        fprintf(stderr, "kernel_launch: unexpected problem shape\n");
        return;
    }
    Params p;
    p.x = d_in[0];
    p.w = d_in[1];
    for (int i = 0; i < NR; ++i) {
        p.out[i] = d_out_all[i];
        p.ws[i] = d_ws_all[i];
    }
    p.rank = rank;
    p.pad = 0;
    p.my_ws = d_ws_all[rank];
    for (int i = 0; i < 8; ++i) p.pace[i] = g_pace[i];
    const int sg[10] = {SG};
    {
        int seq[10], ns = 0;
        for (int d = sg[0]; d > 0; d /= 10) seq[ns++] = d % 10 - 1;
        for (int w = 0; w < NWG; ++w) {
            const int xcd = w & 7, tnl = ((w >> 3) + TROT) & 3, tm = (w >> 5) & 1, ks = w >> 6;
            int key[16];
            for (int k = 0; k < 16; ++k) {
                const int ph = k + sg[2] * ks + sg[3] * xcd;
                const int mine = seq[(ph + sg[1] * tm) % ns], other = seq[(ph + sg[1] * (1 - tm)) % ns];
                int lead = 0;
                if (mine == tnl && other == tnl) lead = ((ph / ns + tm) & 1) ? sg[4] : sg[6];
                else if (mine == tnl) lead = sg[4];
                else if (other == tnl) lead = 0;
                else lead = (((ph + tnl * sg[7]) >> sg[8]) + tm + sg[9]) & 1 ? sg[5] : 0;
                key[k] = (4 * ((k + ROT2 * ks + ROT3 * xcd) & 15) - 4 * lead - (lead ? 2 : 0)) * 16 + k;
            }
            unsigned long long t = 0;
            for (int k = 0; k < 16; ++k) {
                int pos = 0;
                for (int q = 0; q < 16; ++q) pos += key[q] < key[k];
                t |= (unsigned long long)k << (4 * pos);
            }
            p.ord[w] = t;
        }
    }
    hipLaunchKernelGGL(gemm_a2a_kernel, dim3(NWG), dim3(NTHREADS), 0, stream, p);
    HIP_CHECK(hipGetLastError());
}
```

```cpp
#include <hip/hip_runtime.h>
#include <rccl/rccl.h>
#include <cstdio>
#include <cstdint>

#define HIP_CHECK(expr) do { hipError_t _e = (expr); if (_e != hipSuccess) { \
    fprintf(stderr, "HIP error %s at %s:%d\n", hipGetErrorString(_e), __FILE__, __LINE__); \
    return; } } while (0)

#define SWAPX 0
#define TLW 2
#define SWIDX 0
#define PH1 19
#define SPEC 1
#define SPERIOD 256
#define WBMODE 3
#define XST_AUX 0
#define WBDELAY 48
#define POLLGATE 3
#define RECORD 1
#define SGRACE 0
#define XSLEEP 0
#define XSLEEP1 0
#define XSLEEP2 0
#define SPRIO 0
#define SWQUIET 1
#define SPINS 0
#define LPRIO 0
#define LATEHEAT 1
#define PREF 7
#define FSLEEP 16
#define PHDEL 2
#define LATEZ 8
#define TOUCH 0
#define TSLEEP 2
#define TMASK 2
#define TROT 0
#define ZTH 4.9f
#define SG 13, 0, 2, 1, 2, 1, 3, 0, 1, 0
#define ROT2 3
#define ROT3 12
namespace {

typedef __attribute__((ext_vector_type(4))) short s16x4;
typedef __attribute__((ext_vector_type(8))) short s16x8;
typedef __attribute__((ext_vector_type(2))) __bf16 bf16x2;
typedef __attribute__((ext_vector_type(2))) float f32x2;
typedef __attribute__((ext_vector_type(4))) float f32x4;
typedef __attribute__((ext_vector_type(2))) unsigned u32x2;
typedef __attribute__((ext_vector_type(4))) unsigned u32x4;
typedef __attribute__((address_space(3))) s16x4* lds_s16x4_ptr;
typedef __attribute__((address_space(3))) s16x8* lds_s16x8_ptr;
typedef __attribute__((address_space(3))) f32x4* lds_f32x4_ptr;
typedef __attribute__((address_space(3))) volatile unsigned* lds_u32_ptr;

constexpr int NR = 8;
constexpr int M_LOC = 512;
constexpr int K_TOT = 4096;
constexpr int N_TOT = 4096;
constexpr int N_LOC = N_TOT / NR;
constexpr int BM = 256, BN = 128, BK = 64;
constexpr int NS = 3;
constexpr int A_BYTES = BM * BK * 2;
constexpr int B_BYTES = BK * BN * 2;
constexpr int STAGE_BYTES = A_BYTES + B_BYTES;
constexpr int NSTEP = 16;
constexpr int NWG = 256;
constexpr int NTHREADS = 512;
constexpr int LDS_CALL = NS * STAGE_BYTES;
constexpr int LDS_RED = LDS_CALL + 16;
constexpr int LDS_HF = LDS_RED + 16;
constexpr int LDS_RDY = LDS_CALL + 4;
constexpr int LDS_L16 = LDS_CALL + 8;
constexpr int LDS_DUMMY = LDS_HF + 32;
constexpr int LDS_BYTES = LDS_DUMMY + 256;
constexpr int LDS_HO = 2 * STAGE_BYTES;

constexpr size_t WS_XG = 0;
constexpr size_t WS_LGF = WS_XG + 2 * 8192;
constexpr size_t WS_LGS = WS_LGF + 16384;
constexpr size_t WS_DONE = WS_LGS + 16384;
constexpr size_t WS_CTR = WS_DONE + NWG * 128;
constexpr size_t WS_DIAG = WS_CTR + NWG * 128;
constexpr size_t WS_XB = 256 * 1024;
#define XB_STRIDE 24832
constexpr size_t WS_END = WS_XB + (size_t)NWG * XB_STRIDE;
static_assert(WS_DIAG + NWG * 16 * 8 <= WS_XB, "");

constexpr unsigned POLL_LIMIT = 1u << 20;

struct Params {
    const void* x;
    const void* w;
    void* my_ws;
    int rank;
    int pad;
    int pace[8];
    void* out[NR];
    void* ws[NR];
    unsigned long long ord[NWG];
};

__device__ __forceinline__ void glds16(unsigned voff, const void* sbase, unsigned lds_dst) {
    asm volatile("s_mov_b32 m0, %2\n\ts_nop 0\n\tglobal_load_lds_dwordx4 %0, %1"
                 :: "v"(voff), "s"(sbase), "s"(lds_dst) : "memory");
}
#define WAIT_VM(n) asm volatile("s_waitcnt vmcnt(%0)" :: "i"(n) : "memory")
#define WAIT_LGKM0() asm volatile("s_waitcnt lgkmcnt(0)" ::: "memory")
#define BARRIER() asm volatile("s_barrier" ::: "memory")
#define SB() __builtin_amdgcn_sched_barrier(0)
#define MFMA(acc, a, b) asm volatile("v_mfma_f32_16x16x32_bf16 %0, %1, %2, %0" : "+v"(acc) : "v"(a), "v"(b))
#define MFMA0(acc, a, b) asm volatile("v_mfma_f32_16x16x32_bf16 %0, %1, %2, 0" : "+v"(acc) : "v"(a), "v"(b))

__device__ __forceinline__ unsigned wave_max_l63(unsigned v) {
    v = max(v, (unsigned)__builtin_amdgcn_update_dpp(0, (int)v, 0x111, 0xf, 0xf, false));
    v = max(v, (unsigned)__builtin_amdgcn_update_dpp(0, (int)v, 0x112, 0xf, 0xf, false));
    v = max(v, (unsigned)__builtin_amdgcn_update_dpp(0, (int)v, 0x114, 0xf, 0xf, false));
    v = max(v, (unsigned)__builtin_amdgcn_update_dpp(0, (int)v, 0x118, 0xf, 0xf, false));
    v = max(v, (unsigned)__builtin_amdgcn_update_dpp(0, (int)v, 0x142, 0xa, 0xf, false));
    v = max(v, (unsigned)__builtin_amdgcn_update_dpp(0, (int)v, 0x143, 0xc, 0xf, false));
    return v;
}
__device__ __forceinline__ float wave_sum_l63(float v) {
    v += __int_as_float(__builtin_amdgcn_update_dpp(0, __float_as_int(v), 0x111, 0xf, 0xf, false));
    v += __int_as_float(__builtin_amdgcn_update_dpp(0, __float_as_int(v), 0x112, 0xf, 0xf, false));
    v += __int_as_float(__builtin_amdgcn_update_dpp(0, __float_as_int(v), 0x114, 0xf, 0xf, false));
    v += __int_as_float(__builtin_amdgcn_update_dpp(0, __float_as_int(v), 0x118, 0xf, 0xf, false));
    v += __int_as_float(__builtin_amdgcn_update_dpp(0, __float_as_int(v), 0x142, 0xa, 0xf, false));
    v += __int_as_float(__builtin_amdgcn_update_dpp(0, __float_as_int(v), 0x143, 0xc, 0xf, false));
    return v;
}
__device__ __forceinline__ unsigned wave_max_u32(unsigned v) { return (unsigned)__builtin_amdgcn_readlane((int)wave_max_l63(v), 63); }
__device__ __forceinline__ void publish_l63(const char* ws, const char* ws2, unsigned off, unsigned v, unsigned call) {
    asm volatile("s_lshl_b64 exec, 1, 63\n\tglobal_store_dwordx2 %0, %1, %2\n\tglobal_store_dwordx2 %0, %1, %3 sc1\n\ts_mov_b64 exec, -1"
                 :: "v"(off), "v"((u32x2){v, call}), "s"(ws), "s"(ws2) : "memory", "scc");
}
__device__ __forceinline__ void add2(f32x4 (&y)[4], const u32x4& s0, const u32x4& s1, const u32x4& g0, const u32x4& g1) {
    asm volatile("s_nop 1\n\tv_mfma_f32_16x16x32_bf16 %0, %4, %6, %0\n\tv_mfma_f32_16x16x32_bf16 %1, %5, %6, %1\n\t"
                 "v_mfma_f32_16x16x32_bf16 %2, %4, %7, %2\n\tv_mfma_f32_16x16x32_bf16 %3, %5, %7, %3\n\ts_nop 7"
                 : "+v"(y[0]), "+v"(y[1]), "+v"(y[2]), "+v"(y[3]) : "v"(s0), "v"(s1), "v"(g0), "v"(g1));
}
__device__ __forceinline__ void make_sel(int lane, u32x4& s0, u32x4& s1) {
    const unsigned one = (lane >> 4) == ((lane & 15) >> 2) ? (lane & 1 ? 0x3f800000u : 0x3f80u) : 0u;
    const unsigned x0 = lane & 2 ? 0u : one, x1 = lane & 2 ? one : 0u;
    s0 = (u32x4){x0, x1, 0u, 0u};
    s1 = (u32x4){0u, 0u, x0, x1};
    asm volatile("s_nop 1" : "+v"(s0), "+v"(s1));
}

__device__ __forceinline__ unsigned pk_bf16(float a, float b) {
    const bf16x2 v = __builtin_convertvector((f32x2){a, b}, bf16x2);
    return __builtin_bit_cast(unsigned, v);
}
__device__ __forceinline__ unsigned pk_bf16_tag(float a, float b, unsigned tag) {
    const unsigned t = (__float_as_uint(a) + ((1u - tag) << 16)) & 0xfffe0000u;
    return (pk_bf16(__uint_as_float(t), b) & ~1u) | tag;
}
__device__ __forceinline__ u32x4 pack_granule(const f32x4& a, const f32x4& b, unsigned tag) {
    u32x4 g;
    g[0] = pk_bf16_tag(a[0], a[1], tag);
    g[1] = pk_bf16(a[2], a[3]);
    g[2] = pk_bf16(b[0], b[1]);
    g[3] = pk_bf16(b[2], b[3]);
    return g;
}
__device__ __forceinline__ float add_lo(unsigned w, float c) {
    unsigned one_lo = 0x00003f80u;
    asm("" : "+s"(one_lo));
    return __builtin_amdgcn_fdot2_f32_bf16(__builtin_bit_cast(bf16x2, w), __builtin_bit_cast(bf16x2, one_lo), c, false);
}
__device__ __forceinline__ float add_hi(unsigned w, float c) {
    return __builtin_amdgcn_fdot2_f32_bf16(__builtin_bit_cast(bf16x2, w), __builtin_bit_cast(bf16x2, 0x3f800000u), c, false);
}
__device__ __forceinline__ void add_granule(f32x4& a, f32x4& b, const u32x4& g) {
    a[0] = add_lo(g[0], a[0]); a[1] = add_hi(g[0], a[1]); a[2] = add_lo(g[1], a[2]); a[3] = add_hi(g[1], a[3]);
    b[0] = add_lo(g[2], b[0]); b[1] = add_hi(g[2], b[1]); b[2] = add_lo(g[3], b[2]); b[3] = add_hi(g[3], b[3]);
}
__device__ __forceinline__ void tr4(unsigned& r0, unsigned& r1, unsigned& r2, unsigned& r3) {
    u32x2 t;
    t = __builtin_amdgcn_permlane32_swap(r0, r2, false, false); r0 = t[0]; r2 = t[1];
    t = __builtin_amdgcn_permlane32_swap(r1, r3, false, false); r1 = t[0]; r3 = t[1];
    t = __builtin_amdgcn_permlane16_swap(r0, r1, false, false); r0 = t[0]; r1 = t[1];
    t = __builtin_amdgcn_permlane16_swap(r2, r3, false, false); r2 = t[0]; r3 = t[1];
}

#define NCYC 2
#define STAMPT(i, t) do {} while (0)
#define STAMP(i) STAMPT(i, 256)

__device__ __forceinline__ unsigned wait_amax(unsigned lds_base, unsigned call) {
    for (unsigned it = 0;; ++it) {
        u32x2 r;
        asm volatile("ds_read_b64 %0, %1\n\ts_waitcnt lgkmcnt(0)" : "=v"(r) : "v"(lds_base + LDS_RED) : "memory");
        if (__builtin_amdgcn_readfirstlane(r[1]) == call) return __builtin_amdgcn_readfirstlane(r[0]);
        if (it > 64 * POLL_LIMIT) return 0xffffffffu;
        if (SPINS) __builtin_amdgcn_s_sleep(4);
    }
}
__device__ __forceinline__ float inv_scale(unsigned gmax) {
    const float scale = __uint_as_float(gmax) / 448.0f;
    return scale > 0.f ? 1.0f / scale : 0.f;
}

__device__ __forceinline__ char* xg_entry(const Params& p, int lane, int g) {
    char* b = (char*)p.ws[0];
#pragma unroll
    for (int i = 1; i < NR; ++i) {
        char* w = (char*)p.ws[i];
        asm("" : "+s"(w));
        b = (lane >> 3) == i ? w : b;
    }
    return b + WS_XG + (lane & 7) * 1024 + (p.rank * 8 + g) * 16;
}

template <int V> struct IntC { static constexpr int value = V; };

__global__ void __launch_bounds__(NTHREADS, 2) gemm_a2a_kernel(const Params p) {
    __shared__ __attribute__((aligned(1024))) char lds[LDS_BYTES];
    const int tid = threadIdx.x;
    const int lane = tid & 63;
    const int wave = __builtin_amdgcn_readfirstlane(tid >> 6);
    const int wg = blockIdx.x;
    const int rank = p.rank;
    char* const my_ws = (char*)p.my_ws;
    const char* const gx = (const char*)p.x;
    const char* const gw = (const char*)p.w;
    const unsigned long long otab = p.ord[wg];

    const int xcd = wg & 7, idx = wg >> 3;
    const int tnl = (idx + TROT) & 3, tm = (idx >> 2) & 1, ks = idx >> 3;
    const int row0 = tm * BM;
    const int col0 = xcd * N_LOC + tnl * BN;

    const int of = 0;
    const int hi = wave & 1;
    unsigned voffA[8], voffB[4];
    {
        unsigned ve = (unsigned)(((wave & 2) * 64 + hi * 16 + (lane >> 3)) * (K_TOT * 2) + (((lane & 7) ^ (lane >> 4)) * 16));
        asm("" : "+v"(ve));
        unsigned vo = (ve ^ 64u) + 8 * (K_TOT * 2);
        asm("" : "+v"(vo));
#pragma unroll
        for (int i = 0; i < 8; ++i) voffA[i] = (i & 1 ? vo : ve) | (unsigned)((((i >> 1) ^ 3 ^ ks) & 3) * 32 * (K_TOT * 2));
        unsigned vb = (unsigned)(((wave & 3) * 16 + (lane >> 4)) * (N_TOT * 2) + (((lane & 15) ^ (2 * (lane >> 4))) * 16));
        asm("" : "+v"(vb));
        voffB[0] = vb; voffB[1] = vb | 4 * (N_TOT * 2); voffB[2] = vb ^ (8 * (N_TOT * 2) + 128); voffB[3] = vb ^ (12 * (N_TOT * 2) + 128);
    }
    asm volatile("" : "+v"(voffA[0]), "+v"(voffA[1]), "+v"(voffA[2]), "+v"(voffA[3]), "+v"(voffA[4]), "+v"(voffA[5]), "+v"(voffA[6]), "+v"(voffA[7]),
                      "+v"(voffB[0]), "+v"(voffB[1]), "+v"(voffB[2]), "+v"(voffB[3]));
    unsigned offA = row0 * (K_TOT * 2) + ks * NSTEP * (BK * 2), offB = col0 * 2 + ks * NSTEP * (BK * N_TOT * 2);
    unsigned dA = (unsigned)(uintptr_t)lds + wave * 8192, dB = (unsigned)(uintptr_t)lds + A_BYTES + wave * 4096;
    asm volatile("" : "+s"(offA), "+s"(offB), "+s"(dA), "+s"(dB));
    if (wave >= 4) {
        u32x4 ra[2];
#pragma unroll
        for (int i = 0; i < 8; ++i) ra[i >> 2][i & 3] = ((__umul24(tid, 0x9e3779u + i * 0x2c1b3u) + 0x5bd1e9u * i) & 0x80ff80ffu) | 0x3f003f00u;
        f32x4 d[4] = {};
#pragma unroll 1
        for (int r = 0; r < PH1; ++r) {
#pragma unroll
            for (int i = 0; i < 8; ++i) MFMA(d[i & 3], ra[i & 1], ra[(i >> 1) & 1]);
        }
        asm volatile("s_nop 7\n\ts_nop 7" :: "v"(d[0]), "v"(d[1]), "v"(d[2]), "v"(d[3]));
    }
    asm volatile("" :: "s"(gx), "s"(gw), "s"(my_ws), "s"(rank), "s"(otab));
    auto ktile = [&](int s) {
        return ((int)(otab >> (4 * (s & 15))) + of) & 15;
    };
    unsigned* ctr = (unsigned*)(my_ws + WS_CTR + (size_t)wg * 128);
    const unsigned lds_base = (unsigned)(uintptr_t)lds;

    if (__builtin_expect(wave < 4, 1)) {
        if (LPRIO) __builtin_amdgcn_s_setprio(LPRIO);
        const char* const ga = gx + offA;
        const char* const gb = gw + offB;
        auto issue_at = [&](unsigned ka, unsigned kb, unsigned so) {
            const char* a = ga + ka * (BK * 2);
            const char* b = gb + kb * (BK * N_TOT * 2);
#pragma unroll
            for (int j = 0; j < 4; ++j) {
                glds16(voffB[j], b, dB + so + j * 1024);
                glds16(voffA[2 * j], a, dA + so + 2 * j * 1024);
                glds16(voffA[2 * j + 1], a, dA + so + (2 * j + 1) * 1024);
            }
        };
        {
            const char* b = gb + (unsigned)ktile(0) * (BK * N_TOT * 2);
#pragma unroll
            for (int j = 0; j < 4; ++j) glds16(voffB[j], b, dB + j * 1024);
        }
        if (!hi) {
            const char* a = ga + (unsigned)ktile(0) * (BK * 2);
#pragma unroll
            for (int j = 0; j < 8; ++j) glds16(voffA[j], a, dA + j * 1024);
        }
        issue_at(ktile(1 - hi), ktile(1), STAGE_BYTES);
        int ordB = ktile(lane), ordA = ktile(lane - hi);
        asm volatile("" : "+v"(ordA), "+v"(ordB));
        auto issue = [&](int u, unsigned so) { issue_at(__builtin_amdgcn_readlane(ordA, u), __builtin_amdgcn_readlane(ordB, u), so); };
        unsigned so = 2 * STAGE_BYTES;
#define TLS(k)
        for (int k = 0; k < NSTEP - 2; ++k) {
            WAIT_VM(12);
            BARRIER();
            TLS(k);
            issue(k + 2, so);
            so = (so == 2 * STAGE_BYTES) ? 0 : so + STAGE_BYTES;
        }
        const bool tch = TOUCH && wave == 0 && (TMASK >> tnl & 1);
        auto touch = [&]() {
            const char* const xb = my_ws + WS_XB + (size_t)wg * XB_STRIDE;
            const unsigned tv = lane * 128;
            asm volatile("s_mov_b32 m0, %4\n\ts_nop 0\n\tglobal_load_lds_dword %0, %1\n\tglobal_load_lds_dword %0, %2\n\tglobal_load_lds_dword %0, %3"
                         :: "v"(tv), "s"(xb), "s"(xb + 8192), "s"(xb + 16384), "s"(lds_base + LDS_DUMMY) : "memory");
        };
        if (TOUCH == 13 && tch) { touch(); WAIT_VM(15); } else WAIT_VM(12);
        BARRIER();
        TLS(14);
        if (TOUCH == 14 && tch) { __builtin_amdgcn_s_sleep(TSLEEP); touch(); }
        if (hi) {
            const char* a = ga + (unsigned)__builtin_amdgcn_readlane(ordA, 16) * (BK * 2);
#pragma unroll
            for (int j = 0; j < 8; ++j) glds16(voffA[j], a, dA + STAGE_BYTES + j * 1024);
        }
        const unsigned call = __builtin_amdgcn_readfirstlane(*(const unsigned*)(lds + LDS_CALL));
        const int wm = wave & 1;
        const unsigned par = call & 1u;
        const unsigned xv = wm * 8192 + lane * 16;
        char* const xge = xg_entry(p, lane, xcd) + par * 8192;
        const int orow = rank * M_LOC + row0 + wm * 128 + ks * 32 + (lane & 15);
        char* const o = (char*)p.out[xcd] + (size_t)orow * N_LOC + tnl * BN + (lane >> 4) * 16;
        char* const o2 = o + 16 * N_LOC;
        char* const dflag = (char*)p.ws[xcd] + WS_DONE + (size_t)(rank * 32 + idx) * 128 + wm * 8;
        const unsigned lg = WS_LGF + xcd * 2048 + 1024 + (idx * 2 + wm) * 8;
        const char* const ws2 = my_ws + (WS_LGS - WS_LGF);
        const char* const gsrc = my_ws + WS_XG + par * 8192 + xcd * 1024 + lane * 16;
        asm volatile("" :: "v"(xge), "v"(o), "v"(o2), "v"(dflag), "v"(lg), "s"(ws2), "v"(xv), "v"(gsrc));
        if (hi) WAIT_VM(8); else if (tch) WAIT_VM(3); else WAIT_VM(0);
        BARRIER();
        TLS(15);
        if (hi) {
            WAIT_VM(0);
            asm volatile("ds_write_b32 %0, %1" :: "v"(lds_base + LDS_L16 + (wave >> 1) * 4), "v"(1u) : "memory");
        }
        if (wave >= 2) {
            const __amdgpu_buffer_rsrc_t xr = __builtin_amdgcn_make_buffer_rsrc(my_ws + WS_XB, 0, NWG * XB_STRIDE, 0x00020000);
            TLS(16);
            u32x4 v[12];
            const unsigned xw = wm * 4096 + lane * 16;
            auto xld = [&](int b) {
#pragma unroll
                for (int q = 0; q < 4; ++q)
                    v[4 * b + q] = __builtin_amdgcn_raw_buffer_load_b128(xr, xw + q * 1024, (wg ^ ((3 - b) << 6)) * XB_STRIDE + b * 8192, 16);
            };
            f32x4 y[2][2][4];
            u32x4 sel0, sel1;
            make_sel(lane, sel0, sel1);
            if (SPRIO) __builtin_amdgcn_s_setprio(SPRIO);
            bool bad = false;
            for (unsigned it = 0;; ++it) {
                u32x2 f;
                asm volatile("ds_read_b64 %0, %1\n\ts_waitcnt lgkmcnt(0)" : "=v"(f) : "v"(lds_base + LDS_HF + wm * 8) : "memory");
                if (__builtin_amdgcn_readfirstlane(f[0] & f[1])) break;
                if (it > 64 * POLL_LIMIT) { bad = true; break; }
                if (SPINS) __builtin_amdgcn_s_sleep(SPINS);
            }
            if (XSLEEP) __builtin_amdgcn_s_sleep(XSLEEP);
            xld(0); xld(1);
            if (XSLEEP1) __builtin_amdgcn_s_sleep(XSLEEP1);
            xld(2);
            for (unsigned it = 0;; ++it) {
                u32x2 f;
                asm volatile("ds_read_b64 %0, %1\n\ts_waitcnt lgkmcnt(0)" : "=v"(f) : "v"(lds_base + LDS_HF + wm * 8) : "memory");
                if (__builtin_amdgcn_readfirstlane(f[0] & f[1] & 2u)) break;
                if (it > 64 * POLL_LIMIT) { bad = true; break; }
            }
#pragma unroll
            for (int n = 0; n < 8; ++n)
                y[0][n >> 2][n & 3] = *(lds_f32x4_ptr)(uintptr_t)(lds_base + LDS_HO + xv + n * 1024);
            unsigned pend = 7;
            const unsigned frdy = lds_base + LDS_RDY + wm;
            auto pass = [&]() {
#pragma unroll
                for (int b = 0; b < 3; ++b)
                    if (pend >> b & 1) {
                        const unsigned tags = v[4 * b][0] & v[4 * b + 1][0] & v[4 * b + 2][0] & v[4 * b + 3][0];
                        if (__all(tags & 1u)) {
                            pend &= ~(1u << b);
                            asm volatile("ds_write_b8 %0, %1" :: "v"(frdy), "v"(pend ^ 7u) : "memory");
                        }
                    }
#pragma unroll
                for (int b = 0; b < 3; ++b)
                    if (pend >> b & 1) xld(b);
            };
            pass();
            for (unsigned it = 0; pend; ++it) {
                if ((it & 255u) == 255u) asm volatile("buffer_wbl2 sc1\n\ts_waitcnt vmcnt(0)" ::: "memory");
                pass();
                if (it > POLL_LIMIT) { bad = true; pend = 0; }
            }
            unsigned pav = *(lds_u32_ptr)(uintptr_t)(lds_base + LDS_RED);
#pragma unroll
            for (int q = 0; q < 12; ++q) {
                MFMA(y[0][(q >> 1) & 1][2 * (q & 1)], sel0, v[q]);
                MFMA(y[0][(q >> 1) & 1][2 * (q & 1) + 1], sel1, v[q]);
            }
            asm volatile("s_nop 7" : "+v"(y[0][0][0]), "+v"(y[0][0][1]), "+v"(y[0][0][2]), "+v"(y[0][0][3]),
                                     "+v"(y[0][1][0]), "+v"(y[0][1][1]), "+v"(y[0][1][2]), "+v"(y[0][1][3]));
            float am = 0.f;
#pragma unroll
            for (int i = 0; i < 8; ++i)
#pragma unroll
                for (int e = 0; e < 4; ++e) am = fmaxf(am, fabsf(y[0][i >> 2][i & 3][e]));
            const unsigned av = wave_max_l63(__float_as_uint(am) | (bad ? 0xffffffffu : 0u));
            const float tv = wave_sum_l63(am) * (ZTH / (2.327f * 64.f));
            publish_l63(my_ws, ws2, lg, av, call);
            unsigned amb = (unsigned)__builtin_amdgcn_readlane((int)av, 63);
            unsigned told = 0;
            {
                asm volatile("" : "+v"(pav));
                const unsigned pa = __builtin_amdgcn_readfirstlane(pav);
                if (amb > max(pa, (unsigned)__builtin_amdgcn_readlane(__float_as_int(tv), 63))) {
                    told = amb;
                    asm volatile("global_store_dwordx2 %0, %1, off offset:8 sc0 sc1" :: "v"(xge), "v"(((unsigned long long)call << 32) | amb) : "memory");
                }
            }
            if (lane == 0) asm volatile("ds_add_u32 %0, %1" :: "v"(lds_base + LDS_RED + 8), "v"(1u) : "memory");
            STAMPT(7, 128);
            auto send = [&](auto hc, unsigned val) {
                constexpr int h = decltype(hc)::value;
                const float inv = inv_scale(val);
                char* const dst = h ? o2 : o;
                u32x4 q[2];
#pragma unroll
                for (int c = 0; c < 2; ++c) {
                    unsigned r[4];
#pragma unroll
                    for (int j = 0; j < 4; ++j) {
                        int w = __builtin_amdgcn_cvt_pk_fp8_f32(y[h][c][j][0] * inv, y[h][c][j][1] * inv, 0, false);
                        w = __builtin_amdgcn_cvt_pk_fp8_f32(y[h][c][j][2] * inv, y[h][c][j][3] * inv, w, true);
                        r[j] = val == 0xffffffffu ? 0x7f7f7f7fu : (unsigned)w;
                    }
                    tr4(r[0], r[1], r[2], r[3]);
                    q[c] = (u32x4){r[0], r[1], r[2], r[3]};
                }
                asm volatile("global_store_dwordx4 %0, %1, off sc0 sc1\n\t"
                             "global_store_dwordx4 %0, %2, off offset:64 sc0 sc1\n\ts_nop 1" :: "v"(dst), "v"(q[0]), "v"(q[1]) : "memory");
            };
            unsigned s0 = 0xfffffffeu, s1 = 0xfffffffdu, flagged = 0xfffffffeu;
            bool have1 = false;
            unsigned ts = (unsigned)__builtin_amdgcn_s_memrealtime() - SGRACE;
            ts += (2u * SPERIOD + (unsigned)((idx * 2 + wm) * SPERIOD >> 6) - ts % SPERIOD) % SPERIOD;
            for (unsigned it = 0;; ++it) {
                u32x2 r, f;
                unsigned long long now;
                asm volatile("ds_read_b64 %0, %3\n\tds_read_b64 %1, %4\n\ts_memrealtime %2\n\ts_waitcnt lgkmcnt(0)"
                             : "=&v"(r), "=&v"(f), "=s"(now) : "v"(lds_base + LDS_RED), "v"(lds_base + LDS_HF + 16 + wm * 8) : "memory");
                unsigned pa = __builtin_amdgcn_readfirstlane(r[0]);
                const unsigned st = __builtin_amdgcn_readfirstlane(r[1]);
                bool fin = st == call;
                if (it > 64 * POLL_LIMIT) { fin = true; pa = 0xffffffffu; }
                if (!have1) {
                    const unsigned f0 = __builtin_amdgcn_readfirstlane(f[0]), f1 = __builtin_amdgcn_readfirstlane(f[1]);
                    if (f0 & f1 & 0x80000000u) {
                        const unsigned fm = max(f0, f1);
                        const unsigned a1 = fm == 0xffffffffu ? fm : fm & 0x7fffffffu;
                        if (pa != 0u) told = max(told, a1);
                        amb = max(amb, a1);
                        have1 = true;
                        if (lane == 0) asm volatile("ds_add_u32 %0, %1" :: "v"(lds_base + LDS_RED + 8), "v"(16u) : "memory");
#pragma unroll
                        for (int n = 0; n < 8; ++n)
                            y[1][n >> 2][n & 3] = *(lds_f32x4_ptr)(uintptr_t)(lds_base + LDS_HO + 16384 + xv + n * 1024);
                    }
                }
                if (!have1 && pa != 0xffffffffu) fin = false;
                if (RECORD && pa != 0u && amb > max(pa, told)) {
                    told = amb;
                    asm volatile("global_store_dwordx2 %0, %1, off offset:8 sc0 sc1" :: "v"(xge), "v"(((unsigned long long)call << 32) | amb) : "memory");
                }
                const unsigned val = fin ? pa : max(pa, amb);
                bool go = fin;
                if (SPEC && (int)((unsigned)now - ts) >= 0) {
                    go = go || (st == ~call && pa != 0u && (RECORD || pa >= amb));
                    if ((int)((unsigned)now - ts) >= SGRACE || (go && (s0 != val || (s1 != val && have1)))) ts += SPERIOD;
                }
                if (go && s0 != val) {
                    send(IntC<0>{}, val);
                    s0 = val;
                    go = fin;
                }
                if (go && s1 != val && (have1 || fin)) {
                    send(IntC<1>{}, val);
                    s1 = val;
                }
                if (s0 == s1 && s0 != flagged) {
                    flagged = s0;
                    asm volatile("s_waitcnt vmcnt(0)" ::: "memory");
                    STAMPT(4, 128);
                    if (lane == 0 && xcd != rank)
                        asm volatile("global_store_dwordx2 %0, %1, off sc0 sc1"
                                     :: "v"(dflag), "v"(((unsigned long long)call << 32) | s0) : "memory");
                }
                if (fin) break;
            }
            return;
        }
        if (wave == 0) {
            u32x4 gv;
            unsigned best = 0;
            auto gstate = [&]() {
                const bool ok = gv[1] == call, started = gv[3] == call;
                const unsigned long long seen = __ballot(started);
                best = max(best, max(ok ? gv[0] : 0u, started ? gv[2] : 0u));
                const unsigned gmax = wave_max_u32(best);
                const unsigned st = __all(ok) ? call : ((seen >> (8 * xcd)) & 0xffu) ? ~call : 0u;
                asm volatile("ds_write_b64 %0, %1" :: "v"(lds_base + LDS_RED), "v"(((unsigned long long)st << 32) | gmax) : "memory");
                return st == call;
            };
            if (idx != SWIDX || !SWQUIET) {
                asm volatile("global_load_dwordx4 %0, %1, off sc0 sc1\n\ts_waitcnt vmcnt(0)" : "=&v"(gv) : "v"(gsrc) : "memory");
                gstate();
            }
            if (idx == SWIDX) {
                const char* lgs = my_ws + WS_LGF + xcd * 2048 + lane * 16;
                const char* lgt = my_ws + WS_LGF + xcd * 2048 + 1024 + lane * 8;
                unsigned m0 = 0, m1 = 0, m2 = 0;
                bool ok0 = false, ok1 = false, ok2 = false;
                unsigned it = 0;
                for (;;) {
                    const unsigned alt = ((it & 63u) == 63u) ? (WS_LGS - WS_LGF) : 0;
                    u32x4 v;
                    u32x2 u;
                    asm volatile("global_load_dwordx4 %0, %2, off sc1\n\tglobal_load_dwordx2 %1, %3, off sc1\n\ts_waitcnt vmcnt(0)"
                                 : "=&v"(v), "=&v"(u) : "v"(lgs + alt), "v"(lgt + alt) : "memory");
                    if (v[1] == call) { m0 = v[0]; ok0 = true; }
                    if (v[3] == call) { m1 = v[2]; ok1 = true; }
                    if (u[1] == call) { m2 = u[0]; ok2 = true; }
                    if (__all(ok0 && ok1 && ok2)) break;
                    if (++it > 4 * POLL_LIMIT) { m0 = 0xffffffffu; break; }
                }
                const unsigned m = wave_max_u32(max(max(m0, m1), m2));
                asm volatile("global_store_dwordx2 %0, %1, off sc0 sc1"
                             :: "v"(xge), "v"(((unsigned long long)call << 32) | m) : "memory");
                STAMPT(6, 0);
            }
            for (unsigned it = 0; POLLGATE && it < 64 * POLL_LIMIT; ++it) {
                unsigned c;
                asm volatile("ds_read_b32 %0, %1\n\ts_waitcnt lgkmcnt(0)" : "=v"(c) : "v"(lds_base + (POLLGATE >= 3 ? LDS_RDY : LDS_RED + 8)) : "memory");
                c = __builtin_amdgcn_readfirstlane(c);
                if (POLLGATE == 3 ? c == 0x0707u : POLLGATE == 4 ? __builtin_popcount(c) >= 4 : POLLGATE == 5 ? c != 0u : POLLGATE == 1 ? (c & 15u) == 2u : c == 34u) break;
                if (SPINS) __builtin_amdgcn_s_sleep(2);
            }
            for (unsigned it = 0;;) {
                asm volatile("global_load_dwordx4 %0, %1, off sc0 sc1\n\ts_waitcnt vmcnt(0)"
                             : "=&v"(gv) : "v"(gsrc) : "memory");
                if (++it > 4 * POLL_LIMIT) { gv[0] = 0xffffffffu; gv[1] = call; }
                if (gstate()) break;
                if ((it & 255u) == 0u) asm volatile("buffer_wbl2 sc1\n\ts_waitcnt vmcnt(0)" ::: "memory");
            }
            return;
        }
        if ((wg >> 5) == rank) return;
        const unsigned long long want = ((unsigned long long)call << 32) | wait_amax(lds_base, call);
        __builtin_amdgcn_s_sleep(FSLEEP);
        const char* f = my_ws + WS_DONE + (size_t)wg * 128 + (lane & 1) * 8;
        for (unsigned it = 0;; ++it) {
            unsigned long long v;
            asm volatile("global_load_dwordx2 %0, %1, off sc0 sc1\n\ts_waitcnt vmcnt(0)" : "=&v"(v) : "v"(f) : "memory");
            if (__all(v == want)) break;
            if (it > POLL_LIMIT) {
                if (lane == 0)
                    __hip_atomic_store((unsigned char*)p.out[rank] + (size_t)rank * M_LOC * N_LOC,
                                       (unsigned char)0x7f, __ATOMIC_RELAXED, __HIP_MEMORY_SCOPE_SYSTEM);
                break;
            }
        }
        STAMPT(5, 64);
        return;
    }
    const int cw = wave - 4;
    const int wm = cw >> 1, wn = cw & 1;
    unsigned call = *(const __attribute__((address_space(4))) unsigned*)(uintptr_t)ctr;
    char* wsp[NR];
#pragma unroll
    for (int i = 0; i < NR; ++i) wsp[i] = (char*)p.ws[i];
    unsigned rdA[2], rdB[4];
#pragma unroll
    for (int kk = 0; kk < 2; ++kk) {
        const int r = lane & 15;
        const int c = 4 * kk + (lane >> 4);
        rdA[kk] = lds_base + (unsigned)(wm * 16384 + r * 128 + ((c ^ ((r >> 1) & 7)) * 16));
    }
#pragma unroll
    for (int cn = 0; cn < 4; ++cn) {
        const int g = lane >> 4, q = (lane & 15) >> 2, pp = lane & 3;
        const int k = 8 * g + q;
        const int h = (k & 3) | (((k >> 3) & 1) << 2);
        const int u = (wn * 4 + cn) ^ h;
        rdB[cn] = lds_base + (unsigned)(A_BYTES + k * 256 + u * 32 + pp * 8);
    }
    f32x4 acc[8][4];
#pragma unroll
    for (int i = 0; i < LATEZ; ++i) {
#pragma unroll
        for (int j = 0; j < 4; ++j) acc[i][j] = (f32x4){0.f, 0.f, 0.f, 0.f};
        asm volatile("s_nop 1" : "+v"(acc[i][0]), "+v"(acc[i][1]), "+v"(acc[i][2]), "+v"(acc[i][3]));
    }
    s16x8 aq[2][2];
    s16x4 bl[2][4][2], bh[2][4][2];
    asm volatile("" : "+v"(rdA[0]), "+v"(rdA[1]), "+v"(rdB[0]), "+v"(rdB[1]), "+v"(rdB[2]), "+v"(rdB[3]));
    __builtin_amdgcn_s_sleep(PHDEL);
    asm volatile("" : "+s"(call), "+s"(wsp[0]), "+s"(wsp[1]), "+s"(wsp[2]), "+s"(wsp[3]), "+s"(wsp[4]), "+s"(wsp[5]), "+s"(wsp[6]), "+s"(wsp[7]));

#define BFR(s, n, kk) __builtin_shufflevector(bl[s][n][kk], bh[s][n][kk], 0, 1, 2, 3, 4, 5, 6, 7)
#define RDA(kk, mt) (*(lds_s16x8_ptr)(uintptr_t)(rdA[kk] + (mt) * 2048))
#define RDB(kk, n, h) __builtin_amdgcn_ds_read_tr16_b64_v4i16((lds_s16x4_ptr)(uintptr_t)(rdB[n] + (kk) * 8192 + (h) * 1024))
    auto rdb = [&](auto sc, int j) {
        constexpr int s = decltype(sc)::value;
        const int n = j >> 2, kk = (j >> 1) & 1;
        if (j & 1) bh[s][n][kk] = RDB(kk, n, 1); else bl[s][n][kk] = RDB(kk, n, 0);
    };
    auto grp = [&](auto mtc, auto abc, auto bsc, int na, auto&& fill) {
        constexpr int mt = decltype(mtc)::value, ab = decltype(abc)::value, bs = decltype(bsc)::value;
#pragma unroll
        for (int i = 0; i < 8; ++i) {
            const int nt = (mt & 1) ? 3 - (i >> 1) : i >> 1, kk = (i & 1) ^ (mt & 1);
            MFMA(acc[mt][nt], BFR(bs, nt, kk), aq[ab][kk]);
            SB();
            if (i < 2 && na >= 0) aq[1 - ab][i] = RDA(i, na);
            fill(i);
            SB();
        }
    };
#define I(v) IntC<v>{}
    auto none = [](int) {};
    unsigned slot = 0, sd = 0;
    auto next_slot = [&](int i) {
        if (i == 0) { sd = slot == NS - 1 ? (unsigned)(-(NS - 1) * STAGE_BYTES) : (unsigned)STAGE_BYTES; slot = slot == NS - 1 ? 0 : slot + 1; }
        else if (i < 5) rdB[i - 1] += sd;
        else rdA[i - 5] += sd;
    };
    auto interval = [&](auto sc, int k) {
        constexpr int s = decltype(sc)::value;
        BARRIER();
        SB();
        grp(I(3), I(1), I(1 - s), 4, [&](int i) { if (i >= 2 && i < 6) rdb(sc, i - 2); });
        grp(I(4), I(0), I(1 - s), 5, [&](int i) { if (i >= 2 && i < 6) rdb(sc, i + 2); });
        grp(I(5), I(1), I(1 - s), 6, [&](int i) { if (i >= 2 && i < 6) rdb(sc, i + 6); });
        grp(I(6), I(0), I(1 - s), 7, [&](int i) { if (i >= 2 && i < 6) rdb(sc, i + 10); });
        grp(I(7), I(1), I(1 - s), 0, none);
        grp(I(0), I(0), sc, 1, none);
        grp(I(1), I(1), sc, 2, [&](int i) { if (i >= 2 && i < 7) next_slot(i - 2); });
        grp(I(2), I(0), sc, 3, [&](int i) { if (i == 2 || i == 4) next_slot(4 + (i >> 1)); });
        WAIT_LGKM0();
        SB();
    };
    BARRIER();
    SB();
    aq[0][0] = RDA(0, 0);
    rdb(I(0), 0); rdb(I(0), 1);
    SB();
    aq[0][1] = RDA(1, 0);
    SB();
#pragma unroll
    for (int j = 2; j < 16; ++j) { rdb(I(0), j); if (j & 1) SB(); }
    grp(I(0), I(0), I(0), 1, none);
    grp(I(1), I(1), I(0), 2, [&](int i) { if (i >= 2 && i < 7) next_slot(i - 2); });
    grp(I(2), I(0), I(0), 3, [&](int i) { if (i == 2 || i == 4) next_slot(4 + (i >> 1)); });
#pragma unroll
    for (int i = LATEZ; i < 8; ++i) {
#pragma unroll
        for (int j = 0; j < 4; ++j) acc[i][j] = (f32x4){0.f, 0.f, 0.f, 0.f};
        asm volatile("s_nop 1" : "+v"(acc[i][0]), "+v"(acc[i][1]), "+v"(acc[i][2]), "+v"(acc[i][3]));
    }
    const unsigned xv = cw * 4096 + lane * 16;
    const __amdgpu_buffer_rsrc_t xr = __builtin_amdgcn_make_buffer_rsrc(my_ws + WS_XB, 0, NWG * XB_STRIDE, 0x00020000);
    const unsigned xo = wg * XB_STRIDE;
    const unsigned xw = cw * 2048 + lane * 16;
    unsigned lg = WS_LGF + xcd * 2048 + (idx * 4 + cw) * 8;
    const char* ws2 = my_ws + (WS_LGS - WS_LGF);
    unsigned xp[3];
#pragma unroll
    for (int b = 0; b < 3; ++b) xp[b] = (wg ^ ((3 - b) << 6)) * XB_STRIDE + b * 8192;
    unsigned fl16 = lds_base + LDS_L16 + wm * 4, fhf = lds_base + LDS_HF + cw * 4;
    asm volatile("" : "+v"(lg), "+s"(ws2), "+s"(xp[0]), "+s"(xp[1]), "+s"(xp[2]), "+v"(fl16), "+v"(fhf));
    u32x4 sel0, sel1;
    make_sel(lane, sel0, sel1);
    unsigned tmask = 0xfffe0000u;
    ++call;
    if (tid < 256 + 4) *(u32x4*)(lds + LDS_CALL + lane * 16) = (u32x4){lane ? 0u : call, 0u, 0u, 0u};
    const unsigned par = call & 1u;
    char* xge = wsp[0];
#pragma unroll
    for (int i = 1; i < NR; ++i) xge = (lane >> 3) == i ? wsp[i] : xge;
    xge += WS_XG + (lane & 7) * 1024 + (rank * 8 + xcd) * 16 + par * 8192;
    asm volatile("" : "+v"(tmask), "+v"(xge) : "v"(xv), "v"(xw));
    if (cw == 0 && idx == SWIDX) {
        asm volatile("global_store_dwordx2 %0, %1, off offset:8 sc0 sc1" :: "v"(xge), "v"((unsigned long long)call << 32) : "memory");
    }
    WAIT_LGKM0();
    SB();
    for (int k = 1; k < NSTEP - 1; k += 2) {
        interval(I(1), k);
        interval(I(0), k + 1);
    }
    u32x4 gp[3][2];
    auto xst = [&](auto mtc, int i) {
        constexpr int mt = decltype(mtc)::value;
        const int nt = (i >= 4) != (mt & 1) ? 2 : 0;
        const int j = i & 3;
        u32x4& g = gp[mt & 3][nt >> 1];
        if (j == 1) {
            g[0] = (__float_as_uint(acc[mt][nt][0]) + (mt < 4 ? 0u : 0x10000u)) & tmask;
            g[1] = pk_bf16(acc[mt][nt][2], acc[mt][nt][3]);
        } else if (j == 2) {
            g[0] = pk_bf16(__uint_as_float(g[0]), acc[mt][nt][1]);
            g[0] = mt < 4 ? g[0] | 1u : g[0] & ~1u;
        } else if (j == 3) {
            g[2] = pk_bf16(acc[mt][nt + 1][0], acc[mt][nt + 1][1]);
            g[3] = pk_bf16(acc[mt][nt + 1][2], acc[mt][nt + 1][3]);
            if (mt < 4) __builtin_amdgcn_raw_buffer_store_b128(g, xr, xw + (nt >> 1) * 1024, xo + mt * 8192, XST_AUX);
            else asm volatile("" : "+v"(g));
        }
    };
    u32x4 v[6];
    auto xld = [&](int b, int q) {
        v[2 * b + q] = __builtin_amdgcn_raw_buffer_load_b128(xr, xw + q * 1024, xo + b * 8192, 16);
    };
    unsigned ho = lds_base + LDS_HO + xv;
    asm("" : "+v"(ho));
    constexpr int S1 = STAGE_BYTES / 2048;
    BARRIER();
    SB();
    grp(I(3), I(1), I(0), 4, [&](int i) { rdb(I(1), i); });
    grp(I(4), I(0), I(0), 0, [&](int i) { rdb(I(1), i + 8); });
    grp(I(0), I(1), I(1), 1, none);
    grp(I(1), I(0), I(1), 2, [&](int i) { xst(I(0), i); });
    grp(I(2), I(1), I(1), 3, [&](int i) { xst(I(1), i); });
    unsigned l16;
    grp(I(3), I(0), I(1), 5, [&](int i) { xst(I(2), i); if (i == PREF) *(lds_u32_ptr)(uintptr_t)fhf = 1u; });
    grp(I(5), I(1), I(0), 6, [&](int i) {
        if (i >= 1 && i < 5) *(lds_f32x4_ptr)(uintptr_t)(ho + (4 - i) * 1024) = acc[3][4 - i];
        if (i == 5) *(lds_u32_ptr)(uintptr_t)fhf = 3u;
    });
    grp(I(6), I(0), I(0), 7, [&](int i) { if (i == 0) l16 = *(lds_u32_ptr)(uintptr_t)fl16;
                                          if (i == 6) l16 = __builtin_amdgcn_readfirstlane(l16); });
    if (__builtin_expect(l16 == 0u, 0))
        for (unsigned it = 0; it < 64 * POLL_LIMIT; ++it)
            if (__builtin_amdgcn_readfirstlane(*(lds_u32_ptr)(uintptr_t)fl16)) break;
    SB();
    grp(I(7), I(1), I(0), S1 + 4, none);
    grp(I(4), I(0), I(1), S1 + 5, none);
    grp(I(5), I(1), I(1), S1 + 6, [&](int i) { xst(I(4), i); });
    grp(I(6), I(0), I(1), S1 + 7, [&](int i) { xst(I(5), i); });
    grp(I(7), I(1), I(1), -1, [&](int i) { xst(I(6), i); });
    asm volatile("s_nop 5" : "+v"(acc[7][0]), "+v"(acc[7][1]), "+v"(acc[7][2]), "+v"(acc[7][3]));
    SB();
    STAMP(1);
    bool bad = false;
    for (unsigned it = 0, todo = 7; todo; ++it) {
        unsigned n = (__builtin_amdgcn_readfirstlane(*(lds_u32_ptr)(uintptr_t)(lds_base + LDS_RDY)) >> (wm * 8)) & todo;
        if (it > 64 * POLL_LIMIT) { n = todo; bad = true; }
#pragma unroll
        for (int b = 0; b < 3; ++b)
            if (n >> b & 1) {
#pragma unroll
                for (int q = 0; q < 2; ++q)
                    __builtin_amdgcn_raw_buffer_store_b128(gp[b][q], xr, xw + q * 1024, xp[b], XST_AUX);
            }
        todo &= ~n;
    }
    if (XSLEEP2) __builtin_amdgcn_s_sleep(XSLEEP2);
    unsigned pend = 7;
    auto pass = [&]() {
#pragma unroll
        for (int b = 0; b < 3; ++b)
            if (pend >> b & 1) { xld(b, 0); xld(b, 1); }
#pragma unroll
        for (int b = 0; b < 3; ++b)
            if (pend >> b & 1) {
                if (__all(((v[2 * b][0] | v[2 * b + 1][0]) & 1u) == 0u)) {
                    pend &= ~(1u << b);
                    add2(acc[7], sel0, sel1, v[2 * b], v[2 * b + 1]);
                }
            }
    };
    pass();
    for (unsigned it = 0; pend; ++it) {
        if ((it & 255u) == 255u) asm volatile("buffer_wbl2 sc1\n\ts_waitcnt vmcnt(0)" ::: "memory");
        asm volatile("" ::: "memory");
        pass();
        if (it > POLL_LIMIT) { bad = true; pend = 0; }
    }
    unsigned pav = *(lds_u32_ptr)(uintptr_t)(lds_base + LDS_RED);
    float am = 0.f;
#pragma unroll
    for (int i = 0; i < 4; ++i)
#pragma unroll
        for (int e = 0; e < 4; ++e) am = fmaxf(am, fabsf(acc[7][i][e]));
    const unsigned av = wave_max_l63(__float_as_uint(am) | (bad ? 0xffffffffu : 0u));
    const float tv = wave_sum_l63(am) * (ZTH / (2.05f * 64.f));
    publish_l63(my_ws, ws2, lg, av, call);
    const unsigned amb = (unsigned)__builtin_amdgcn_readlane((int)av, 63);
    {
        asm volatile("" : "+v"(pav));
        const unsigned pa = __builtin_amdgcn_readfirstlane(pav);
        if (RECORD && amb > (pa != 0u ? pa : (unsigned)__builtin_amdgcn_readlane(__float_as_int(tv), 63)))
            asm volatile("global_store_dwordx2 %0, %1, off offset:8 sc0 sc1" :: "v"(xge), "v"(((unsigned long long)call << 32) | amb) : "memory");
    }
#pragma unroll
    for (int n = 0; n < 4; ++n) *(lds_f32x4_ptr)(uintptr_t)(ho + 16384 + n * 1024) = acc[7][n];
    asm volatile("" ::: "memory");
    *(lds_u32_ptr)(uintptr_t)(fhf + 16) = amb | 0x80000000u;
    STAMP(2);
    if (tid == 256) __hip_atomic_store(ctr, call, __ATOMIC_RELAXED, __HIP_MEMORY_SCOPE_AGENT);
    if (WBMODE != 2 && cw == 2 && idx == 1) {
        wait_amax(lds_base, call);
        __builtin_amdgcn_s_sleep(WBDELAY);
        asm volatile("buffer_wbl2 sc1" ::: "memory");
    }
}

}

#define PACE 0, 0, -1, 0, 0, 0, 0, 0
static const int g_pace[8] = {PACE};
extern "C" void kernel_launch(int rank, int nranks, ncclComm_t comm,
                              hipStream_t stream,
                              void* const* d_in, const int* in_sizes, int n_in,
                              void* d_out, int out_size,
                              const int* in_dtypes, int out_dtype,
                              void* const* d_out_all,
                              void* const* d_ws_all, size_t ws_bytes) {
    (void)comm; (void)in_dtypes; (void)out_dtype; (void)d_out;
    if (nranks != NR || n_in < 2 || (unsigned)rank >= (unsigned)NR || ws_bytes < WS_END ||
        in_sizes[0] != M_LOC * K_TOT || in_sizes[1] != K_TOT * N_TOT || out_size != N_TOT * N_LOC) {
        fprintf(stderr, "kernel_launch: unexpected problem shape\n");
        return;
    }
    Params p;
    p.x = d_in[0];
    p.w = d_in[1];
    for (int i = 0; i < NR; ++i) {
        p.out[i] = d_out_all[i];
        p.ws[i] = d_ws_all[i];
    }
    p.rank = rank;
    p.pad = 0;
    p.my_ws = d_ws_all[rank];
    for (int i = 0; i < 8; ++i) p.pace[i] = g_pace[i];
    const int sg[10] = {SG};
    {
        int seq[10], ns = 0;
        for (int d = sg[0]; d > 0; d /= 10) seq[ns++] = d % 10 - 1;
        for (int w = 0; w < NWG; ++w) {
            const int xcd = w & 7, tnl = ((w >> 3) + TROT) & 3, tm = (w >> 5) & 1, ks = w >> 6;
            int key[16];
            for (int k = 0; k < 16; ++k) {
                const int ph = k + sg[2] * ks + sg[3] * xcd;
                const int mine = seq[(ph + sg[1] * tm) % ns], other = seq[(ph + sg[1] * (1 - tm)) % ns];
                int lead = 0;
                if (mine == tnl && other == tnl) lead = ((ph / ns + tm) & 1) ? sg[4] : sg[6];
                else if (mine == tnl) lead = sg[4];
                else if (other == tnl) lead = 0;
                else lead = (((ph + tnl * sg[7]) >> sg[8]) + tm + sg[9]) & 1 ? sg[5] : 0;
                key[k] = (4 * ((k + ROT2 * ks + ROT3 * xcd) & 15) - 4 * lead - (lead ? 2 : 0)) * 16 + k;
            }
            unsigned long long t = 0;
            for (int k = 0; k < 16; ++k) {
                int pos = 0;
                for (int q = 0; q < 16; ++q) pos += key[q] < key[k];
                t |= (unsigned long long)k << (4 * pos);
            }
            p.ord[w] = t;
        }
    }
    hipLaunchKernelGGL(gemm_a2a_kernel, dim3(NWG), dim3(NTHREADS), 0, stream, p);
    HIP_CHECK(hipGetLastError());
}
```

```cpp
#include <hip/hip_runtime.h>
#include <rccl/rccl.h>
#include <cstdio>
#include <cstdint>

#define HIP_CHECK(expr) do { hipError_t _e = (expr); if (_e != hipSuccess) { \
    fprintf(stderr, "HIP error %s at %s:%d\n", hipGetErrorString(_e), __FILE__, __LINE__); \
    return; } } while (0)

#define SWAPX 0
#define TLW 2
#define SWIDX 0
#define PH1 19
#define SPEC 1
#define SPERIOD 256
#define WBMODE 3
#define XST_AUX 0
#define WBDELAY 48
#define POLLGATE 3
#define RECORD 1
#define SGRACE 0
#define XSLEEP 0
#define XSLEEP1 0
#define XSLEEP2 0
#define SPRIO 0
#define SWQUIET 1
#define SPINS 0
#define LPRIO 0
#define LATEHEAT 1
#define PREF 7
#define FSLEEP 16
#define PHDEL 2
#define LATEZ 8
#define TOUCH 0
#define TSLEEP 2
#define TMASK 2
#define TROT 0
#define ZZ 4.0f
#define ZTH2 4.4f
#define ZTH 4.9f
#define SG 13, 0, 2, 1, 2, 1, 3, 0, 1, 0
#define ROT2 3
#define ROT3 12
namespace {

typedef __attribute__((ext_vector_type(4))) short s16x4;
typedef __attribute__((ext_vector_type(8))) short s16x8;
typedef __attribute__((ext_vector_type(2))) __bf16 bf16x2;
typedef __attribute__((ext_vector_type(2))) float f32x2;
typedef __attribute__((ext_vector_type(4))) float f32x4;
typedef __attribute__((ext_vector_type(2))) unsigned u32x2;
typedef __attribute__((ext_vector_type(4))) unsigned u32x4;
typedef __attribute__((address_space(3))) s16x4* lds_s16x4_ptr;
typedef __attribute__((address_space(3))) s16x8* lds_s16x8_ptr;
typedef __attribute__((address_space(3))) f32x4* lds_f32x4_ptr;
typedef __attribute__((address_space(3))) volatile unsigned* lds_u32_ptr;

constexpr int NR = 8;
constexpr int M_LOC = 512;
constexpr int K_TOT = 4096;
constexpr int N_TOT = 4096;
constexpr int N_LOC = N_TOT / NR;
constexpr int BM = 256, BN = 128, BK = 64;
constexpr int NS = 3;
constexpr int A_BYTES = BM * BK * 2;
constexpr int B_BYTES = BK * BN * 2;
constexpr int STAGE_BYTES = A_BYTES + B_BYTES;
constexpr int NSTEP = 16;
constexpr int NWG = 256;
constexpr int NTHREADS = 512;
constexpr int LDS_CALL = NS * STAGE_BYTES;
constexpr int LDS_RED = LDS_CALL + 16;
constexpr int LDS_HF = LDS_RED + 16;
constexpr int LDS_RDY = LDS_CALL + 4;
constexpr int LDS_L16 = LDS_CALL + 8;
constexpr int LDS_DUMMY = LDS_HF + 32;
constexpr int LDS_BYTES = LDS_DUMMY + 256;
constexpr int LDS_HO = 2 * STAGE_BYTES;

constexpr size_t WS_XG = 0;
constexpr size_t WS_LGF = WS_XG + 2 * 8192;
constexpr size_t WS_LGS = WS_LGF + 16384;
constexpr size_t WS_DONE = WS_LGS + 16384;
constexpr size_t WS_CTR = WS_DONE + NWG * 128;
constexpr size_t WS_DIAG = WS_CTR + NWG * 128;
constexpr size_t WS_XB = 256 * 1024;
#define XB_STRIDE 24832
constexpr size_t WS_END = WS_XB + (size_t)NWG * XB_STRIDE;
static_assert(WS_DIAG + NWG * 16 * 8 <= WS_XB, "");

constexpr unsigned POLL_LIMIT = 1u << 20;

struct Params {
    const void* x;
    const void* w;
    void* my_ws;
    int rank;
    int pad;
    int pace[8];
    void* out[NR];
    void* ws[NR];
    unsigned long long ord[NWG];
};

__device__ __forceinline__ void glds16(unsigned voff, const void* sbase, unsigned lds_dst) {
    asm volatile("s_mov_b32 m0, %2\n\ts_nop 0\n\tglobal_load_lds_dwordx4 %0, %1"
                 :: "v"(voff), "s"(sbase), "s"(lds_dst) : "memory");
}
#define WAIT_VM(n) asm volatile("s_waitcnt vmcnt(%0)" :: "i"(n) : "memory")
#define WAIT_LGKM0() asm volatile("s_waitcnt lgkmcnt(0)" ::: "memory")
#define BARRIER() asm volatile("s_barrier" ::: "memory")
#define SB() __builtin_amdgcn_sched_barrier(0)
#define MFMA(acc, a, b) asm volatile("v_mfma_f32_16x16x32_bf16 %0, %1, %2, %0" : "+v"(acc) : "v"(a), "v"(b))
#define MFMA0(acc, a, b) asm volatile("v_mfma_f32_16x16x32_bf16 %0, %1, %2, 0" : "+v"(acc) : "v"(a), "v"(b))

__device__ __forceinline__ unsigned wave_max_l63(unsigned v) {
    v = max(v, (unsigned)__builtin_amdgcn_update_dpp(0, (int)v, 0x111, 0xf, 0xf, false));
    v = max(v, (unsigned)__builtin_amdgcn_update_dpp(0, (int)v, 0x112, 0xf, 0xf, false));
    v = max(v, (unsigned)__builtin_amdgcn_update_dpp(0, (int)v, 0x114, 0xf, 0xf, false));
    v = max(v, (unsigned)__builtin_amdgcn_update_dpp(0, (int)v, 0x118, 0xf, 0xf, false));
    v = max(v, (unsigned)__builtin_amdgcn_update_dpp(0, (int)v, 0x142, 0xa, 0xf, false));
    v = max(v, (unsigned)__builtin_amdgcn_update_dpp(0, (int)v, 0x143, 0xc, 0xf, false));
    return v;
}
__device__ __forceinline__ u32x2 early_bar(float sum, float sumsq, float unit) {
    const float mean = sum * (1.f / 64.f);
    const unsigned tz = __float_as_uint(mean + ZZ * __builtin_amdgcn_sqrtf(fabsf(sumsq * (1.f / 64.f) - mean * mean)));
    u32x2 r = {max(__float_as_uint(mean * (ZTH / unit)), tz), max(__float_as_uint(mean * (ZTH2 / unit)), tz)};
    r = (u32x2){(unsigned)__builtin_amdgcn_readlane((int)r[0], 63), (unsigned)__builtin_amdgcn_readlane((int)r[1], 63)};
    asm volatile("" : "+s"(r));
    return r;
}
__device__ __forceinline__ void wave_red3(unsigned& m, float& s, float& q) {
#define RED3(c) "v_max_u32_dpp %0, %0, %0 " c "\n\tv_add_f32_dpp %1, %1, %1 " c "\n\tv_add_f32_dpp %2, %2, %2 " c "\n\t"
    asm volatile("s_nop 1\n\t"
                 RED3("row_shr:1 row_mask:0xf bank_mask:0xf bound_ctrl:1") RED3("row_shr:2 row_mask:0xf bank_mask:0xf bound_ctrl:1")
                 RED3("row_shr:4 row_mask:0xf bank_mask:0xf bound_ctrl:1") RED3("row_shr:8 row_mask:0xf bank_mask:0xf bound_ctrl:1")
                 RED3("row_bcast:15 row_mask:0xa bank_mask:0xf") RED3("row_bcast:31 row_mask:0xc bank_mask:0xf") "s_nop 0"
                 : "+v"(m), "+v"(s), "+v"(q));
#undef RED3
}
__device__ __forceinline__ unsigned wave_max_u32(unsigned v) { return (unsigned)__builtin_amdgcn_readlane((int)wave_max_l63(v), 63); }
__device__ __forceinline__ void publish_l63(const char* ws, const char* ws2, unsigned off, unsigned v, unsigned call) {
    asm volatile("s_lshl_b64 exec, 1, 63\n\tglobal_store_dwordx2 %0, %1, %2\n\tglobal_store_dwordx2 %0, %1, %3 sc1\n\ts_mov_b64 exec, -1"
                 :: "v"(off), "v"((u32x2){v, call}), "s"(ws), "s"(ws2) : "memory", "scc");
}
__device__ __forceinline__ void add2(f32x4 (&y)[4], const u32x4& s0, const u32x4& s1, const u32x4& g0, const u32x4& g1) {
    asm volatile("s_nop 1\n\tv_mfma_f32_16x16x32_bf16 %0, %4, %6, %0\n\tv_mfma_f32_16x16x32_bf16 %1, %5, %6, %1\n\t"
                 "v_mfma_f32_16x16x32_bf16 %2, %4, %7, %2\n\tv_mfma_f32_16x16x32_bf16 %3, %5, %7, %3\n\ts_nop 7"
                 : "+v"(y[0]), "+v"(y[1]), "+v"(y[2]), "+v"(y[3]) : "v"(s0), "v"(s1), "v"(g0), "v"(g1));
}
__device__ __forceinline__ void make_sel(int lane, u32x4& s0, u32x4& s1) {
    const unsigned one = (lane >> 4) == ((lane & 15) >> 2) ? (lane & 1 ? 0x3f800000u : 0x3f80u) : 0u;
    const unsigned x0 = lane & 2 ? 0u : one, x1 = lane & 2 ? one : 0u;
    s0 = (u32x4){x0, x1, 0u, 0u};
    s1 = (u32x4){0u, 0u, x0, x1};
    asm volatile("s_nop 1" : "+v"(s0), "+v"(s1));
}

__device__ __forceinline__ unsigned pk_bf16(float a, float b) {
    const bf16x2 v = __builtin_convertvector((f32x2){a, b}, bf16x2);
    return __builtin_bit_cast(unsigned, v);
}
__device__ __forceinline__ unsigned pk_bf16_tag(float a, float b, unsigned tag) {
    const unsigned t = (__float_as_uint(a) + ((1u - tag) << 16)) & 0xfffe0000u;
    return (pk_bf16(__uint_as_float(t), b) & ~1u) | tag;
}
__device__ __forceinline__ u32x4 pack_granule(const f32x4& a, const f32x4& b, unsigned tag) {
    u32x4 g;
    g[0] = pk_bf16_tag(a[0], a[1], tag);
    g[1] = pk_bf16(a[2], a[3]);
    g[2] = pk_bf16(b[0], b[1]);
    g[3] = pk_bf16(b[2], b[3]);
    return g;
}
__device__ __forceinline__ float add_lo(unsigned w, float c) {
    unsigned one_lo = 0x00003f80u;
    asm("" : "+s"(one_lo));
    return __builtin_amdgcn_fdot2_f32_bf16(__builtin_bit_cast(bf16x2, w), __builtin_bit_cast(bf16x2, one_lo), c, false);
}
__device__ __forceinline__ float add_hi(unsigned w, float c) {
    return __builtin_amdgcn_fdot2_f32_bf16(__builtin_bit_cast(bf16x2, w), __builtin_bit_cast(bf16x2, 0x3f800000u), c, false);
}
__device__ __forceinline__ void add_granule(f32x4& a, f32x4& b, const u32x4& g) {
    a[0] = add_lo(g[0], a[0]); a[1] = add_hi(g[0], a[1]); a[2] = add_lo(g[1], a[2]); a[3] = add_hi(g[1], a[3]);
    b[0] = add_lo(g[2], b[0]); b[1] = add_hi(g[2], b[1]); b[2] = add_lo(g[3], b[2]); b[3] = add_hi(g[3], b[3]);
}
__device__ __forceinline__ void tr4(unsigned& r0, unsigned& r1, unsigned& r2, unsigned& r3) {
    u32x2 t;
    t = __builtin_amdgcn_permlane32_swap(r0, r2, false, false); r0 = t[0]; r2 = t[1];
    t = __builtin_amdgcn_permlane32_swap(r1, r3, false, false); r1 = t[0]; r3 = t[1];
    t = __builtin_amdgcn_permlane16_swap(r0, r1, false, false); r0 = t[0]; r1 = t[1];
    t = __builtin_amdgcn_permlane16_swap(r2, r3, false, false); r2 = t[0]; r3 = t[1];
}

#define NCYC 2
#define STAMPT(i, t) do {} while (0)
#define STAMP(i) STAMPT(i, 256)

__device__ __forceinline__ unsigned wait_amax(unsigned lds_base, unsigned call) {
    for (unsigned it = 0;; ++it) {
        u32x2 r;
        asm volatile("ds_read_b64 %0, %1\n\ts_waitcnt lgkmcnt(0)" : "=v"(r) : "v"(lds_base + LDS_RED) : "memory");
        if (__builtin_amdgcn_readfirstlane(r[1]) == call) return __builtin_amdgcn_readfirstlane(r[0]);
        if (it > 64 * POLL_LIMIT) return 0xffffffffu;
        if (SPINS) __builtin_amdgcn_s_sleep(4);
    }
}
__device__ __forceinline__ float inv_scale(unsigned gmax) {
    const float scale = __uint_as_float(gmax) / 448.0f;
    return scale > 1e-38f ? 1.0f / scale : 0.f;
}

__device__ __forceinline__ char* xg_entry(const Params& p, int lane, int g) {
    char* b = (char*)p.ws[0];
#pragma unroll
    for (int i = 1; i < NR; ++i) {
        char* w = (char*)p.ws[i];
        asm("" : "+s"(w));
        b = (lane >> 3) == i ? w : b;
    }
    return b + WS_XG + (lane & 7) * 1024 + (p.rank * 8 + g) * 16;
}

template <int V> struct IntC { static constexpr int value = V; };

__global__ void __launch_bounds__(NTHREADS, 2) gemm_a2a_kernel(const Params p) {
    __shared__ __attribute__((aligned(1024))) char lds[LDS_BYTES];
    const int tid = threadIdx.x;
    const int lane = tid & 63;
    const int wave = __builtin_amdgcn_readfirstlane(tid >> 6);
    const int wg = blockIdx.x;
    const int rank = p.rank;
    char* const my_ws = (char*)p.my_ws;
    const char* const gx = (const char*)p.x;
    const char* const gw = (const char*)p.w;
    const unsigned long long otab = p.ord[wg];

    const int xcd = wg & 7, idx = wg >> 3;
    const int tnl = (idx + TROT) & 3, tm = (idx >> 2) & 1, ks = idx >> 3;
    const int row0 = tm * BM;
    const int col0 = xcd * N_LOC + tnl * BN;

    const int of = 0;
    const int hi = wave & 1;
    unsigned voffA[8], voffB[4];
    {
        unsigned ve = (unsigned)(((wave & 2) * 64 + hi * 16 + (lane >> 3)) * (K_TOT * 2) + (((lane & 7) ^ (lane >> 4)) * 16));
        asm("" : "+v"(ve));
        unsigned vo = (ve ^ 64u) + 8 * (K_TOT * 2);
        asm("" : "+v"(vo));
#pragma unroll
        for (int i = 0; i < 8; ++i) voffA[i] = (i & 1 ? vo : ve) | (unsigned)((((i >> 1) ^ 3 ^ ks) & 3) * 32 * (K_TOT * 2));
        unsigned vb = (unsigned)(((wave & 3) * 16 + (lane >> 4)) * (N_TOT * 2) + (((lane & 15) ^ (2 * (lane >> 4))) * 16));
        asm("" : "+v"(vb));
        voffB[0] = vb; voffB[1] = vb | 4 * (N_TOT * 2); voffB[2] = vb ^ (8 * (N_TOT * 2) + 128); voffB[3] = vb ^ (12 * (N_TOT * 2) + 128);
    }
    asm volatile("" : "+v"(voffA[0]), "+v"(voffA[1]), "+v"(voffA[2]), "+v"(voffA[3]), "+v"(voffA[4]), "+v"(voffA[5]), "+v"(voffA[6]), "+v"(voffA[7]),
                      "+v"(voffB[0]), "+v"(voffB[1]), "+v"(voffB[2]), "+v"(voffB[3]));
    unsigned offA = row0 * (K_TOT * 2) + ks * NSTEP * (BK * 2), offB = col0 * 2 + ks * NSTEP * (BK * N_TOT * 2);
    unsigned dA = (unsigned)(uintptr_t)lds + wave * 8192, dB = (unsigned)(uintptr_t)lds + A_BYTES + wave * 4096;
    asm volatile("" : "+s"(offA), "+s"(offB), "+s"(dA), "+s"(dB));
    if (wave >= 4) {
        u32x4 ra[2];
#pragma unroll
        for (int i = 0; i < 8; ++i) ra[i >> 2][i & 3] = ((__umul24(tid, 0x9e3779u + i * 0x2c1b3u) + 0x5bd1e9u * i) & 0x80ff80ffu) | 0x3f003f00u;
        f32x4 d[4] = {};
#pragma unroll 1
        for (int r = 0; r < PH1; ++r) {
#pragma unroll
            for (int i = 0; i < 8; ++i) MFMA(d[i & 3], ra[i & 1], ra[(i >> 1) & 1]);
        }
        asm volatile("s_nop 7\n\ts_nop 7" :: "v"(d[0]), "v"(d[1]), "v"(d[2]), "v"(d[3]));
    }
    asm volatile("" :: "s"(gx), "s"(gw), "s"(my_ws), "s"(rank), "s"(otab));
    auto ktile = [&](int s) {
        return ((int)(otab >> (4 * (s & 15))) + of) & 15;
    };
    unsigned* ctr = (unsigned*)(my_ws + WS_CTR + (size_t)wg * 128);
    const unsigned lds_base = (unsigned)(uintptr_t)lds;

    if (__builtin_expect(wave < 4, 1)) {
        if (LPRIO) __builtin_amdgcn_s_setprio(LPRIO);
        const char* const ga = gx + offA;
        const char* const gb = gw + offB;
        auto issue_at = [&](unsigned ka, unsigned kb, unsigned so) {
            const char* a = ga + ka * (BK * 2);
            const char* b = gb + kb * (BK * N_TOT * 2);
#pragma unroll
            for (int j = 0; j < 4; ++j) {
                glds16(voffB[j], b, dB + so + j * 1024);
                glds16(voffA[2 * j], a, dA + so + 2 * j * 1024);
                glds16(voffA[2 * j + 1], a, dA + so + (2 * j + 1) * 1024);
            }
        };
        {
            const char* b = gb + (unsigned)ktile(0) * (BK * N_TOT * 2);
#pragma unroll
            for (int j = 0; j < 4; ++j) glds16(voffB[j], b, dB + j * 1024);
        }
        if (!hi) {
            const char* a = ga + (unsigned)ktile(0) * (BK * 2);
#pragma unroll
            for (int j = 0; j < 8; ++j) glds16(voffA[j], a, dA + j * 1024);
        }
        issue_at(ktile(1 - hi), ktile(1), STAGE_BYTES);
        int ordB = ktile(lane), ordA = ktile(lane - hi);
        asm volatile("" : "+v"(ordA), "+v"(ordB));
        auto issue = [&](int u, unsigned so) { issue_at(__builtin_amdgcn_readlane(ordA, u), __builtin_amdgcn_readlane(ordB, u), so); };
        unsigned so = 2 * STAGE_BYTES;
#define TLS(k)
        for (int k = 0; k < NSTEP - 2; ++k) {
            WAIT_VM(12);
            BARRIER();
            TLS(k);
            issue(k + 2, so);
            so = (so == 2 * STAGE_BYTES) ? 0 : so + STAGE_BYTES;
        }
        const bool tch = TOUCH && wave == 0 && (TMASK >> tnl & 1);
        auto touch = [&]() {
            const char* const xb = my_ws + WS_XB + (size_t)wg * XB_STRIDE;
            const unsigned tv = lane * 128;
            asm volatile("s_mov_b32 m0, %4\n\ts_nop 0\n\tglobal_load_lds_dword %0, %1\n\tglobal_load_lds_dword %0, %2\n\tglobal_load_lds_dword %0, %3"
                         :: "v"(tv), "s"(xb), "s"(xb + 8192), "s"(xb + 16384), "s"(lds_base + LDS_DUMMY) : "memory");
        };
        if (TOUCH == 13 && tch) { touch(); WAIT_VM(15); } else WAIT_VM(12);
        BARRIER();
        TLS(14);
        if (TOUCH == 14 && tch) { __builtin_amdgcn_s_sleep(TSLEEP); touch(); }
        if (hi) {
            const char* a = ga + (unsigned)__builtin_amdgcn_readlane(ordA, 16) * (BK * 2);
#pragma unroll
            for (int j = 0; j < 8; ++j) glds16(voffA[j], a, dA + STAGE_BYTES + j * 1024);
        }
        const unsigned call = __builtin_amdgcn_readfirstlane(*(const unsigned*)(lds + LDS_CALL));
        const int wm = wave & 1;
        const unsigned par = call & 1u;
        const unsigned xv = wm * 8192 + lane * 16;
        char* const xge = xg_entry(p, lane, xcd) + par * 8192;
        const int orow = rank * M_LOC + row0 + wm * 128 + ks * 32 + (lane & 15);
        char* const o = (char*)p.out[xcd] + (size_t)orow * N_LOC + tnl * BN + (lane >> 4) * 16;
        char* const o2 = o + 16 * N_LOC;
        char* const dflag = (char*)p.ws[xcd] + WS_DONE + (size_t)(rank * 32 + idx) * 128 + wm * 8;
        const unsigned lg = WS_LGF + xcd * 2048 + 1024 + (idx * 2 + wm) * 8;
        const char* const ws2 = my_ws + (WS_LGS - WS_LGF);
        const char* const gsrc = my_ws + WS_XG + par * 8192 + xcd * 1024 + lane * 16;
        asm volatile("" :: "v"(xge), "v"(o), "v"(o2), "v"(dflag), "v"(lg), "s"(ws2), "v"(xv), "v"(gsrc));
        if (hi) WAIT_VM(8); else if (tch) WAIT_VM(3); else WAIT_VM(0);
        BARRIER();
        TLS(15);
        if (hi) {
            WAIT_VM(0);
            asm volatile("ds_write_b32 %0, %1" :: "v"(lds_base + LDS_L16 + (wave >> 1) * 4), "v"(1u) : "memory");
        }
        if (wave >= 2) {
            const __amdgpu_buffer_rsrc_t xr = __builtin_amdgcn_make_buffer_rsrc(my_ws + WS_XB, 0, NWG * XB_STRIDE, 0x00020000);
            TLS(16);
            u32x4 v[12];
            const unsigned xw = wm * 4096 + lane * 16;
            auto xld = [&](int b) {
#pragma unroll
                for (int q = 0; q < 4; ++q)
                    v[4 * b + q] = __builtin_amdgcn_raw_buffer_load_b128(xr, xw + q * 1024, (wg ^ ((3 - b) << 6)) * XB_STRIDE + b * 8192, 16);
            };
            f32x4 y[2][2][4];
            u32x4 sel0, sel1;
            make_sel(lane, sel0, sel1);
            if (SPRIO) __builtin_amdgcn_s_setprio(SPRIO);
            bool bad = false;
            for (unsigned it = 0;; ++it) {
                u32x2 f;
                asm volatile("ds_read_b64 %0, %1\n\ts_waitcnt lgkmcnt(0)" : "=v"(f) : "v"(lds_base + LDS_HF + wm * 8) : "memory");
                if (__builtin_amdgcn_readfirstlane(f[0] & f[1])) break;
                if (it > 64 * POLL_LIMIT) { bad = true; break; }
                if (SPINS) __builtin_amdgcn_s_sleep(SPINS);
            }
            if (XSLEEP) __builtin_amdgcn_s_sleep(XSLEEP);
            xld(0); xld(1);
            if (XSLEEP1) __builtin_amdgcn_s_sleep(XSLEEP1);
            xld(2);
            for (unsigned it = 0;; ++it) {
                u32x2 f;
                asm volatile("ds_read_b64 %0, %1\n\ts_waitcnt lgkmcnt(0)" : "=v"(f) : "v"(lds_base + LDS_HF + wm * 8) : "memory");
                if (__builtin_amdgcn_readfirstlane(f[0] & f[1] & 2u)) break;
                if (it > 64 * POLL_LIMIT) { bad = true; break; }
            }
#pragma unroll
            for (int n = 0; n < 8; ++n)
                y[0][n >> 2][n & 3] = *(lds_f32x4_ptr)(uintptr_t)(lds_base + LDS_HO + xv + n * 1024);
            unsigned pend = 7;
            const unsigned frdy = lds_base + LDS_RDY + wm;
            auto pass = [&]() {
#pragma unroll
                for (int b = 0; b < 3; ++b)
                    if (pend >> b & 1) {
                        const unsigned tags = v[4 * b][0] & v[4 * b + 1][0] & v[4 * b + 2][0] & v[4 * b + 3][0];
                        if (__all(tags & 1u)) {
                            pend &= ~(1u << b);
                            asm volatile("ds_write_b8 %0, %1" :: "v"(frdy), "v"(pend ^ 7u) : "memory");
                        }
                    }
#pragma unroll
                for (int b = 0; b < 3; ++b)
                    if (pend >> b & 1) xld(b);
            };
            pass();
            for (unsigned it = 0; pend; ++it) {
                if ((it & 255u) == 255u) asm volatile("buffer_wbl2 sc1\n\ts_waitcnt vmcnt(0)" ::: "memory");
                pass();
                if (it > POLL_LIMIT) { bad = true; pend = 0; }
            }
            unsigned pav = *(lds_u32_ptr)(uintptr_t)(lds_base + LDS_RED);
#pragma unroll
            for (int q = 0; q < 12; ++q) {
                MFMA(y[0][(q >> 1) & 1][2 * (q & 1)], sel0, v[q]);
                MFMA(y[0][(q >> 1) & 1][2 * (q & 1) + 1], sel1, v[q]);
            }
            asm volatile("s_nop 7" : "+v"(y[0][0][0]), "+v"(y[0][0][1]), "+v"(y[0][0][2]), "+v"(y[0][0][3]),
                                     "+v"(y[0][1][0]), "+v"(y[0][1][1]), "+v"(y[0][1][2]), "+v"(y[0][1][3]));
            float am = 0.f;
#pragma unroll
            for (int i = 0; i < 8; ++i)
#pragma unroll
                for (int e = 0; e < 4; ++e) am = fmaxf(am, fabsf(y[0][i >> 2][i & 3][e]));
            unsigned av = __float_as_uint(am) | (bad ? 0xffffffffu : 0u);
            float asq = am * am;
            wave_red3(av, am, asq);
            publish_l63(my_ws, ws2, lg, av, call);
            const u32x2 tv = early_bar(am, asq, 2.327f);
            unsigned amb = (unsigned)__builtin_amdgcn_readlane((int)av, 63);
            unsigned told = tv[1];
            {
                asm volatile("" : "+v"(pav));
                const unsigned pa = __builtin_amdgcn_readfirstlane(pav);
                if (amb > max(pa, tv[0])) {
                    told = amb;
                    asm volatile("global_store_dwordx2 %0, %1, off offset:8 sc0 sc1" :: "v"(xge), "v"(((unsigned long long)call << 32) | amb) : "memory");
                }
            }
            if (lane == 0) asm volatile("ds_add_u32 %0, %1" :: "v"(lds_base + LDS_RED + 8), "v"(1u) : "memory");
            STAMPT(7, 128);
            auto send = [&](auto hc, unsigned val) {
                constexpr int h = decltype(hc)::value;
                const float inv = inv_scale(val);
                char* const dst = h ? o2 : o;
                u32x4 q[2];
#pragma unroll
                for (int c = 0; c < 2; ++c) {
                    unsigned r[4];
#pragma unroll
                    for (int j = 0; j < 4; ++j) {
                        int w = __builtin_amdgcn_cvt_pk_fp8_f32(y[h][c][j][0] * inv, y[h][c][j][1] * inv, 0, false);
                        w = __builtin_amdgcn_cvt_pk_fp8_f32(y[h][c][j][2] * inv, y[h][c][j][3] * inv, w, true);
                        r[j] = val == 0xffffffffu ? 0x7f7f7f7fu : (unsigned)w;
                    }
                    tr4(r[0], r[1], r[2], r[3]);
                    q[c] = (u32x4){r[0], r[1], r[2], r[3]};
                }
                asm volatile("global_store_dwordx4 %0, %1, off sc0 sc1\n\t"
                             "global_store_dwordx4 %0, %2, off offset:64 sc0 sc1\n\ts_nop 1" :: "v"(dst), "v"(q[0]), "v"(q[1]) : "memory");
            };
            unsigned s0 = 0xfffffffeu, s1 = 0xfffffffdu, flagged = 0xfffffffeu;
            bool have1 = false;
            unsigned ts = (unsigned)__builtin_amdgcn_s_memrealtime() - SGRACE;
            ts += (2u * SPERIOD + (unsigned)((idx * 2 + wm) * SPERIOD >> 6) - ts % SPERIOD) % SPERIOD;
            for (unsigned it = 0;; ++it) {
                u32x2 r, f;
                unsigned long long now;
                asm volatile("ds_read_b64 %0, %3\n\tds_read_b64 %1, %4\n\ts_memrealtime %2\n\ts_waitcnt lgkmcnt(0)"
                             : "=&v"(r), "=&v"(f), "=s"(now) : "v"(lds_base + LDS_RED), "v"(lds_base + LDS_HF + 16 + wm * 8) : "memory");
                unsigned pa = __builtin_amdgcn_readfirstlane(r[0]);
                const unsigned st = __builtin_amdgcn_readfirstlane(r[1]);
                bool fin = st == call;
                if (it > 64 * POLL_LIMIT) { fin = true; pa = 0xffffffffu; }
                if (!have1) {
                    const unsigned f0 = __builtin_amdgcn_readfirstlane(f[0]), f1 = __builtin_amdgcn_readfirstlane(f[1]);
                    if (f0 & f1 & 0x80000000u) {
                        const unsigned fm = max(f0, f1);
                        const unsigned a1 = fm == 0xffffffffu ? fm : fm & 0x7fffffffu;
                        if (pa != 0u) told = max(told, a1);
                        amb = max(amb, a1);
                        have1 = true;
                        if (lane == 0) asm volatile("ds_add_u32 %0, %1" :: "v"(lds_base + LDS_RED + 8), "v"(16u) : "memory");
#pragma unroll
                        for (int n = 0; n < 8; ++n)
                            y[1][n >> 2][n & 3] = *(lds_f32x4_ptr)(uintptr_t)(lds_base + LDS_HO + 16384 + xv + n * 1024);
                    }
                }
                if (!have1 && pa != 0xffffffffu) fin = false;
                if (RECORD && pa != 0u && amb > max(pa, told)) {
                    told = amb;
                    asm volatile("global_store_dwordx2 %0, %1, off offset:8 sc0 sc1" :: "v"(xge), "v"(((unsigned long long)call << 32) | amb) : "memory");
                }
                const unsigned val = fin ? pa : max(pa, amb);
                bool go = fin;
                if (SPEC && (int)((unsigned)now - ts) >= 0) {
                    go = go || (st == ~call && pa != 0u && (RECORD || pa >= amb));
                    if ((int)((unsigned)now - ts) >= SGRACE || (go && (s0 != val || (s1 != val && have1)))) ts += SPERIOD;
                }
                if (go && s0 != val) {
                    send(IntC<0>{}, val);
                    s0 = val;
                    go = fin;
                }
                if (go && s1 != val && (have1 || fin)) {
                    send(IntC<1>{}, val);
                    s1 = val;
                }
                if (s0 == s1 && s0 != flagged) {
                    flagged = s0;
                    asm volatile("s_waitcnt vmcnt(0)" ::: "memory");
                    STAMPT(4, 128);
                    if (lane == 0 && xcd != rank)
                        asm volatile("global_store_dwordx2 %0, %1, off sc0 sc1"
                                     :: "v"(dflag), "v"(((unsigned long long)call << 32) | s0) : "memory");
                }
                if (fin) break;
            }
            return;
        }
        if (wave == 0) {
            u32x4 gv;
            unsigned best = 0;
            auto gstate = [&]() {
                const bool ok = gv[1] == call, started = gv[3] == call;
                const unsigned long long seen = __ballot(started);
                best = max(best, max(ok ? gv[0] : 0u, started ? gv[2] : 0u));
                const unsigned gmax = wave_max_u32(best);
                const unsigned st = __all(ok) ? call : ((seen >> (8 * xcd)) & 0xffu) ? ~call : 0u;
                asm volatile("ds_write_b64 %0, %1" :: "v"(lds_base + LDS_RED), "v"(((unsigned long long)st << 32) | gmax) : "memory");
                return st == call;
            };
            if (idx != SWIDX || !SWQUIET) {
                asm volatile("global_load_dwordx4 %0, %1, off sc0 sc1\n\ts_waitcnt vmcnt(0)" : "=&v"(gv) : "v"(gsrc) : "memory");
                gstate();
            }
            if (idx == SWIDX) {
                const char* lgs = my_ws + WS_LGF + xcd * 2048 + lane * 16;
                const char* lgt = my_ws + WS_LGF + xcd * 2048 + 1024 + lane * 8;
                unsigned m0 = 0, m1 = 0, m2 = 0;
                bool ok0 = false, ok1 = false, ok2 = false;
                unsigned it = 0;
                for (;;) {
                    const unsigned alt = ((it & 63u) == 63u) ? (WS_LGS - WS_LGF) : 0;
                    u32x4 v;
                    u32x2 u;
                    asm volatile("global_load_dwordx4 %0, %2, off sc1\n\tglobal_load_dwordx2 %1, %3, off sc1\n\ts_waitcnt vmcnt(0)"
                                 : "=&v"(v), "=&v"(u) : "v"(lgs + alt), "v"(lgt + alt) : "memory");
                    if (v[1] == call) { m0 = v[0]; ok0 = true; }
                    if (v[3] == call) { m1 = v[2]; ok1 = true; }
                    if (u[1] == call) { m2 = u[0]; ok2 = true; }
                    if (__all(ok0 && ok1 && ok2)) break;
                    if (++it > 4 * POLL_LIMIT) { m0 = 0xffffffffu; break; }
                }
                const unsigned m = wave_max_u32(max(max(m0, m1), m2));
                asm volatile("global_store_dwordx2 %0, %1, off sc0 sc1"
                             :: "v"(xge), "v"(((unsigned long long)call << 32) | m) : "memory");
                STAMPT(6, 0);
            }
            for (unsigned it = 0; POLLGATE && it < 64 * POLL_LIMIT; ++it) {
                unsigned c;
                asm volatile("ds_read_b32 %0, %1\n\ts_waitcnt lgkmcnt(0)" : "=v"(c) : "v"(lds_base + (POLLGATE >= 3 ? LDS_RDY : LDS_RED + 8)) : "memory");
                c = __builtin_amdgcn_readfirstlane(c);
                if (POLLGATE == 3 ? c == 0x0707u : POLLGATE == 4 ? __builtin_popcount(c) >= 4 : POLLGATE == 5 ? c != 0u : POLLGATE == 1 ? (c & 15u) == 2u : c == 34u) break;
                if (SPINS) __builtin_amdgcn_s_sleep(2);
            }
            for (unsigned it = 0;;) {
                asm volatile("global_load_dwordx4 %0, %1, off sc0 sc1\n\ts_waitcnt vmcnt(0)"
                             : "=&v"(gv) : "v"(gsrc) : "memory");
                if (++it > 4 * POLL_LIMIT) { gv[0] = 0xffffffffu; gv[1] = call; }
                if (gstate()) break;
                if ((it & 255u) == 0u) asm volatile("buffer_wbl2 sc1\n\ts_waitcnt vmcnt(0)" ::: "memory");
            }
            return;
        }
        if ((wg >> 5) == rank) return;
        const unsigned long long want = ((unsigned long long)call << 32) | wait_amax(lds_base, call);
        __builtin_amdgcn_s_sleep(FSLEEP);
        const char* f = my_ws + WS_DONE + (size_t)wg * 128 + (lane & 1) * 8;
        for (unsigned it = 0;; ++it) {
            unsigned long long v;
            asm volatile("global_load_dwordx2 %0, %1, off sc0 sc1\n\ts_waitcnt vmcnt(0)" : "=&v"(v) : "v"(f) : "memory");
            if (__all(v == want)) break;
            if (it > POLL_LIMIT) {
                if (lane == 0)
                    __hip_atomic_store((unsigned char*)p.out[rank] + (size_t)rank * M_LOC * N_LOC,
                                       (unsigned char)0x7f, __ATOMIC_RELAXED, __HIP_MEMORY_SCOPE_SYSTEM);
                break;
            }
        }
        STAMPT(5, 64);
        return;
    }
    const int cw = wave - 4;
    const int wm = cw >> 1, wn = cw & 1;
    unsigned call = *(const __attribute__((address_space(4))) unsigned*)(uintptr_t)ctr;
    char* wsp[NR];
#pragma unroll
    for (int i = 0; i < NR; ++i) wsp[i] = (char*)p.ws[i];
    unsigned rdA[2], rdB[4];
#pragma unroll
    for (int kk = 0; kk < 2; ++kk) {
        const int r = lane & 15;
        const int c = 4 * kk + (lane >> 4);
        rdA[kk] = lds_base + (unsigned)(wm * 16384 + r * 128 + ((c ^ ((r >> 1) & 7)) * 16));
    }
#pragma unroll
    for (int cn = 0; cn < 4; ++cn) {
        const int g = lane >> 4, q = (lane & 15) >> 2, pp = lane & 3;
        const int k = 8 * g + q;
        const int h = (k & 3) | (((k >> 3) & 1) << 2);
        const int u = (wn * 4 + cn) ^ h;
        rdB[cn] = lds_base + (unsigned)(A_BYTES + k * 256 + u * 32 + pp * 8);
    }
    f32x4 acc[8][4];
#pragma unroll
    for (int i = 0; i < LATEZ; ++i) {
#pragma unroll
        for (int j = 0; j < 4; ++j) acc[i][j] = (f32x4){0.f, 0.f, 0.f, 0.f};
        asm volatile("s_nop 1" : "+v"(acc[i][0]), "+v"(acc[i][1]), "+v"(acc[i][2]), "+v"(acc[i][3]));
    }
    s16x8 aq[2][2];
    s16x4 bl[2][4][2], bh[2][4][2];
    asm volatile("" : "+v"(rdA[0]), "+v"(rdA[1]), "+v"(rdB[0]), "+v"(rdB[1]), "+v"(rdB[2]), "+v"(rdB[3]));
    __builtin_amdgcn_s_sleep(PHDEL);
    asm volatile("" : "+s"(call), "+s"(wsp[0]), "+s"(wsp[1]), "+s"(wsp[2]), "+s"(wsp[3]), "+s"(wsp[4]), "+s"(wsp[5]), "+s"(wsp[6]), "+s"(wsp[7]));

#define BFR(s, n, kk) __builtin_shufflevector(bl[s][n][kk], bh[s][n][kk], 0, 1, 2, 3, 4, 5, 6, 7)
#define RDA(kk, mt) (*(lds_s16x8_ptr)(uintptr_t)(rdA[kk] + (mt) * 2048))
#define RDB(kk, n, h) __builtin_amdgcn_ds_read_tr16_b64_v4i16((lds_s16x4_ptr)(uintptr_t)(rdB[n] + (kk) * 8192 + (h) * 1024))
    auto rdb = [&](auto sc, int j) {
        constexpr int s = decltype(sc)::value;
        const int n = j >> 2, kk = (j >> 1) & 1;
        if (j & 1) bh[s][n][kk] = RDB(kk, n, 1); else bl[s][n][kk] = RDB(kk, n, 0);
    };
    auto grp = [&](auto mtc, auto abc, auto bsc, int na, auto&& fill) {
        constexpr int mt = decltype(mtc)::value, ab = decltype(abc)::value, bs = decltype(bsc)::value;
#pragma unroll
        for (int i = 0; i < 8; ++i) {
            const int nt = (mt & 1) ? 3 - (i >> 1) : i >> 1, kk = (i & 1) ^ (mt & 1);
            MFMA(acc[mt][nt], BFR(bs, nt, kk), aq[ab][kk]);
            SB();
            if (i < 2 && na >= 0) aq[1 - ab][i] = RDA(i, na);
            fill(i);
            SB();
        }
    };
#define I(v) IntC<v>{}
    auto none = [](int) {};
    unsigned slot = 0, sd = 0;
    auto next_slot = [&](int i) {
        if (i == 0) { sd = slot == NS - 1 ? (unsigned)(-(NS - 1) * STAGE_BYTES) : (unsigned)STAGE_BYTES; slot = slot == NS - 1 ? 0 : slot + 1; }
        else if (i < 5) rdB[i - 1] += sd;
        else rdA[i - 5] += sd;
    };
    auto interval = [&](auto sc, int k) {
        constexpr int s = decltype(sc)::value;
        BARRIER();
        SB();
        grp(I(3), I(1), I(1 - s), 4, [&](int i) { if (i >= 2 && i < 6) rdb(sc, i - 2); });
        grp(I(4), I(0), I(1 - s), 5, [&](int i) { if (i >= 2 && i < 6) rdb(sc, i + 2); });
        grp(I(5), I(1), I(1 - s), 6, [&](int i) { if (i >= 2 && i < 6) rdb(sc, i + 6); });
        grp(I(6), I(0), I(1 - s), 7, [&](int i) { if (i >= 2 && i < 6) rdb(sc, i + 10); });
        grp(I(7), I(1), I(1 - s), 0, none);
        grp(I(0), I(0), sc, 1, none);
        grp(I(1), I(1), sc, 2, [&](int i) { if (i >= 2 && i < 7) next_slot(i - 2); });
        grp(I(2), I(0), sc, 3, [&](int i) { if (i == 2 || i == 4) next_slot(4 + (i >> 1)); });
        WAIT_LGKM0();
        SB();
    };
    BARRIER();
    SB();
    aq[0][0] = RDA(0, 0);
    rdb(I(0), 0); rdb(I(0), 1);
    SB();
    aq[0][1] = RDA(1, 0);
    SB();
#pragma unroll
    for (int j = 2; j < 16; ++j) { rdb(I(0), j); if (j & 1) SB(); }
    grp(I(0), I(0), I(0), 1, none);
    grp(I(1), I(1), I(0), 2, [&](int i) { if (i >= 2 && i < 7) next_slot(i - 2); });
    grp(I(2), I(0), I(0), 3, [&](int i) { if (i == 2 || i == 4) next_slot(4 + (i >> 1)); });
#pragma unroll
    for (int i = LATEZ; i < 8; ++i) {
#pragma unroll
        for (int j = 0; j < 4; ++j) acc[i][j] = (f32x4){0.f, 0.f, 0.f, 0.f};
        asm volatile("s_nop 1" : "+v"(acc[i][0]), "+v"(acc[i][1]), "+v"(acc[i][2]), "+v"(acc[i][3]));
    }
    const unsigned xv = cw * 4096 + lane * 16;
    const __amdgpu_buffer_rsrc_t xr = __builtin_amdgcn_make_buffer_rsrc(my_ws + WS_XB, 0, NWG * XB_STRIDE, 0x00020000);
    const unsigned xo = wg * XB_STRIDE;
    const unsigned xw = cw * 2048 + lane * 16;
    unsigned lg = WS_LGF + xcd * 2048 + (idx * 4 + cw) * 8;
    const char* ws2 = my_ws + (WS_LGS - WS_LGF);
    unsigned xp[3];
#pragma unroll
    for (int b = 0; b < 3; ++b) xp[b] = (wg ^ ((3 - b) << 6)) * XB_STRIDE + b * 8192;
    unsigned fl16 = lds_base + LDS_L16 + wm * 4, fhf = lds_base + LDS_HF + cw * 4;
    asm volatile("" : "+v"(lg), "+s"(ws2), "+s"(xp[0]), "+s"(xp[1]), "+s"(xp[2]), "+v"(fl16), "+v"(fhf));
    u32x4 sel0, sel1;
    make_sel(lane, sel0, sel1);
    unsigned tmask = 0xfffe0000u;
    ++call;
    if (tid < 256 + 4) *(u32x4*)(lds + LDS_CALL + lane * 16) = (u32x4){lane ? 0u : call, 0u, 0u, 0u};
    const unsigned par = call & 1u;
    char* xge = wsp[0];
#pragma unroll
    for (int i = 1; i < NR; ++i) xge = (lane >> 3) == i ? wsp[i] : xge;
    xge += WS_XG + (lane & 7) * 1024 + (rank * 8 + xcd) * 16 + par * 8192;
    asm volatile("" : "+v"(tmask), "+v"(xge) : "v"(xv), "v"(xw));
    if (cw == 0 && idx == SWIDX) {
        asm volatile("global_store_dwordx2 %0, %1, off offset:8 sc0 sc1" :: "v"(xge), "v"((unsigned long long)call << 32) : "memory");
    }
    WAIT_LGKM0();
    SB();
    for (int k = 1; k < NSTEP - 1; k += 2) {
        interval(I(1), k);
        interval(I(0), k + 1);
    }
    u32x4 gp[3][2];
    auto xst = [&](auto mtc, int i) {
        constexpr int mt = decltype(mtc)::value;
        const int nt = (i >= 4) != (mt & 1) ? 2 : 0;
        const int j = i & 3;
        u32x4& g = gp[mt & 3][nt >> 1];
        if (j == 1) {
            g[0] = (__float_as_uint(acc[mt][nt][0]) + (mt < 4 ? 0u : 0x10000u)) & tmask;
            g[1] = pk_bf16(acc[mt][nt][2], acc[mt][nt][3]);
        } else if (j == 2) {
            g[0] = pk_bf16(__uint_as_float(g[0]), acc[mt][nt][1]);
            g[0] = mt < 4 ? g[0] | 1u : g[0] & ~1u;
        } else if (j == 3) {
            g[2] = pk_bf16(acc[mt][nt + 1][0], acc[mt][nt + 1][1]);
            g[3] = pk_bf16(acc[mt][nt + 1][2], acc[mt][nt + 1][3]);
            if (mt < 4) __builtin_amdgcn_raw_buffer_store_b128(g, xr, xw + (nt >> 1) * 1024, xo + mt * 8192, XST_AUX);
            else asm volatile("" : "+v"(g));
        }
    };
    u32x4 v[6];
    auto xld = [&](int b, int q) {
        v[2 * b + q] = __builtin_amdgcn_raw_buffer_load_b128(xr, xw + q * 1024, xo + b * 8192, 16);
    };
    unsigned ho = lds_base + LDS_HO + xv;
    asm("" : "+v"(ho));
    constexpr int S1 = STAGE_BYTES / 2048;
    BARRIER();
    SB();
    grp(I(3), I(1), I(0), 4, [&](int i) { rdb(I(1), i); });
    grp(I(4), I(0), I(0), 0, [&](int i) { rdb(I(1), i + 8); });
    grp(I(0), I(1), I(1), 1, none);
    grp(I(1), I(0), I(1), 2, [&](int i) { xst(I(0), i); });
    grp(I(2), I(1), I(1), 3, [&](int i) { xst(I(1), i); });
    unsigned l16;
    grp(I(3), I(0), I(1), 5, [&](int i) { xst(I(2), i); if (i == PREF) *(lds_u32_ptr)(uintptr_t)fhf = 1u; });
    grp(I(5), I(1), I(0), 6, [&](int i) {
        if (i >= 1 && i < 5) *(lds_f32x4_ptr)(uintptr_t)(ho + (4 - i) * 1024) = acc[3][4 - i];
        if (i == 5) *(lds_u32_ptr)(uintptr_t)fhf = 3u;
    });
    grp(I(6), I(0), I(0), 7, [&](int i) { if (i == 0) l16 = *(lds_u32_ptr)(uintptr_t)fl16;
                                          if (i == 6) l16 = __builtin_amdgcn_readfirstlane(l16); });
    if (__builtin_expect(l16 == 0u, 0))
        for (unsigned it = 0; it < 64 * POLL_LIMIT; ++it)
            if (__builtin_amdgcn_readfirstlane(*(lds_u32_ptr)(uintptr_t)fl16)) break;
    SB();
    grp(I(7), I(1), I(0), S1 + 4, none);
    grp(I(4), I(0), I(1), S1 + 5, none);
    grp(I(5), I(1), I(1), S1 + 6, [&](int i) { xst(I(4), i); });
    grp(I(6), I(0), I(1), S1 + 7, [&](int i) { xst(I(5), i); });
    grp(I(7), I(1), I(1), -1, [&](int i) { xst(I(6), i); });
    asm volatile("s_nop 5" : "+v"(acc[7][0]), "+v"(acc[7][1]), "+v"(acc[7][2]), "+v"(acc[7][3]));
    SB();
    STAMP(1);
    bool bad = false;
    for (unsigned it = 0, todo = 7; todo; ++it) {
        unsigned n = (__builtin_amdgcn_readfirstlane(*(lds_u32_ptr)(uintptr_t)(lds_base + LDS_RDY)) >> (wm * 8)) & todo;
        if (it > 64 * POLL_LIMIT) { n = todo; bad = true; }
#pragma unroll
        for (int b = 0; b < 3; ++b)
            if (n >> b & 1) {
#pragma unroll
                for (int q = 0; q < 2; ++q)
                    __builtin_amdgcn_raw_buffer_store_b128(gp[b][q], xr, xw + q * 1024, xp[b], XST_AUX);
            }
        todo &= ~n;
    }
    if (XSLEEP2) __builtin_amdgcn_s_sleep(XSLEEP2);
    unsigned pend = 7;
    auto pass = [&]() {
#pragma unroll
        for (int b = 0; b < 3; ++b)
            if (pend >> b & 1) { xld(b, 0); xld(b, 1); }
#pragma unroll
        for (int b = 0; b < 3; ++b)
            if (pend >> b & 1) {
                if (__all(((v[2 * b][0] | v[2 * b + 1][0]) & 1u) == 0u)) {
                    pend &= ~(1u << b);
                    add2(acc[7], sel0, sel1, v[2 * b], v[2 * b + 1]);
                }
            }
    };
    pass();
    for (unsigned it = 0; pend; ++it) {
        if ((it & 255u) == 255u) asm volatile("buffer_wbl2 sc1\n\ts_waitcnt vmcnt(0)" ::: "memory");
        asm volatile("" ::: "memory");
        pass();
        if (it > POLL_LIMIT) { bad = true; pend = 0; }
    }
    unsigned pav = *(lds_u32_ptr)(uintptr_t)(lds_base + LDS_RED);
    float am = 0.f;
#pragma unroll
    for (int i = 0; i < 4; ++i)
#pragma unroll
        for (int e = 0; e < 4; ++e) am = fmaxf(am, fabsf(acc[7][i][e]));
    unsigned av = __float_as_uint(am) | (bad ? 0xffffffffu : 0u);
    float asq = am * am;
    wave_red3(av, am, asq);
    publish_l63(my_ws, ws2, lg, av, call);
    const u32x2 tv = early_bar(am, asq, 2.05f);
    const unsigned amb = (unsigned)__builtin_amdgcn_readlane((int)av, 63);
    {
        asm volatile("" : "+v"(pav));
        const unsigned pa = __builtin_amdgcn_readfirstlane(pav);
        if (RECORD && amb > (pa != 0u ? max(pa, tv[1]) : tv[0]))
            asm volatile("global_store_dwordx2 %0, %1, off offset:8 sc0 sc1" :: "v"(xge), "v"(((unsigned long long)call << 32) | amb) : "memory");
    }
#pragma unroll
    for (int n = 0; n < 4; ++n) *(lds_f32x4_ptr)(uintptr_t)(ho + 16384 + n * 1024) = acc[7][n];
    asm volatile("" ::: "memory");
    *(lds_u32_ptr)(uintptr_t)(fhf + 16) = amb | 0x80000000u;
    STAMP(2);
    if (tid == 256) __hip_atomic_store(ctr, call, __ATOMIC_RELAXED, __HIP_MEMORY_SCOPE_AGENT);
    if (WBMODE != 2 && cw == 2 && idx == 1) {
        wait_amax(lds_base, call);
        __builtin_amdgcn_s_sleep(WBDELAY);
        asm volatile("buffer_wbl2 sc1" ::: "memory");
    }
}

}

#define PACE 0, 0, -1, 0, 0, 0, 0, 0
static const int g_pace[8] = {PACE};
extern "C" void kernel_launch(int rank, int nranks, ncclComm_t comm,
                              hipStream_t stream,
                              void* const* d_in, const int* in_sizes, int n_in,
                              void* d_out, int out_size,
                              const int* in_dtypes, int out_dtype,
                              void* const* d_out_all,
                              void* const* d_ws_all, size_t ws_bytes) {
    (void)comm; (void)in_dtypes; (void)out_dtype; (void)d_out;
    if (nranks != NR || n_in < 2 || (unsigned)rank >= (unsigned)NR || ws_bytes < WS_END ||
        in_sizes[0] != M_LOC * K_TOT || in_sizes[1] != K_TOT * N_TOT || out_size != N_TOT * N_LOC) {
        fprintf(stderr, "kernel_launch: unexpected problem shape\n");
        return;
    }
    Params p;
    p.x = d_in[0];
    p.w = d_in[1];
    for (int i = 0; i < NR; ++i) {
        p.out[i] = d_out_all[i];
        p.ws[i] = d_ws_all[i];
    }
    p.rank = rank;
    p.pad = 0;
    p.my_ws = d_ws_all[rank];
    for (int i = 0; i < 8; ++i) p.pace[i] = g_pace[i];
    const int sg[10] = {SG};
    {
        int seq[10], ns = 0;
        for (int d = sg[0]; d > 0; d /= 10) seq[ns++] = d % 10 - 1;
        for (int w = 0; w < NWG; ++w) {
            const int xcd = w & 7, tnl = ((w >> 3) + TROT) & 3, tm = (w >> 5) & 1, ks = w >> 6;
            int key[16];
            for (int k = 0; k < 16; ++k) {
                const int ph = k + sg[2] * ks + sg[3] * xcd;
                const int mine = seq[(ph + sg[1] * tm) % ns], other = seq[(ph + sg[1] * (1 - tm)) % ns];
                int lead = 0;
                if (mine == tnl && other == tnl) lead = ((ph / ns + tm) & 1) ? sg[4] : sg[6];
                else if (mine == tnl) lead = sg[4];
                else if (other == tnl) lead = 0;
                else lead = (((ph + tnl * sg[7]) >> sg[8]) + tm + sg[9]) & 1 ? sg[5] : 0;
                key[k] = (4 * ((k + ROT2 * ks + ROT3 * xcd) & 15) - 4 * lead - (lead ? 2 : 0)) * 16 + k;
            }
            unsigned long long t = 0;
            for (int k = 0; k < 16; ++k) {
                int pos = 0;
                for (int q = 0; q < 16; ++q) pos += key[q] < key[k];
                t |= (unsigned long long)k << (4 * pos);
            }
            p.ord[w] = t;
        }
    }
    hipLaunchKernelGGL(gemm_a2a_kernel, dim3(NWG), dim3(NTHREADS), 0, stream, p);
    HIP_CHECK(hipGetLastError());
}
```

```cpp
#include <hip/hip_runtime.h>
#include <rccl/rccl.h>
#include <cstdio>
#include <cstdint>

#define HIP_CHECK(expr) do { hipError_t _e = (expr); if (_e != hipSuccess) { \
    fprintf(stderr, "HIP error %s at %s:%d\n", hipGetErrorString(_e), __FILE__, __LINE__); \
    return; } } while (0)

#define SWAPX 0
#define TLW 2
#define SWIDX 0
#define PH1 19
#define SPEC 1
#define SPERIOD 256
#define WBMODE 3
#define XST_AUX 0
#define WBDELAY 48
#define POLLGATE 3
#define RECORD 1
#define SGRACE 0
#define XSLEEP 0
#define XSLEEP1 0
#define XSLEEP2 0
#define SPRIO 0
#define SWQUIET 1
#define SPINS 0
#define LPRIO 0
#define LATEHEAT 1
#define PREF 7
#define FSLEEP 16
#define PHDEL 2
#define LATEZ 8
#define TOUCH 0
#define TSLEEP 2
#define TMASK 2
#define TROT 0
#define ZZ 4.0f
#define ZTH2 4.4f
#define ZTH 4.9f
#define SG 13, 0, 2, 1, 2, 1, 3, 0, 1, 0
#define ROT2 3
#define ROT3 12
namespace {

typedef __attribute__((ext_vector_type(4))) short s16x4;
typedef __attribute__((ext_vector_type(8))) short s16x8;
typedef __attribute__((ext_vector_type(2))) __bf16 bf16x2;
typedef __attribute__((ext_vector_type(2))) float f32x2;
typedef __attribute__((ext_vector_type(4))) float f32x4;
typedef __attribute__((ext_vector_type(2))) unsigned u32x2;
typedef __attribute__((ext_vector_type(4))) unsigned u32x4;
typedef __attribute__((address_space(3))) s16x4* lds_s16x4_ptr;
typedef __attribute__((address_space(3))) s16x8* lds_s16x8_ptr;
typedef __attribute__((address_space(3))) f32x4* lds_f32x4_ptr;
typedef __attribute__((address_space(3))) volatile unsigned* lds_u32_ptr;

constexpr int NR = 8;
constexpr int M_LOC = 512;
constexpr int K_TOT = 4096;
constexpr int N_TOT = 4096;
constexpr int N_LOC = N_TOT / NR;
constexpr int BM = 256, BN = 128, BK = 64;
constexpr int NS = 3;
constexpr int A_BYTES = BM * BK * 2;
constexpr int B_BYTES = BK * BN * 2;
constexpr int STAGE_BYTES = A_BYTES + B_BYTES;
constexpr int NSTEP = 16;
constexpr int NWG = 256;
constexpr int NTHREADS = 512;
constexpr int LDS_CALL = NS * STAGE_BYTES;
constexpr int LDS_RED = LDS_CALL + 16;
constexpr int LDS_HF = LDS_RED + 16;
constexpr int LDS_RDY = LDS_CALL + 4;
constexpr int LDS_L16 = LDS_CALL + 8;
constexpr int LDS_DUMMY = LDS_HF + 32;
constexpr int LDS_BYTES = LDS_DUMMY + 256;
constexpr int LDS_HO = 2 * STAGE_BYTES;

constexpr size_t WS_XG = 0;
constexpr size_t WS_LGF = WS_XG + 2 * 8192;
constexpr size_t WS_LGS = WS_LGF + 16384;
constexpr size_t WS_DONE = WS_LGS + 16384;
constexpr size_t WS_CTR = WS_DONE + NWG * 128;
constexpr size_t WS_DIAG = WS_CTR + NWG * 128;
constexpr size_t WS_XB = 256 * 1024;
#define XB_STRIDE 24832
constexpr size_t WS_END = WS_XB + (size_t)NWG * XB_STRIDE;
static_assert(WS_DIAG + NWG * 16 * 8 <= WS_XB, "");

constexpr unsigned POLL_LIMIT = 1u << 20;

struct Params {
    const void* x;
    const void* w;
    void* my_ws;
    int rank;
    int pad;
    int pace[8];
    void* out[NR];
    void* ws[NR];
    unsigned long long ord[NWG];
};

__device__ __forceinline__ void glds16(unsigned voff, const void* sbase, unsigned lds_dst) {
    asm volatile("s_mov_b32 m0, %2\n\ts_nop 0\n\tglobal_load_lds_dwordx4 %0, %1"
                 :: "v"(voff), "s"(sbase), "s"(lds_dst) : "memory");
}
#define WAIT_VM(n) asm volatile("s_waitcnt vmcnt(%0)" :: "i"(n) : "memory")
#define WAIT_LGKM0() asm volatile("s_waitcnt lgkmcnt(0)" ::: "memory")
#define BARRIER() asm volatile("s_barrier" ::: "memory")
#define SB() __builtin_amdgcn_sched_barrier(0)
#define MFMA(acc, a, b) asm volatile("v_mfma_f32_16x16x32_bf16 %0, %1, %2, %0" : "+v"(acc) : "v"(a), "v"(b))
#define MFMA0(acc, a, b) asm volatile("v_mfma_f32_16x16x32_bf16 %0, %1, %2, 0" : "+v"(acc) : "v"(a), "v"(b))

__device__ __forceinline__ unsigned wave_max_l63(unsigned v) {
    v = max(v, (unsigned)__builtin_amdgcn_update_dpp(0, (int)v, 0x111, 0xf, 0xf, false));
    v = max(v, (unsigned)__builtin_amdgcn_update_dpp(0, (int)v, 0x112, 0xf, 0xf, false));
    v = max(v, (unsigned)__builtin_amdgcn_update_dpp(0, (int)v, 0x114, 0xf, 0xf, false));
    v = max(v, (unsigned)__builtin_amdgcn_update_dpp(0, (int)v, 0x118, 0xf, 0xf, false));
    v = max(v, (unsigned)__builtin_amdgcn_update_dpp(0, (int)v, 0x142, 0xa, 0xf, false));
    v = max(v, (unsigned)__builtin_amdgcn_update_dpp(0, (int)v, 0x143, 0xc, 0xf, false));
    return v;
}
__device__ __forceinline__ u32x2 early_bar(float sum, float sumsq, float unit) {
    const float mean = sum * (1.f / 64.f);
    const unsigned tz = __float_as_uint(mean + ZZ * __builtin_amdgcn_sqrtf(fabsf(sumsq * (1.f / 64.f) - mean * mean)));
    u32x2 r = {max(__float_as_uint(mean * (ZTH / unit)), tz), max(__float_as_uint(mean * (ZTH2 / unit)), tz)};
    r = (u32x2){(unsigned)__builtin_amdgcn_readlane((int)r[0], 63), (unsigned)__builtin_amdgcn_readlane((int)r[1], 63)};
    asm volatile("" : "+s"(r));
    return r;
}
__device__ __forceinline__ void wave_red3(unsigned& m, float& s, float& q) {
#define RED3(c) "v_max_u32_dpp %0, %0, %0 " c "\n\tv_add_f32_dpp %1, %1, %1 " c "\n\tv_add_f32_dpp %2, %2, %2 " c "\n\t"
    asm volatile("s_nop 1\n\t"
                 RED3("row_shr:1 row_mask:0xf bank_mask:0xf bound_ctrl:1") RED3("row_shr:2 row_mask:0xf bank_mask:0xf bound_ctrl:1")
                 RED3("row_shr:4 row_mask:0xf bank_mask:0xf bound_ctrl:1") RED3("row_shr:8 row_mask:0xf bank_mask:0xf bound_ctrl:1")
                 RED3("row_bcast:15 row_mask:0xa bank_mask:0xf") RED3("row_bcast:31 row_mask:0xc bank_mask:0xf") "s_nop 0"
                 : "+v"(m), "+v"(s), "+v"(q));
#undef RED3
}
__device__ __forceinline__ unsigned wave_max_u32(unsigned v) { return (unsigned)__builtin_amdgcn_readlane((int)wave_max_l63(v), 63); }
__device__ __forceinline__ void publish_l63(const char* ws, const char* ws2, unsigned off, unsigned v, unsigned call) {
    asm volatile("s_lshl_b64 exec, 1, 63\n\tglobal_store_dwordx2 %0, %1, %2\n\tglobal_store_dwordx2 %0, %1, %3 sc1\n\ts_mov_b64 exec, -1"
                 :: "v"(off), "v"((u32x2){v, call}), "s"(ws), "s"(ws2) : "memory", "scc");
}
__device__ __forceinline__ void add2(f32x4 (&y)[4], const u32x4& s0, const u32x4& s1, const u32x4& g0, const u32x4& g1) {
    asm volatile("s_nop 1\n\tv_mfma_f32_16x16x32_bf16 %0, %4, %6, %0\n\tv_mfma_f32_16x16x32_bf16 %1, %5, %6, %1\n\t"
                 "v_mfma_f32_16x16x32_bf16 %2, %4, %7, %2\n\tv_mfma_f32_16x16x32_bf16 %3, %5, %7, %3\n\ts_nop 7"
                 : "+v"(y[0]), "+v"(y[1]), "+v"(y[2]), "+v"(y[3]) : "v"(s0), "v"(s1), "v"(g0), "v"(g1));
}
__device__ __forceinline__ void make_sel(int lane, u32x4& s0, u32x4& s1) {
    const unsigned one = (lane >> 4) == ((lane & 15) >> 2) ? (lane & 1 ? 0x3f800000u : 0x3f80u) : 0u;
    const unsigned x0 = lane & 2 ? 0u : one, x1 = lane & 2 ? one : 0u;
    s0 = (u32x4){x0, x1, 0u, 0u};
    s1 = (u32x4){0u, 0u, x0, x1};
    asm volatile("s_nop 1" : "+v"(s0), "+v"(s1));
}

__device__ __forceinline__ unsigned pk_bf16(float a, float b) {
    const bf16x2 v = __builtin_convertvector((f32x2){a, b}, bf16x2);
    return __builtin_bit_cast(unsigned, v);
}
__device__ __forceinline__ unsigned pk_bf16_tag(float a, float b, unsigned tag) {
    const unsigned t = (__float_as_uint(a) + ((1u - tag) << 16)) & 0xfffe0000u;
    return (pk_bf16(__uint_as_float(t), b) & ~1u) | tag;
}
__device__ __forceinline__ u32x4 pack_granule(const f32x4& a, const f32x4& b, unsigned tag) {
    u32x4 g;
    g[0] = pk_bf16_tag(a[0], a[1], tag);
    g[1] = pk_bf16(a[2], a[3]);
    g[2] = pk_bf16(b[0], b[1]);
    g[3] = pk_bf16(b[2], b[3]);
    return g;
}
__device__ __forceinline__ float add_lo(unsigned w, float c) {
    unsigned one_lo = 0x00003f80u;
    asm("" : "+s"(one_lo));
    return __builtin_amdgcn_fdot2_f32_bf16(__builtin_bit_cast(bf16x2, w), __builtin_bit_cast(bf16x2, one_lo), c, false);
}
__device__ __forceinline__ float add_hi(unsigned w, float c) {
    return __builtin_amdgcn_fdot2_f32_bf16(__builtin_bit_cast(bf16x2, w), __builtin_bit_cast(bf16x2, 0x3f800000u), c, false);
}
__device__ __forceinline__ void add_granule(f32x4& a, f32x4& b, const u32x4& g) {
    a[0] = add_lo(g[0], a[0]); a[1] = add_hi(g[0], a[1]); a[2] = add_lo(g[1], a[2]); a[3] = add_hi(g[1], a[3]);
    b[0] = add_lo(g[2], b[0]); b[1] = add_hi(g[2], b[1]); b[2] = add_lo(g[3], b[2]); b[3] = add_hi(g[3], b[3]);
}
__device__ __forceinline__ void tr4(unsigned& r0, unsigned& r1, unsigned& r2, unsigned& r3) {
    u32x2 t;
    t = __builtin_amdgcn_permlane32_swap(r0, r2, false, false); r0 = t[0]; r2 = t[1];
    t = __builtin_amdgcn_permlane32_swap(r1, r3, false, false); r1 = t[0]; r3 = t[1];
    t = __builtin_amdgcn_permlane16_swap(r0, r1, false, false); r0 = t[0]; r1 = t[1];
    t = __builtin_amdgcn_permlane16_swap(r2, r3, false, false); r2 = t[0]; r3 = t[1];
}

#define NCYC 2
#define STAMPT(i, t) do {} while (0)
#define STAMP(i) STAMPT(i, 256)

__device__ __forceinline__ unsigned wait_amax(unsigned lds_base, unsigned call) {
    for (unsigned it = 0;; ++it) {
        u32x2 r;
        asm volatile("ds_read_b64 %0, %1\n\ts_waitcnt lgkmcnt(0)" : "=v"(r) : "v"(lds_base + LDS_RED) : "memory");
        if (__builtin_amdgcn_readfirstlane(r[1]) == call) return __builtin_amdgcn_readfirstlane(r[0]);
        if (it > 64 * POLL_LIMIT) return 0xffffffffu;
        if (SPINS) __builtin_amdgcn_s_sleep(4);
    }
}
__device__ __forceinline__ float inv_scale(unsigned gmax) {
    const float scale = __uint_as_float(gmax) / 448.0f;
    return scale > 1e-38f ? 1.0f / scale : 0.f;
}

__device__ __forceinline__ char* xg_entry(const Params& p, int lane, int g) {
    char* b = (char*)p.ws[0];
#pragma unroll
    for (int i = 1; i < NR; ++i) {
        char* w = (char*)p.ws[i];
        asm("" : "+s"(w));
        b = (lane >> 3) == i ? w : b;
    }
    return b + WS_XG + (lane & 7) * 1024 + (p.rank * 8 + g) * 16;
}

template <int V> struct IntC { static constexpr int value = V; };

__global__ void __launch_bounds__(NTHREADS, 2) gemm_a2a_kernel(const Params p) {
    __shared__ __attribute__((aligned(1024))) char lds[LDS_BYTES];
    const int tid = threadIdx.x;
    const int lane = tid & 63;
    const int wave = __builtin_amdgcn_readfirstlane(tid >> 6);
    const int wg = blockIdx.x;
    const int rank = p.rank;
    char* const my_ws = (char*)p.my_ws;
    const char* const gx = (const char*)p.x;
    const char* const gw = (const char*)p.w;
    const unsigned long long otab = p.ord[wg];

    const int xcd = wg & 7, idx = wg >> 3;
    const int tnl = (idx + TROT) & 3, tm = (idx >> 2) & 1, ks = idx >> 3;
    const int row0 = tm * BM;
    const int col0 = xcd * N_LOC + tnl * BN;

    const int of = 0;
    const int hi = wave & 1;
    unsigned voffA[8], voffB[4];
    {
        unsigned ve = (unsigned)(((wave & 2) * 64 + hi * 16 + (lane >> 3)) * (K_TOT * 2) + (((lane & 7) ^ (lane >> 4)) * 16));
        asm("" : "+v"(ve));
        unsigned vo = (ve ^ 64u) + 8 * (K_TOT * 2);
        asm("" : "+v"(vo));
#pragma unroll
        for (int i = 0; i < 8; ++i) voffA[i] = (i & 1 ? vo : ve) | (unsigned)((((i >> 1) ^ 3 ^ ks) & 3) * 32 * (K_TOT * 2));
        unsigned vb = (unsigned)(((wave & 3) * 16 + (lane >> 4)) * (N_TOT * 2) + (((lane & 15) ^ (2 * (lane >> 4))) * 16));
        asm("" : "+v"(vb));
        voffB[0] = vb; voffB[1] = vb | 4 * (N_TOT * 2); voffB[2] = vb ^ (8 * (N_TOT * 2) + 128); voffB[3] = vb ^ (12 * (N_TOT * 2) + 128);
    }
    asm volatile("" : "+v"(voffA[0]), "+v"(voffA[1]), "+v"(voffA[2]), "+v"(voffA[3]), "+v"(voffA[4]), "+v"(voffA[5]), "+v"(voffA[6]), "+v"(voffA[7]),
                      "+v"(voffB[0]), "+v"(voffB[1]), "+v"(voffB[2]), "+v"(voffB[3]));
    unsigned offA = row0 * (K_TOT * 2) + ks * NSTEP * (BK * 2), offB = col0 * 2 + ks * NSTEP * (BK * N_TOT * 2);
    unsigned dA = (unsigned)(uintptr_t)lds + wave * 8192, dB = (unsigned)(uintptr_t)lds + A_BYTES + wave * 4096;
    asm volatile("" : "+s"(offA), "+s"(offB), "+s"(dA), "+s"(dB));
    if (wave >= 4) {
        u32x4 ra[2];
#pragma unroll
        for (int i = 0; i < 8; ++i) ra[i >> 2][i & 3] = ((__umul24(tid, 0x9e3779u + i * 0x2c1b3u) + 0x5bd1e9u * i) & 0x80ff80ffu) | 0x3f003f00u;
        f32x4 d[4] = {};
#pragma unroll 1
        for (int r = 0; r < PH1; ++r) {
#pragma unroll
            for (int i = 0; i < 8; ++i) MFMA(d[i & 3], ra[i & 1], ra[(i >> 1) & 1]);
        }
        asm volatile("s_nop 7\n\ts_nop 7" :: "v"(d[0]), "v"(d[1]), "v"(d[2]), "v"(d[3]));
    }
    asm volatile("" :: "s"(gx), "s"(gw), "s"(my_ws), "s"(rank), "s"(otab));
    auto ktile = [&](int s) {
        return ((int)(otab >> (4 * (s & 15))) + of) & 15;
    };
    unsigned* ctr = (unsigned*)(my_ws + WS_CTR + (size_t)wg * 128);
    const unsigned lds_base = (unsigned)(uintptr_t)lds;

    if (__builtin_expect(wave < 4, 1)) {
        if (LPRIO) __builtin_amdgcn_s_setprio(LPRIO);
        const char* const ga = gx + offA;
        const char* const gb = gw + offB;
        auto issue_at = [&](unsigned ka, unsigned kb, unsigned so) {
            const char* a = ga + ka * (BK * 2);
            const char* b = gb + kb * (BK * N_TOT * 2);
#pragma unroll
            for (int j = 0; j < 4; ++j) {
                glds16(voffB[j], b, dB + so + j * 1024);
                glds16(voffA[2 * j], a, dA + so + 2 * j * 1024);
                glds16(voffA[2 * j + 1], a, dA + so + (2 * j + 1) * 1024);
            }
        };
        {
            const char* b = gb + (unsigned)ktile(0) * (BK * N_TOT * 2);
#pragma unroll
            for (int j = 0; j < 4; ++j) glds16(voffB[j], b, dB + j * 1024);
        }
        if (!hi) {
            const char* a = ga + (unsigned)ktile(0) * (BK * 2);
#pragma unroll
            for (int j = 0; j < 8; ++j) glds16(voffA[j], a, dA + j * 1024);
        }
        issue_at(ktile(1 - hi), ktile(1), STAGE_BYTES);
        int ordB = ktile(lane), ordA = ktile(lane - hi);
        asm volatile("" : "+v"(ordA), "+v"(ordB));
        auto issue = [&](int u, unsigned so) { issue_at(__builtin_amdgcn_readlane(ordA, u), __builtin_amdgcn_readlane(ordB, u), so); };
        unsigned so = 2 * STAGE_BYTES;
#define TLS(k)
        for (int k = 0; k < NSTEP - 2; ++k) {
            WAIT_VM(12);
            BARRIER();
            TLS(k);
            issue(k + 2, so);
            so = (so == 2 * STAGE_BYTES) ? 0 : so + STAGE_BYTES;
        }
        const bool tch = TOUCH && wave == 0 && (TMASK >> tnl & 1);
        auto touch = [&]() {
            const char* const xb = my_ws + WS_XB + (size_t)wg * XB_STRIDE;
            const unsigned tv = lane * 128;
            asm volatile("s_mov_b32 m0, %4\n\ts_nop 0\n\tglobal_load_lds_dword %0, %1\n\tglobal_load_lds_dword %0, %2\n\tglobal_load_lds_dword %0, %3"
                         :: "v"(tv), "s"(xb), "s"(xb + 8192), "s"(xb + 16384), "s"(lds_base + LDS_DUMMY) : "memory");
        };
        if (TOUCH == 13 && tch) { touch(); WAIT_VM(15); } else WAIT_VM(12);
        BARRIER();
        TLS(14);
        if (TOUCH == 14 && tch) { __builtin_amdgcn_s_sleep(TSLEEP); touch(); }
        if (hi) {
            const char* a = ga + (unsigned)__builtin_amdgcn_readlane(ordA, 16) * (BK * 2);
#pragma unroll
            for (int j = 0; j < 8; ++j) glds16(voffA[j], a, dA + STAGE_BYTES + j * 1024);
        }
        const unsigned call = __builtin_amdgcn_readfirstlane(*(const unsigned*)(lds + LDS_CALL));
        const int wm = wave & 1;
        const unsigned par = call & 1u;
        const unsigned xv = wm * 8192 + lane * 16;
        char* const xge = xg_entry(p, lane, xcd) + par * 8192;
        const int orow = rank * M_LOC + row0 + wm * 128 + ks * 32 + (lane & 15);
        char* const o = (char*)p.out[xcd] + (size_t)orow * N_LOC + tnl * BN + (lane >> 4) * 16;
        char* const o2 = o + 16 * N_LOC;
        char* const dflag = (char*)p.ws[xcd] + WS_DONE + (size_t)(rank * 32 + idx) * 128 + wm * 8;
        const unsigned lg = WS_LGF + xcd * 2048 + 1024 + (idx * 2 + wm) * 8;
        const char* const ws2 = my_ws + (WS_LGS - WS_LGF);
        const char* const gsrc = my_ws + WS_XG + par * 8192 + xcd * 1024 + lane * 16;
        asm volatile("" :: "v"(xge), "v"(o), "v"(o2), "v"(dflag), "v"(lg), "s"(ws2), "v"(xv), "v"(gsrc));
        if (hi) WAIT_VM(8); else if (tch) WAIT_VM(3); else WAIT_VM(0);
        BARRIER();
        TLS(15);
        if (hi) {
            WAIT_VM(0);
            asm volatile("ds_write_b32 %0, %1" :: "v"(lds_base + LDS_L16 + (wave >> 1) * 4), "v"(1u) : "memory");
        }
        if (wave >= 2) {
            const __amdgpu_buffer_rsrc_t xr = __builtin_amdgcn_make_buffer_rsrc(my_ws + WS_XB, 0, NWG * XB_STRIDE, 0x00020000);
            TLS(16);
            u32x4 v[12];
            const unsigned xw = wm * 4096 + lane * 16;
            auto xld = [&](int b) {
#pragma unroll
                for (int q = 0; q < 4; ++q)
                    v[4 * b + q] = __builtin_amdgcn_raw_buffer_load_b128(xr, xw + q * 1024, (wg ^ ((3 - b) << 6)) * XB_STRIDE + b * 8192, 16);
            };
            f32x4 y[2][2][4];
            u32x4 sel0, sel1;
            make_sel(lane, sel0, sel1);
            if (SPRIO) __builtin_amdgcn_s_setprio(SPRIO);
            bool bad = false;
            for (unsigned it = 0;; ++it) {
                u32x2 f;
                asm volatile("ds_read_b64 %0, %1\n\ts_waitcnt lgkmcnt(0)" : "=v"(f) : "v"(lds_base + LDS_HF + wm * 8) : "memory");
                if (__builtin_amdgcn_readfirstlane(f[0] & f[1])) break;
                if (it > 64 * POLL_LIMIT) { bad = true; break; }
                if (SPINS) __builtin_amdgcn_s_sleep(SPINS);
            }
            if (XSLEEP) __builtin_amdgcn_s_sleep(XSLEEP);
            xld(0); xld(1);
            if (XSLEEP1) __builtin_amdgcn_s_sleep(XSLEEP1);
            xld(2);
            for (unsigned it = 0;; ++it) {
                u32x2 f;
                asm volatile("ds_read_b64 %0, %1\n\ts_waitcnt lgkmcnt(0)" : "=v"(f) : "v"(lds_base + LDS_HF + wm * 8) : "memory");
                if (__builtin_amdgcn_readfirstlane(f[0] & f[1] & 2u)) break;
                if (it > 64 * POLL_LIMIT) { bad = true; break; }
            }
#pragma unroll
            for (int n = 0; n < 8; ++n)
                y[0][n >> 2][n & 3] = *(lds_f32x4_ptr)(uintptr_t)(lds_base + LDS_HO + xv + n * 1024);
            unsigned pend = 7;
            const unsigned frdy = lds_base + LDS_RDY + wm;
            auto pass = [&]() {
#pragma unroll
                for (int b = 0; b < 3; ++b)
                    if (pend >> b & 1) {
                        const unsigned tags = v[4 * b][0] & v[4 * b + 1][0] & v[4 * b + 2][0] & v[4 * b + 3][0];
                        if (__all(tags & 1u)) {
                            pend &= ~(1u << b);
                            asm volatile("ds_write_b8 %0, %1" :: "v"(frdy), "v"(pend ^ 7u) : "memory");
                        }
                    }
#pragma unroll
                for (int b = 0; b < 3; ++b)
                    if (pend >> b & 1) xld(b);
            };
            pass();
            for (unsigned it = 0; pend; ++it) {
                if ((it & 255u) == 255u) asm volatile("buffer_wbl2 sc1\n\ts_waitcnt vmcnt(0)" ::: "memory");
                pass();
                if (it > POLL_LIMIT) { bad = true; pend = 0; }
            }
            unsigned pav = *(lds_u32_ptr)(uintptr_t)(lds_base + LDS_RED);
#pragma unroll
            for (int q = 0; q < 12; ++q) {
                MFMA(y[0][(q >> 1) & 1][2 * (q & 1)], sel0, v[q]);
                MFMA(y[0][(q >> 1) & 1][2 * (q & 1) + 1], sel1, v[q]);
            }
            asm volatile("s_nop 7" : "+v"(y[0][0][0]), "+v"(y[0][0][1]), "+v"(y[0][0][2]), "+v"(y[0][0][3]),
                                     "+v"(y[0][1][0]), "+v"(y[0][1][1]), "+v"(y[0][1][2]), "+v"(y[0][1][3]));
            float am = 0.f;
#pragma unroll
            for (int i = 0; i < 8; ++i)
#pragma unroll
                for (int e = 0; e < 4; ++e) am = fmaxf(am, fabsf(y[0][i >> 2][i & 3][e]));
            unsigned av = __float_as_uint(am) | (bad ? 0xffffffffu : 0u);
            float asq = am * am;
            wave_red3(av, am, asq);
            publish_l63(my_ws, ws2, lg, av, call);
            const u32x2 tv = early_bar(am, asq, 2.327f);
            unsigned amb = (unsigned)__builtin_amdgcn_readlane((int)av, 63);
            unsigned told = tv[1];
            {
                asm volatile("" : "+v"(pav));
                const unsigned pa = __builtin_amdgcn_readfirstlane(pav);
                if (amb > max(pa, tv[0])) {
                    told = amb;
                    asm volatile("global_store_dwordx2 %0, %1, off offset:8 sc0 sc1" :: "v"(xge), "v"(((unsigned long long)call << 32) | amb) : "memory");
                }
            }
            if (lane == 0) asm volatile("ds_add_u32 %0, %1" :: "v"(lds_base + LDS_RED + 8), "v"(1u) : "memory");
            STAMPT(7, 128);
            auto send = [&](auto hc, unsigned val) {
                constexpr int h = decltype(hc)::value;
                const float inv = inv_scale(val);
                char* const dst = h ? o2 : o;
                u32x4 q[2];
#pragma unroll
                for (int c = 0; c < 2; ++c) {
                    unsigned r[4];
#pragma unroll
                    for (int j = 0; j < 4; ++j) {
                        int w = __builtin_amdgcn_cvt_pk_fp8_f32(y[h][c][j][0] * inv, y[h][c][j][1] * inv, 0, false);
                        w = __builtin_amdgcn_cvt_pk_fp8_f32(y[h][c][j][2] * inv, y[h][c][j][3] * inv, w, true);
                        r[j] = val == 0xffffffffu ? 0x7f7f7f7fu : (unsigned)w;
                    }
                    tr4(r[0], r[1], r[2], r[3]);
                    q[c] = (u32x4){r[0], r[1], r[2], r[3]};
                }
                asm volatile("global_store_dwordx4 %0, %1, off sc0 sc1\n\t"
                             "global_store_dwordx4 %0, %2, off offset:64 sc0 sc1\n\ts_nop 1" :: "v"(dst), "v"(q[0]), "v"(q[1]) : "memory");
            };
            unsigned s0 = 0xfffffffeu, s1 = 0xfffffffdu, flagged = 0xfffffffeu;
            bool have1 = false;
            unsigned ts = (unsigned)__builtin_amdgcn_s_memrealtime() - SGRACE;
            ts += (2u * SPERIOD + (unsigned)((idx * 2 + wm) * SPERIOD >> 6) - ts % SPERIOD) % SPERIOD;
            for (unsigned it = 0;; ++it) {
                u32x2 r, f;
                unsigned long long now;
                asm volatile("ds_read_b64 %0, %3\n\tds_read_b64 %1, %4\n\ts_memrealtime %2\n\ts_waitcnt lgkmcnt(0)"
                             : "=&v"(r), "=&v"(f), "=s"(now) : "v"(lds_base + LDS_RED), "v"(lds_base + LDS_HF + 16 + wm * 8) : "memory");
                unsigned pa = __builtin_amdgcn_readfirstlane(r[0]);
                const unsigned st = __builtin_amdgcn_readfirstlane(r[1]);
                bool fin = st == call;
                if (it > 64 * POLL_LIMIT) { fin = true; pa = 0xffffffffu; }
                if (!have1) {
                    const unsigned f0 = __builtin_amdgcn_readfirstlane(f[0]), f1 = __builtin_amdgcn_readfirstlane(f[1]);
                    if (f0 & f1 & 0x80000000u) {
                        const unsigned fm = max(f0, f1);
                        const unsigned a1 = fm == 0xffffffffu ? fm : fm & 0x7fffffffu;
                        if (pa != 0u) told = max(told, a1);
                        amb = max(amb, a1);
                        have1 = true;
                        if (lane == 0) asm volatile("ds_add_u32 %0, %1" :: "v"(lds_base + LDS_RED + 8), "v"(16u) : "memory");
#pragma unroll
                        for (int n = 0; n < 8; ++n)
                            y[1][n >> 2][n & 3] = *(lds_f32x4_ptr)(uintptr_t)(lds_base + LDS_HO + 16384 + xv + n * 1024);
                    }
                }
                if (!have1 && pa != 0xffffffffu) fin = false;
                if (RECORD && pa != 0u && amb > max(pa, told)) {
                    told = amb;
                    asm volatile("global_store_dwordx2 %0, %1, off offset:8 sc0 sc1" :: "v"(xge), "v"(((unsigned long long)call << 32) | amb) : "memory");
                }
                const unsigned val = fin ? pa : max(pa, amb);
                bool go = fin;
                if (SPEC && (int)((unsigned)now - ts) >= 0) {
                    go = go || (st == ~call && pa != 0u && (RECORD || pa >= amb));
                    if ((int)((unsigned)now - ts) >= SGRACE || (go && (s0 != val || (s1 != val && have1)))) ts += SPERIOD;
                }
                if (go && s0 != val) {
                    send(IntC<0>{}, val);
                    s0 = val;
                    go = fin;
                }
                if (go && s1 != val && (have1 || fin)) {
                    send(IntC<1>{}, val);
                    s1 = val;
                }
                if (s0 == s1 && s0 != flagged) {
                    flagged = s0;
                    asm volatile("s_waitcnt vmcnt(0)" ::: "memory");
                    STAMPT(4, 128);
                    if (lane == 0 && xcd != rank)
                        asm volatile("global_store_dwordx2 %0, %1, off sc0 sc1"
                                     :: "v"(dflag), "v"(((unsigned long long)call << 32) | s0) : "memory");
                }
                if (fin) break;
            }
            return;
        }
        if (wave == 0) {
            u32x4 gv;
            unsigned gmax = 0;
            auto gstate = [&]() {
                const bool ok = gv[1] == call, started = gv[3] == call;
                const unsigned long long seen = __ballot(started);
                const unsigned cand = max(ok ? gv[0] : 0u, started ? gv[2] : 0u);
                for (unsigned long long m = __ballot(cand > gmax); m; m = __ballot(cand > gmax))
                    gmax = (unsigned)__builtin_amdgcn_readlane((int)cand, __builtin_ctzll(m));
                const unsigned st = __all(ok) ? call : ((seen >> (8 * xcd)) & 0xffu) ? ~call : 0u;
                asm volatile("ds_write_b64 %0, %1" :: "v"(lds_base + LDS_RED), "v"(((unsigned long long)st << 32) | gmax) : "memory");
                return st == call;
            };
            if (idx != SWIDX || !SWQUIET) {
                asm volatile("global_load_dwordx4 %0, %1, off sc0 sc1\n\ts_waitcnt vmcnt(0)" : "=&v"(gv) : "v"(gsrc) : "memory");
                gstate();
            }
            if (idx == SWIDX) {
                const char* lgs = my_ws + WS_LGF + xcd * 2048 + lane * 16;
                const char* lgt = my_ws + WS_LGF + xcd * 2048 + 1024 + lane * 8;
                unsigned m0 = 0, m1 = 0, m2 = 0;
                bool ok0 = false, ok1 = false, ok2 = false;
                unsigned it = 0;
                for (;;) {
                    const unsigned alt = ((it & 63u) == 63u) ? (WS_LGS - WS_LGF) : 0;
                    u32x4 v;
                    u32x2 u;
                    asm volatile("global_load_dwordx4 %0, %2, off sc1\n\tglobal_load_dwordx2 %1, %3, off sc1\n\ts_waitcnt vmcnt(0)"
                                 : "=&v"(v), "=&v"(u) : "v"(lgs + alt), "v"(lgt + alt) : "memory");
                    if (v[1] == call) { m0 = v[0]; ok0 = true; }
                    if (v[3] == call) { m1 = v[2]; ok1 = true; }
                    if (u[1] == call) { m2 = u[0]; ok2 = true; }
                    if (__all(ok0 && ok1 && ok2)) break;
                    if (++it > 4 * POLL_LIMIT) { m0 = 0xffffffffu; break; }
                }
                const unsigned m = wave_max_u32(max(max(m0, m1), m2));
                asm volatile("global_store_dwordx2 %0, %1, off sc0 sc1"
                             :: "v"(xge), "v"(((unsigned long long)call << 32) | m) : "memory");
                STAMPT(6, 0);
            }
            for (unsigned it = 0; POLLGATE && it < 64 * POLL_LIMIT; ++it) {
                unsigned c;
                asm volatile("ds_read_b32 %0, %1\n\ts_waitcnt lgkmcnt(0)" : "=v"(c) : "v"(lds_base + (POLLGATE >= 3 ? LDS_RDY : LDS_RED + 8)) : "memory");
                c = __builtin_amdgcn_readfirstlane(c);
                if (POLLGATE == 3 ? c == 0x0707u : POLLGATE == 4 ? __builtin_popcount(c) >= 4 : POLLGATE == 5 ? c != 0u : POLLGATE == 1 ? (c & 15u) == 2u : c == 34u) break;
                if (SPINS) __builtin_amdgcn_s_sleep(2);
            }
            for (unsigned it = 0;;) {
                asm volatile("global_load_dwordx4 %0, %1, off sc0 sc1\n\ts_waitcnt vmcnt(0)"
                             : "=&v"(gv) : "v"(gsrc) : "memory");
                if (++it > 4 * POLL_LIMIT) { gv[0] = 0xffffffffu; gv[1] = call; }
                if (gstate()) break;
                if ((it & 255u) == 0u) asm volatile("buffer_wbl2 sc1\n\ts_waitcnt vmcnt(0)" ::: "memory");
            }
            return;
        }
        if ((wg >> 5) == rank) return;
        const unsigned long long want = ((unsigned long long)call << 32) | wait_amax(lds_base, call);
        __builtin_amdgcn_s_sleep(FSLEEP);
        const char* f = my_ws + WS_DONE + (size_t)wg * 128 + (lane & 1) * 8;
        for (unsigned it = 0;; ++it) {
            unsigned long long v;
            asm volatile("global_load_dwordx2 %0, %1, off sc0 sc1\n\ts_waitcnt vmcnt(0)" : "=&v"(v) : "v"(f) : "memory");
            if (__all(v == want)) break;
            if (it > POLL_LIMIT) {
                if (lane == 0)
                    __hip_atomic_store((unsigned char*)p.out[rank] + (size_t)rank * M_LOC * N_LOC,
                                       (unsigned char)0x7f, __ATOMIC_RELAXED, __HIP_MEMORY_SCOPE_SYSTEM);
                break;
            }
        }
        STAMPT(5, 64);
        return;
    }
    const int cw = wave - 4;
    const int wm = cw >> 1, wn = cw & 1;
    unsigned call = *(const __attribute__((address_space(4))) unsigned*)(uintptr_t)ctr;
    char* wsp[NR];
#pragma unroll
    for (int i = 0; i < NR; ++i) wsp[i] = (char*)p.ws[i];
    unsigned rdA[2], rdB[4];
#pragma unroll
    for (int kk = 0; kk < 2; ++kk) {
        const int r = lane & 15;
        const int c = 4 * kk + (lane >> 4);
        rdA[kk] = lds_base + (unsigned)(wm * 16384 + r * 128 + ((c ^ ((r >> 1) & 7)) * 16));
    }
#pragma unroll
    for (int cn = 0; cn < 4; ++cn) {
        const int g = lane >> 4, q = (lane & 15) >> 2, pp = lane & 3;
        const int k = 8 * g + q;
        const int h = (k & 3) | (((k >> 3) & 1) << 2);
        const int u = (wn * 4 + cn) ^ h;
        rdB[cn] = lds_base + (unsigned)(A_BYTES + k * 256 + u * 32 + pp * 8);
    }
    f32x4 acc[8][4];
#pragma unroll
    for (int i = 0; i < LATEZ; ++i) {
#pragma unroll
        for (int j = 0; j < 4; ++j) acc[i][j] = (f32x4){0.f, 0.f, 0.f, 0.f};
        asm volatile("s_nop 1" : "+v"(acc[i][0]), "+v"(acc[i][1]), "+v"(acc[i][2]), "+v"(acc[i][3]));
    }
    s16x8 aq[2][2];
    s16x4 bl[2][4][2], bh[2][4][2];
    asm volatile("" : "+v"(rdA[0]), "+v"(rdA[1]), "+v"(rdB[0]), "+v"(rdB[1]), "+v"(rdB[2]), "+v"(rdB[3]));
    __builtin_amdgcn_s_sleep(PHDEL);
    asm volatile("" : "+s"(call), "+s"(wsp[0]), "+s"(wsp[1]), "+s"(wsp[2]), "+s"(wsp[3]), "+s"(wsp[4]), "+s"(wsp[5]), "+s"(wsp[6]), "+s"(wsp[7]));

#define BFR(s, n, kk) __builtin_shufflevector(bl[s][n][kk], bh[s][n][kk], 0, 1, 2, 3, 4, 5, 6, 7)
#define RDA(kk, mt) (*(lds_s16x8_ptr)(uintptr_t)(rdA[kk] + (mt) * 2048))
#define RDB(kk, n, h) __builtin_amdgcn_ds_read_tr16_b64_v4i16((lds_s16x4_ptr)(uintptr_t)(rdB[n] + (kk) * 8192 + (h) * 1024))
    auto rdb = [&](auto sc, int j) {
        constexpr int s = decltype(sc)::value;
        const int n = j >> 2, kk = (j >> 1) & 1;
        if (j & 1) bh[s][n][kk] = RDB(kk, n, 1); else bl[s][n][kk] = RDB(kk, n, 0);
    };
    auto grp = [&](auto mtc, auto abc, auto bsc, int na, auto&& fill) {
        constexpr int mt = decltype(mtc)::value, ab = decltype(abc)::value, bs = decltype(bsc)::value;
#pragma unroll
        for (int i = 0; i < 8; ++i) {
            const int nt = (mt & 1) ? 3 - (i >> 1) : i >> 1, kk = (i & 1) ^ (mt & 1);
            MFMA(acc[mt][nt], BFR(bs, nt, kk), aq[ab][kk]);
            SB();
            if (i < 2 && na >= 0) aq[1 - ab][i] = RDA(i, na);
            fill(i);
            SB();
        }
    };
#define I(v) IntC<v>{}
    auto none = [](int) {};
    unsigned slot = 0, sd = 0;
    auto next_slot = [&](int i) {
        if (i == 0) { sd = slot == NS - 1 ? (unsigned)(-(NS - 1) * STAGE_BYTES) : (unsigned)STAGE_BYTES; slot = slot == NS - 1 ? 0 : slot + 1; }
        else if (i < 5) rdB[i - 1] += sd;
        else rdA[i - 5] += sd;
    };
    auto interval = [&](auto sc, int k) {
        constexpr int s = decltype(sc)::value;
        BARRIER();
        SB();
        grp(I(3), I(1), I(1 - s), 4, [&](int i) { if (i >= 2 && i < 6) rdb(sc, i - 2); });
        grp(I(4), I(0), I(1 - s), 5, [&](int i) { if (i >= 2 && i < 6) rdb(sc, i + 2); });
        grp(I(5), I(1), I(1 - s), 6, [&](int i) { if (i >= 2 && i < 6) rdb(sc, i + 6); });
        grp(I(6), I(0), I(1 - s), 7, [&](int i) { if (i >= 2 && i < 6) rdb(sc, i + 10); });
        grp(I(7), I(1), I(1 - s), 0, none);
        grp(I(0), I(0), sc, 1, none);
        grp(I(1), I(1), sc, 2, [&](int i) { if (i >= 2 && i < 7) next_slot(i - 2); });
        grp(I(2), I(0), sc, 3, [&](int i) { if (i == 2 || i == 4) next_slot(4 + (i >> 1)); });
        WAIT_LGKM0();
        SB();
    };
    BARRIER();
    SB();
    aq[0][0] = RDA(0, 0);
    rdb(I(0), 0); rdb(I(0), 1);
    SB();
    aq[0][1] = RDA(1, 0);
    SB();
#pragma unroll
    for (int j = 2; j < 16; ++j) { rdb(I(0), j); if (j & 1) SB(); }
    grp(I(0), I(0), I(0), 1, none);
    grp(I(1), I(1), I(0), 2, [&](int i) { if (i >= 2 && i < 7) next_slot(i - 2); });
    grp(I(2), I(0), I(0), 3, [&](int i) { if (i == 2 || i == 4) next_slot(4 + (i >> 1)); });
#pragma unroll
    for (int i = LATEZ; i < 8; ++i) {
#pragma unroll
        for (int j = 0; j < 4; ++j) acc[i][j] = (f32x4){0.f, 0.f, 0.f, 0.f};
        asm volatile("s_nop 1" : "+v"(acc[i][0]), "+v"(acc[i][1]), "+v"(acc[i][2]), "+v"(acc[i][3]));
    }
    const unsigned xv = cw * 4096 + lane * 16;
    const __amdgpu_buffer_rsrc_t xr = __builtin_amdgcn_make_buffer_rsrc(my_ws + WS_XB, 0, NWG * XB_STRIDE, 0x00020000);
    const unsigned xo = wg * XB_STRIDE;
    const unsigned xw = cw * 2048 + lane * 16;
    unsigned lg = WS_LGF + xcd * 2048 + (idx * 4 + cw) * 8;
    const char* ws2 = my_ws + (WS_LGS - WS_LGF);
    unsigned xp[3];
#pragma unroll
    for (int b = 0; b < 3; ++b) xp[b] = (wg ^ ((3 - b) << 6)) * XB_STRIDE + b * 8192;
    unsigned fl16 = lds_base + LDS_L16 + wm * 4, fhf = lds_base + LDS_HF + cw * 4;
    asm volatile("" : "+v"(lg), "+s"(ws2), "+s"(xp[0]), "+s"(xp[1]), "+s"(xp[2]), "+v"(fl16), "+v"(fhf));
    u32x4 sel0, sel1;
    make_sel(lane, sel0, sel1);
    unsigned tmask = 0xfffe0000u;
    ++call;
    if (tid < 256 + 4) *(u32x4*)(lds + LDS_CALL + lane * 16) = (u32x4){lane ? 0u : call, 0u, 0u, 0u};
    const unsigned par = call & 1u;
    char* xge = wsp[0];
#pragma unroll
    for (int i = 1; i < NR; ++i) xge = (lane >> 3) == i ? wsp[i] : xge;
    xge += WS_XG + (lane & 7) * 1024 + (rank * 8 + xcd) * 16 + par * 8192;
    asm volatile("" : "+v"(tmask), "+v"(xge) : "v"(xv), "v"(xw));
    if (cw == 0 && idx == SWIDX) {
        asm volatile("global_store_dwordx2 %0, %1, off offset:8 sc0 sc1" :: "v"(xge), "v"((unsigned long long)call << 32) : "memory");
    }
    WAIT_LGKM0();
    SB();
    for (int k = 1; k < NSTEP - 1; k += 2) {
        interval(I(1), k);
        interval(I(0), k + 1);
    }
    u32x4 gp[3][2];
    auto xst = [&](auto mtc, int i) {
        constexpr int mt = decltype(mtc)::value;
        const int nt = (i >= 4) != (mt & 1) ? 2 : 0;
        const int j = i & 3;
        u32x4& g = gp[mt & 3][nt >> 1];
        if (j == 1) {
            g[0] = (__float_as_uint(acc[mt][nt][0]) + (mt < 4 ? 0u : 0x10000u)) & tmask;
            g[1] = pk_bf16(acc[mt][nt][2], acc[mt][nt][3]);
        } else if (j == 2) {
            g[0] = pk_bf16(__uint_as_float(g[0]), acc[mt][nt][1]);
            g[0] = mt < 4 ? g[0] | 1u : g[0] & ~1u;
        } else if (j == 3) {
            g[2] = pk_bf16(acc[mt][nt + 1][0], acc[mt][nt + 1][1]);
            g[3] = pk_bf16(acc[mt][nt + 1][2], acc[mt][nt + 1][3]);
            if (mt < 4) __builtin_amdgcn_raw_buffer_store_b128(g, xr, xw + (nt >> 1) * 1024, xo + mt * 8192, XST_AUX);
            else asm volatile("" : "+v"(g));
        }
    };
    u32x4 v[6];
    auto xld = [&](int b, int q) {
        v[2 * b + q] = __builtin_amdgcn_raw_buffer_load_b128(xr, xw + q * 1024, xo + b * 8192, 16);
    };
    unsigned ho = lds_base + LDS_HO + xv;
    asm("" : "+v"(ho));
    constexpr int S1 = STAGE_BYTES / 2048;
    BARRIER();
    SB();
    grp(I(3), I(1), I(0), 4, [&](int i) { rdb(I(1), i); });
    grp(I(4), I(0), I(0), 0, [&](int i) { rdb(I(1), i + 8); });
    grp(I(0), I(1), I(1), 1, none);
    grp(I(1), I(0), I(1), 2, [&](int i) { xst(I(0), i); });
    grp(I(2), I(1), I(1), 3, [&](int i) { xst(I(1), i); });
    unsigned l16;
    grp(I(3), I(0), I(1), 5, [&](int i) { xst(I(2), i); if (i == PREF) *(lds_u32_ptr)(uintptr_t)fhf = 1u; });
    grp(I(5), I(1), I(0), 6, [&](int i) {
        if (i >= 1 && i < 5) *(lds_f32x4_ptr)(uintptr_t)(ho + (4 - i) * 1024) = acc[3][4 - i];
        if (i == 5) *(lds_u32_ptr)(uintptr_t)fhf = 3u;
    });
    grp(I(6), I(0), I(0), 7, [&](int i) { if (i == 0) l16 = *(lds_u32_ptr)(uintptr_t)fl16;
                                          if (i == 6) l16 = __builtin_amdgcn_readfirstlane(l16); });
    if (__builtin_expect(l16 == 0u, 0))
        for (unsigned it = 0; it < 64 * POLL_LIMIT; ++it)
            if (__builtin_amdgcn_readfirstlane(*(lds_u32_ptr)(uintptr_t)fl16)) break;
    SB();
    grp(I(7), I(1), I(0), S1 + 4, none);
    grp(I(4), I(0), I(1), S1 + 5, none);
    grp(I(5), I(1), I(1), S1 + 6, [&](int i) { xst(I(4), i); });
    grp(I(6), I(0), I(1), S1 + 7, [&](int i) { xst(I(5), i); });
    grp(I(7), I(1), I(1), -1, [&](int i) { xst(I(6), i); });
    asm volatile("s_nop 5" : "+v"(acc[7][0]), "+v"(acc[7][1]), "+v"(acc[7][2]), "+v"(acc[7][3]));
    SB();
    STAMP(1);
    bool bad = false;
    for (unsigned it = 0, todo = 7; todo; ++it) {
        unsigned n = (__builtin_amdgcn_readfirstlane(*(lds_u32_ptr)(uintptr_t)(lds_base + LDS_RDY)) >> (wm * 8)) & todo;
        if (it > 64 * POLL_LIMIT) { n = todo; bad = true; }
#pragma unroll
        for (int b = 0; b < 3; ++b)
            if (n >> b & 1) {
#pragma unroll
                for (int q = 0; q < 2; ++q)
                    __builtin_amdgcn_raw_buffer_store_b128(gp[b][q], xr, xw + q * 1024, xp[b], XST_AUX);
            }
        todo &= ~n;
    }
    if (XSLEEP2) __builtin_amdgcn_s_sleep(XSLEEP2);
    unsigned pend = 7;
    auto pass = [&]() {
#pragma unroll
        for (int b = 0; b < 3; ++b)
            if (pend >> b & 1) { xld(b, 0); xld(b, 1); }
#pragma unroll
        for (int b = 0; b < 3; ++b)
            if (pend >> b & 1) {
                if (__all(((v[2 * b][0] | v[2 * b + 1][0]) & 1u) == 0u)) {
                    pend &= ~(1u << b);
                    add2(acc[7], sel0, sel1, v[2 * b], v[2 * b + 1]);
                }
            }
    };
    pass();
    for (unsigned it = 0; pend; ++it) {
        if ((it & 255u) == 255u) asm volatile("buffer_wbl2 sc1\n\ts_waitcnt vmcnt(0)" ::: "memory");
        asm volatile("" ::: "memory");
        pass();
        if (it > POLL_LIMIT) { bad = true; pend = 0; }
    }
    unsigned pav = *(lds_u32_ptr)(uintptr_t)(lds_base + LDS_RED);
    float am = 0.f;
#pragma unroll
    for (int i = 0; i < 4; ++i)
#pragma unroll
        for (int e = 0; e < 4; ++e) am = fmaxf(am, fabsf(acc[7][i][e]));
    unsigned av = __float_as_uint(am) | (bad ? 0xffffffffu : 0u);
    float asq = am * am;
    wave_red3(av, am, asq);
    publish_l63(my_ws, ws2, lg, av, call);
    const u32x2 tv = early_bar(am, asq, 2.05f);
    const unsigned amb = (unsigned)__builtin_amdgcn_readlane((int)av, 63);
    {
        asm volatile("" : "+v"(pav));
        const unsigned pa = __builtin_amdgcn_readfirstlane(pav);
        if (RECORD && amb > (pa != 0u ? max(pa, tv[1]) : tv[0]))
            asm volatile("global_store_dwordx2 %0, %1, off offset:8 sc0 sc1" :: "v"(xge), "v"(((unsigned long long)call << 32) | amb) : "memory");
    }
#pragma unroll
    for (int n = 0; n < 4; ++n) *(lds_f32x4_ptr)(uintptr_t)(ho + 16384 + n * 1024) = acc[7][n];
    asm volatile("" ::: "memory");
    *(lds_u32_ptr)(uintptr_t)(fhf + 16) = amb | 0x80000000u;
    STAMP(2);
    if (tid == 256) __hip_atomic_store(ctr, call, __ATOMIC_RELAXED, __HIP_MEMORY_SCOPE_AGENT);
    if (WBMODE != 2 && cw == 2 && idx == 1) {
        wait_amax(lds_base, call);
        __builtin_amdgcn_s_sleep(WBDELAY);
        asm volatile("buffer_wbl2 sc1" ::: "memory");
    }
}

}

#define PACE 0, 0, -1, 0, 0, 0, 0, 0
static const int g_pace[8] = {PACE};
extern "C" void kernel_launch(int rank, int nranks, ncclComm_t comm,
                              hipStream_t stream,
                              void* const* d_in, const int* in_sizes, int n_in,
                              void* d_out, int out_size,
                              const int* in_dtypes, int out_dtype,
                              void* const* d_out_all,
                              void* const* d_ws_all, size_t ws_bytes) {
    (void)comm; (void)in_dtypes; (void)out_dtype; (void)d_out;
    if (nranks != NR || n_in < 2 || (unsigned)rank >= (unsigned)NR || ws_bytes < WS_END ||
        in_sizes[0] != M_LOC * K_TOT || in_sizes[1] != K_TOT * N_TOT || out_size != N_TOT * N_LOC) {
        fprintf(stderr, "kernel_launch: unexpected problem shape\n");
        return;
    }
    Params p;
    p.x = d_in[0];
    p.w = d_in[1];
    for (int i = 0; i < NR; ++i) {
        p.out[i] = d_out_all[i];
        p.ws[i] = d_ws_all[i];
    }
    p.rank = rank;
    p.pad = 0;
    p.my_ws = d_ws_all[rank];
    for (int i = 0; i < 8; ++i) p.pace[i] = g_pace[i];
    const int sg[10] = {SG};
    {
        int seq[10], ns = 0;
        for (int d = sg[0]; d > 0; d /= 10) seq[ns++] = d % 10 - 1;
        for (int w = 0; w < NWG; ++w) {
            const int xcd = w & 7, tnl = ((w >> 3) + TROT) & 3, tm = (w >> 5) & 1, ks = w >> 6;
            int key[16];
            for (int k = 0; k < 16; ++k) {
                const int ph = k + sg[2] * ks + sg[3] * xcd;
                const int mine = seq[(ph + sg[1] * tm) % ns], other = seq[(ph + sg[1] * (1 - tm)) % ns];
                int lead = 0;
                if (mine == tnl && other == tnl) lead = ((ph / ns + tm) & 1) ? sg[4] : sg[6];
                else if (mine == tnl) lead = sg[4];
                else if (other == tnl) lead = 0;
                else lead = (((ph + tnl * sg[7]) >> sg[8]) + tm + sg[9]) & 1 ? sg[5] : 0;
                key[k] = (4 * ((k + ROT2 * ks + ROT3 * xcd) & 15) - 4 * lead - (lead ? 2 : 0)) * 16 + k;
            }
            unsigned long long t = 0;
            for (int k = 0; k < 16; ++k) {
                int pos = 0;
                for (int q = 0; q < 16; ++q) pos += key[q] < key[k];
                t |= (unsigned long long)k << (4 * pos);
            }
            p.ord[w] = t;
        }
    }
    hipLaunchKernelGGL(gemm_a2a_kernel, dim3(NWG), dim3(NTHREADS), 0, stream, p);
    HIP_CHECK(hipGetLastError());
}
```
